```python
import numpy as np

kernel_name = "gpu_gemm_rs_m4096_k4096_n16384_f32_q8"


ABSMAX_RTOL = 0.02

COMPUTE_DTYPE = "float32"
EPILOGUE = "quant_int8"
IN_QUANT = None
OUT_QUANT = "int8"
IN_DTYPES = {"x": "float32", "w_mat": "float32"}
WORLD_SIZE = 8

def setup_inputs():

    x = np.random.randn(4096, 4096).astype(np.float32)
    w_mat = (np.random.randn(4096, 16384) / np.sqrt(4096)).astype(np.float32)
    return {"x": x, "w_mat": w_mat}

def _quant_dequant(y, amax):
    scale = amax / 127.0
    q = np.clip(np.round(y / scale), -127.0, 127.0)
    return (q * scale).astype(np.float32)
def _epilogue(y):
    return _quant_dequant(y, np.abs(y).max())

def reference(x, w_mat):
    y = (x @ w_mat).astype(np.float32)
    return _epilogue(y)
```

```cpp
#include <hip/hip_runtime.h>
#include <rccl/rccl.h>
#include <cstdio>
#include <cstdint>

#define HIP_CHECK(expr) do { hipError_t _e = (expr); if (_e != hipSuccess) { \
    fprintf(stderr, "HIP error %s at %s:%d\n", hipGetErrorString(_e), __FILE__, __LINE__); \
    return; } } while (0)

typedef __attribute__((ext_vector_type(8))) short bf16x8;
typedef __attribute__((ext_vector_type(4))) float f32x4;
typedef __attribute__((ext_vector_type(4))) unsigned u32x4;
typedef unsigned long long u64;
#define GAS __attribute__((address_space(1)))
typedef GAS char gchar;
typedef GAS unsigned gu32;
typedef GAS unsigned long long gu64;
typedef GAS u32x4 gu32x4;

namespace {
constexpr int NR = 8;
constexpr int KSH = 512;
constexpr int NTOT = 16384;
constexpr int NLOC = 2048;
constexpr int MSH = 512;
constexpr int NSTAGE = 8;
constexpr int XU = 32;
constexpr int WU = 16;
constexpr int UNIT = 16384;
constexpr int NWG = 256;
constexpr int NSTEP = 64;
constexpr int NSLOT = 3;
constexpr int SLOT = 3 * UNIT;
constexpr int NTHREADS = 768;

constexpr size_t OFF_TICKET = 0;
constexpr size_t OFF_TMO = 128;
constexpr size_t OFF_AMAX = 256;
constexpr size_t OFF_ARRIVE = 384;
constexpr size_t OFF_SLOTS = 512;
constexpr size_t OFF_OUTF = 1024;
constexpr size_t OFF_XF = 4096;
constexpr size_t OFF_WF = OFF_XF + XU * 64 * 4;
constexpr size_t OFF_XS = 1u << 20;
constexpr size_t OFF_WS = OFF_XS + (size_t)NR * NSTAGE * XU * UNIT;
constexpr size_t WS_NEED = OFF_WS + (size_t)NR * NSTAGE * WU * UNIT;

constexpr u64 DEADLINE_TICKS = 40000000ull;

constexpr int C_CALL = 0, C_FULL = 2, C_DONE = 8, C_WMAX = 16, C_GMAX = 24, C_DEAD = 25;
constexpr int LDS_CTL = NSLOT * SLOT;
constexpr int LDS_BYTES = LDS_CTL + 256;
}

struct Params {
  const float* x;
  const float* w;
  void* const* out_all;
  void* const* ws_all;
  int rank;
};

__device__ __forceinline__ unsigned pk_bf16(float a, float b) {
  typedef __attribute__((ext_vector_type(2))) float f2;
  typedef __attribute__((ext_vector_type(2))) __bf16 b2;
  f2 v = {a, b};
  b2 r = __builtin_convertvector(v, b2);
  return __builtin_bit_cast(unsigned, r);
}

__device__ __forceinline__ void glds4k(const void* g, unsigned ldsaddr) {
  unsigned keep;
  asm volatile(
      "s_mov_b32 %0, m0\n\ts_mov_b32 m0, %2\n\ts_nop 0\n\t"
      "global_load_lds_dwordx4 %1, off\n\t"
      "global_load_lds_dwordx4 %1, off offset:1024\n\t"
      "global_load_lds_dwordx4 %1, off offset:2048\n\t"
      "global_load_lds_dwordx4 %1, off offset:3072\n\t"
      "s_mov_b32 m0, %0"
      : "=&s"(keep) : "v"(g), "s"(ldsaddr) : "memory");
}

__device__ __forceinline__ unsigned lds_ld(unsigned addr) {
  unsigned v;
  asm volatile("ds_read_b32 %0, %1\n\ts_waitcnt lgkmcnt(0)" : "=v"(v) : "v"(addr) : "memory");
  return v;
}
__device__ __forceinline__ unsigned lds_ld_min2(unsigned addr) {
  typedef __attribute__((ext_vector_type(2))) unsigned u32x2;
  u32x2 v;
  asm volatile("ds_read_b64 %0, %1\n\ts_waitcnt lgkmcnt(0)" : "=v"(v) : "v"(addr) : "memory");
  return v[0] < v[1] ? v[0] : v[1];
}
__device__ __forceinline__ void lds_st(unsigned addr, unsigned v) {
  asm volatile("ds_write_b32 %0, %1" :: "v"(addr), "v"(v) : "memory");
}

__device__ __forceinline__ void st16(gchar* p, u32x4 v) {
  *(gu32x4*)p = v;
}

__device__ __forceinline__ void publish_u32(gu32* flag, unsigned val, bool leader) {
  asm volatile("s_waitcnt vmcnt(0)" ::: "memory");
  __builtin_amdgcn_fence(__ATOMIC_RELEASE, "");
  asm volatile("s_waitcnt vmcnt(0)" ::: "memory");
  if (leader) __hip_atomic_store(flag, val, __ATOMIC_RELAXED, __HIP_MEMORY_SCOPE_SYSTEM);
}

__global__ __launch_bounds__(NTHREADS, 3) void fused_gemm_q8(Params P) {
  __shared__ __attribute__((aligned(1024))) char lds[LDS_BYTES];
  const unsigned ldsbase = (unsigned)(uintptr_t)lds;
  const unsigned ctl = ldsbase + LDS_CTL;

  const int tid = threadIdx.x;
  const int lane = tid & 63;
  const int wave = __builtin_amdgcn_readfirstlane(tid >> 6);
  const int rank = P.rank;
  char* const myws = static_cast<char*>(P.ws_all[rank]);
  unsigned* const tmo = reinterpret_cast<unsigned*>(myws + OFF_TMO);

  if (tid < 64) {
    unsigned v = 0;
    if (tid == 0) {
      unsigned t = __hip_atomic_fetch_add(reinterpret_cast<unsigned*>(myws + OFF_TICKET), 1u,
                                          __ATOMIC_RELAXED, __HIP_MEMORY_SCOPE_AGENT);
      v = t / NWG + 1;
    }
    lds_st(ctl + tid * 4, v);
  }
  asm volatile("s_waitcnt lgkmcnt(0)" ::: "memory");
  __syncthreads();
  const unsigned call = __builtin_amdgcn_readfirstlane(lds_ld(ctl + C_CALL * 4));

  const int bx = blockIdx.x & 7, bj = blockIdx.x >> 3;
  const int ti = 4 * (bx >> 1) + (bj & 3);
  const int tj = 8 * (bx & 1) + (bj >> 2);

  f32x4 acc[4][4];
  const int wm = wave & 3, wn = (wave >> 2) & 1;

  if (wave < 8) {
#pragma unroll
    for (int a = 0; a < 4; ++a)
#pragma unroll
      for (int b = 0; b < 4; ++b) acc[a][b] = f32x4{0.f, 0.f, 0.f, 0.f};
    const int xoff = (wm >> 1) * UNIT + ((wm & 1) * 4) * 1024 + lane * 16;
    const int woff = 2 * UNIT + (wn * 4) * 1024 + lane * 16;
    int slot = 0;
    for (int t = 0; t < NSTEP; ++t) {
      while (true) {
        if (lds_ld_min2(ctl + C_FULL * 4) > (unsigned)t) break;
        __builtin_amdgcn_s_sleep(1);
      }
      const char* sb = lds + slot * SLOT;
      bf16x8 xf[2][4], wf[2][4];
#pragma unroll
      for (int kb = 0; kb < 2; ++kb) {
#pragma unroll
        for (int i = 0; i < 4; ++i) {
          wf[kb][i] = *reinterpret_cast<const bf16x8*>(sb + woff + (kb * 8 + i) * 1024);
          xf[kb][i] = *reinterpret_cast<const bf16x8*>(sb + xoff + (kb * 8 + i) * 1024);
        }
      }
#pragma unroll
      for (int mt = 0; mt < 4; ++mt)
#pragma unroll
        for (int nt = 0; nt < 4; ++nt)
          acc[mt][nt] = __builtin_amdgcn_mfma_f32_16x16x32_bf16(wf[0][nt], xf[0][mt], acc[mt][nt], 0, 0, 0);
      asm volatile("s_waitcnt lgkmcnt(0)" ::: "memory");
      if (lane == 0) lds_st(ctl + (C_DONE + wave) * 4, (unsigned)(t + 1));
#pragma unroll
      for (int mt = 0; mt < 4; ++mt)
#pragma unroll
        for (int nt = 0; nt < 4; ++nt)
          acc[mt][nt] = __builtin_amdgcn_mfma_f32_16x16x32_bf16(wf[1][nt], xf[1][mt], acc[mt][nt], 0, 0, 0);
      slot = (slot == NSLOT - 1) ? 0 : slot + 1;
    }
    float m = 0.f;
#pragma unroll
    for (int a = 0; a < 4; ++a)
#pragma unroll
      for (int b = 0; b < 4; ++b)
#pragma unroll
        for (int j = 0; j < 4; ++j) m = fmaxf(m, fabsf(acc[a][b][j]));
#pragma unroll
    for (int o = 32; o >= 1; o >>= 1) m = fmaxf(m, __shfl_xor(m, o));
    if (lane == 0) lds_st(ctl + (C_WMAX + wave) * 4, __float_as_uint(m));
  } else if (wave < 10) {
    const int L = wave - 8;
    const unsigned* pxf = reinterpret_cast<const unsigned*>(myws + OFF_XF) + (2 * ti + L) * 64 + lane;
    const unsigned* pwf = reinterpret_cast<const unsigned*>(myws + OFF_WF) + tj * 64 + lane;
    u64 ready = 0;
    bool dead = false;
    const u64 t0 = __builtin_amdgcn_s_memrealtime();
    int slot = 0;
    for (int t = 0; t < NSTEP; ++t) {
      int p, s;
      if (t < NSTAGE) { p = rank; s = t; }
      else { s = (t - NSTAGE) / 7; p = (rank + 1 + (t - NSTAGE) % 7) & 7; }
      const int bit = p * 8 + s;
      if (!((ready >> bit) & 1) && !dead) {
        asm volatile("s_waitcnt vmcnt(0)" ::: "memory");
        if (lane == 0) lds_st(ctl + (C_FULL + L) * 4, (unsigned)t);
        while (true) {
          unsigned fx, fw;
          asm volatile("global_load_dword %0, %2, off sc0 sc1\n\tglobal_load_dword %1, %3, off sc0 sc1\n\t"
                       "s_waitcnt vmcnt(0)"
                       : "=&v"(fx), "=&v"(fw) : "v"(pxf), "v"(pwf) : "memory");
          ready = __ballot(fx >= call && fw >= call);
          if ((ready >> bit) & 1) break;
          if (__hip_atomic_load(tmo, __ATOMIC_RELAXED, __HIP_MEMORY_SCOPE_AGENT) == call ||
              __builtin_amdgcn_s_memrealtime() - t0 > DEADLINE_TICKS) {
            if (lane == 0) __hip_atomic_store(tmo, call, __ATOMIC_RELAXED, __HIP_MEMORY_SCOPE_AGENT);
            dead = true;
            break;
          }
          __builtin_amdgcn_s_sleep(2);
        }
        asm volatile("buffer_inv sc0 sc1" ::: "memory");
      }
      if (t >= NSLOT) {
        while (true) {
          unsigned d = lds_ld(ctl + (C_DONE + (lane & 7)) * 4);
          if (__all(d >= (unsigned)(t - NSLOT + 1))) break;
          __builtin_amdgcn_s_sleep(1);
        }
      }
      unsigned fx, fw;
      asm volatile("global_load_dword %0, %2, off sc0 sc1\n\tglobal_load_dword %1, %3, off sc0 sc1"
                   : "=&v"(fx), "=&v"(fw) : "v"(pxf), "v"(pwf) : "memory");
      const char* gx = myws + OFF_XS + ((size_t)((p * NSTAGE + s) * XU + 2 * ti + L)) * UNIT + lane * 16;
      const char* gw = myws + OFF_WS + ((size_t)((p * NSTAGE + s) * WU + tj)) * UNIT + L * 8192 + lane * 16;
      const unsigned lx = ldsbase + slot * SLOT + L * UNIT;
      const unsigned lw = ldsbase + slot * SLOT + 2 * UNIT + L * 8192;
#pragma unroll
      for (int i = 0; i < 4; ++i) glds4k(gx + i * 4096, lx + i * 4096);
#pragma unroll
      for (int i = 0; i < 2; ++i) glds4k(gw + i * 4096, lw + i * 4096);
      asm volatile("s_waitcnt vmcnt(24)" : "+v"(fx), "+v"(fw) :: "memory");
      if (lane == 0) lds_st(ctl + (C_FULL + L) * 4, (unsigned)t);
      u64 nr = __ballot(fx >= call && fw >= call);
      if (nr & ~ready) {
        ready |= nr;
        asm volatile("buffer_inv sc0 sc1" ::: "memory");
      }
      slot = (slot == NSLOT - 1) ? 0 : slot + 1;
    }
    asm volatile("s_waitcnt vmcnt(0)" ::: "memory");
    if (lane == 0) lds_st(ctl + (C_FULL + L) * 4, (unsigned)NSTEP);
  } else {
    const int sw = blockIdx.x * 2 + (wave - 10);
    for (int i = 0; i < 6; ++i) {
      const int u = sw + 512 * i;
      int d, s, item;
      if (u < 336) { s = 0; d = (rank + 1 + u % 7) & 7; item = u / 7; }
      else if (u < 720) { int j = u - 336; s = j / 48; item = j % 48; d = rank; }
      else { int j = u - 720; s = 1 + j / 336; j %= 336; d = (rank + 1 + j % 7) & 7; item = j / 7; }
      gchar* dws = (gchar*)(P.ws_all[(unsigned)d < (unsigned)NR ? d : rank]);
      const int q = item / 3, rr = item - 3 * q;
      if (rr != 2) {
        const int xu = 2 * q + rr;
        const float* src = P.x + (size_t)(xu * 128 + (lane & 15)) * KSH + s * 64 + 8 * (lane >> 4);
        gchar* dst = dws + OFF_XS + ((size_t)((rank * NSTAGE + s) * XU + xu)) * UNIT + lane * 16;
#pragma unroll
        for (int kb = 0; kb < 2; ++kb) {
          f32x4 a[8], b[8];
#pragma unroll
          for (int blk = 0; blk < 8; ++blk) {
            const float* sp = src + (size_t)blk * 16 * KSH + kb * 32;
            a[blk] = *reinterpret_cast<const f32x4*>(sp);
            b[blk] = *reinterpret_cast<const f32x4*>(sp + 4);
          }
#pragma unroll
          for (int blk = 0; blk < 8; ++blk) {
            u32x4 v = {pk_bf16(a[blk][0], a[blk][1]), pk_bf16(a[blk][2], a[blk][3]),
                       pk_bf16(b[blk][0], b[blk][1]), pk_bf16(b[blk][2], b[blk][3])};
            st16(dst + (kb * 8 + blk) * 1024, v);
          }
        }
        gu32* flag = (gu32*)(dws + OFF_XF) + xu * 64 + rank * 8 + s;
        publish_u32(flag, call, lane == 0);
      } else {
        const int wu = q;
        const float* src = P.w + (size_t)(s * 64 + 8 * (lane >> 4)) * NTOT + d * NLOC + wu * 128 +
                           ((lane & 15) >> 2) * 16 + (lane & 3);
        gchar* dst = dws + OFF_WS + ((size_t)((rank * NSTAGE + s) * WU + wu)) * UNIT + lane * 16;
#pragma unroll
        for (int g = 0; g < 4; ++g) {
          const int kb = g >> 1, hf = g & 1;
          float v[4][8];
#pragma unroll
          for (int nt = 0; nt < 4; ++nt)
#pragma unroll
            for (int j = 0; j < 8; ++j)
              v[nt][j] = src[(size_t)(kb * 32 + j) * NTOT + hf * 64 + nt * 4];
#pragma unroll
          for (int nt = 0; nt < 4; ++nt) {
            u32x4 o = {pk_bf16(v[nt][0], v[nt][1]), pk_bf16(v[nt][2], v[nt][3]),
                       pk_bf16(v[nt][4], v[nt][5]), pk_bf16(v[nt][6], v[nt][7])};
            st16(dst + (kb * 8 + hf * 4 + nt) * 1024, o);
          }
        }
        gu32* flag = (gu32*)(dws + OFF_WF) + wu * 64 + rank * 8 + s;
        publish_u32(flag, call, lane == 0);
      }
    }
  }

  asm volatile("s_waitcnt lgkmcnt(0)" ::: "memory");
  __syncthreads();

  if (wave == 0) {
    float m = __uint_as_float(lds_ld(ctl + (C_WMAX + (lane & 7)) * 4));
#pragma unroll
    for (int o = 4; o >= 1; o >>= 1) m = fmaxf(m, __shfl_xor(m, o));
    unsigned last = 0;
    if (lane == 0) {
      u64 v = ((u64)call << 32) | (u64)__float_as_uint(m);
      u64 old = __hip_atomic_fetch_max(reinterpret_cast<u64*>(myws + OFF_AMAX), v, __ATOMIC_RELAXED,
                                       __HIP_MEMORY_SCOPE_AGENT);
      asm volatile("s_waitcnt vmcnt(0)" :: "v"(old) : "memory");
      unsigned tk = __hip_atomic_fetch_add(reinterpret_cast<unsigned*>(myws + OFF_ARRIVE), 1u,
                                           __ATOMIC_RELAXED, __HIP_MEMORY_SCOPE_AGENT);
      last = (tk == call * NWG - 1) ? 1u : 0u;
    }
    last = __builtin_amdgcn_readfirstlane(last);
    if (last) {
      u64 v = __hip_atomic_load(reinterpret_cast<u64*>(myws + OFF_AMAX), __ATOMIC_RELAXED,
                                __HIP_MEMORY_SCOPE_AGENT);
      if (lane < NR) {
        gu64* dstp = (gu64*)((gchar*)(P.ws_all[lane]) + OFF_SLOTS) + rank;
        __hip_atomic_store(dstp, v, __ATOMIC_RELAXED, __HIP_MEMORY_SCOPE_SYSTEM);
      }
    }
    const u64* slots = reinterpret_cast<const u64*>(myws + OFF_SLOTS) + (lane & 7);
    const u64 t0 = __builtin_amdgcn_s_memrealtime();
    u64 v;
    while (true) {
      v = __hip_atomic_load(slots, __ATOMIC_RELAXED, __HIP_MEMORY_SCOPE_SYSTEM);
      if (__all((unsigned)(v >> 32) >= call)) break;
      if (__hip_atomic_load(tmo, __ATOMIC_RELAXED, __HIP_MEMORY_SCOPE_AGENT) == call ||
          __builtin_amdgcn_s_memrealtime() - t0 > DEADLINE_TICKS) {
        if (lane == 0) __hip_atomic_store(tmo, call, __ATOMIC_RELAXED, __HIP_MEMORY_SCOPE_AGENT);
        break;
      }
      __builtin_amdgcn_s_sleep(1);
    }
    float g = __uint_as_float((unsigned)v);
#pragma unroll
    for (int o = 4; o >= 1; o >>= 1) g = fmaxf(g, __shfl_xor(g, o));
    unsigned dead = (__hip_atomic_load(tmo, __ATOMIC_RELAXED, __HIP_MEMORY_SCOPE_AGENT) == call) ? 1u : 0u;
    if (lane == 0) { lds_st(ctl + C_GMAX * 4, __float_as_uint(g)); lds_st(ctl + C_DEAD * 4, dead); }
  }
  asm volatile("s_waitcnt lgkmcnt(0)" ::: "memory");
  __syncthreads();

  const int owner = ti >> 1;
  if (wave < 8) {
    const float amax = __uint_as_float(lds_ld(ctl + C_GMAX * 4));
    const bool dead = lds_ld(ctl + C_DEAD * 4) != 0;
    const float scale = amax / 127.0f;
    gchar* ob = (gchar*)(P.out_all[(unsigned)owner < (unsigned)NR ? owner : 0]);
    const int row0 = (ti & 1) * 256 + wm * 64 + (lane & 15);
    const int col = rank * NLOC + tj * 128 + wn * 64 + (lane >> 4) * 16;
#pragma unroll
    for (int mt = 0; mt < 4; ++mt) {
      u32x4 o;
#pragma unroll
      for (int nt = 0; nt < 4; ++nt) {
        unsigned pk = 0;
#pragma unroll
        for (int j = 0; j < 4; ++j) {
          float q = (scale > 0.f) ? rintf(acc[mt][nt][j] / scale) : 0.f;
          q = fminf(fmaxf(q, -127.f), 127.f);
          int qi = dead ? -128 : (int)q;
          pk |= ((unsigned)qi & 0xffu) << (8 * j);
        }
        o[nt] = pk;
      }
      *(gu32x4*)(ob + (size_t)(row0 + mt * 16) * NTOT + col) = o;
    }
    asm volatile("s_waitcnt vmcnt(0)" ::: "memory");
  }
  asm volatile("s_waitcnt lgkmcnt(0)" ::: "memory");
  __syncthreads();
  if (wave == 0) {
    __builtin_amdgcn_fence(__ATOMIC_RELEASE, "");
    asm volatile("s_waitcnt vmcnt(0)" ::: "memory");
    if (lane == 0) {
      gu32* f = (gu32*)((gchar*)(P.ws_all[(unsigned)owner < (unsigned)NR ? owner : 0]) + OFF_OUTF) +
                    rank * 32 + (ti & 1) * 16 + tj;
      __hip_atomic_store(f, call, __ATOMIC_RELAXED, __HIP_MEMORY_SCOPE_SYSTEM);
    }
    if (blockIdx.x == 0) {
      const unsigned* of = reinterpret_cast<const unsigned*>(myws + OFF_OUTF) + lane * 4;
      const u64 t0 = __builtin_amdgcn_s_memrealtime();
      while (true) {
        bool ok = true;
#pragma unroll
        for (int j = 0; j < 4; ++j)
          ok &= __hip_atomic_load(of + j, __ATOMIC_RELAXED, __HIP_MEMORY_SCOPE_SYSTEM) >= call;
        if (__all(ok)) break;
        if (__hip_atomic_load(tmo, __ATOMIC_RELAXED, __HIP_MEMORY_SCOPE_AGENT) == call ||
            __builtin_amdgcn_s_memrealtime() - t0 > DEADLINE_TICKS) {
          if (lane == 0) {
            __hip_atomic_store(tmo, call, __ATOMIC_RELAXED, __HIP_MEMORY_SCOPE_AGENT);
            ((gchar*)(P.out_all[rank]))[rank * NLOC] = (char)-128;
          }
          break;
        }
        __builtin_amdgcn_s_sleep(1);
      }
      __builtin_amdgcn_fence(__ATOMIC_ACQUIRE, "");
    }
  }
}

extern "C" void kernel_launch(int rank, int nranks, ncclComm_t comm, hipStream_t stream,
                              void* const* d_in, const int* in_sizes, int n_in, void* d_out,
                              int out_size, const int* in_dtypes, int out_dtype,
                              void* const* d_out_all, void* const* d_ws_all, size_t ws_bytes) {
  if (nranks != NR || ws_bytes < WS_NEED || n_in < 2) {
    fprintf(stderr, "unexpected configuration\n");
    return;
  }
  Params P;
  P.x = static_cast<const float*>(d_in[0]);
  P.w = static_cast<const float*>(d_in[1]);
  P.out_all = d_out_all;
  P.ws_all = d_ws_all;
  P.rank = rank;
  hipLaunchKernelGGL(fused_gemm_q8, dim3(NWG), dim3(NTHREADS), 0, stream, P);
  HIP_CHECK(hipGetLastError());
}
```

```cpp
#include <hip/hip_runtime.h>
#include <rccl/rccl.h>
#include <cstdio>
#include <cstdint>
#define ST_SC 1
#define NO_REL 1

#define HIP_CHECK(expr) do { hipError_t _e = (expr); if (_e != hipSuccess) { \
    fprintf(stderr, "HIP error %s at %s:%d\n", hipGetErrorString(_e), __FILE__, __LINE__); \
    return; } } while (0)

typedef __attribute__((ext_vector_type(8))) short bf16x8;
typedef __attribute__((ext_vector_type(4))) float f32x4;
typedef __attribute__((ext_vector_type(4))) unsigned u32x4;
typedef unsigned long long u64;
#define GAS __attribute__((address_space(1)))
typedef GAS char gchar;
typedef GAS unsigned gu32;
typedef GAS unsigned long long gu64;
typedef GAS u32x4 gu32x4;

namespace {
constexpr int NR = 8;
constexpr int KSH = 512;
constexpr int NTOT = 16384;
constexpr int NLOC = 2048;
constexpr int MSH = 512;
constexpr int NSTAGE = 8;
constexpr int XU = 32;
constexpr int WU = 16;
constexpr int UNIT = 16384;
constexpr int NWG = 256;
constexpr int NUNIT = NR * NSTAGE * (XU + WU);
constexpr int NSTEP = 64;
constexpr int NSTEP_RUN = NSTEP;
constexpr int NSLOT = 3;
constexpr int SLOT = 3 * UNIT;
#define NSEND 2
constexpr int NTHREADS = 640 + 64 * NSEND;

constexpr size_t OFF_TICKET = 0;
constexpr size_t OFF_TMO = 128;
constexpr size_t OFF_AMAX = 256;
constexpr size_t OFF_ARRIVE = 384;
constexpr size_t OFF_SQ = 640;
constexpr size_t OFF_SLOTS = 512;
constexpr size_t OFF_OUTF = 1024;
constexpr size_t OFF_XF = 4096;
constexpr size_t OFF_WF = OFF_XF + XU * 64 * 4;
constexpr size_t OFF_XS = 1u << 20;
constexpr size_t OFF_WS = OFF_XS + (size_t)NR * NSTAGE * XU * UNIT;
constexpr size_t WS_NEED = OFF_WS + (size_t)NR * NSTAGE * WU * UNIT;

constexpr u64 DEADLINE_TICKS = 40000000ull;

constexpr int C_CALL = 0, C_FULL = 2, C_DONE = 8, C_WMAX = 16, C_GMAX = 24, C_DEAD = 25, C_Q = 32;
constexpr int LDS_CTL = NSLOT * SLOT;
constexpr int LDS_BYTES = LDS_CTL + 512;
}

#define TS(i) do {} while (0)

#define INVMODE 0
#define ACQ_INV() asm volatile("" ::: "memory")
#define LDPOL " sc1"
#define POLLSLEEP 2
struct Params {
  const float* x;
  const float* w;
  void* const* out_all;
  void* const* ws_all;
  int rank;
};

__device__ __forceinline__ unsigned pk_bf16(float a, float b) {
  typedef __attribute__((ext_vector_type(2))) float f2;
  typedef __attribute__((ext_vector_type(2))) __bf16 b2;
  f2 v = {a, b};
  b2 r = __builtin_convertvector(v, b2);
  return __builtin_bit_cast(unsigned, r);
}

__device__ __forceinline__ void glds4k(const void* g, unsigned ldsaddr) {
  unsigned keep;
  asm volatile(
      "s_mov_b32 %0, m0\n\ts_mov_b32 m0, %2\n\ts_nop 0\n\t"
      "global_load_lds_dwordx4 %1, off" LDPOL "\n\t"
      "global_load_lds_dwordx4 %1, off offset:1024" LDPOL "\n\t"
      "global_load_lds_dwordx4 %1, off offset:2048" LDPOL "\n\t"
      "global_load_lds_dwordx4 %1, off offset:3072" LDPOL "\n\t"
      "s_mov_b32 m0, %0"
      : "=&s"(keep) : "v"(g), "s"(ldsaddr) : "memory");
}

__device__ __forceinline__ unsigned lds_ld(unsigned addr) {
  unsigned v;
  asm volatile("ds_read_b32 %0, %1\n\ts_waitcnt lgkmcnt(0)" : "=v"(v) : "v"(addr) : "memory");
  return v;
}
__device__ __forceinline__ unsigned lds_ld_min2(unsigned addr) {
  typedef __attribute__((ext_vector_type(2))) unsigned u32x2;
  u32x2 v;
  asm volatile("ds_read_b64 %0, %1\n\ts_waitcnt lgkmcnt(0)" : "=v"(v) : "v"(addr) : "memory");
  return v[0] < v[1] ? v[0] : v[1];
}
__device__ __forceinline__ void lds_st(unsigned addr, unsigned v) {
  asm volatile("ds_write_b32 %0, %1" :: "v"(addr), "v"(v) : "memory");
}

__device__ __forceinline__ void st16(gchar* p, u32x4 v) {
  asm volatile("global_store_dwordx4 %0, %1, off sc0 sc1\n\ts_nop 1" :: "v"(p), "v"(v) : "memory");
}

__device__ __forceinline__ void publish_u32(gu32* flag, unsigned val, bool leader) {
  asm volatile("s_waitcnt vmcnt(0)" ::: "memory");
  if (leader) __hip_atomic_store(flag, val, __ATOMIC_RELAXED, __HIP_MEMORY_SCOPE_SYSTEM);
}

__global__ __launch_bounds__(NTHREADS, (NTHREADS + 255) / 256) void fused_gemm_q8(Params P) {
  __shared__ __attribute__((aligned(1024))) char lds[LDS_BYTES];
  const unsigned ldsbase = (unsigned)(uintptr_t)lds;
  const unsigned ctl = ldsbase + LDS_CTL;

  const int tid = threadIdx.x;
  const int lane = tid & 63;
  const int wave = __builtin_amdgcn_readfirstlane(tid >> 6);
  const int rank = P.rank;
  char* const myws = static_cast<char*>(P.ws_all[rank]);
  unsigned* const tmo = reinterpret_cast<unsigned*>(myws + OFF_TMO);

  if (tid < 128) {
    unsigned v = 0;
    if (tid == 0) {
      unsigned t = __hip_atomic_fetch_add(reinterpret_cast<unsigned*>(myws + OFF_TICKET), 1u,
                                          __ATOMIC_RELAXED, __HIP_MEMORY_SCOPE_AGENT);
      v = t / NWG + 1;
    }
    lds_st(ctl + tid * 4, v);
  }
  asm volatile("s_waitcnt lgkmcnt(0)" ::: "memory");
  __syncthreads();
  if (wave == 0) TS(0);
  const unsigned call = __builtin_amdgcn_readfirstlane(lds_ld(ctl + C_CALL * 4));

  const int bx = blockIdx.x & 7, bj = blockIdx.x >> 3;
  const int ti = 4 * (bx >> 1) + (bj & 3);
  const int tj = 8 * (bx & 1) + (bj >> 2);

  f32x4 acc[4][4];
  const int wm = wave & 3, wn = (wave >> 2) & 1;

  if (wave < 8) {
#pragma unroll
    for (int a = 0; a < 4; ++a)
#pragma unroll
      for (int b = 0; b < 4; ++b) acc[a][b] = f32x4{0.f, 0.f, 0.f, 0.f};
    const int xoff = (wm >> 1) * UNIT + ((wm & 1) * 4) * 1024 + lane * 16;
    const int woff = 2 * UNIT + (wn * 4) * 1024 + lane * 16;
    int slot = 0;
    for (int t = 0; t < NSTEP_RUN; ++t) {
      while (true) {
        if (lds_ld_min2(ctl + C_FULL * 4) > (unsigned)t) break;
        __builtin_amdgcn_s_sleep(1);
      }
      const char* sb = lds + slot * SLOT;
      bf16x8 xf[2][4], wf[2][4];
#pragma unroll
      for (int kb = 0; kb < 2; ++kb) {
#pragma unroll
        for (int i = 0; i < 4; ++i) {
          wf[kb][i] = *reinterpret_cast<const bf16x8*>(sb + woff + (kb * 8 + i) * 1024);
          xf[kb][i] = *reinterpret_cast<const bf16x8*>(sb + xoff + (kb * 8 + i) * 1024);
        }
      }
#pragma unroll
      for (int mt = 0; mt < 4; ++mt)
#pragma unroll
        for (int nt = 0; nt < 4; ++nt)
          acc[mt][nt] = __builtin_amdgcn_mfma_f32_16x16x32_bf16(wf[0][nt], xf[0][mt], acc[mt][nt], 0, 0, 0);
      asm volatile("s_waitcnt lgkmcnt(0)" ::: "memory");
      if (lane == 0) lds_st(ctl + (C_DONE + wave) * 4, (unsigned)(t + 1));
#pragma unroll
      for (int mt = 0; mt < 4; ++mt)
#pragma unroll
        for (int nt = 0; nt < 4; ++nt)
          acc[mt][nt] = __builtin_amdgcn_mfma_f32_16x16x32_bf16(wf[1][nt], xf[1][mt], acc[mt][nt], 0, 0, 0);
      slot = (slot == NSLOT - 1) ? 0 : slot + 1;
    }
    float m = 0.f;
#pragma unroll
    for (int a = 0; a < 4; ++a)
#pragma unroll
      for (int b = 0; b < 4; ++b)
#pragma unroll
        for (int j = 0; j < 4; ++j) m = fmaxf(m, fabsf(acc[a][b][j]));
#pragma unroll
    for (int o = 32; o >= 1; o >>= 1) m = fmaxf(m, __shfl_xor(m, o));
    if (lane == 0) lds_st(ctl + (C_WMAX + wave) * 4, __float_as_uint(m));
    if (wave == 0) TS(12);
  } else if (wave < 10) {
    const int L = wave - 8;
    int slot = 0;
    if (L == 0) {
      const unsigned* pf0 = reinterpret_cast<const unsigned*>(myws + OFF_XF) + (2 * ti) * 64 + lane;
      const unsigned* pf2 = reinterpret_cast<const unsigned*>(myws + OFF_WF) + tj * 64 + lane;
      u64 ready = 0, issued = 0;
      bool dead = false;
      unsigned inv = 0;
      const u64 t0 = __builtin_amdgcn_s_memrealtime();
      for (int t = 0; t < NSTEP_RUN; ++t) {
        if (!(ready & ~issued)) {
          asm volatile("s_waitcnt vmcnt(0)" ::: "memory");
          if (lane == 0) lds_st(ctl + C_FULL * 4, (unsigned)t);
          while (!dead) {
            unsigned f0, f1, f2;
            asm volatile("global_load_dword %0, %3, off sc0 sc1\n\tglobal_load_dword %1, %3, off offset:256 sc0 sc1\n\t"
                         "global_load_dword %2, %4, off sc0 sc1\n\ts_waitcnt vmcnt(0)"
                         : "=&v"(f0), "=&v"(f1), "=&v"(f2) : "v"(pf0), "v"(pf2) : "memory");
            ready = __ballot(f0 >= call && f1 >= call && f2 >= call);
            if (ready & ~issued) break;
            if (__hip_atomic_load(tmo, __ATOMIC_RELAXED, __HIP_MEMORY_SCOPE_AGENT) == call ||
                __builtin_amdgcn_s_memrealtime() - t0 > DEADLINE_TICKS) {
              if (lane == 0) __hip_atomic_store(tmo, call, __ATOMIC_RELAXED, __HIP_MEMORY_SCOPE_AGENT);
              dead = true;
            }
            __builtin_amdgcn_s_sleep(POLLSLEEP);
          }
          if (dead) ready = ~0ull;
          ACQ_INV();
          inv = 1;
        }
        const int bit = __builtin_ctzll(ready & ~issued);
        issued |= 1ull << bit;
        if (lane == 0) lds_st(ctl + (C_Q + t) * 4, (unsigned)(bit + 1) | (inv << 8));
        inv = 0;
        if (t >= NSLOT) {
          while (true) {
            unsigned d = lds_ld(ctl + (C_DONE + (lane & 7)) * 4);
            if (__all(d >= (unsigned)(t - NSLOT + 1))) break;
            __builtin_amdgcn_s_sleep(1);
          }
        }
        unsigned f0, f1, f2;
        asm volatile("global_load_dword %0, %3, off sc0 sc1\n\tglobal_load_dword %1, %3, off offset:256 sc0 sc1\n\t"
                     "global_load_dword %2, %4, off sc0 sc1"
                     : "=&v"(f0), "=&v"(f1), "=&v"(f2) : "v"(pf0), "v"(pf2) : "memory");
        const char* gx = myws + OFF_XS + ((size_t)(bit * XU + 2 * ti)) * UNIT + lane * 16;
        const char* gw = myws + OFF_WS + ((size_t)(bit * WU + tj)) * UNIT + lane * 16;
        const unsigned lx = ldsbase + slot * SLOT;
        const unsigned lw = ldsbase + slot * SLOT + 2 * UNIT;
#pragma unroll
        for (int i = 0; i < 4; ++i) glds4k(gx + i * 4096, lx + i * 4096);
#pragma unroll
        for (int i = 0; i < 2; ++i) glds4k(gw + i * 4096, lw + i * 4096);
        asm volatile("s_waitcnt vmcnt(24)" : "+v"(f0), "+v"(f1), "+v"(f2) :: "memory");
        if (lane == 0) lds_st(ctl + C_FULL * 4, (unsigned)t);
        u64 nr = __ballot(f0 >= call && f1 >= call && f2 >= call);
        if (nr & ~ready) {
          ready |= nr;
          ACQ_INV();
          inv = 1;
        }
        slot = (slot == NSLOT - 1) ? 0 : slot + 1;
      }
    } else {
      for (int t = 0; t < NSTEP_RUN; ++t) {
        unsigned e = lds_ld(ctl + (C_Q + t) * 4);
        if (e == 0) {
          asm volatile("s_waitcnt vmcnt(0)" ::: "memory");
          if (lane == 0) lds_st(ctl + (C_FULL + 1) * 4, (unsigned)t);
          while ((e = lds_ld(ctl + (C_Q + t) * 4)) == 0) __builtin_amdgcn_s_sleep(1);
        }
        e = __builtin_amdgcn_readfirstlane(e);
        if (e >> 8) ACQ_INV();
        const int bit = (int)(e & 0xff) - 1;
        if (t >= NSLOT) {
          while (true) {
            unsigned d = lds_ld(ctl + (C_DONE + (lane & 7)) * 4);
            if (__all(d >= (unsigned)(t - NSLOT + 1))) break;
            __builtin_amdgcn_s_sleep(1);
          }
        }
        const char* gx = myws + OFF_XS + ((size_t)(bit * XU + 2 * ti + 1)) * UNIT + lane * 16;
        const char* gw = myws + OFF_WS + ((size_t)(bit * WU + tj)) * UNIT + 8192 + lane * 16;
        const unsigned lx = ldsbase + slot * SLOT + UNIT;
        const unsigned lw = ldsbase + slot * SLOT + 2 * UNIT + 8192;
#pragma unroll
        for (int i = 0; i < 4; ++i) glds4k(gx + i * 4096, lx + i * 4096);
#pragma unroll
        for (int i = 0; i < 2; ++i) glds4k(gw + i * 4096, lw + i * 4096);
        asm volatile("s_waitcnt vmcnt(24)" ::: "memory");
        if (lane == 0) lds_st(ctl + (C_FULL + 1) * 4, (unsigned)t);
        slot = (slot == NSLOT - 1) ? 0 : slot + 1;
      }
    }
    asm volatile("s_waitcnt vmcnt(0)" ::: "memory");
    if (lane == 0) lds_st(ctl + (C_FULL + L) * 4, (unsigned)NSTEP);
    if (L == 0) TS(11);
  } else {
    unsigned* const sq = reinterpret_cast<unsigned*>(myws + OFF_SQ);
    const unsigned qbase = (call - 1) * (NUNIT + NSEND * NWG);
    unsigned tk = 0;
    if (lane == 0) tk = __hip_atomic_fetch_add(sq, 1u, __ATOMIC_RELAXED, __HIP_MEMORY_SCOPE_AGENT);
    for (int i = 0;; ++i) {
      const int u = (int)(__builtin_amdgcn_readfirstlane(tk) - qbase);
      if ((unsigned)u >= (unsigned)NUNIT) break;
      if (lane == 0) tk = __hip_atomic_fetch_add(sq, 1u, __ATOMIC_RELAXED, __HIP_MEMORY_SCOPE_AGENT);
      int d, s, item;
      if (u < 336) { s = 0; d = (rank + 1 + u % 7) & 7; item = u / 7; }
      else if (u < 720) { int j = u - 336; s = j / 48; item = j % 48; d = rank; }
      else { int j = u - 720; s = 1 + j / 336; j %= 336; d = (rank + 1 + j % 7) & 7; item = j / 7; }
      gchar* dws = (gchar*)(P.ws_all[(unsigned)d < (unsigned)NR ? d : rank]);
      const int q = item / 3, rr = item - 3 * q;
      if (rr != 2) {
        const int xu = 2 * q + rr;
        const float* src = P.x + (size_t)(xu * 128 + (lane & 15)) * KSH + s * 64 + 8 * (lane >> 4);
        gchar* dst = dws + OFF_XS + ((size_t)((rank * NSTAGE + s) * XU + xu)) * UNIT + lane * 16;
#pragma unroll
        for (int kb = 0; kb < 2; ++kb) {
          f32x4 a[8], b[8];
#pragma unroll
          for (int blk = 0; blk < 8; ++blk) {
            const float* sp = src + (size_t)blk * 16 * KSH + kb * 32;
            a[blk] = *reinterpret_cast<const f32x4*>(sp);
            b[blk] = *reinterpret_cast<const f32x4*>(sp + 4);
          }
#pragma unroll
          for (int blk = 0; blk < 8; ++blk) {
            u32x4 v = {pk_bf16(a[blk][0], a[blk][1]), pk_bf16(a[blk][2], a[blk][3]),
                       pk_bf16(b[blk][0], b[blk][1]), pk_bf16(b[blk][2], b[blk][3])};
            st16(dst + (kb * 8 + blk) * 1024, v);
          }
        }
        gu32* flag = (gu32*)(dws + OFF_XF) + xu * 64 + rank * 8 + s;
        publish_u32(flag, call, lane == 0);
      } else {
        const int wu = q;
        const float* src = P.w + (size_t)(s * 64 + 8 * (lane >> 4)) * NTOT + d * NLOC + wu * 128 +
                           ((lane & 15) >> 2) * 16 + (lane & 3);
        gchar* dst = dws + OFF_WS + ((size_t)((rank * NSTAGE + s) * WU + wu)) * UNIT + lane * 16;
#pragma unroll
        for (int g = 0; g < 4; ++g) {
          const int kb = g >> 1, hf = g & 1;
          float v[4][8];
#pragma unroll
          for (int nt = 0; nt < 4; ++nt)
#pragma unroll
            for (int j = 0; j < 8; ++j)
              v[nt][j] = src[(size_t)(kb * 32 + j) * NTOT + hf * 64 + nt * 4];
#pragma unroll
          for (int nt = 0; nt < 4; ++nt) {
            u32x4 o = {pk_bf16(v[nt][0], v[nt][1]), pk_bf16(v[nt][2], v[nt][3]),
                       pk_bf16(v[nt][4], v[nt][5]), pk_bf16(v[nt][6], v[nt][7])};
            st16(dst + (kb * 8 + hf * 4 + nt) * 1024, o);
          }
        }
        gu32* flag = (gu32*)(dws + OFF_WF) + wu * 64 + rank * 8 + s;
        publish_u32(flag, call, lane == 0);
      }
      if (wave == 10 && i < 12) TS(20 + i);
    }
    if (wave < 12) TS(1 + wave - 10);
  }

  asm volatile("s_waitcnt lgkmcnt(0)" ::: "memory");
  __syncthreads();

  if (wave == 0) {
    TS(13);
    float m = __uint_as_float(lds_ld(ctl + (C_WMAX + (lane & 7)) * 4));
#pragma unroll
    for (int o = 4; o >= 1; o >>= 1) m = fmaxf(m, __shfl_xor(m, o));
    unsigned last = 0;
    if (lane == 0) {
      u64 v = ((u64)call << 32) | (u64)__float_as_uint(m);
      u64 old = __hip_atomic_fetch_max(reinterpret_cast<u64*>(myws + OFF_AMAX), v, __ATOMIC_RELAXED,
                                       __HIP_MEMORY_SCOPE_AGENT);
      asm volatile("s_waitcnt vmcnt(0)" :: "v"(old) : "memory");
      unsigned tk = __hip_atomic_fetch_add(reinterpret_cast<unsigned*>(myws + OFF_ARRIVE), 1u,
                                           __ATOMIC_RELAXED, __HIP_MEMORY_SCOPE_AGENT);
      last = (tk == call * NWG - 1) ? 1u : 0u;
    }
    last = __builtin_amdgcn_readfirstlane(last);
    if (last) {
      u64 v = __hip_atomic_load(reinterpret_cast<u64*>(myws + OFF_AMAX), __ATOMIC_RELAXED,
                                __HIP_MEMORY_SCOPE_AGENT);
      if (lane < NR) {
        gu64* dstp = (gu64*)((gchar*)(P.ws_all[lane]) + OFF_SLOTS) + rank;
        __hip_atomic_store(dstp, v, __ATOMIC_RELAXED, __HIP_MEMORY_SCOPE_SYSTEM);
      }
    }
    const u64* slots = reinterpret_cast<const u64*>(myws + OFF_SLOTS) + (lane & 7);
    const u64 t0 = __builtin_amdgcn_s_memrealtime();
    u64 v;
    while (true) {
      v = __hip_atomic_load(slots, __ATOMIC_RELAXED, __HIP_MEMORY_SCOPE_SYSTEM);
      if (__all((unsigned)(v >> 32) >= call)) break;
      if (__hip_atomic_load(tmo, __ATOMIC_RELAXED, __HIP_MEMORY_SCOPE_AGENT) == call ||
          __builtin_amdgcn_s_memrealtime() - t0 > DEADLINE_TICKS) {
        if (lane == 0) __hip_atomic_store(tmo, call, __ATOMIC_RELAXED, __HIP_MEMORY_SCOPE_AGENT);
        break;
      }
      __builtin_amdgcn_s_sleep(1);
    }
    float g = __uint_as_float((unsigned)v);
#pragma unroll
    for (int o = 4; o >= 1; o >>= 1) g = fmaxf(g, __shfl_xor(g, o));
    unsigned dead = (__hip_atomic_load(tmo, __ATOMIC_RELAXED, __HIP_MEMORY_SCOPE_AGENT) == call) ? 1u : 0u;
    if (lane == 0) { lds_st(ctl + C_GMAX * 4, __float_as_uint(g)); lds_st(ctl + C_DEAD * 4, dead); }
  }
  asm volatile("s_waitcnt lgkmcnt(0)" ::: "memory");
  __syncthreads();

  if (wave == 0) TS(14);
  const int owner = ti >> 1;
  if (wave < 8) {
    const float amax = __uint_as_float(lds_ld(ctl + C_GMAX * 4));
    const bool dead = lds_ld(ctl + C_DEAD * 4) != 0;
    const float scale = amax / 127.0f;
    gchar* ob = (gchar*)(P.out_all[(unsigned)owner < (unsigned)NR ? owner : 0]);
    const int row0 = (ti & 1) * 256 + wm * 64 + (lane & 15);
    const int col = rank * NLOC + tj * 128 + wn * 64 + (lane >> 4) * 16;
#pragma unroll
    for (int mt = 0; mt < 4; ++mt) {
      u32x4 o;
#pragma unroll
      for (int nt = 0; nt < 4; ++nt) {
        unsigned pk = 0;
#pragma unroll
        for (int j = 0; j < 4; ++j) {
          float q = (scale > 0.f) ? rintf(acc[mt][nt][j] / scale) : 0.f;
          q = fminf(fmaxf(q, -127.f), 127.f);
          int qi = dead ? -128 : (int)q;
          pk |= ((unsigned)qi & 0xffu) << (8 * j);
        }
        o[nt] = pk;
      }
      st16(ob + (size_t)(row0 + mt * 16) * NTOT + col, o);
    }
    asm volatile("s_waitcnt vmcnt(0)" ::: "memory");
  }
  asm volatile("s_waitcnt lgkmcnt(0)" ::: "memory");
  __syncthreads();
  if (wave == 0) {
    TS(15);
    if (lane == 0) {
      gu32* f = (gu32*)((gchar*)(P.ws_all[(unsigned)owner < (unsigned)NR ? owner : 0]) + OFF_OUTF) +
                    rank * 32 + (ti & 1) * 16 + tj;
      __hip_atomic_store(f, call, __ATOMIC_RELAXED, __HIP_MEMORY_SCOPE_SYSTEM);
    }
    if (blockIdx.x == 0) {
      const unsigned* of = reinterpret_cast<const unsigned*>(myws + OFF_OUTF) + lane * 4;
      const u64 t0 = __builtin_amdgcn_s_memrealtime();
      while (true) {
        bool ok = true;
#pragma unroll
        for (int j = 0; j < 4; ++j)
          ok &= __hip_atomic_load(of + j, __ATOMIC_RELAXED, __HIP_MEMORY_SCOPE_SYSTEM) >= call;
        if (__all(ok)) break;
        if (__hip_atomic_load(tmo, __ATOMIC_RELAXED, __HIP_MEMORY_SCOPE_AGENT) == call ||
            __builtin_amdgcn_s_memrealtime() - t0 > DEADLINE_TICKS) {
          if (lane == 0) {
            __hip_atomic_store(tmo, call, __ATOMIC_RELAXED, __HIP_MEMORY_SCOPE_AGENT);
            ((gchar*)(P.out_all[rank]))[rank * NLOC] = (char)-128;
          }
          break;
        }
        __builtin_amdgcn_s_sleep(1);
      }
      __builtin_amdgcn_fence(__ATOMIC_ACQUIRE, "");
      TS(16);
    }
  }
}

extern "C" void kernel_launch(int rank, int nranks, ncclComm_t comm, hipStream_t stream,
                              void* const* d_in, const int* in_sizes, int n_in, void* d_out,
                              int out_size, const int* in_dtypes, int out_dtype,
                              void* const* d_out_all, void* const* d_ws_all, size_t ws_bytes) {
  if (nranks != NR || ws_bytes < WS_NEED || n_in < 2) {
    fprintf(stderr, "unexpected configuration\n");
    return;
  }
  Params P;
  P.x = static_cast<const float*>(d_in[0]);
  P.w = static_cast<const float*>(d_in[1]);
  P.out_all = d_out_all;
  P.ws_all = d_ws_all;
  P.rank = rank;
  hipLaunchKernelGGL(fused_gemm_q8, dim3(NWG), dim3(NTHREADS), 0, stream, P);
  HIP_CHECK(hipGetLastError());
}
```

```cpp
#include <hip/hip_runtime.h>
#include <rccl/rccl.h>
#include <cstdio>
#include <cstdint>

#define HIP_CHECK(expr) do { hipError_t _e = (expr); if (_e != hipSuccess) { \
    fprintf(stderr, "HIP error %s at %s:%d\n", hipGetErrorString(_e), __FILE__, __LINE__); \
    return; } } while (0)

typedef __attribute__((ext_vector_type(8))) short bf16x8;
typedef __attribute__((ext_vector_type(4))) float f32x4;
typedef __attribute__((ext_vector_type(4))) unsigned u32x4;
typedef unsigned long long u64;
#define GAS __attribute__((address_space(1)))
typedef GAS char gchar;
typedef GAS unsigned gu32;
typedef GAS unsigned long long gu64;
typedef GAS u32x4 gu32x4;

namespace {
constexpr int NR = 8;
constexpr int KSH = 512;
constexpr int NTOT = 16384;
constexpr int NLOC = 2048;
constexpr int MSH = 512;
constexpr int NSTAGE = 8;
constexpr int XU = 32;
constexpr int WU = 16;
constexpr int UNIT = 16384;
constexpr int NWG = 256;
constexpr int NUNIT = NR * NSTAGE * (XU + WU);
constexpr int NSTEP = 64;
constexpr int NSLOT = 3;
constexpr int SLOT = 3 * UNIT;
#define NSEND 1
constexpr int NTHREADS = 640 + 64 * NSEND;

constexpr size_t OFF_TICKET = 0;
constexpr size_t OFF_TMO = 128;
constexpr size_t OFF_AMAX = 256;
constexpr size_t OFF_ARRIVE = 384;
constexpr size_t OFF_SQ = 640;
constexpr size_t OFF_SLOTS = 512;
constexpr size_t OFF_OUTF = 1024;
constexpr size_t OFF_XF = 4096;
constexpr size_t OFF_WF = OFF_XF + XU * 64 * 4;
constexpr size_t OFF_XS = 1u << 20;
constexpr size_t OFF_WS = OFF_XS + (size_t)NR * NSTAGE * XU * UNIT;
constexpr size_t WS_NEED = OFF_WS + (size_t)NR * NSTAGE * WU * UNIT;

constexpr u64 DEADLINE_TICKS = 40000000ull;

constexpr int C_CALL = 0, C_FULL = 2, C_DONE = 8, C_WMAX = 16, C_GMAX = 24, C_DEAD = 25, C_Q = 32;
constexpr int LDS_CTL = NSLOT * SLOT;
constexpr int LDS_BYTES = LDS_CTL + 512;
}

#define TS(i) do {} while (0)

#define LDPOL " sc0 sc1"

struct Params {
  const float* x;
  const float* w;
  void* const* out_all;
  void* const* ws_all;
  int rank;
};

__device__ __forceinline__ unsigned pk_bf16(float a, float b) {
  typedef __attribute__((ext_vector_type(2))) float f2;
  typedef __attribute__((ext_vector_type(2))) __bf16 b2;
  f2 v = {a, b};
  b2 r = __builtin_convertvector(v, b2);
  return __builtin_bit_cast(unsigned, r);
}

__device__ __forceinline__ void glds4k(const void* g, unsigned ldsaddr) {
  unsigned keep;
  asm volatile(
      "s_mov_b32 %0, m0\n\ts_mov_b32 m0, %2\n\ts_nop 0\n\t"
      "global_load_lds_dwordx4 %1, off" LDPOL "\n\t"
      "global_load_lds_dwordx4 %1, off offset:1024" LDPOL "\n\t"
      "global_load_lds_dwordx4 %1, off offset:2048" LDPOL "\n\t"
      "global_load_lds_dwordx4 %1, off offset:3072" LDPOL "\n\t"
      "s_mov_b32 m0, %0"
      : "=&s"(keep) : "v"(g), "s"(ldsaddr) : "memory");
}

__device__ __forceinline__ unsigned lds_ld(unsigned addr) {
  unsigned v;
  asm volatile("ds_read_b32 %0, %1\n\ts_waitcnt lgkmcnt(0)" : "=v"(v) : "v"(addr) : "memory");
  return v;
}
__device__ __forceinline__ unsigned lds_ld_min2(unsigned addr) {
  typedef __attribute__((ext_vector_type(2))) unsigned u32x2;
  u32x2 v;
  asm volatile("ds_read_b64 %0, %1\n\ts_waitcnt lgkmcnt(0)" : "=v"(v) : "v"(addr) : "memory");
  return v[0] < v[1] ? v[0] : v[1];
}
__device__ __forceinline__ void lds_st(unsigned addr, unsigned v) {
  asm volatile("ds_write_b32 %0, %1" :: "v"(addr), "v"(v) : "memory");
}

__device__ __forceinline__ void st16(gchar* p, u32x4 v) {
  asm volatile("global_store_dwordx4 %0, %1, off sc0 sc1\n\ts_nop 1" :: "v"(p), "v"(v) : "memory");
}

__device__ __forceinline__ void publish_u32(gu32* flag, unsigned val, bool leader) {
  asm volatile("s_waitcnt vmcnt(0)" ::: "memory");
  if (leader) __hip_atomic_store(flag, val, __ATOMIC_RELAXED, __HIP_MEMORY_SCOPE_SYSTEM);
}

__global__ __launch_bounds__(NTHREADS, (NTHREADS + 255) / 256) void fused_gemm_q8(Params P) {
  __shared__ __attribute__((aligned(1024))) char lds[LDS_BYTES];
  const unsigned ldsbase = (unsigned)(uintptr_t)lds;
  const unsigned ctl = ldsbase + LDS_CTL;

  const int tid = threadIdx.x;
  const int lane = tid & 63;
  const int wave = __builtin_amdgcn_readfirstlane(tid >> 6);
  const int rank = P.rank;
  char* const myws = static_cast<char*>(P.ws_all[rank]);
  unsigned* const tmo = reinterpret_cast<unsigned*>(myws + OFF_TMO);

  if (tid < 128) {
    unsigned v = 0;
    if (tid == 0) {
      unsigned t = __hip_atomic_fetch_add(reinterpret_cast<unsigned*>(myws + OFF_TICKET), 1u,
                                          __ATOMIC_RELAXED, __HIP_MEMORY_SCOPE_AGENT);
      v = t / NWG + 1;
    }
    lds_st(ctl + tid * 4, v);
  }
  asm volatile("s_waitcnt lgkmcnt(0)" ::: "memory");
  __syncthreads();
  if (wave == 0) TS(0);
  const unsigned call = __builtin_amdgcn_readfirstlane(lds_ld(ctl + C_CALL * 4));

  const int bx = blockIdx.x & 7, bj = blockIdx.x >> 3;
  const int ti = 4 * (bx >> 1) + (bj & 3);
  const int tj = 8 * (bx & 1) + (bj >> 2);

  f32x4 acc[4][4];
  const int wm = wave & 3, wn = (wave >> 2) & 1;

  if (wave < 8) {
#pragma unroll
    for (int a = 0; a < 4; ++a)
#pragma unroll
      for (int b = 0; b < 4; ++b) acc[a][b] = f32x4{0.f, 0.f, 0.f, 0.f};
    const int xoff = (wm >> 1) * UNIT + ((wm & 1) * 4) * 1024 + lane * 16;
    const int woff = 2 * UNIT + (wn * 4) * 1024 + lane * 16;
    int slot = 0;
    for (int t = 0; t < NSTEP; ++t) {
      while (true) {
        if (lds_ld_min2(ctl + C_FULL * 4) > (unsigned)t) break;
        __builtin_amdgcn_s_sleep(1);
      }
      const char* sb = lds + slot * SLOT;
      bf16x8 xf[2][4], wf[2][4];
#pragma unroll
      for (int kb = 0; kb < 2; ++kb) {
#pragma unroll
        for (int i = 0; i < 4; ++i) {
          wf[kb][i] = *reinterpret_cast<const bf16x8*>(sb + woff + (kb * 8 + i) * 1024);
          xf[kb][i] = *reinterpret_cast<const bf16x8*>(sb + xoff + (kb * 8 + i) * 1024);
        }
      }
#pragma unroll
      for (int mt = 0; mt < 4; ++mt)
#pragma unroll
        for (int nt = 0; nt < 4; ++nt)
          acc[mt][nt] = __builtin_amdgcn_mfma_f32_16x16x32_bf16(wf[0][nt], xf[0][mt], acc[mt][nt], 0, 0, 0);
      asm volatile("s_waitcnt lgkmcnt(0)" ::: "memory");
      if (lane == 0) lds_st(ctl + (C_DONE + wave) * 4, (unsigned)(t + 1));
#pragma unroll
      for (int mt = 0; mt < 4; ++mt)
#pragma unroll
        for (int nt = 0; nt < 4; ++nt)
          acc[mt][nt] = __builtin_amdgcn_mfma_f32_16x16x32_bf16(wf[1][nt], xf[1][mt], acc[mt][nt], 0, 0, 0);
      slot = (slot == NSLOT - 1) ? 0 : slot + 1;
    }
    float m = 0.f;
#pragma unroll
    for (int a = 0; a < 4; ++a)
#pragma unroll
      for (int b = 0; b < 4; ++b)
#pragma unroll
        for (int j = 0; j < 4; ++j) m = fmaxf(m, fabsf(acc[a][b][j]));
#pragma unroll
    for (int o = 32; o >= 1; o >>= 1) m = fmaxf(m, __shfl_xor(m, o));
    if (lane == 0) lds_st(ctl + (C_WMAX + wave) * 4, __float_as_uint(m));
    if (wave == 0) TS(12);
  } else if (wave < 10) {
    const int L = wave - 8;
    int slot = 0;
    if (L == 0) {
      const unsigned* pf0 = reinterpret_cast<const unsigned*>(myws + OFF_XF) + (2 * ti) * 64 + lane;
      const unsigned* pf2 = reinterpret_cast<const unsigned*>(myws + OFF_WF) + tj * 64 + lane;
      u64 ready = 0, issued = 0;
      bool dead = false;
      unsigned spins = 0;
      const u64 t0 = __builtin_amdgcn_s_memrealtime();
      for (int t = 0; t < NSTEP; ++t) {
        if (!(ready & ~issued)) {
          asm volatile("s_waitcnt vmcnt(0)" ::: "memory");
          if (lane == 0) lds_st(ctl + C_FULL * 4, (unsigned)t);
          while (!dead) {
            unsigned f0, f1, f2;
            asm volatile("global_load_dword %0, %3, off sc0 sc1\n\tglobal_load_dword %1, %3, off offset:256 sc0 sc1\n\t"
                         "global_load_dword %2, %4, off sc0 sc1\n\ts_waitcnt vmcnt(0)"
                         : "=&v"(f0), "=&v"(f1), "=&v"(f2) : "v"(pf0), "v"(pf2) : "memory");
            ready = __ballot(f0 >= call && f1 >= call && f2 >= call);
            if (ready & ~issued) break;
            if ((++spins & 15) == 0 &&
                (__hip_atomic_load(tmo, __ATOMIC_RELAXED, __HIP_MEMORY_SCOPE_AGENT) == call ||
                 __builtin_amdgcn_s_memrealtime() - t0 > DEADLINE_TICKS)) {
              if (lane == 0) __hip_atomic_store(tmo, call, __ATOMIC_RELAXED, __HIP_MEMORY_SCOPE_AGENT);
              dead = true;
            }
            __builtin_amdgcn_s_sleep(2);
          }
          if (dead) ready = ~0ull;
        }
        const int bit = __builtin_ctzll(ready & ~issued);
        issued |= 1ull << bit;
        if (lane == 0) lds_st(ctl + (C_Q + t) * 4, (unsigned)(bit + 1));
        if (t >= NSLOT) {
          while (true) {
            unsigned d = lds_ld(ctl + (C_DONE + (lane & 7)) * 4);
            if (__all(d >= (unsigned)(t - NSLOT + 1))) break;
            __builtin_amdgcn_s_sleep(1);
          }
        }
        unsigned f0, f1, f2;
        asm volatile("global_load_dword %0, %3, off sc0 sc1\n\tglobal_load_dword %1, %3, off offset:256 sc0 sc1\n\t"
                     "global_load_dword %2, %4, off sc0 sc1"
                     : "=&v"(f0), "=&v"(f1), "=&v"(f2) : "v"(pf0), "v"(pf2) : "memory");
        const char* gx = myws + OFF_XS + ((size_t)(bit * XU + 2 * ti)) * UNIT + lane * 16;
        const char* gw = myws + OFF_WS + ((size_t)(bit * WU + tj)) * UNIT + lane * 16;
        const unsigned lx = ldsbase + slot * SLOT;
        const unsigned lw = ldsbase + slot * SLOT + 2 * UNIT;
#pragma unroll
        for (int i = 0; i < 4; ++i) glds4k(gx + i * 4096, lx + i * 4096);
#pragma unroll
        for (int i = 0; i < 2; ++i) glds4k(gw + i * 4096, lw + i * 4096);
        asm volatile("s_waitcnt vmcnt(24)" : "+v"(f0), "+v"(f1), "+v"(f2) :: "memory");
        if (lane == 0) lds_st(ctl + C_FULL * 4, (unsigned)t);
        ready |= __ballot(f0 >= call && f1 >= call && f2 >= call);
        slot = (slot == NSLOT - 1) ? 0 : slot + 1;
      }
    } else {
      for (int t = 0; t < NSTEP; ++t) {
        unsigned e = lds_ld(ctl + (C_Q + t) * 4);
        if (e == 0) {
          asm volatile("s_waitcnt vmcnt(0)" ::: "memory");
          if (lane == 0) lds_st(ctl + (C_FULL + 1) * 4, (unsigned)t);
          while ((e = lds_ld(ctl + (C_Q + t) * 4)) == 0) __builtin_amdgcn_s_sleep(1);
        }
        e = __builtin_amdgcn_readfirstlane(e);
        const int bit = (int)e - 1;
        if (t >= NSLOT) {
          while (true) {
            unsigned d = lds_ld(ctl + (C_DONE + (lane & 7)) * 4);
            if (__all(d >= (unsigned)(t - NSLOT + 1))) break;
            __builtin_amdgcn_s_sleep(1);
          }
        }
        const char* gx = myws + OFF_XS + ((size_t)(bit * XU + 2 * ti + 1)) * UNIT + lane * 16;
        const char* gw = myws + OFF_WS + ((size_t)(bit * WU + tj)) * UNIT + 8192 + lane * 16;
        const unsigned lx = ldsbase + slot * SLOT + UNIT;
        const unsigned lw = ldsbase + slot * SLOT + 2 * UNIT + 8192;
#pragma unroll
        for (int i = 0; i < 4; ++i) glds4k(gx + i * 4096, lx + i * 4096);
#pragma unroll
        for (int i = 0; i < 2; ++i) glds4k(gw + i * 4096, lw + i * 4096);
        asm volatile("s_waitcnt vmcnt(24)" ::: "memory");
        if (lane == 0) lds_st(ctl + (C_FULL + 1) * 4, (unsigned)t);
        slot = (slot == NSLOT - 1) ? 0 : slot + 1;
      }
    }
    asm volatile("s_waitcnt vmcnt(0)" ::: "memory");
    if (lane == 0) lds_st(ctl + (C_FULL + L) * 4, (unsigned)NSTEP);
    if (L == 0) TS(11);
  } else {
    unsigned* const sq = reinterpret_cast<unsigned*>(myws + OFF_SQ);
    const unsigned qbase = (call - 1) * NUNIT - NSEND * NWG;
    unsigned tk = qbase + blockIdx.x * NSEND + (wave - 10);
    for (int i = 0;; ++i) {
      const int u = (int)(__builtin_amdgcn_readfirstlane(tk) - qbase);
      if ((unsigned)u >= (unsigned)NUNIT) break;
      if (lane == 0) tk = __hip_atomic_fetch_add(sq, 1u, __ATOMIC_RELAXED, __HIP_MEMORY_SCOPE_AGENT);
      int d, s, item;
      if (u < 336) { s = 0; d = (rank + 1 + u % 7) & 7; item = u / 7; }
      else if (u < 720) { int j = u - 336; s = j / 48; item = j % 48; d = rank; }
      else { int j = u - 720; s = 1 + j / 336; j %= 336; d = (rank + 1 + j % 7) & 7; item = j / 7; }
      gchar* dws = (gchar*)(P.ws_all[(unsigned)d < (unsigned)NR ? d : rank]);
      const int q = item / 3, rr = item - 3 * q;
      if (rr != 2) {
        const int xu = 2 * q + rr;
        const float* src = P.x + (size_t)(xu * 128 + (lane & 15)) * KSH + s * 64 + 8 * (lane >> 4);
        gchar* dst = dws + OFF_XS + ((size_t)((rank * NSTAGE + s) * XU + xu)) * UNIT + lane * 16;
#pragma unroll
        for (int kb = 0; kb < 2; ++kb) {
          f32x4 a[8], b[8];
#pragma unroll
          for (int blk = 0; blk < 8; ++blk) {
            const float* sp = src + (size_t)blk * 16 * KSH + kb * 32;
            a[blk] = *reinterpret_cast<const f32x4*>(sp);
            b[blk] = *reinterpret_cast<const f32x4*>(sp + 4);
          }
#pragma unroll
          for (int blk = 0; blk < 8; ++blk) {
            u32x4 v = {pk_bf16(a[blk][0], a[blk][1]), pk_bf16(a[blk][2], a[blk][3]),
                       pk_bf16(b[blk][0], b[blk][1]), pk_bf16(b[blk][2], b[blk][3])};
            st16(dst + (kb * 8 + blk) * 1024, v);
          }
        }
        gu32* flag = (gu32*)(dws + OFF_XF) + xu * 64 + rank * 8 + s;
        publish_u32(flag, call, lane == 0);
      } else {
        const int wu = q;
        const float* src = P.w + (size_t)(s * 64 + 8 * (lane >> 4)) * NTOT + d * NLOC + wu * 128 +
                           ((lane & 15) >> 2) * 16 + (lane & 3);
        gchar* dst = dws + OFF_WS + ((size_t)((rank * NSTAGE + s) * WU + wu)) * UNIT + lane * 16;
#pragma unroll
        for (int g = 0; g < 4; ++g) {
          const int kb = g >> 1, hf = g & 1;
          float v[4][8];
#pragma unroll
          for (int nt = 0; nt < 4; ++nt)
#pragma unroll
            for (int j = 0; j < 8; ++j)
              v[nt][j] = src[(size_t)(kb * 32 + j) * NTOT + hf * 64 + nt * 4];
#pragma unroll
          for (int nt = 0; nt < 4; ++nt) {
            u32x4 o = {pk_bf16(v[nt][0], v[nt][1]), pk_bf16(v[nt][2], v[nt][3]),
                       pk_bf16(v[nt][4], v[nt][5]), pk_bf16(v[nt][6], v[nt][7])};
            st16(dst + (kb * 8 + hf * 4 + nt) * 1024, o);
          }
        }
        gu32* flag = (gu32*)(dws + OFF_WF) + wu * 64 + rank * 8 + s;
        publish_u32(flag, call, lane == 0);
      }
      if (wave == 10 && i < 12) TS(20 + i);
    }
    if (wave < 12) TS(1 + wave - 10);
  }

  asm volatile("s_waitcnt lgkmcnt(0)" ::: "memory");
  __syncthreads();

  if (wave == 0) {
    TS(13);
    float m = __uint_as_float(lds_ld(ctl + (C_WMAX + (lane & 7)) * 4));
#pragma unroll
    for (int o = 4; o >= 1; o >>= 1) m = fmaxf(m, __shfl_xor(m, o));
    unsigned last = 0;
    if (lane == 0) {
      u64 v = ((u64)call << 32) | (u64)__float_as_uint(m);
      u64 old = __hip_atomic_fetch_max(reinterpret_cast<u64*>(myws + OFF_AMAX), v, __ATOMIC_RELAXED,
                                       __HIP_MEMORY_SCOPE_AGENT);
      asm volatile("s_waitcnt vmcnt(0)" :: "v"(old) : "memory");
      unsigned tk = __hip_atomic_fetch_add(reinterpret_cast<unsigned*>(myws + OFF_ARRIVE), 1u,
                                           __ATOMIC_RELAXED, __HIP_MEMORY_SCOPE_AGENT);
      last = (tk == call * NWG - 1) ? 1u : 0u;
    }
    last = __builtin_amdgcn_readfirstlane(last);
    if (last) {
      u64 v = __hip_atomic_load(reinterpret_cast<u64*>(myws + OFF_AMAX), __ATOMIC_RELAXED,
                                __HIP_MEMORY_SCOPE_AGENT);
      if (lane < NR) {
        gu64* dstp = (gu64*)((gchar*)(P.ws_all[lane]) + OFF_SLOTS) + rank;
        __hip_atomic_store(dstp, v, __ATOMIC_RELAXED, __HIP_MEMORY_SCOPE_SYSTEM);
      }
    }
    const u64* slots = reinterpret_cast<const u64*>(myws + OFF_SLOTS) + (lane & 7);
    const u64 t0 = __builtin_amdgcn_s_memrealtime();
    u64 v;
    while (true) {
      v = __hip_atomic_load(slots, __ATOMIC_RELAXED, __HIP_MEMORY_SCOPE_SYSTEM);
      if (__all((unsigned)(v >> 32) >= call)) break;
      if (__hip_atomic_load(tmo, __ATOMIC_RELAXED, __HIP_MEMORY_SCOPE_AGENT) == call ||
          __builtin_amdgcn_s_memrealtime() - t0 > DEADLINE_TICKS) {
        if (lane == 0) __hip_atomic_store(tmo, call, __ATOMIC_RELAXED, __HIP_MEMORY_SCOPE_AGENT);
        break;
      }
      __builtin_amdgcn_s_sleep(1);
    }
    float g = __uint_as_float((unsigned)v);
#pragma unroll
    for (int o = 4; o >= 1; o >>= 1) g = fmaxf(g, __shfl_xor(g, o));
    unsigned dead = (__hip_atomic_load(tmo, __ATOMIC_RELAXED, __HIP_MEMORY_SCOPE_AGENT) == call) ? 1u : 0u;
    if (lane == 0) { lds_st(ctl + C_GMAX * 4, __float_as_uint(g)); lds_st(ctl + C_DEAD * 4, dead); }
  }
  asm volatile("s_waitcnt lgkmcnt(0)" ::: "memory");
  __syncthreads();

  if (wave == 0) TS(14);
  const int owner = ti >> 1;
  if (wave < 8) {
    const float amax = __uint_as_float(lds_ld(ctl + C_GMAX * 4));
    const bool dead = lds_ld(ctl + C_DEAD * 4) != 0;
    const float scale = amax / 127.0f;
    const float inv = scale > 0.f ? 1.0f / scale : 0.f;
    gchar* ob = (gchar*)(P.out_all[(unsigned)owner < (unsigned)NR ? owner : 0]);
    const int row0 = (ti & 1) * 256 + wm * 64 + (lane & 15);
    const int col = rank * NLOC + tj * 128 + wn * 64 + (lane >> 4) * 16;
#pragma unroll
    for (int mt = 0; mt < 4; ++mt) {
      u32x4 o;
#pragma unroll
      for (int nt = 0; nt < 4; ++nt) {
        unsigned pk = 0;
#pragma unroll
        for (int j = 0; j < 4; ++j) {
          float q = rintf(acc[mt][nt][j] * inv);
          q = fminf(fmaxf(q, -127.f), 127.f);
          int qi = dead ? -128 : (int)q;
          pk |= ((unsigned)qi & 0xffu) << (8 * j);
        }
        o[nt] = pk;
      }
      st16(ob + (size_t)(row0 + mt * 16) * NTOT + col, o);
    }
    if (wave == 0) TS(17);
    asm volatile("s_waitcnt vmcnt(0)" ::: "memory");
    if (wave == 0) TS(18);
  }
  asm volatile("s_waitcnt lgkmcnt(0)" ::: "memory");
  __syncthreads();
  if (wave == 0) {
    TS(15);
    if (lane == 0) {
      gu32* f = (gu32*)((gchar*)(P.ws_all[(unsigned)owner < (unsigned)NR ? owner : 0]) + OFF_OUTF) +
                    rank * 32 + (ti & 1) * 16 + tj;
      __hip_atomic_store(f, call, __ATOMIC_RELAXED, __HIP_MEMORY_SCOPE_SYSTEM);
    }
    if (blockIdx.x == 0) {
      const unsigned* of = reinterpret_cast<const unsigned*>(myws + OFF_OUTF) + lane * 4;
      const u64 t0 = __builtin_amdgcn_s_memrealtime();
      while (true) {
        bool ok = true;
#pragma unroll
        for (int j = 0; j < 4; ++j)
          ok &= __hip_atomic_load(of + j, __ATOMIC_RELAXED, __HIP_MEMORY_SCOPE_SYSTEM) >= call;
        if (__all(ok)) break;
        if (__hip_atomic_load(tmo, __ATOMIC_RELAXED, __HIP_MEMORY_SCOPE_AGENT) == call ||
            __builtin_amdgcn_s_memrealtime() - t0 > DEADLINE_TICKS) {
          if (lane == 0) {
            __hip_atomic_store(tmo, call, __ATOMIC_RELAXED, __HIP_MEMORY_SCOPE_AGENT);
            ((gchar*)(P.out_all[rank]))[rank * NLOC] = (char)-128;
          }
          break;
        }
        __builtin_amdgcn_s_sleep(1);
      }
      __builtin_amdgcn_fence(__ATOMIC_ACQUIRE, "");
      TS(16);
    }
  }
}

extern "C" void kernel_launch(int rank, int nranks, ncclComm_t comm, hipStream_t stream,
                              void* const* d_in, const int* in_sizes, int n_in, void* d_out,
                              int out_size, const int* in_dtypes, int out_dtype,
                              void* const* d_out_all, void* const* d_ws_all, size_t ws_bytes) {
  if (nranks != NR || ws_bytes < WS_NEED || n_in < 2) {
    fprintf(stderr, "unexpected configuration\n");
    return;
  }
  Params P;
  P.x = static_cast<const float*>(d_in[0]);
  P.w = static_cast<const float*>(d_in[1]);
  P.out_all = d_out_all;
  P.ws_all = d_ws_all;
  P.rank = rank;
  hipLaunchKernelGGL(fused_gemm_q8, dim3(NWG), dim3(NTHREADS), 0, stream, P);
  HIP_CHECK(hipGetLastError());
}
```

```cpp
#include <hip/hip_runtime.h>
#include <rccl/rccl.h>
#include <cstdio>
#include <cstdint>

#define HIP_CHECK(expr) do { hipError_t _e = (expr); if (_e != hipSuccess) { \
    fprintf(stderr, "HIP error %s at %s:%d\n", hipGetErrorString(_e), __FILE__, __LINE__); \
    return; } } while (0)

typedef __attribute__((ext_vector_type(8))) short bf16x8;
typedef __attribute__((ext_vector_type(4))) float f32x4;
typedef __attribute__((ext_vector_type(4))) unsigned u32x4;
typedef unsigned long long u64;
#define GAS __attribute__((address_space(1)))
typedef GAS char gchar;
typedef GAS unsigned gu32;
typedef GAS unsigned long long gu64;
typedef GAS u32x4 gu32x4;

namespace {
constexpr int NR = 8;
constexpr int KSH = 512;
constexpr int NTOT = 16384;
constexpr int NLOC = 2048;
constexpr int MSH = 512;
constexpr int NSTAGE = 8;
constexpr int XU = 32;
constexpr int WU = 16;
constexpr int UNIT = 16384;
constexpr int NWG = 256;
constexpr int SU = 4;
constexpr int NUNIT = NR * NSTAGE * (XU + WU);
#define U1V 2560
constexpr int U1 = U1V;
constexpr int NITEM = U1 + 4 * (NUNIT - U1);
#define XB 2
constexpr int NSTEP = 64;
constexpr int NSLOT = 3;
constexpr int SLOT = 3 * UNIT;
#define NSEND 2
constexpr int NTHREADS = 640 + 64 * NSEND;

constexpr size_t OFF_TICKET = 0;
constexpr size_t OFF_TMO = 128;
constexpr size_t OFF_AMAX = 256;
constexpr size_t OFF_ARRIVE = 384;
constexpr size_t OFF_SLOTS = 512;
constexpr size_t OFF_OUTF = 1024;
constexpr size_t OFF_SQ = 2048;
constexpr size_t OFF_XF = 4096;
constexpr size_t OFF_WF = OFF_XF + XU * 64 * SU * 4;
constexpr size_t OFF_XS = 1u << 20;
constexpr size_t OFF_WS = OFF_XS + (size_t)NR * NSTAGE * XU * UNIT;
constexpr size_t WS_NEED = OFF_WS + (size_t)NR * NSTAGE * WU * UNIT;

constexpr u64 DEADLINE_TICKS = 40000000ull;

constexpr int C_CALL = 0, C_FULL = 2, C_DONE = 8, C_WMAX = 16, C_GMAX = 24, C_DEAD = 25, C_Q = 32;
constexpr int LDS_CTL = NSLOT * SLOT;
constexpr int LDS_BYTES = LDS_CTL + 512;
}

#define TS(i) do {} while (0)

#define LDPOL " sc0 sc1"

struct Params {
  const float* x;
  const float* w;
  void* const* out_all;
  void* const* ws_all;
  int rank;
};

__device__ __forceinline__ unsigned pk_bf16(float a, float b) {
  typedef __attribute__((ext_vector_type(2))) float f2;
  typedef __attribute__((ext_vector_type(2))) __bf16 b2;
  f2 v = {a, b};
  b2 r = __builtin_convertvector(v, b2);
  return __builtin_bit_cast(unsigned, r);
}

__device__ __forceinline__ void glds4k(const void* g, unsigned ldsaddr) {
  unsigned keep;
  asm volatile(
      "s_mov_b32 %0, m0\n\ts_mov_b32 m0, %2\n\ts_nop 0\n\t"
      "global_load_lds_dwordx4 %1, off" LDPOL "\n\t"
      "global_load_lds_dwordx4 %1, off offset:1024" LDPOL "\n\t"
      "global_load_lds_dwordx4 %1, off offset:2048" LDPOL "\n\t"
      "global_load_lds_dwordx4 %1, off offset:3072" LDPOL "\n\t"
      "s_mov_b32 m0, %0"
      : "=&s"(keep) : "v"(g), "s"(ldsaddr) : "memory");
}

__device__ __forceinline__ unsigned min12(u32x4 a, u32x4 b, u32x4 c) {
  unsigned m0 = min(min(a[0], a[1]), min(a[2], a[3]));
  unsigned m1 = min(min(b[0], b[1]), min(b[2], b[3]));
  unsigned m2 = min(min(c[0], c[1]), min(c[2], c[3]));
  return min(m0, min(m1, m2));
}
__device__ __forceinline__ unsigned lds_ld(unsigned addr) {
  unsigned v;
  asm volatile("ds_read_b32 %0, %1\n\ts_waitcnt lgkmcnt(0)" : "=v"(v) : "v"(addr) : "memory");
  return v;
}
__device__ __forceinline__ unsigned lds_ld_min2(unsigned addr) {
  typedef __attribute__((ext_vector_type(2))) unsigned u32x2;
  u32x2 v;
  asm volatile("ds_read_b64 %0, %1\n\ts_waitcnt lgkmcnt(0)" : "=v"(v) : "v"(addr) : "memory");
  return v[0] < v[1] ? v[0] : v[1];
}
__device__ __forceinline__ void lds_st(unsigned addr, unsigned v) {
  asm volatile("ds_write_b32 %0, %1" :: "v"(addr), "v"(v) : "memory");
}

__device__ __forceinline__ void st16(gchar* p, u32x4 v) {
  asm volatile("global_store_dwordx4 %0, %1, off sc0 sc1\n\ts_nop 1" :: "v"(p), "v"(v) : "memory");
}

__device__ __forceinline__ void publish_u32(gu32* flag, unsigned val, bool leader) {
  asm volatile("s_waitcnt vmcnt(0)" ::: "memory");
  if (leader) __hip_atomic_store(flag, val, __ATOMIC_RELAXED, __HIP_MEMORY_SCOPE_SYSTEM);
}

__global__ __launch_bounds__(NTHREADS, (NTHREADS + 255) / 256) void fused_gemm_q8(Params P) {
  __shared__ __attribute__((aligned(1024))) char lds[LDS_BYTES];
  const unsigned ldsbase = (unsigned)(uintptr_t)lds;
  const unsigned ctl = ldsbase + LDS_CTL;

  const int tid = threadIdx.x;
  const int lane = tid & 63;
  const int wave = __builtin_amdgcn_readfirstlane(tid >> 6);
  const int rank = P.rank;
  char* const myws = static_cast<char*>(P.ws_all[rank]);
  unsigned* const tmo = reinterpret_cast<unsigned*>(myws + OFF_TMO);

  if (tid < 128) {
    unsigned v = 0;
    if (tid == 0) {
      unsigned t = __hip_atomic_fetch_add(reinterpret_cast<unsigned*>(myws + OFF_TICKET), 1u,
                                          __ATOMIC_RELAXED, __HIP_MEMORY_SCOPE_AGENT);
      v = t / NWG + 1;
    }
    lds_st(ctl + tid * 4, v);
  }
  asm volatile("s_waitcnt lgkmcnt(0)" ::: "memory");
  __syncthreads();
  if (wave == 0) TS(0);
  const unsigned call = __builtin_amdgcn_readfirstlane(lds_ld(ctl + C_CALL * 4));

  const int bx = blockIdx.x & 7, bj = blockIdx.x >> 3;
  const int ti = 2 * (bj & 7) + (bx & 1);
  const int tj = 4 * (bx >> 1) + (bj >> 3);

  f32x4 acc[4][4];
  const int wm = wave & 3, wn = (wave >> 2) & 1;

  if (wave < 8) {
#pragma unroll
    for (int a = 0; a < 4; ++a)
#pragma unroll
      for (int b = 0; b < 4; ++b) acc[a][b] = f32x4{0.f, 0.f, 0.f, 0.f};
    const int xoff = (wm >> 1) * UNIT + ((wm & 1) * 4) * 1024 + lane * 16;
    const int woff = 2 * UNIT + (wn * 4) * 1024 + lane * 16;
    int slot = 0;
    for (int t = 0; t < NSTEP; ++t) {
      while (true) {
        if (lds_ld_min2(ctl + C_FULL * 4) > (unsigned)t) break;
        __builtin_amdgcn_s_sleep(1);
      }
      const char* sb = lds + slot * SLOT;
      bf16x8 xf[2][4], wf[2][4];
#pragma unroll
      for (int kb = 0; kb < 2; ++kb) {
#pragma unroll
        for (int i = 0; i < 4; ++i) {
          wf[kb][i] = *reinterpret_cast<const bf16x8*>(sb + woff + (kb * 8 + i) * 1024);
          xf[kb][i] = *reinterpret_cast<const bf16x8*>(sb + xoff + (kb * 8 + i) * 1024);
        }
      }
#pragma unroll
      for (int mt = 0; mt < 4; ++mt)
#pragma unroll
        for (int nt = 0; nt < 4; ++nt)
          acc[mt][nt] = __builtin_amdgcn_mfma_f32_16x16x32_bf16(wf[0][nt], xf[0][mt], acc[mt][nt], 0, 0, 0);
      asm volatile("s_waitcnt lgkmcnt(0)" ::: "memory");
      if (lane == 0) lds_st(ctl + (C_DONE + wave) * 4, (unsigned)(t + 1));
#pragma unroll
      for (int mt = 0; mt < 4; ++mt)
#pragma unroll
        for (int nt = 0; nt < 4; ++nt)
          acc[mt][nt] = __builtin_amdgcn_mfma_f32_16x16x32_bf16(wf[1][nt], xf[1][mt], acc[mt][nt], 0, 0, 0);
      slot = (slot == NSLOT - 1) ? 0 : slot + 1;
    }
    float m = 0.f;
#pragma unroll
    for (int a = 0; a < 4; ++a)
#pragma unroll
      for (int b = 0; b < 4; ++b)
#pragma unroll
        for (int j = 0; j < 4; ++j) m = fmaxf(m, fabsf(acc[a][b][j]));
#pragma unroll
    for (int o = 32; o >= 1; o >>= 1) m = fmaxf(m, __shfl_xor(m, o));
    if (lane == 0) lds_st(ctl + (C_WMAX + wave) * 4, __float_as_uint(m));
    if (wave == 0) TS(12);
  } else if (wave < 10) {
    const int L = wave - 8;
    int slot = 0;

    if (L == 0) {
      const unsigned* pf0 = reinterpret_cast<const unsigned*>(myws + OFF_XF) + ((2 * ti) * 64 + lane) * SU;
      const unsigned* pf2 = reinterpret_cast<const unsigned*>(myws + OFF_WF) + (tj * 64 + lane) * SU;
      u64 ready = 0, issued = 0;
      bool dead = false;
      unsigned spins = 0;
      const u64 t0 = __builtin_amdgcn_s_memrealtime();
      for (int t = 0; t < NSTEP; ++t) {
        if (!(ready & ~issued)) {
          asm volatile("s_waitcnt vmcnt(0)" ::: "memory");
          if (lane == 0) lds_st(ctl + C_FULL * 4, (unsigned)t);
          while (!dead) {
            u32x4 f0, f1, f2;
            asm volatile("global_load_dwordx4 %0, %3, off sc0 sc1\n\tglobal_load_dwordx4 %1, %3, off offset:1024 sc0 sc1\n\t"
                         "global_load_dwordx4 %2, %4, off sc0 sc1\n\ts_waitcnt vmcnt(0)"
                         : "=&v"(f0), "=&v"(f1), "=&v"(f2) : "v"(pf0), "v"(pf2) : "memory");
            ready = __ballot(min12(f0, f1, f2) >= call);
            if (ready & ~issued) break;
            if ((++spins & 15) == 0 &&
                (__hip_atomic_load(tmo, __ATOMIC_RELAXED, __HIP_MEMORY_SCOPE_AGENT) == call ||
                 __builtin_amdgcn_s_memrealtime() - t0 > DEADLINE_TICKS)) {
              if (lane == 0) __hip_atomic_store(tmo, call, __ATOMIC_RELAXED, __HIP_MEMORY_SCOPE_AGENT);
              dead = true;
            }
            __builtin_amdgcn_s_sleep(2);
          }
          if (dead) ready = ~0ull;
        }
        const int bit = __builtin_ctzll(ready & ~issued);
        issued |= 1ull << bit;
        if (lane == 0) lds_st(ctl + (C_Q + t) * 4, (unsigned)(bit + 1));
        if (t >= NSLOT) {
          while (true) {
            unsigned d = lds_ld(ctl + (C_DONE + (lane & 7)) * 4);
            if (__all(d >= (unsigned)(t - NSLOT + 1))) break;
            __builtin_amdgcn_s_sleep(1);
          }
        }
        u32x4 f0, f1, f2;
        asm volatile("global_load_dwordx4 %0, %3, off sc0 sc1\n\tglobal_load_dwordx4 %1, %3, off offset:1024 sc0 sc1\n\t"
                     "global_load_dwordx4 %2, %4, off sc0 sc1"
                     : "=&v"(f0), "=&v"(f1), "=&v"(f2) : "v"(pf0), "v"(pf2) : "memory");
        const char* gx = myws + OFF_XS + ((size_t)(bit * XU + 2 * ti)) * UNIT + lane * 16;
        const char* gw = myws + OFF_WS + ((size_t)(bit * WU + tj)) * UNIT + lane * 16;
        const unsigned lx = ldsbase + slot * SLOT;
        const unsigned lw = ldsbase + slot * SLOT + 2 * UNIT;
#pragma unroll
        for (int i = 0; i < 4; ++i) glds4k(gx + i * 4096, lx + i * 4096);
#pragma unroll
        for (int i = 0; i < 2; ++i) glds4k(gw + i * 4096, lw + i * 4096);
        asm volatile("s_waitcnt vmcnt(24)" : "+v"(f0), "+v"(f1), "+v"(f2) :: "memory");
        if (lane == 0) lds_st(ctl + C_FULL * 4, (unsigned)t);
        ready |= __ballot(min12(f0, f1, f2) >= call);
        slot = (slot == NSLOT - 1) ? 0 : slot + 1;
      }
    } else {
      for (int t = 0; t < NSTEP; ++t) {
        unsigned e = lds_ld(ctl + (C_Q + t) * 4);
        if (e == 0) {
          asm volatile("s_waitcnt vmcnt(0)" ::: "memory");
          if (lane == 0) lds_st(ctl + (C_FULL + 1) * 4, (unsigned)t);
          while ((e = lds_ld(ctl + (C_Q + t) * 4)) == 0) __builtin_amdgcn_s_sleep(1);
        }
        e = __builtin_amdgcn_readfirstlane(e);
        const int bit = (int)e - 1;
        if (t >= NSLOT) {
          while (true) {
            unsigned d = lds_ld(ctl + (C_DONE + (lane & 7)) * 4);
            if (__all(d >= (unsigned)(t - NSLOT + 1))) break;
            __builtin_amdgcn_s_sleep(1);
          }
        }
        const char* gx = myws + OFF_XS + ((size_t)(bit * XU + 2 * ti + 1)) * UNIT + lane * 16;
        const char* gw = myws + OFF_WS + ((size_t)(bit * WU + tj)) * UNIT + 8192 + lane * 16;
        const unsigned lx = ldsbase + slot * SLOT + UNIT;
        const unsigned lw = ldsbase + slot * SLOT + 2 * UNIT + 8192;
#pragma unroll
        for (int i = 0; i < 4; ++i) glds4k(gx + i * 4096, lx + i * 4096);
#pragma unroll
        for (int i = 0; i < 2; ++i) glds4k(gw + i * 4096, lw + i * 4096);
        asm volatile("s_waitcnt vmcnt(24)" ::: "memory");
        if (lane == 0) lds_st(ctl + (C_FULL + 1) * 4, (unsigned)t);
        slot = (slot == NSLOT - 1) ? 0 : slot + 1;
      }
    }
    asm volatile("s_waitcnt vmcnt(0)" ::: "memory");
    if (lane == 0) lds_st(ctl + (C_FULL + L) * 4, (unsigned)NSTEP);
    if (L == 0) TS(11);
  } else {
    __builtin_amdgcn_s_setprio(3);
    unsigned* const sq = reinterpret_cast<unsigned*>(myws + OFF_SQ);
    const unsigned qbase = (call - 1) * NITEM - NSEND * NWG;
    unsigned tk = qbase + blockIdx.x * NSEND + (wave - 10);
    for (int i = 0;; ++i) {
      const int tq = (int)(__builtin_amdgcn_readfirstlane(tk) - qbase);
      if ((unsigned)tq >= (unsigned)NITEM) break;
      if (lane == 0) tk = __hip_atomic_fetch_add(sq, 1u, __ATOMIC_RELAXED, __HIP_MEMORY_SCOPE_AGENT);
      int u, f0, f1;
      if (tq < U1) { u = tq; f0 = 0; f1 = 16; }
      else { u = U1 + ((tq - U1) >> 2); f0 = ((tq - U1) & 3) * 4; f1 = f0 + 4; }
      int d, s, item;
      if (u < 336) { s = 0; d = (rank + 1 + u % 7) & 7; item = u / 7; }
      else if (u < 720) { int j = u - 336; s = j / 48; item = j % 48; d = rank; }
      else { int j = u - 720; s = 1 + j / 336; j %= 336; d = (rank + 1 + j % 7) & 7; item = j / 7; }
      gchar* dws = (gchar*)(P.ws_all[(unsigned)d < (unsigned)NR ? d : rank]);
      const int q = item / 3, rr = item - 3 * q;
      gchar* flag;
      if (rr != 2) {
        const int xu = 2 * q + rr;
        const float* src = P.x + (size_t)(xu * 128 + (lane & 15)) * KSH + s * 64 + 8 * (lane >> 4);
        gchar* dst = dws + OFF_XS + ((size_t)((rank * NSTAGE + s) * XU + xu)) * UNIT + lane * 16;
        for (int f = f0; f < f1; f += XB) {
          f32x4 a[XB], b[XB];
#pragma unroll
          for (int j = 0; j < XB; ++j) {
            const float* sp = src + (size_t)((f + j) & 7) * 16 * KSH + ((f + j) >> 3) * 32;
            a[j] = *reinterpret_cast<const f32x4*>(sp);
            b[j] = *reinterpret_cast<const f32x4*>(sp + 4);
          }
#pragma unroll
          for (int j = 0; j < XB; ++j) {
            u32x4 v = {pk_bf16(a[j][0], a[j][1]), pk_bf16(a[j][2], a[j][3]),
                       pk_bf16(b[j][0], b[j][1]), pk_bf16(b[j][2], b[j][3])};
            st16(dst + (f + j) * 1024, v);
          }
        }
        flag = dws + OFF_XF + (xu * 64 + rank * 8 + s) * 16;
      } else {
        const int wu = q;
        const float* src = P.w + (size_t)(s * 64 + 8 * (lane >> 4)) * NTOT + d * NLOC + wu * 128 +
                           ((lane & 15) >> 2) * 16 + (lane & 3);
        gchar* dst = dws + OFF_WS + ((size_t)((rank * NSTAGE + s) * WU + wu)) * UNIT + lane * 16;
        for (int f = f0; f < f1; f += XB) {
          float v[XB][8];
#pragma unroll
          for (int jj = 0; jj < XB; ++jj) {
            const int kb = (f + jj) >> 3, nb = (f + jj) & 7;
            const float* sp = src + (size_t)(kb * 32) * NTOT + (nb >> 2) * 64 + (nb & 3) * 4;
#pragma unroll
            for (int j = 0; j < 8; ++j) v[jj][j] = sp[(size_t)j * NTOT];
          }
#pragma unroll
          for (int jj = 0; jj < XB; ++jj) {
            u32x4 o = {pk_bf16(v[jj][0], v[jj][1]), pk_bf16(v[jj][2], v[jj][3]),
                       pk_bf16(v[jj][4], v[jj][5]), pk_bf16(v[jj][6], v[jj][7])};
            st16(dst + (f + jj) * 1024, o);
          }
        }
        flag = dws + OFF_WF + (wu * 64 + rank * 8 + s) * 16;
      }
      asm volatile("s_waitcnt vmcnt(0)" ::: "memory");
      {
        const int fl = lane * 4;
        if (lane < 4 && fl >= f0 && fl < f1)
          __hip_atomic_store((gu32*)flag + lane, call, __ATOMIC_RELAXED, __HIP_MEMORY_SCOPE_SYSTEM);
      }
      if (wave == 10 && i < 12) TS(20 + i);
    }
    if (wave < 12) TS(1 + wave - 10);
  }

  asm volatile("s_waitcnt lgkmcnt(0)" ::: "memory");
  __syncthreads();

  if (wave == 0) {
    TS(13);
    float m = __uint_as_float(lds_ld(ctl + (C_WMAX + (lane & 7)) * 4));
#pragma unroll
    for (int o = 4; o >= 1; o >>= 1) m = fmaxf(m, __shfl_xor(m, o));
    unsigned last = 0;
    if (lane == 0) {
      u64 v = ((u64)call << 32) | (u64)__float_as_uint(m);
      u64 old = __hip_atomic_fetch_max(reinterpret_cast<u64*>(myws + OFF_AMAX), v, __ATOMIC_RELAXED,
                                       __HIP_MEMORY_SCOPE_AGENT);
      asm volatile("s_waitcnt vmcnt(0)" :: "v"(old) : "memory");
      unsigned tk = __hip_atomic_fetch_add(reinterpret_cast<unsigned*>(myws + OFF_ARRIVE), 1u,
                                           __ATOMIC_RELAXED, __HIP_MEMORY_SCOPE_AGENT);
      last = (tk == call * NWG - 1) ? 1u : 0u;
    }
    last = __builtin_amdgcn_readfirstlane(last);
    if (last) {
      u64 v = __hip_atomic_load(reinterpret_cast<u64*>(myws + OFF_AMAX), __ATOMIC_RELAXED,
                                __HIP_MEMORY_SCOPE_AGENT);
      if (lane < NR) {
        gu64* dstp = (gu64*)((gchar*)(P.ws_all[lane]) + OFF_SLOTS) + rank;
        __hip_atomic_store(dstp, v, __ATOMIC_RELAXED, __HIP_MEMORY_SCOPE_SYSTEM);
      }
    }
    const u64* slots = reinterpret_cast<const u64*>(myws + OFF_SLOTS) + (lane & 7);
    const u64 t0 = __builtin_amdgcn_s_memrealtime();
    u64 v;
    while (true) {
      v = __hip_atomic_load(slots, __ATOMIC_RELAXED, __HIP_MEMORY_SCOPE_SYSTEM);
      if (__all((unsigned)(v >> 32) >= call)) break;
      if (__hip_atomic_load(tmo, __ATOMIC_RELAXED, __HIP_MEMORY_SCOPE_AGENT) == call ||
          __builtin_amdgcn_s_memrealtime() - t0 > DEADLINE_TICKS) {
        if (lane == 0) __hip_atomic_store(tmo, call, __ATOMIC_RELAXED, __HIP_MEMORY_SCOPE_AGENT);
        break;
      }
      __builtin_amdgcn_s_sleep(1);
    }
    float g = __uint_as_float((unsigned)v);
#pragma unroll
    for (int o = 4; o >= 1; o >>= 1) g = fmaxf(g, __shfl_xor(g, o));
    unsigned dead = (__hip_atomic_load(tmo, __ATOMIC_RELAXED, __HIP_MEMORY_SCOPE_AGENT) == call) ? 1u : 0u;
    if (lane == 0) { lds_st(ctl + C_GMAX * 4, __float_as_uint(g)); lds_st(ctl + C_DEAD * 4, dead); }
  }
  asm volatile("s_waitcnt lgkmcnt(0)" ::: "memory");
  __syncthreads();

  if (wave == 0) TS(14);
  const int owner = ti >> 1;
  if (wave < 8) {
    const float amax = __uint_as_float(lds_ld(ctl + C_GMAX * 4));
    const bool dead = lds_ld(ctl + C_DEAD * 4) != 0;
    const float scale = amax / 127.0f;
    const float inv = scale > 0.f ? 1.0f / scale : 0.f;
    gchar* ob = (gchar*)(P.out_all[(unsigned)owner < (unsigned)NR ? owner : 0]);
    const int row0 = (ti & 1) * 256 + wm * 64 + (lane & 15);
    const int col = rank * NLOC + tj * 128 + wn * 64 + (lane >> 4) * 16;
#pragma unroll
    for (int mt = 0; mt < 4; ++mt) {
      u32x4 o;
#pragma unroll
      for (int nt = 0; nt < 4; ++nt) {
        unsigned pk = 0;
#pragma unroll
        for (int j = 0; j < 4; ++j) {
          float q = rintf(acc[mt][nt][j] * inv);
          q = fminf(fmaxf(q, -127.f), 127.f);
          int qi = dead ? -128 : (int)q;
          pk |= ((unsigned)qi & 0xffu) << (8 * j);
        }
        o[nt] = pk;
      }
      st16(ob + (size_t)(row0 + mt * 16) * NTOT + col, o);
    }
    if (wave == 0) TS(17);
    asm volatile("s_waitcnt vmcnt(0)" ::: "memory");
    if (wave == 0) TS(18);
  }
  asm volatile("s_waitcnt lgkmcnt(0)" ::: "memory");
  __syncthreads();
  if (wave == 0) {
    TS(15);
    if (lane == 0) {
      gu32* f = (gu32*)((gchar*)(P.ws_all[(unsigned)owner < (unsigned)NR ? owner : 0]) + OFF_OUTF) +
                    rank * 32 + (ti & 1) * 16 + tj;
      __hip_atomic_store(f, call, __ATOMIC_RELAXED, __HIP_MEMORY_SCOPE_SYSTEM);
    }
    if (blockIdx.x == 0) {
      const unsigned* of = reinterpret_cast<const unsigned*>(myws + OFF_OUTF) + lane * 4;
      const u64 t0 = __builtin_amdgcn_s_memrealtime();
      while (true) {
        bool ok = true;
#pragma unroll
        for (int j = 0; j < 4; ++j)
          ok &= __hip_atomic_load(of + j, __ATOMIC_RELAXED, __HIP_MEMORY_SCOPE_SYSTEM) >= call;
        if (__all(ok)) break;
        if (__hip_atomic_load(tmo, __ATOMIC_RELAXED, __HIP_MEMORY_SCOPE_AGENT) == call ||
            __builtin_amdgcn_s_memrealtime() - t0 > DEADLINE_TICKS) {
          if (lane == 0) {
            __hip_atomic_store(tmo, call, __ATOMIC_RELAXED, __HIP_MEMORY_SCOPE_AGENT);
            ((gchar*)(P.out_all[rank]))[rank * NLOC] = (char)-128;
          }
          break;
        }
        __builtin_amdgcn_s_sleep(1);
      }
      __builtin_amdgcn_fence(__ATOMIC_ACQUIRE, "");
      TS(16);
    }
  }
}

extern "C" void kernel_launch(int rank, int nranks, ncclComm_t comm, hipStream_t stream,
                              void* const* d_in, const int* in_sizes, int n_in, void* d_out,
                              int out_size, const int* in_dtypes, int out_dtype,
                              void* const* d_out_all, void* const* d_ws_all, size_t ws_bytes) {
  if (nranks != NR || ws_bytes < WS_NEED || n_in < 2) {
    fprintf(stderr, "unexpected configuration\n");
    return;
  }
  Params P;
  P.x = static_cast<const float*>(d_in[0]);
  P.w = static_cast<const float*>(d_in[1]);
  P.out_all = d_out_all;
  P.ws_all = d_ws_all;
  P.rank = rank;
  hipLaunchKernelGGL(fused_gemm_q8, dim3(NWG), dim3(NTHREADS), 0, stream, P);
  HIP_CHECK(hipGetLastError());
}
```

```cpp
#include <hip/hip_runtime.h>
#include <rccl/rccl.h>
#include <cstdio>
#include <cstdint>
#include <type_traits>

#define HIP_CHECK(expr) do { hipError_t _e = (expr); if (_e != hipSuccess) { \
    fprintf(stderr, "HIP error %s at %s:%d\n", hipGetErrorString(_e), __FILE__, __LINE__); \
    return; } } while (0)

typedef __attribute__((ext_vector_type(8))) short bf16x8;
typedef __attribute__((ext_vector_type(4))) float f32x4;
typedef __attribute__((ext_vector_type(4))) unsigned u32x4;
typedef unsigned long long u64;
#define GAS __attribute__((address_space(1)))
typedef GAS char gchar;
typedef GAS unsigned gu32;
typedef GAS unsigned long long gu64;
typedef GAS u32x4 gu32x4;

namespace {
constexpr int NR = 8;
constexpr int KSH = 512;
constexpr int NTOT = 16384;
constexpr int NLOC = 2048;
constexpr int MSH = 512;
constexpr int NSTAGE = 8;
constexpr int XU = 32;
constexpr int WU = 16;
constexpr int UNIT = 16384;
constexpr int NWG = 256;
constexpr int SU = 4;
constexpr int NUNIT = NR * NSTAGE * (XU + WU);
#define U1V 2560
constexpr int U1 = U1V;
constexpr int NITEM = U1 + 4 * (NUNIT - U1);
#define THR 3
#define LAUX 17
#define EXP 0
#define LPRIO 1
#define PSLEEP 2
constexpr int NSTEP = 64;
constexpr int NSLOT = 3;
constexpr int SLOT = 3 * UNIT;
#define NSEND 2
#define NLOADV 6
constexpr int NLOAD = NLOADV;
constexpr int QPL = 12 / NLOAD;
constexpr int QE = NLOAD == 4 ? 64 : 128;
constexpr int W_SEND = 8 + NLOAD;
constexpr int NTHREADS = 64 * (W_SEND + NSEND);
constexpr unsigned F_RAW = 1u << 12;
constexpr int MAXESC = 16;

constexpr size_t OFF_TICKET = 0;
constexpr size_t OFF_TMO = 128;
constexpr size_t OFF_AMAX = 256;
constexpr size_t OFF_ARRIVE = 384;
constexpr size_t OFF_SLOTS = 512;
constexpr size_t OFF_OUTF = 1024;
constexpr size_t OFF_SQ = 2048;
constexpr size_t OFF_XF = 4096;
constexpr size_t OFF_WF = OFF_XF + XU * 64 * SU * 4;
constexpr size_t OFF_XS = 1u << 20;
constexpr size_t OFF_WS = OFF_XS + (size_t)NR * NSTAGE * XU * UNIT;
constexpr size_t WS_NEED = OFF_WS + (size_t)NR * NSTAGE * WU * UNIT;

constexpr u64 DEADLINE_TICKS = 40000000ull;

constexpr int C_CALL = 0, C_FULL = 8, C_DONE = 16, C_WMAX = 24, C_GMAX = 32, C_DEAD = 33;
constexpr int LDS_CTL = NSLOT * SLOT;
constexpr int LDS_Q = LDS_CTL + 512;
constexpr int LDS_BYTES = LDS_Q + 64 * QE;
}

#define TS(i) do {} while (0)
#define TV(i, v) do {} while (0)
#define CYC() 0ull

struct Params {
  const float* x;
  const float* w;
  void* const* out_all;
  void* const* ws_all;
  int rank;
};

__device__ __forceinline__ unsigned pk_bf16(float a, float b) {
  typedef __attribute__((ext_vector_type(2))) float f2;
  typedef __attribute__((ext_vector_type(2))) __bf16 b2;
  f2 v = {a, b};
  b2 r = __builtin_convertvector(v, b2);
  return __builtin_bit_cast(unsigned, r);
}

__device__ __forceinline__ unsigned lds_ld(unsigned addr) {
  unsigned v;
  asm volatile("ds_read_b32 %0, %1\n\ts_waitcnt lgkmcnt(0)" : "=v"(v) : "v"(addr) : "memory");
  return v;
}
__device__ __forceinline__ unsigned lds_ld_min4(unsigned addr) {
  u32x4 v;
  if (NLOAD == 4) {
    asm volatile("ds_read_b128 %0, %1\n\ts_waitcnt lgkmcnt(0)" : "=v"(v) : "v"(addr) : "memory");
    return min(min(v[0], v[1]), min(v[2], v[3]));
  }
  u32x4 w;
  asm volatile("ds_read_b128 %0, %2\n\tds_read_b128 %1, %2 offset:16\n\ts_waitcnt lgkmcnt(0)" : "=&v"(v), "=&v"(w) : "v"(addr) : "memory");
  return min(min(min(v[0], v[1]), min(v[2], v[3])), min(w[0], w[1]));
}
__device__ __forceinline__ void lds_st16(unsigned addr, u32x4 v) {
  asm volatile("ds_write_b128 %0, %1" :: "v"(addr), "v"(v) : "memory");
}
__device__ __forceinline__ void lds_st1(unsigned addr, unsigned v) {
  asm volatile("ds_write_b8 %0, %1" :: "v"(addr), "v"(v) : "memory");
}
__device__ __forceinline__ void lds_st(unsigned addr, unsigned v) {
  asm volatile("ds_write_b32 %0, %1" :: "v"(addr), "v"(v) : "memory");
}

__device__ __forceinline__ void st16(gchar* p, u32x4 v) {
  asm volatile("global_store_dwordx4 %0, %1, off sc0 sc1\n\ts_nop 1" :: "v"(p), "v"(v) : "memory");
}

__device__ __forceinline__ void st4(gchar* p, unsigned v) {
  asm volatile("global_store_dword %0, %1, off sc0 sc1\n\ts_nop 1" :: "v"(p), "v"(v) : "memory");
}

__device__ __forceinline__ u32x4 unpack8(unsigned l0, unsigned l1, unsigned n, unsigned base4) {
  const unsigned h0 = ((n & 0x07070707u) + base4) | ((n & 0x08080808u) << 4);
  const unsigned h1 = (((n >> 4) & 0x07070707u) + base4) | (n & 0x80808080u);
  return u32x4{__builtin_amdgcn_perm(h0, l0, 0x05010400u), __builtin_amdgcn_perm(h0, l0, 0x07030602u),
               __builtin_amdgcn_perm(h1, l1, 0x05010400u), __builtin_amdgcn_perm(h1, l1, 0x07030602u)};
}

__device__ __forceinline__ unsigned send_block(const unsigned (&p)[4][4], float m, gchar* dst, int lane) {
  int mi = (int)__float_as_uint(m);
  mi = max(mi, __builtin_amdgcn_update_dpp(mi, mi, 0x111, 0xf, 0xf, false));
  mi = max(mi, __builtin_amdgcn_update_dpp(mi, mi, 0x112, 0xf, 0xf, false));
  mi = max(mi, __builtin_amdgcn_update_dpp(mi, mi, 0x114, 0xf, 0xf, false));
  mi = max(mi, __builtin_amdgcn_update_dpp(mi, mi, 0x118, 0xf, 0xf, false));
  mi = max(mi, __builtin_amdgcn_update_dpp(mi, mi, 0x142, 0xa, 0xf, false));
  mi = max(mi, __builtin_amdgcn_update_dpp(mi, mi, 0x143, 0xc, 0xf, false));
  const unsigned top = (pk_bf16(__uint_as_float((unsigned)__builtin_amdgcn_readlane(mi, 63)), 0.f) >> 8) & 0x7fu;
  const unsigned base = __builtin_amdgcn_readfirstlane(top > 7u ? top - 7u : 0u);
  const unsigned base4 = base * 0x01010101u;
  unsigned lo[4][2], hi[4][2], nb[4], aa = ~0u, ao = 0u;
#pragma unroll
  for (int j = 0; j < 4; ++j) {
    lo[j][0] = __builtin_amdgcn_perm(p[j][1], p[j][0], 0x06040200u);
    hi[j][0] = __builtin_amdgcn_perm(p[j][1], p[j][0], 0x07050301u);
    lo[j][1] = __builtin_amdgcn_perm(p[j][3], p[j][2], 0x06040200u);
    hi[j][1] = __builtin_amdgcn_perm(p[j][3], p[j][2], 0x07050301u);
    const unsigned u0 = (hi[j][0] | 0x80808080u) - base4;
    const unsigned u1 = (hi[j][1] | 0x80808080u) - base4;
    aa &= u0 & u1;
    ao |= u0 | u1;
    nb[j] = (u0 & 0x07070707u) | ((hi[j][0] >> 4) & 0x08080808u) | ((u1 & 0x07070707u) << 4) |
            (hi[j][1] & 0x80808080u);
  }
  unsigned info = base;
  if (__ballot(((~aa & 0x80808080u) | (ao & 0x78787878u)) != 0u)) {
    unsigned em[4][2], c = 0;
#pragma unroll
    for (int j = 0; j < 4; ++j)
#pragma unroll
      for (int h = 0; h < 2; ++h) {
        const unsigned u = (hi[j][h] | 0x80808080u) - base4;
        em[j][h] = (~u | ((u & 0x78787878u) + 0x78787878u)) & 0x80808080u;
        c += __builtin_popcount(em[j][h]);
      }
    unsigned total = 0;
#pragma unroll
    for (int b = 0; b < 6; ++b) total += (unsigned)__builtin_popcountll(__ballot((c >> b) & 1u)) << b;
    if (total > (unsigned)MAXESC) {
#pragma unroll
      for (int j = 0; j < 4; ++j) {
        st16(dst + j * 1024, u32x4{p[j][0], p[j][1], p[j][2], p[j][3]});
        if (j == 1) asm volatile("s_waitcnt vmcnt(0)" ::: "memory");
      }
      return F_RAW;
    }
    unsigned n = 0;
#pragma unroll
    for (int j = 0; j < 4; ++j)
#pragma unroll
      for (int h = 0; h < 2; ++h) {
        if (__ballot(em[j][h] != 0u) == 0) continue;
        for (int b = 0; b < 4; ++b) {
          const bool on = (em[j][h] >> (8 * b + 7)) & 1u;
          const u64 bal = __ballot(on);
          const unsigned idx = n + __builtin_amdgcn_mbcnt_hi((unsigned)(bal >> 32), __builtin_amdgcn_mbcnt_lo((unsigned)bal, 0u));
          if (on) st4(dst - lane * 16 + 3072 + idx * 4,
                      (unsigned)(j * 512 + lane * 8 + h * 4 + b) | (((hi[j][h] >> (8 * b)) & 0xffu) << 16));
          n += (unsigned)__builtin_popcountll(bal);
        }
      }
    info |= total << 7;
  }
  st16(dst, u32x4{lo[0][0], lo[0][1], lo[1][0], lo[1][1]});
  st16(dst + 1024, u32x4{lo[2][0], lo[2][1], lo[3][0], lo[3][1]});
  asm volatile("s_waitcnt vmcnt(1)" ::: "memory");
  st16(dst + 2048, u32x4{nb[0], nb[1], nb[2], nb[3]});
  return info;
}

struct LBuf {
  u32x4 a[QPL], b[QPL], n[QPL];
  unsigned e[QPL], info[QPL], so[QPL];
};

__global__ __launch_bounds__(NTHREADS, (NTHREADS + 255) / 256) void fused_gemm_q8(Params P) {
  __shared__ __attribute__((aligned(1024))) char lds[LDS_BYTES];
  const unsigned ldsbase = (unsigned)(uintptr_t)lds;
  const unsigned ctl = ldsbase + LDS_CTL;

  const int tid = threadIdx.x;
  const int lane = tid & 63;
  const int wave = __builtin_amdgcn_readfirstlane(tid >> 6);
  const int rank = P.rank;
  char* const myws = static_cast<char*>(P.ws_all[rank]);
  unsigned* const tmo = reinterpret_cast<unsigned*>(myws + OFF_TMO);

  if (tid >= 128 && tid < 128 + 4 * QE) lds_st16(ldsbase + LDS_Q + (tid - 128) * 16, u32x4{0u, 0u, 0u, 0u});
  if (tid < 128) {
    unsigned v = 0;
    if (tid == 0) {
      unsigned t = __hip_atomic_fetch_add(reinterpret_cast<unsigned*>(myws + OFF_TICKET), 1u,
                                          __ATOMIC_RELAXED, __HIP_MEMORY_SCOPE_AGENT);
      v = t / NWG + 1;
    }
    lds_st(ctl + tid * 4, v);
  }
  asm volatile("s_waitcnt lgkmcnt(0)" ::: "memory");
  __syncthreads();
  if (wave == 0) TS(0);
  const unsigned call = __builtin_amdgcn_readfirstlane(lds_ld(ctl + C_CALL * 4));

  const int bx = blockIdx.x & 7, bj = blockIdx.x >> 3;
  const int ti = 2 * (bj & 7) + (bx & 1);
  const int tj = 4 * (bx >> 1) + (bj >> 3);

  const unsigned tag = call << 13;
  f32x4 acc[4][4];
  const int wm = wave & 3, wn = (wave >> 2) & 1;

  if (wave < 8) {
#pragma unroll
    for (int a = 0; a < 4; ++a)
#pragma unroll
      for (int b = 0; b < 4; ++b) acc[a][b] = f32x4{0.f, 0.f, 0.f, 0.f};
    const int xoff = (wm >> 1) * UNIT + ((wm & 1) * 4) * 1024 + lane * 16;
    const int woff = 2 * UNIT + (wn * 4) * 1024 + lane * 16;
    int slot = 0;
    const unsigned* pf0 = reinterpret_cast<const unsigned*>(myws + OFF_XF) + ((2 * ti) * 64 + lane) * SU;
    const unsigned* pf2 = reinterpret_cast<const unsigned*>(myws + OFF_WF) + (tj * 64 + lane) * SU;
    u64 seen = 0;
    unsigned cnt = wave == 7 ? 0u : (unsigned)NSTEP, spins = 0;
    bool dead = false;
    const u64 t0 = __builtin_amdgcn_s_memrealtime();
    for (int t = 0; t < NSTEP; ++t) {
      while (true) {
        if (lds_ld_min4(ctl + C_FULL * 4) > (unsigned)t) break;
        if (cnt >= (unsigned)NSTEP) { __builtin_amdgcn_s_sleep(1); continue; }
        u32x4 f0, f1, f2;
        asm volatile("global_load_dwordx4 %0, %3, off sc0 sc1\n\tglobal_load_dwordx4 %1, %3, off offset:1024 sc0 sc1\n\t"
                     "global_load_dwordx4 %2, %4, off sc0 sc1\n\ts_waitcnt vmcnt(0)"
                     : "=&v"(f0), "=&v"(f1), "=&v"(f2) : "v"(pf0), "v"(pf2) : "memory");
        unsigned x = 0;
#pragma unroll
        for (int j = 0; j < 4; ++j) x |= (f0[j] ^ tag) | (f1[j] ^ tag) | (f2[j] ^ tag);
        const u64 nm = __ballot(dead || (x >> 13) == 0u) & ~seen;
        if (nm) {
          if ((nm >> lane) & 1) {
            const unsigned pos = cnt + __builtin_amdgcn_mbcnt_hi((unsigned)(nm >> 32), __builtin_amdgcn_mbcnt_lo((unsigned)nm, 0u));
            const unsigned ea = ldsbase + LDS_Q + pos * QE;
            if (dead) {
#pragma unroll
              for (int j = 0; j < 4; ++j) { f0[j] = tag | F_RAW; f1[j] = tag | F_RAW; f2[j] = tag | F_RAW; }
            }
            const unsigned s1 = (unsigned)lane + 1;
            if (NLOAD == 4) {
              lds_st16(ea, u32x4{f0[0], f0[1], f0[2], s1});
              lds_st16(ea + 16, u32x4{f0[3], f1[0], f1[1], s1});
              lds_st16(ea + 32, u32x4{f1[2], f1[3], f2[0], s1});
              lds_st16(ea + 48, u32x4{f2[1], f2[2], f2[3], s1});
            } else {
              lds_st16(ea, u32x4{f0[0], f0[1], tag, s1});
              lds_st16(ea + 16, u32x4{f0[2], f0[3], tag, s1});
              lds_st16(ea + 32, u32x4{f1[0], f1[1], tag, s1});
              lds_st16(ea + 48, u32x4{f1[2], f1[3], tag, s1});
              lds_st16(ea + 64, u32x4{f2[0], f2[1], tag, s1});
              lds_st16(ea + 80, u32x4{f2[2], f2[3], tag, s1});
            }
          }
          cnt += (unsigned)__builtin_popcountll(nm);
          seen |= nm;
        } else if ((++spins & 15) == 0 &&
                   (__hip_atomic_load(tmo, __ATOMIC_RELAXED, __HIP_MEMORY_SCOPE_AGENT) == call ||
                    __builtin_amdgcn_s_memrealtime() - t0 > DEADLINE_TICKS)) {
          if (lane == 0) __hip_atomic_store(tmo, call, __ATOMIC_RELAXED, __HIP_MEMORY_SCOPE_AGENT);
          dead = true;
        }
      }
      const char* sb = lds + slot * SLOT;
      bf16x8 xf[2][4], wf[2][4];
#pragma unroll
      for (int kb = 0; kb < 2; ++kb) {
#pragma unroll
        for (int i = 0; i < 4; ++i) {
          wf[kb][i] = *reinterpret_cast<const bf16x8*>(sb + woff + (kb * 8 + i) * 1024);
          xf[kb][i] = *reinterpret_cast<const bf16x8*>(sb + xoff + (kb * 8 + i) * 1024);
        }
      }
#pragma unroll
      for (int mt = 0; mt < 4; ++mt)
#pragma unroll
        for (int nt = 0; nt < 4; ++nt)
          acc[mt][nt] = __builtin_amdgcn_mfma_f32_16x16x32_bf16(wf[0][nt], xf[0][mt], acc[mt][nt], 0, 0, 0);
      asm volatile("s_waitcnt lgkmcnt(0)" ::: "memory");
      if (lane == 0) lds_st(ctl + (C_DONE + wave) * 4, (unsigned)(t + 1));
#pragma unroll
      for (int mt = 0; mt < 4; ++mt)
#pragma unroll
        for (int nt = 0; nt < 4; ++nt)
          acc[mt][nt] = __builtin_amdgcn_mfma_f32_16x16x32_bf16(wf[1][nt], xf[1][mt], acc[mt][nt], 0, 0, 0);
      slot = (slot == NSLOT - 1) ? 0 : slot + 1;
    }
    float m = 0.f;
#pragma unroll
    for (int a = 0; a < 4; ++a)
#pragma unroll
      for (int b = 0; b < 4; ++b)
#pragma unroll
        for (int j = 0; j < 4; ++j) m = fmaxf(m, fabsf(acc[a][b][j]));
#pragma unroll
    for (int o = 32; o >= 1; o >>= 1) m = fmaxf(m, __shfl_xor(m, o));
    if (lane == 0) lds_st(ctl + (C_WMAX + wave) * 4, __float_as_uint(m));
    if (wave == 0) TS(12);
    if (wave == 7) TS(19);
  } else if (wave < W_SEND) {
    __builtin_amdgcn_s_setprio(LPRIO);
    const int L = wave - 8;
    const __amdgpu_buffer_rsrc_t rs = __builtin_amdgcn_make_buffer_rsrc(myws, 0, 0x7fffffff, 0x00020000);
    unsigned c0[QPL], cs[QPL];
#pragma unroll
    for (int k = 0; k < QPL; ++k) {
      const int g = QPL * L + k;
      c0[k] = g < 8 ? (unsigned)OFF_XS + (unsigned)(2 * ti) * UNIT + g * 4096
                    : (unsigned)OFF_WS + (unsigned)tj * UNIT + (g - 8) * 4096;
      cs[k] = g < 8 ? XU * UNIT : WU * UNIT;
    }
    unsigned freed = 0;
    auto stage = [&](LBuf& C, int t, int slot, auto do_issue, auto do_proc) {
      unsigned w0 = 0, w1 = 0, w2 = 0, w3 = 1;
      if (do_issue) {
        const unsigned ea = ldsbase + LDS_Q + (t + 2) * QE + L * 16;
        while (true) {
          u32x4 v;
          asm volatile("ds_read_b128 %0, %1\n\ts_waitcnt lgkmcnt(0)" : "=v"(v) : "v"(ea) : "memory");
          w0 = __builtin_amdgcn_readfirstlane(v[0]); w1 = __builtin_amdgcn_readfirstlane(v[1]);
          w2 = __builtin_amdgcn_readfirstlane(v[2]); w3 = __builtin_amdgcn_readfirstlane(v[3]);
          if (((((w0 ^ tag) | (w1 ^ tag) | (w2 ^ tag)) >> 13) == 0u) && w3 != 0u) break;
          __builtin_amdgcn_s_sleep(1);
        }
      }
      if (do_proc) {
        while ((int)freed < t - NSLOT + 1) {
          unsigned d = lds_ld(ctl + (C_DONE + (lane & 7)) * 4);
          d = min(d, (unsigned)__builtin_amdgcn_update_dpp((int)d, (int)d, 0x111, 0xf, 0xf, false));
          d = min(d, (unsigned)__builtin_amdgcn_update_dpp((int)d, (int)d, 0x112, 0xf, 0xf, false));
          d = min(d, (unsigned)__builtin_amdgcn_update_dpp((int)d, (int)d, 0x114, 0xf, 0xf, false));
          freed = (unsigned)__builtin_amdgcn_readlane((int)d, 7);
          if ((int)freed >= t - NSLOT + 1) break;
          __builtin_amdgcn_s_sleep(1);
        }
      }
#pragma unroll
      for (int k = 0; k < QPL; ++k) {
        if (do_proc) {
          const unsigned qb = ldsbase + slot * SLOT + (QPL * L + k) * 4096;
          const unsigned la = qb + lane * 16;
          const unsigned info = C.info[k];
          if (info & F_RAW) {
            const u32x4 v = __builtin_amdgcn_raw_buffer_load_b128(rs, lane * 16 + 3072, C.so[k], LAUX);
            lds_st16(la, C.a[k]);
            lds_st16(la + 1024, C.b[k]);
            lds_st16(la + 2048, C.n[k]);
            lds_st16(la + 3072, v);
          } else {
            const unsigned base4 = (info & 0x7fu) * 0x01010101u;
            lds_st16(la, unpack8(C.a[k][0], C.a[k][1], C.n[k][0], base4));
            lds_st16(la + 1024, unpack8(C.a[k][2], C.a[k][3], C.n[k][1], base4));
            lds_st16(la + 2048, unpack8(C.b[k][0], C.b[k][1], C.n[k][2], base4));
            lds_st16(la + 3072, unpack8(C.b[k][2], C.b[k][3], C.n[k][3], base4));
            const unsigned ne = (info >> 7) & 31u;
            if (ne != 0 && (unsigned)lane < ne && lane < MAXESC)
              lds_st1(qb + (C.e[k] & 0x7ffu) * 2 + 1, C.e[k] >> 16);
          }
        }
        if (do_issue) {
          C.info[k] = k == 0 ? w0 : k == 1 ? w1 : w2;
          C.so[k] = c0[k] + (w3 - 1) * cs[k];
          C.a[k] = __builtin_amdgcn_raw_buffer_load_b128(rs, lane * 16, C.so[k], LAUX);
          C.b[k] = __builtin_amdgcn_raw_buffer_load_b128(rs, lane * 16 + 1024, C.so[k], LAUX);
          C.n[k] = __builtin_amdgcn_raw_buffer_load_b128(rs, lane * 16 + 2048, C.so[k], LAUX);
          C.e[k] = __builtin_amdgcn_raw_buffer_load_b32(rs, (lane & 15) * 4 + 3072, C.so[k], LAUX);
        }
      }
      if (do_proc) {
        if (lane == 0) lds_st(ctl + (C_FULL + L) * 4, (unsigned)(t + 1));
      }
    };
    constexpr std::true_type yes{};
    constexpr std::false_type no{};
    LBuf A, B;
    stage(A, -2, 0, yes, no);
    stage(B, -1, 0, yes, no);
    int slot = 0;
    for (int t = 0; t < NSTEP - 2; t += 2) {
      stage(A, t, slot, yes, yes);
      slot = (slot == NSLOT - 1) ? 0 : slot + 1;
      stage(B, t + 1, slot, yes, yes);
      slot = (slot == NSLOT - 1) ? 0 : slot + 1;
    }
    stage(A, NSTEP - 2, slot, no, yes);
    slot = (slot == NSLOT - 1) ? 0 : slot + 1;
    stage(B, NSTEP - 1, slot, no, yes);
    if (L == 0) TS(11);
  } else {
    __builtin_amdgcn_s_setprio(3);
    unsigned* const sq = reinterpret_cast<unsigned*>(myws + OFF_SQ);
    const unsigned qbase = (call - 1) * NITEM - NSEND * NWG;
    unsigned tk = qbase + blockIdx.x * NSEND + (wave - W_SEND);
    for (int i = 0;; ++i) {
      const int tq = (int)(__builtin_amdgcn_readfirstlane(tk) - qbase);
      if ((unsigned)tq >= (unsigned)NITEM) break;
      if (lane == 0) tk = __hip_atomic_fetch_add(sq, 1u, __ATOMIC_RELAXED, __HIP_MEMORY_SCOPE_AGENT);
      int u, f0, f1;
      if (tq < U1) { u = tq; f0 = 0; f1 = 16; }
      else { u = U1 + ((tq - U1) >> 2); f0 = ((tq - U1) & 3) * 4; f1 = f0 + 4; }
      int d, s, item;
      if (u < 336) { s = 0; d = (rank + 1 + u % 7) & 7; item = u / 7; }
      else if (u < 720) { int j = u - 336; s = j / 48; item = j % 48; d = rank; }
      else { int j = u - 720; s = 1 + j / 336; j %= 336; d = (rank + 1 + j % 7) & 7; item = j / 7; }
      gchar* dws = (gchar*)(P.ws_all[(unsigned)d < (unsigned)NR ? d : rank]);
      const int q = item / 3, rr = item - 3 * q;
      gchar* flag;
      unsigned fv = 0;
      if (rr != 2) {
        const int xu = 2 * q + rr;
        const float* src = P.x + (size_t)(xu * 128 + (lane & 15)) * KSH + s * 64 + 8 * (lane >> 4);
        gchar* dst = dws + OFF_XS + ((size_t)((rank * NSTAGE + s) * XU + xu)) * UNIT + lane * 16;
        for (int f = f0; f < f1; f += 4) {
          f32x4 a[4], b[4];
#pragma unroll
          for (int j = 0; j < 4; ++j) {
            const float* sp = src + (size_t)((f + j) & 7) * 16 * KSH + ((f + j) >> 3) * 32;
            a[j] = *reinterpret_cast<const f32x4*>(sp);
            b[j] = *reinterpret_cast<const f32x4*>(sp + 4);
          }
          unsigned p[4][4];
          float m = 0.f;
#pragma unroll
          for (int j = 0; j < 4; ++j) {
            p[j][0] = pk_bf16(a[j][0], a[j][1]); p[j][1] = pk_bf16(a[j][2], a[j][3]);
            p[j][2] = pk_bf16(b[j][0], b[j][1]); p[j][3] = pk_bf16(b[j][2], b[j][3]);
#pragma unroll
            for (int e = 0; e < 4; ++e) m = fmaxf(m, fmaxf(fabsf(a[j][e]), fabsf(b[j][e])));
          }
          const unsigned info = send_block(p, m, dst + f * 1024, lane);
          if (lane * 4 == f) fv = info;
        }
        flag = dws + OFF_XF + (xu * 64 + rank * 8 + s) * 16;
      } else {
        const int wu = q;
        const float* src = P.w + (size_t)(s * 64 + 8 * (lane >> 4)) * NTOT + d * NLOC + wu * 128 +
                           ((lane & 15) >> 2) * 16 + (lane & 3);
        gchar* dst = dws + OFF_WS + ((size_t)((rank * NSTAGE + s) * WU + wu)) * UNIT + lane * 16;
        for (int f = f0; f < f1; f += 4) {
          float v[4][8];
#pragma unroll
          for (int jj = 0; jj < 4; ++jj) {
            const int kb = (f + jj) >> 3, nb = (f + jj) & 7;
            const float* sp = src + (size_t)(kb * 32) * NTOT + (nb >> 2) * 64 + (nb & 3) * 4;
#pragma unroll
            for (int j = 0; j < 8; ++j) v[jj][j] = sp[(size_t)j * NTOT];
          }
          unsigned p[4][4];
          float m = 0.f;
#pragma unroll
          for (int jj = 0; jj < 4; ++jj) {
#pragma unroll
            for (int e = 0; e < 4; ++e) {
              p[jj][e] = pk_bf16(v[jj][2 * e], v[jj][2 * e + 1]);
              m = fmaxf(m, fmaxf(fabsf(v[jj][2 * e]), fabsf(v[jj][2 * e + 1])));
            }
          }
          const unsigned info = send_block(p, m, dst + f * 1024, lane);
          if (lane * 4 == f) fv = info;
        }
        flag = dws + OFF_WF + (wu * 64 + rank * 8 + s) * 16;
      }
      asm volatile("s_waitcnt vmcnt(0)" ::: "memory");
      {
        const int fl = lane * 4;
        if (lane < 4 && fl >= f0 && fl < f1)
          __hip_atomic_store((gu32*)flag + lane, (call << 13) | fv, __ATOMIC_RELAXED, __HIP_MEMORY_SCOPE_SYSTEM);
      }
      if (wave == W_SEND && i < 12) TS(20 + i);
    }
    if (wave < W_SEND + 2) TS(1 + wave - W_SEND);
  }

  asm volatile("s_waitcnt lgkmcnt(0)" ::: "memory");
  __syncthreads();

  if (wave == 0) {
    TS(13);
    float m = __uint_as_float(lds_ld(ctl + (C_WMAX + (lane & 7)) * 4));
#pragma unroll
    for (int o = 4; o >= 1; o >>= 1) m = fmaxf(m, __shfl_xor(m, o));
    unsigned last = 0;
    if (lane == 0) {
      u64 v = ((u64)call << 32) | (u64)__float_as_uint(m);
      u64 old = __hip_atomic_fetch_max(reinterpret_cast<u64*>(myws + OFF_AMAX), v, __ATOMIC_RELAXED,
                                       __HIP_MEMORY_SCOPE_AGENT);
      asm volatile("s_waitcnt vmcnt(0)" :: "v"(old) : "memory");
      unsigned tk = __hip_atomic_fetch_add(reinterpret_cast<unsigned*>(myws + OFF_ARRIVE), 1u,
                                           __ATOMIC_RELAXED, __HIP_MEMORY_SCOPE_AGENT);
      last = (tk == call * NWG - 1) ? 1u : 0u;
    }
    last = __builtin_amdgcn_readfirstlane(last);
    if (last) {
      u64 v = __hip_atomic_load(reinterpret_cast<u64*>(myws + OFF_AMAX), __ATOMIC_RELAXED,
                                __HIP_MEMORY_SCOPE_AGENT);
      if (lane < NR) {
        gu64* dstp = (gu64*)((gchar*)(P.ws_all[lane]) + OFF_SLOTS) + rank;
        __hip_atomic_store(dstp, v, __ATOMIC_RELAXED, __HIP_MEMORY_SCOPE_SYSTEM);
      }
    }
    const u64* slots = reinterpret_cast<const u64*>(myws + OFF_SLOTS) + (lane & 7);
    const u64 t0 = __builtin_amdgcn_s_memrealtime();
    u64 v;
    while (true) {
      v = __hip_atomic_load(slots, __ATOMIC_RELAXED, __HIP_MEMORY_SCOPE_SYSTEM);
      if (__all((unsigned)(v >> 32) >= call)) break;
      if (__hip_atomic_load(tmo, __ATOMIC_RELAXED, __HIP_MEMORY_SCOPE_AGENT) == call ||
          __builtin_amdgcn_s_memrealtime() - t0 > DEADLINE_TICKS) {
        if (lane == 0) __hip_atomic_store(tmo, call, __ATOMIC_RELAXED, __HIP_MEMORY_SCOPE_AGENT);
        break;
      }
      __builtin_amdgcn_s_sleep(1);
    }
    float g = __uint_as_float((unsigned)v);
#pragma unroll
    for (int o = 4; o >= 1; o >>= 1) g = fmaxf(g, __shfl_xor(g, o));
    unsigned dead = (__hip_atomic_load(tmo, __ATOMIC_RELAXED, __HIP_MEMORY_SCOPE_AGENT) == call) ? 1u : 0u;
    if (lane == 0) { lds_st(ctl + C_GMAX * 4, __float_as_uint(g)); lds_st(ctl + C_DEAD * 4, dead); }
  }
  asm volatile("s_waitcnt lgkmcnt(0)" ::: "memory");
  __syncthreads();

  if (wave == 0) TS(14);
  const int owner = ti >> 1;
  if (wave < 8) {
    const float amax = __uint_as_float(lds_ld(ctl + C_GMAX * 4));
    const bool dead = lds_ld(ctl + C_DEAD * 4) != 0;
    const float scale = amax / 127.0f;
    const float inv = scale > 0.f ? 1.0f / scale : 0.f;
    gchar* ob = (gchar*)(P.out_all[(unsigned)owner < (unsigned)NR ? owner : 0]);
    const int row0 = (ti & 1) * 256 + wm * 64 + (lane & 15);
    const int col = rank * NLOC + tj * 128 + wn * 64 + (lane >> 4) * 16;
#pragma unroll
    for (int mt = 0; mt < 4; ++mt) {
      u32x4 o;
#pragma unroll
      for (int nt = 0; nt < 4; ++nt) {
        unsigned pk = 0;
#pragma unroll
        for (int j = 0; j < 4; ++j) {
          float q = rintf(acc[mt][nt][j] * inv);
          q = fminf(fmaxf(q, -127.f), 127.f);
          int qi = dead ? -128 : (int)q;
          pk |= ((unsigned)qi & 0xffu) << (8 * j);
        }
        o[nt] = pk;
      }
      st16(ob + (size_t)(row0 + mt * 16) * NTOT + col, o);
    }
    if (wave == 0) TS(17);
    asm volatile("s_waitcnt vmcnt(0)" ::: "memory");
    if (wave == 0) TS(18);
  }
  asm volatile("s_waitcnt lgkmcnt(0)" ::: "memory");
  __syncthreads();
  if (wave == 0) {
    TS(15);
    if (lane == 0) {
      gu32* f = (gu32*)((gchar*)(P.ws_all[(unsigned)owner < (unsigned)NR ? owner : 0]) + OFF_OUTF) +
                    rank * 32 + (ti & 1) * 16 + tj;
      __hip_atomic_store(f, call, __ATOMIC_RELAXED, __HIP_MEMORY_SCOPE_SYSTEM);
    }
    if (blockIdx.x == 0) {
      const unsigned* of = reinterpret_cast<const unsigned*>(myws + OFF_OUTF) + lane * 4;
      const u64 t0 = __builtin_amdgcn_s_memrealtime();
      while (true) {
        bool ok = true;
#pragma unroll
        for (int j = 0; j < 4; ++j)
          ok &= __hip_atomic_load(of + j, __ATOMIC_RELAXED, __HIP_MEMORY_SCOPE_SYSTEM) >= call;
        if (__all(ok)) break;
        if (__hip_atomic_load(tmo, __ATOMIC_RELAXED, __HIP_MEMORY_SCOPE_AGENT) == call ||
            __builtin_amdgcn_s_memrealtime() - t0 > DEADLINE_TICKS) {
          if (lane == 0) {
            __hip_atomic_store(tmo, call, __ATOMIC_RELAXED, __HIP_MEMORY_SCOPE_AGENT);
            ((gchar*)(P.out_all[rank]))[rank * NLOC] = (char)-128;
          }
          break;
        }
        __builtin_amdgcn_s_sleep(1);
      }
      __builtin_amdgcn_fence(__ATOMIC_ACQUIRE, "");
      TS(16);
    }
  }
}

extern "C" void kernel_launch(int rank, int nranks, ncclComm_t comm, hipStream_t stream,
                              void* const* d_in, const int* in_sizes, int n_in, void* d_out,
                              int out_size, const int* in_dtypes, int out_dtype,
                              void* const* d_out_all, void* const* d_ws_all, size_t ws_bytes) {
  if (nranks != NR || ws_bytes < WS_NEED || n_in < 2) {
    fprintf(stderr, "unexpected configuration\n");
    return;
  }
  Params P;
  P.x = static_cast<const float*>(d_in[0]);
  P.w = static_cast<const float*>(d_in[1]);
  P.out_all = d_out_all;
  P.ws_all = d_ws_all;
  P.rank = rank;
  hipLaunchKernelGGL(fused_gemm_q8, dim3(NWG), dim3(NTHREADS), 0, stream, P);
  HIP_CHECK(hipGetLastError());
}
```

```cpp
#include <hip/hip_runtime.h>
#include <rccl/rccl.h>
#include <cstdio>
#include <cstdint>
#include <type_traits>

#define HIP_CHECK(expr) do { hipError_t _e = (expr); if (_e != hipSuccess) { \
    fprintf(stderr, "HIP error %s at %s:%d\n", hipGetErrorString(_e), __FILE__, __LINE__); \
    return; } } while (0)

typedef __attribute__((ext_vector_type(8))) short bf16x8;
typedef __attribute__((ext_vector_type(4))) float f32x4;
typedef __attribute__((ext_vector_type(4))) unsigned u32x4;
typedef unsigned long long u64;
#define GAS __attribute__((address_space(1)))
typedef GAS char gchar;
typedef GAS unsigned gu32;
typedef GAS unsigned long long gu64;
typedef GAS u32x4 gu32x4;

namespace {
constexpr int NR = 8;
constexpr int KSH = 512;
constexpr int NTOT = 16384;
constexpr int NLOC = 2048;
constexpr int MSH = 512;
constexpr int NSTAGE = 8;
constexpr int XU = 32;
constexpr int WU = 16;
constexpr int UNIT = 16384;
constexpr int NWG = 256;
constexpr int SU = 4;
constexpr int NUNIT = NR * NSTAGE * (XU + WU);
#define U1V 2560
constexpr int U1 = U1V;
constexpr int NITEM = U1 + 4 * (NUNIT - U1);
#define THR 3
#define LAUX 17
#define EXP 0
#define LPRIO 1
#define LOCRAW 1
#define NBUF 3
#define PSLEEP 2
constexpr int NSTEP = 64;
constexpr int NSLOT = 3;
constexpr int SLOT = 3 * UNIT;
#define NSEND 2
#define NLOADV 6
constexpr int NLOAD = NLOADV;
constexpr int QPL = 12 / NLOAD;
constexpr int QE = NLOAD == 4 ? 64 : 128;
constexpr int W_SEND = 8 + NLOAD;
constexpr int NTHREADS = 64 * (W_SEND + NSEND);
constexpr unsigned F_RAW = 1u << 12;
constexpr int MAXESC = 16;

constexpr size_t OFF_TICKET = 0;
constexpr size_t OFF_TMO = 128;
constexpr size_t OFF_AMAX = 256;
constexpr size_t OFF_ARRIVE = 384;
constexpr size_t OFF_SLOTS = 512;
constexpr size_t OFF_OUTF = 1024;
constexpr size_t OFF_SQ = 2048;
constexpr size_t OFF_XF = 4096;
constexpr size_t OFF_WF = OFF_XF + XU * 64 * SU * 4;
constexpr size_t OFF_XS = 1u << 20;
constexpr size_t OFF_WS = OFF_XS + (size_t)NR * NSTAGE * XU * UNIT;
constexpr size_t WS_NEED = OFF_WS + (size_t)NR * NSTAGE * WU * UNIT;

constexpr u64 DEADLINE_TICKS = 40000000ull;

constexpr int C_CALL = 0, C_FULL = 8, C_DONE = 16, C_WMAX = 24, C_GMAX = 32, C_DEAD = 33;
constexpr int LDS_CTL = NSLOT * SLOT;
constexpr int LDS_Q = LDS_CTL + 512;
constexpr int LDS_BYTES = LDS_Q + 64 * QE;
}

#define TS(i) do {} while (0)
#define TV(i, v) do {} while (0)
#define CYC() 0ull

struct Params {
  const float* x;
  const float* w;
  void* const* out_all;
  void* const* ws_all;
  int rank;
};

__device__ __forceinline__ unsigned pk_bf16(float a, float b) {
  typedef __attribute__((ext_vector_type(2))) float f2;
  typedef __attribute__((ext_vector_type(2))) __bf16 b2;
  f2 v = {a, b};
  b2 r = __builtin_convertvector(v, b2);
  return __builtin_bit_cast(unsigned, r);
}

__device__ __forceinline__ unsigned lds_ld(unsigned addr) {
  unsigned v;
  asm volatile("ds_read_b32 %0, %1\n\ts_waitcnt lgkmcnt(0)" : "=v"(v) : "v"(addr) : "memory");
  return v;
}
__device__ __forceinline__ unsigned lds_ld_min4(unsigned addr) {
  u32x4 v;
  if (NLOAD == 4) {
    asm volatile("ds_read_b128 %0, %1\n\ts_waitcnt lgkmcnt(0)" : "=v"(v) : "v"(addr) : "memory");
    return min(min(v[0], v[1]), min(v[2], v[3]));
  }
  u32x4 w;
  asm volatile("ds_read_b128 %0, %2\n\tds_read_b128 %1, %2 offset:16\n\ts_waitcnt lgkmcnt(0)" : "=&v"(v), "=&v"(w) : "v"(addr) : "memory");
  return min(min(min(v[0], v[1]), min(v[2], v[3])), min(w[0], w[1]));
}
__device__ __forceinline__ void lds_st16(unsigned addr, u32x4 v) {
  asm volatile("ds_write_b128 %0, %1" :: "v"(addr), "v"(v) : "memory");
}
__device__ __forceinline__ void lds_st1(unsigned addr, unsigned v) {
  asm volatile("ds_write_b8 %0, %1" :: "v"(addr), "v"(v) : "memory");
}
__device__ __forceinline__ void lds_st(unsigned addr, unsigned v) {
  asm volatile("ds_write_b32 %0, %1" :: "v"(addr), "v"(v) : "memory");
}

__device__ __forceinline__ void st16(gchar* p, u32x4 v) {
  asm volatile("global_store_dwordx4 %0, %1, off sc0 sc1\n\ts_nop 1" :: "v"(p), "v"(v) : "memory");
}

__device__ __forceinline__ void st4(gchar* p, unsigned v) {
  asm volatile("global_store_dword %0, %1, off sc0 sc1\n\ts_nop 1" :: "v"(p), "v"(v) : "memory");
}

__device__ __forceinline__ u32x4 unpack8(unsigned l0, unsigned l1, unsigned n, unsigned base4) {
  const unsigned h0 = ((n & 0x07070707u) + base4) | ((n & 0x08080808u) << 4);
  const unsigned h1 = (((n >> 4) & 0x07070707u) + base4) | (n & 0x80808080u);
  return u32x4{__builtin_amdgcn_perm(h0, l0, 0x05010400u), __builtin_amdgcn_perm(h0, l0, 0x07030602u),
               __builtin_amdgcn_perm(h1, l1, 0x05010400u), __builtin_amdgcn_perm(h1, l1, 0x07030602u)};
}

__device__ __forceinline__ unsigned send_block(const unsigned (&p)[4][4], float m, gchar* dst, int lane, bool plain) {
  if (plain) {
#pragma unroll
    for (int j = 0; j < 4; ++j) st16(dst + j * 1024, u32x4{p[j][0], p[j][1], p[j][2], p[j][3]});
    return F_RAW;
  }
  int mi = (int)__float_as_uint(m);
  mi = max(mi, __builtin_amdgcn_update_dpp(mi, mi, 0x111, 0xf, 0xf, false));
  mi = max(mi, __builtin_amdgcn_update_dpp(mi, mi, 0x112, 0xf, 0xf, false));
  mi = max(mi, __builtin_amdgcn_update_dpp(mi, mi, 0x114, 0xf, 0xf, false));
  mi = max(mi, __builtin_amdgcn_update_dpp(mi, mi, 0x118, 0xf, 0xf, false));
  mi = max(mi, __builtin_amdgcn_update_dpp(mi, mi, 0x142, 0xa, 0xf, false));
  mi = max(mi, __builtin_amdgcn_update_dpp(mi, mi, 0x143, 0xc, 0xf, false));
  const unsigned top = (pk_bf16(__uint_as_float((unsigned)__builtin_amdgcn_readlane(mi, 63)), 0.f) >> 8) & 0x7fu;
  const unsigned base = __builtin_amdgcn_readfirstlane(top > 7u ? top - 7u : 0u);
  const unsigned base4 = base * 0x01010101u;
  unsigned lo[4][2], hi[4][2], nb[4], aa = ~0u, ao = 0u;
#pragma unroll
  for (int j = 0; j < 4; ++j) {
    lo[j][0] = __builtin_amdgcn_perm(p[j][1], p[j][0], 0x06040200u);
    hi[j][0] = __builtin_amdgcn_perm(p[j][1], p[j][0], 0x07050301u);
    lo[j][1] = __builtin_amdgcn_perm(p[j][3], p[j][2], 0x06040200u);
    hi[j][1] = __builtin_amdgcn_perm(p[j][3], p[j][2], 0x07050301u);
    const unsigned u0 = (hi[j][0] | 0x80808080u) - base4;
    const unsigned u1 = (hi[j][1] | 0x80808080u) - base4;
    aa &= u0 & u1;
    ao |= u0 | u1;
    nb[j] = (u0 & 0x07070707u) | ((hi[j][0] >> 4) & 0x08080808u) | ((u1 & 0x07070707u) << 4) |
            (hi[j][1] & 0x80808080u);
  }
  unsigned info = base;
  if (__ballot(((~aa & 0x80808080u) | (ao & 0x78787878u)) != 0u)) {
    unsigned em[4][2], c = 0;
#pragma unroll
    for (int j = 0; j < 4; ++j)
#pragma unroll
      for (int h = 0; h < 2; ++h) {
        const unsigned u = (hi[j][h] | 0x80808080u) - base4;
        em[j][h] = (~u | ((u & 0x78787878u) + 0x78787878u)) & 0x80808080u;
        c += __builtin_popcount(em[j][h]);
      }
    unsigned total = 0;
#pragma unroll
    for (int b = 0; b < 6; ++b) total += (unsigned)__builtin_popcountll(__ballot((c >> b) & 1u)) << b;
    if (total > (unsigned)MAXESC) {
#pragma unroll
      for (int j = 0; j < 4; ++j) {
        st16(dst + j * 1024, u32x4{p[j][0], p[j][1], p[j][2], p[j][3]});
        if (j == 1) asm volatile("s_waitcnt vmcnt(0)" ::: "memory");
      }
      return F_RAW;
    }
    unsigned n = 0;
#pragma unroll
    for (int j = 0; j < 4; ++j)
#pragma unroll
      for (int h = 0; h < 2; ++h) {
        if (__ballot(em[j][h] != 0u) == 0) continue;
        for (int b = 0; b < 4; ++b) {
          const bool on = (em[j][h] >> (8 * b + 7)) & 1u;
          const u64 bal = __ballot(on);
          const unsigned idx = n + __builtin_amdgcn_mbcnt_hi((unsigned)(bal >> 32), __builtin_amdgcn_mbcnt_lo((unsigned)bal, 0u));
          if (on) st4(dst - lane * 16 + 3072 + idx * 4,
                      (unsigned)(j * 512 + lane * 8 + h * 4 + b) | (((hi[j][h] >> (8 * b)) & 0xffu) << 16));
          n += (unsigned)__builtin_popcountll(bal);
        }
      }
    info |= total << 7;
  }
  st16(dst, u32x4{lo[0][0], lo[0][1], lo[1][0], lo[1][1]});
  st16(dst + 1024, u32x4{lo[2][0], lo[2][1], lo[3][0], lo[3][1]});
  asm volatile("s_waitcnt vmcnt(1)" ::: "memory");
  st16(dst + 2048, u32x4{nb[0], nb[1], nb[2], nb[3]});
  return info;
}

struct LBuf {
  u32x4 a[QPL], b[QPL], n[QPL], e[QPL];
  unsigned info[QPL];
};

__global__ __launch_bounds__(NTHREADS, (NTHREADS + 255) / 256) void fused_gemm_q8(Params P) {
  __shared__ __attribute__((aligned(1024))) char lds[LDS_BYTES];
  const unsigned ldsbase = (unsigned)(uintptr_t)lds;
  const unsigned ctl = ldsbase + LDS_CTL;

  const int tid = threadIdx.x;
  const int lane = tid & 63;
  const int wave = __builtin_amdgcn_readfirstlane(tid >> 6);
  const int rank = P.rank;
  char* const myws = static_cast<char*>(P.ws_all[rank]);
  unsigned* const tmo = reinterpret_cast<unsigned*>(myws + OFF_TMO);

  if (tid >= 128 && tid < 128 + 4 * QE) lds_st16(ldsbase + LDS_Q + (tid - 128) * 16, u32x4{0u, 0u, 0u, 0u});
  if (tid < 128) {
    unsigned v = 0;
    if (tid == 0) {
      unsigned t = __hip_atomic_fetch_add(reinterpret_cast<unsigned*>(myws + OFF_TICKET), 1u,
                                          __ATOMIC_RELAXED, __HIP_MEMORY_SCOPE_AGENT);
      v = t / NWG + 1;
    }
    lds_st(ctl + tid * 4, v);
  }
  asm volatile("s_waitcnt lgkmcnt(0)" ::: "memory");
  __syncthreads();
  if (wave == 0) TS(0);
  const unsigned call = __builtin_amdgcn_readfirstlane(lds_ld(ctl + C_CALL * 4));

  const int bx = blockIdx.x & 7, bj = blockIdx.x >> 3;
  const int ti = 2 * (bj & 7) + (bx & 1);
  const int tj = 4 * (bx >> 1) + (bj >> 3);

  const unsigned tag = call << 13;
  f32x4 acc[4][4];
  const int wm = wave & 3, wn = (wave >> 2) & 1;

  if (wave < 8) {
#pragma unroll
    for (int a = 0; a < 4; ++a)
#pragma unroll
      for (int b = 0; b < 4; ++b) acc[a][b] = f32x4{0.f, 0.f, 0.f, 0.f};
    const int xoff = (wm >> 1) * UNIT + ((wm & 1) * 4) * 1024 + lane * 16;
    const int woff = 2 * UNIT + (wn * 4) * 1024 + lane * 16;
    int slot = 0;
    const unsigned* pf0 = reinterpret_cast<const unsigned*>(myws + OFF_XF) + ((2 * ti) * 64 + lane) * SU;
    const unsigned* pf2 = reinterpret_cast<const unsigned*>(myws + OFF_WF) + (tj * 64 + lane) * SU;
    u64 seen = 0;
    unsigned cnt = wave == 7 ? 0u : (unsigned)NSTEP, spins = 0;
    bool dead = false;
    const u64 t0 = __builtin_amdgcn_s_memrealtime();
    for (int t = 0; t < NSTEP; ++t) {
      while (true) {
        if (lds_ld_min4(ctl + C_FULL * 4) > (unsigned)t) break;
        if (cnt >= (unsigned)NSTEP) { __builtin_amdgcn_s_sleep(1); continue; }
        u32x4 f0, f1, f2;
        asm volatile("global_load_dwordx4 %0, %3, off sc0 sc1\n\tglobal_load_dwordx4 %1, %3, off offset:1024 sc0 sc1\n\t"
                     "global_load_dwordx4 %2, %4, off sc0 sc1\n\ts_waitcnt vmcnt(0)"
                     : "=&v"(f0), "=&v"(f1), "=&v"(f2) : "v"(pf0), "v"(pf2) : "memory");
        unsigned x = 0;
#pragma unroll
        for (int j = 0; j < 4; ++j) x |= (f0[j] ^ tag) | (f1[j] ^ tag) | (f2[j] ^ tag);
        const u64 nm = __ballot(dead || (x >> 13) == 0u) & ~seen;
        if (nm) {
          if ((nm >> lane) & 1) {
            const unsigned pos = cnt + __builtin_amdgcn_mbcnt_hi((unsigned)(nm >> 32), __builtin_amdgcn_mbcnt_lo((unsigned)nm, 0u));
            const unsigned ea = ldsbase + LDS_Q + pos * QE;
            if (dead) {
#pragma unroll
              for (int j = 0; j < 4; ++j) { f0[j] = tag | F_RAW; f1[j] = tag | F_RAW; f2[j] = tag | F_RAW; }
            }
            const unsigned s1 = (unsigned)lane + 1;
            if (NLOAD == 4) {
              lds_st16(ea, u32x4{f0[0], f0[1], f0[2], s1});
              lds_st16(ea + 16, u32x4{f0[3], f1[0], f1[1], s1});
              lds_st16(ea + 32, u32x4{f1[2], f1[3], f2[0], s1});
              lds_st16(ea + 48, u32x4{f2[1], f2[2], f2[3], s1});
            } else {
              lds_st16(ea, u32x4{f0[0], f0[1], tag, s1});
              lds_st16(ea + 16, u32x4{f0[2], f0[3], tag, s1});
              lds_st16(ea + 32, u32x4{f1[0], f1[1], tag, s1});
              lds_st16(ea + 48, u32x4{f1[2], f1[3], tag, s1});
              lds_st16(ea + 64, u32x4{f2[0], f2[1], tag, s1});
              lds_st16(ea + 80, u32x4{f2[2], f2[3], tag, s1});
            }
          }
          cnt += (unsigned)__builtin_popcountll(nm);
          seen |= nm;
        } else if ((++spins & 15) == 0 &&
                   (__hip_atomic_load(tmo, __ATOMIC_RELAXED, __HIP_MEMORY_SCOPE_AGENT) == call ||
                    __builtin_amdgcn_s_memrealtime() - t0 > DEADLINE_TICKS)) {
          if (lane == 0) __hip_atomic_store(tmo, call, __ATOMIC_RELAXED, __HIP_MEMORY_SCOPE_AGENT);
          dead = true;
        }
      }
      const char* sb = lds + slot * SLOT;
      bf16x8 xf[2][4], wf[2][4];
#pragma unroll
      for (int kb = 0; kb < 2; ++kb) {
#pragma unroll
        for (int i = 0; i < 4; ++i) {
          wf[kb][i] = *reinterpret_cast<const bf16x8*>(sb + woff + (kb * 8 + i) * 1024);
          xf[kb][i] = *reinterpret_cast<const bf16x8*>(sb + xoff + (kb * 8 + i) * 1024);
        }
      }
#pragma unroll
      for (int mt = 0; mt < 4; ++mt)
#pragma unroll
        for (int nt = 0; nt < 4; ++nt)
          acc[mt][nt] = __builtin_amdgcn_mfma_f32_16x16x32_bf16(wf[0][nt], xf[0][mt], acc[mt][nt], 0, 0, 0);
      asm volatile("s_waitcnt lgkmcnt(0)" ::: "memory");
      if (lane == 0) lds_st(ctl + (C_DONE + wave) * 4, (unsigned)(t + 1));
#pragma unroll
      for (int mt = 0; mt < 4; ++mt)
#pragma unroll
        for (int nt = 0; nt < 4; ++nt)
          acc[mt][nt] = __builtin_amdgcn_mfma_f32_16x16x32_bf16(wf[1][nt], xf[1][mt], acc[mt][nt], 0, 0, 0);
      slot = (slot == NSLOT - 1) ? 0 : slot + 1;
    }
    float m = 0.f;
#pragma unroll
    for (int a = 0; a < 4; ++a)
#pragma unroll
      for (int b = 0; b < 4; ++b)
#pragma unroll
        for (int j = 0; j < 4; ++j) m = fmaxf(m, fabsf(acc[a][b][j]));
#pragma unroll
    for (int o = 32; o >= 1; o >>= 1) m = fmaxf(m, __shfl_xor(m, o));
    if (lane == 0) lds_st(ctl + (C_WMAX + wave) * 4, __float_as_uint(m));
    if (wave == 0) TS(12);
    if (wave == 7) TS(19);
  } else if (wave < W_SEND) {
    __builtin_amdgcn_s_setprio(LPRIO);
    const int L = wave - 8;
    const __amdgpu_buffer_rsrc_t rs = __builtin_amdgcn_make_buffer_rsrc(myws, 0, 0x7fffffff, 0x00020000);
    unsigned c0[QPL], cs[QPL];
#pragma unroll
    for (int k = 0; k < QPL; ++k) {
      const int g = QPL * L + k;
      c0[k] = g < 8 ? (unsigned)OFF_XS + (unsigned)(2 * ti) * UNIT + g * 4096
                    : (unsigned)OFF_WS + (unsigned)tj * UNIT + (g - 8) * 4096;
      cs[k] = g < 8 ? XU * UNIT : WU * UNIT;
    }
    unsigned freed = 0;
    auto stage = [&](LBuf& C, int t, int slot, auto do_issue, auto do_proc) {
      unsigned w0 = 0, w1 = 0, w2 = 0, w3 = 1;
      if (do_issue) {
        const unsigned ea = ldsbase + LDS_Q + (t + NBUF) * QE + L * 16;
        while (true) {
          u32x4 v;
          asm volatile("ds_read_b128 %0, %1\n\ts_waitcnt lgkmcnt(0)" : "=v"(v) : "v"(ea) : "memory");
          w0 = __builtin_amdgcn_readfirstlane(v[0]); w1 = __builtin_amdgcn_readfirstlane(v[1]);
          w2 = __builtin_amdgcn_readfirstlane(v[2]); w3 = __builtin_amdgcn_readfirstlane(v[3]);
          if (((((w0 ^ tag) | (w1 ^ tag) | (w2 ^ tag)) >> 13) == 0u) && w3 != 0u) break;
          __builtin_amdgcn_s_sleep(1);
        }
      }
      if (do_proc) {
        while ((int)freed < t - NSLOT + 1) {
          unsigned d = lds_ld(ctl + (C_DONE + (lane & 7)) * 4);
          d = min(d, (unsigned)__builtin_amdgcn_update_dpp((int)d, (int)d, 0x111, 0xf, 0xf, false));
          d = min(d, (unsigned)__builtin_amdgcn_update_dpp((int)d, (int)d, 0x112, 0xf, 0xf, false));
          d = min(d, (unsigned)__builtin_amdgcn_update_dpp((int)d, (int)d, 0x114, 0xf, 0xf, false));
          freed = (unsigned)__builtin_amdgcn_readlane((int)d, 7);
          if ((int)freed >= t - NSLOT + 1) break;
          __builtin_amdgcn_s_sleep(1);
        }
      }
#pragma unroll
      for (int k = 0; k < QPL; ++k) {
        if (do_proc) {
          const unsigned qb = ldsbase + slot * SLOT + (QPL * L + k) * 4096;
          const unsigned la = qb + lane * 16;
          const unsigned info = C.info[k];
          if (info & F_RAW) {
            lds_st16(la, C.a[k]);
            lds_st16(la + 1024, C.b[k]);
            lds_st16(la + 2048, C.n[k]);
            lds_st16(la + 3072, C.e[k]);
          } else {
            const unsigned base4 = (info & 0x7fu) * 0x01010101u;
            lds_st16(la, unpack8(C.a[k][0], C.a[k][1], C.n[k][0], base4));
            lds_st16(la + 1024, unpack8(C.a[k][2], C.a[k][3], C.n[k][1], base4));
            lds_st16(la + 2048, unpack8(C.b[k][0], C.b[k][1], C.n[k][2], base4));
            lds_st16(la + 3072, unpack8(C.b[k][2], C.b[k][3], C.n[k][3], base4));
            const unsigned ne = (info >> 7) & 31u;
            if (ne != 0 && (unsigned)lane < ne && lane < MAXESC)
              lds_st1(qb + (C.e[k][0] & 0x7ffu) * 2 + 1, C.e[k][0] >> 16);
          }
        }
        if (do_issue) {
          const unsigned inf = k == 0 ? w0 : k == 1 ? w1 : w2;
          const unsigned so = __builtin_amdgcn_readfirstlane(c0[k] + (w3 - 1) * cs[k]);
          C.info[k] = inf;
          {
            const u64 em = (inf & F_RAW) ? ~0ull : (inf & (31u << 7)) ? 0xffffull : 0ull;
            const unsigned vo = (inf & F_RAW) ? lane * 16 : lane * 4;
            u64 keep;
            asm volatile("s_mov_b64 %1, exec\n\ts_mov_b64 exec, %5\n\t"
                         "buffer_load_dwordx4 %0, %2, %3, %4 offen offset:3072 sc0 sc1\n\t"
                         "s_mov_b64 exec, %1"
                         : "+v"(C.e[k]), "=&s"(keep) : "v"(vo), "s"(rs), "s"(so), "s"(em) : "memory");
          }
          C.a[k] = __builtin_amdgcn_raw_buffer_load_b128(rs, lane * 16, so, LAUX);
          C.b[k] = __builtin_amdgcn_raw_buffer_load_b128(rs, lane * 16 + 1024, so, LAUX);
          C.n[k] = __builtin_amdgcn_raw_buffer_load_b128(rs, lane * 16 + 2048, so, LAUX);
        }
      }
      if (do_proc) {
        if (lane == 0) lds_st(ctl + (C_FULL + L) * 4, (unsigned)(t + 1));
      }
    };
    constexpr std::true_type yes{};
    constexpr std::false_type no{};
    static_assert(NSLOT == 3 && NSTEP == 64, "unrolled for three slots");
    LBuf A, B, D;
#pragma unroll
    for (int k = 0; k < QPL; ++k) { A.e[k] = u32x4{0u, 0u, 0u, 0u}; B.e[k] = A.e[k]; D.e[k] = A.e[k]; }
    stage(A, -3, 0, yes, no);
    stage(B, -2, 0, yes, no);
    stage(D, -1, 0, yes, no);
    for (int t = 0; t < 60; t += 3) {
      stage(A, t, 0, yes, yes);
      stage(B, t + 1, 1, yes, yes);
      stage(D, t + 2, 2, yes, yes);
    }
    stage(A, 60, 0, yes, yes);
    stage(B, 61, 1, no, yes);
    stage(D, 62, 2, no, yes);
    stage(A, 63, 0, no, yes);
    if (L == 0) TS(11);
  } else {
    __builtin_amdgcn_s_setprio(3);
    unsigned* const sq = reinterpret_cast<unsigned*>(myws + OFF_SQ);
    const unsigned qbase = (call - 1) * NITEM - NSEND * NWG;
    unsigned tk = qbase + blockIdx.x * NSEND + (wave - W_SEND);
    for (int i = 0;; ++i) {
      const int tq = (int)(__builtin_amdgcn_readfirstlane(tk) - qbase);
      if ((unsigned)tq >= (unsigned)NITEM) break;
      if (lane == 0) tk = __hip_atomic_fetch_add(sq, 1u, __ATOMIC_RELAXED, __HIP_MEMORY_SCOPE_AGENT);
      int u, f0, f1;
      if (tq < U1) { u = tq; f0 = 0; f1 = 16; }
      else { u = U1 + ((tq - U1) >> 2); f0 = ((tq - U1) & 3) * 4; f1 = f0 + 4; }
      int d, s, q, rr;
      if (u >= 336 && u < 720) { const int j = u - 336, it = j % 48; s = j / 48; q = it / 3; rr = it - 3 * q; d = rank; }
      else {
        const int j = u < 336 ? u : u - 384, pr = j % 7, ii = j / 7;
        d = (rank + 1 + pr) & 7;
        const int rot = (pr * 16 + 3) / 7;
        if (ii < 16) { s = 0; q = ii; rr = 2; }
        else if (ii < 352) {
          const int i2 = ii - 16, it = i2 % 48;
          s = i2 / 48; q = it / 3; rr = it - 3 * q;
          if (rr == 2) s += 1; else q = (q + rot) & 15;
        } else { s = 7; q = (((ii - 352) >> 1) + rot) & 15; rr = (ii - 352) & 1; }
      }
      gchar* dws = (gchar*)(P.ws_all[(unsigned)d < (unsigned)NR ? d : rank]);
      gchar* flag;
      unsigned fv = 0;
      if (rr != 2) {
        const int xu = 2 * q + rr;
        const float* src = P.x + (size_t)(xu * 128 + (lane & 15)) * KSH + s * 64 + 8 * (lane >> 4);
        gchar* dst = dws + OFF_XS + ((size_t)((rank * NSTAGE + s) * XU + xu)) * UNIT + lane * 16;
        for (int f = f0; f < f1; f += 4) {
          f32x4 a[4], b[4];
#pragma unroll
          for (int j = 0; j < 4; ++j) {
            const float* sp = src + (size_t)((f + j) & 7) * 16 * KSH + ((f + j) >> 3) * 32;
            a[j] = *reinterpret_cast<const f32x4*>(sp);
            b[j] = *reinterpret_cast<const f32x4*>(sp + 4);
          }
          unsigned p[4][4];
          float m = 0.f;
#pragma unroll
          for (int j = 0; j < 4; ++j) {
            p[j][0] = pk_bf16(a[j][0], a[j][1]); p[j][1] = pk_bf16(a[j][2], a[j][3]);
            p[j][2] = pk_bf16(b[j][0], b[j][1]); p[j][3] = pk_bf16(b[j][2], b[j][3]);
#pragma unroll
            for (int e = 0; e < 4; ++e) m = fmaxf(m, fmaxf(fabsf(a[j][e]), fabsf(b[j][e])));
          }
          const unsigned info = send_block(p, m, dst + f * 1024, lane, LOCRAW && d == rank);
          if (lane * 4 == f) fv = info;
        }
        flag = dws + OFF_XF + (xu * 64 + rank * 8 + s) * 16;
      } else {
        const int wu = q;
        const float* src = P.w + (size_t)(s * 64 + 8 * (lane >> 4)) * NTOT + d * NLOC + wu * 128 +
                           ((lane & 15) >> 2) * 16 + (lane & 3);
        gchar* dst = dws + OFF_WS + ((size_t)((rank * NSTAGE + s) * WU + wu)) * UNIT + lane * 16;
        for (int f = f0; f < f1; f += 4) {
          float v[4][8];
#pragma unroll
          for (int jj = 0; jj < 4; ++jj) {
            const int kb = (f + jj) >> 3, nb = (f + jj) & 7;
            const float* sp = src + (size_t)(kb * 32) * NTOT + (nb >> 2) * 64 + (nb & 3) * 4;
#pragma unroll
            for (int j = 0; j < 8; ++j) v[jj][j] = sp[(size_t)j * NTOT];
          }
          unsigned p[4][4];
          float m = 0.f;
#pragma unroll
          for (int jj = 0; jj < 4; ++jj) {
#pragma unroll
            for (int e = 0; e < 4; ++e) {
              p[jj][e] = pk_bf16(v[jj][2 * e], v[jj][2 * e + 1]);
              m = fmaxf(m, fmaxf(fabsf(v[jj][2 * e]), fabsf(v[jj][2 * e + 1])));
            }
          }
          const unsigned info = send_block(p, m, dst + f * 1024, lane, LOCRAW && d == rank);
          if (lane * 4 == f) fv = info;
        }
        flag = dws + OFF_WF + (wu * 64 + rank * 8 + s) * 16;
      }
      asm volatile("s_waitcnt vmcnt(0)" ::: "memory");
      {
        const int fl = lane * 4;
        if (lane < 4 && fl >= f0 && fl < f1)
          __hip_atomic_store((gu32*)flag + lane, (call << 13) | fv, __ATOMIC_RELAXED, __HIP_MEMORY_SCOPE_SYSTEM);
      }
      if (wave == W_SEND && i < 12) TS(20 + i);
    }
    if (wave < W_SEND + 2) TS(1 + wave - W_SEND);
  }

  asm volatile("s_waitcnt lgkmcnt(0)" ::: "memory");
  __syncthreads();

  if (wave == 0) {
    TS(13);
    float m = __uint_as_float(lds_ld(ctl + (C_WMAX + (lane & 7)) * 4));
#pragma unroll
    for (int o = 4; o >= 1; o >>= 1) m = fmaxf(m, __shfl_xor(m, o));
    unsigned last = 0;
    if (lane == 0) {
      u64 v = ((u64)call << 32) | (u64)__float_as_uint(m);
      u64 old = __hip_atomic_fetch_max(reinterpret_cast<u64*>(myws + OFF_AMAX), v, __ATOMIC_RELAXED,
                                       __HIP_MEMORY_SCOPE_AGENT);
      asm volatile("s_waitcnt vmcnt(0)" :: "v"(old) : "memory");
      unsigned tk = __hip_atomic_fetch_add(reinterpret_cast<unsigned*>(myws + OFF_ARRIVE), 1u,
                                           __ATOMIC_RELAXED, __HIP_MEMORY_SCOPE_AGENT);
      last = (tk == call * NWG - 1) ? 1u : 0u;
    }
    last = __builtin_amdgcn_readfirstlane(last);
    if (last) {
      u64 v = __hip_atomic_load(reinterpret_cast<u64*>(myws + OFF_AMAX), __ATOMIC_RELAXED,
                                __HIP_MEMORY_SCOPE_AGENT);
      if (lane < NR) {
        gu64* dstp = (gu64*)((gchar*)(P.ws_all[lane]) + OFF_SLOTS) + rank;
        __hip_atomic_store(dstp, v, __ATOMIC_RELAXED, __HIP_MEMORY_SCOPE_SYSTEM);
      }
    }
    const u64* slots = reinterpret_cast<const u64*>(myws + OFF_SLOTS) + (lane & 7);
    const u64 t0 = __builtin_amdgcn_s_memrealtime();
    u64 v;
    while (true) {
      v = __hip_atomic_load(slots, __ATOMIC_RELAXED, __HIP_MEMORY_SCOPE_SYSTEM);
      if (__all((unsigned)(v >> 32) >= call)) break;
      if (__hip_atomic_load(tmo, __ATOMIC_RELAXED, __HIP_MEMORY_SCOPE_AGENT) == call ||
          __builtin_amdgcn_s_memrealtime() - t0 > DEADLINE_TICKS) {
        if (lane == 0) __hip_atomic_store(tmo, call, __ATOMIC_RELAXED, __HIP_MEMORY_SCOPE_AGENT);
        break;
      }
      __builtin_amdgcn_s_sleep(1);
    }
    float g = __uint_as_float((unsigned)v);
#pragma unroll
    for (int o = 4; o >= 1; o >>= 1) g = fmaxf(g, __shfl_xor(g, o));
    unsigned dead = (__hip_atomic_load(tmo, __ATOMIC_RELAXED, __HIP_MEMORY_SCOPE_AGENT) == call) ? 1u : 0u;
    if (lane == 0) { lds_st(ctl + C_GMAX * 4, __float_as_uint(g)); lds_st(ctl + C_DEAD * 4, dead); }
  }
  asm volatile("s_waitcnt lgkmcnt(0)" ::: "memory");
  __syncthreads();

  if (wave == 0) TS(14);
  const int owner = ti >> 1;
  if (wave < 8) {
    const float amax = __uint_as_float(lds_ld(ctl + C_GMAX * 4));
    const bool dead = lds_ld(ctl + C_DEAD * 4) != 0;
    const float scale = amax / 127.0f;
    const float inv = scale > 0.f ? 1.0f / scale : 0.f;
    gchar* ob = (gchar*)(P.out_all[(unsigned)owner < (unsigned)NR ? owner : 0]);
    const int row0 = (ti & 1) * 256 + wm * 64 + (lane & 15);
    const int col = rank * NLOC + tj * 128 + wn * 64 + (lane >> 4) * 16;
#pragma unroll
    for (int mt = 0; mt < 4; ++mt) {
      u32x4 o;
#pragma unroll
      for (int nt = 0; nt < 4; ++nt) {
        unsigned pk = 0;
#pragma unroll
        for (int j = 0; j < 4; ++j) {
          float q = rintf(acc[mt][nt][j] * inv);
          q = fminf(fmaxf(q, -127.f), 127.f);
          int qi = dead ? -128 : (int)q;
          pk |= ((unsigned)qi & 0xffu) << (8 * j);
        }
        o[nt] = pk;
      }
      st16(ob + (size_t)(row0 + mt * 16) * NTOT + col, o);
    }
    if (wave == 0) TS(17);
    asm volatile("s_waitcnt vmcnt(0)" ::: "memory");
    if (wave == 0) TS(18);
  }
  asm volatile("s_waitcnt lgkmcnt(0)" ::: "memory");
  __syncthreads();
  if (wave == 0) {
    TS(15);
    if (lane == 0) {
      gu32* f = (gu32*)((gchar*)(P.ws_all[(unsigned)owner < (unsigned)NR ? owner : 0]) + OFF_OUTF) +
                    rank * 32 + (ti & 1) * 16 + tj;
      __hip_atomic_store(f, call, __ATOMIC_RELAXED, __HIP_MEMORY_SCOPE_SYSTEM);
    }
    if (blockIdx.x == 0) {
      const unsigned* of = reinterpret_cast<const unsigned*>(myws + OFF_OUTF) + lane * 4;
      const u64 t0 = __builtin_amdgcn_s_memrealtime();
      while (true) {
        bool ok = true;
#pragma unroll
        for (int j = 0; j < 4; ++j)
          ok &= __hip_atomic_load(of + j, __ATOMIC_RELAXED, __HIP_MEMORY_SCOPE_SYSTEM) >= call;
        if (__all(ok)) break;
        if (__hip_atomic_load(tmo, __ATOMIC_RELAXED, __HIP_MEMORY_SCOPE_AGENT) == call ||
            __builtin_amdgcn_s_memrealtime() - t0 > DEADLINE_TICKS) {
          if (lane == 0) {
            __hip_atomic_store(tmo, call, __ATOMIC_RELAXED, __HIP_MEMORY_SCOPE_AGENT);
            ((gchar*)(P.out_all[rank]))[rank * NLOC] = (char)-128;
          }
          break;
        }
        __builtin_amdgcn_s_sleep(1);
      }
      __builtin_amdgcn_fence(__ATOMIC_ACQUIRE, "");
      TS(16);
    }
  }
}

extern "C" void kernel_launch(int rank, int nranks, ncclComm_t comm, hipStream_t stream,
                              void* const* d_in, const int* in_sizes, int n_in, void* d_out,
                              int out_size, const int* in_dtypes, int out_dtype,
                              void* const* d_out_all, void* const* d_ws_all, size_t ws_bytes) {
  if (nranks != NR || ws_bytes < WS_NEED || n_in < 2) {
    fprintf(stderr, "unexpected configuration\n");
    return;
  }
  Params P;
  P.x = static_cast<const float*>(d_in[0]);
  P.w = static_cast<const float*>(d_in[1]);
  P.out_all = d_out_all;
  P.ws_all = d_ws_all;
  P.rank = rank;
  hipLaunchKernelGGL(fused_gemm_q8, dim3(NWG), dim3(NTHREADS), 0, stream, P);
  HIP_CHECK(hipGetLastError());
}
```

```cpp
#include <hip/hip_runtime.h>
#include <rccl/rccl.h>
#include <cstdio>
#include <cstdint>
#include <type_traits>

#define HIP_CHECK(expr) do { hipError_t _e = (expr); if (_e != hipSuccess) { \
    fprintf(stderr, "HIP error %s at %s:%d\n", hipGetErrorString(_e), __FILE__, __LINE__); \
    return; } } while (0)

typedef __attribute__((ext_vector_type(8))) short bf16x8;
typedef __attribute__((ext_vector_type(4))) float f32x4;
typedef __attribute__((ext_vector_type(4))) unsigned u32x4;
typedef unsigned long long u64;
#define GAS __attribute__((address_space(1)))
typedef GAS char gchar;
typedef GAS unsigned gu32;
typedef GAS unsigned long long gu64;
typedef GAS u32x4 gu32x4;

namespace {
constexpr int NR = 8;
constexpr int KSH = 512;
constexpr int NTOT = 16384;
constexpr int NLOC = 2048;
constexpr int MSH = 512;
constexpr int NSTAGE = 8;
constexpr int XU = 32;
constexpr int WU = 16;
constexpr int UNIT = 16384;
constexpr int NWG = 256;
constexpr int SU = 4;
constexpr int NUNIT = 7 * NSTAGE * (XU + WU);
#define THR 2
#define PSLEEP 16
#define SCS 32
#define F1S 128
#define DSLEEP 16
#define DPRIO 2
#define LDPOL " sc0 sc1"
constexpr int NSTEP = 64;
constexpr int TM = 320;
constexpr int NTI = 13;
constexpr int NCW = NTI * 16;
constexpr int NH = 5;
constexpr int HS = 7 * 4096;
#define NSEND 13
constexpr int NSWG = NWG - NCW;
constexpr int NSC = 15 - NSEND < 4 ? 15 - NSEND : 4;
#define S1V 6
#define RTU 224
#define TKD 1
constexpr int S1 = S1V;
constexpr int NDRAW = NUNIT - S1 * NSWG + NSWG;
static_assert(NSEND + NSC < 16 && RTU <= 224, "sender split");
#define NWAVE 16
#define NDECV 2
constexpr int NDEC = NDECV;
constexpr int W_DEC = 11;
constexpr int NTHREADS = 64 * NWAVE;
constexpr unsigned F_RAW = 1u << 12;
constexpr int MAXESC = 16;
constexpr int NPK = NWAVE - W_DEC - NDEC;
constexpr int NPACK = NSTAGE * (XU + NR * WU) * 4;
#define CPK 0
constexpr int PR0 = NCW * (8 * CPK + NPK);
constexpr int PRN = NCW * NPK;
constexpr int NROUND = 1 + (NPACK - PR0 + PRN - 1) / PRN;
constexpr int NIN = 7 * 384;

constexpr size_t OFF_TICKET = 0;
constexpr size_t OFF_TMO = 128;
constexpr size_t OFF_AMAX = 256;
constexpr size_t OFF_ARRIVE = 384;
constexpr size_t OFF_SLOTS = 512;
constexpr size_t OFF_SQ = 2048;
constexpr size_t OFF_XF = 4096;
constexpr size_t OFF_WF = OFF_XF + XU * 64 * SU * 4;
constexpr size_t OFF_OUTF = 65536;
constexpr size_t OFF_EF = 81920;
constexpr size_t OFF_F1 = 131072;
constexpr size_t OFF_XS = 1u << 20;
constexpr size_t OFF_WS = OFF_XS + (size_t)NR * NSTAGE * XU * UNIT;
constexpr size_t OFF_EX = OFF_WS + (size_t)NR * NSTAGE * WU * UNIT;
constexpr size_t OFF_EW = OFF_EX + (size_t)NSTAGE * XU * UNIT;
constexpr size_t WS_NEED = OFF_EW + (size_t)NR * NSTAGE * WU * UNIT;

constexpr u64 DEADLINE_TICKS = 40000000ull;

constexpr int C_CALL = 0, C_FULL = 2, C_DONE = 8, C_WMAX = 16, C_GMAX = 24, C_DEAD = 25, C_NEXT = 26, C_Q = 32;
constexpr int LDS_CTL = NH * HS;
constexpr int LDS_BYTES = LDS_CTL + 512;
}

#define T2(a, i) do {} while (0)
#define TS(i) do {} while (0)
#define TV(i, v) do {} while (0)
#define CYC() 0ull

struct Params {
  const float* x;
  const float* w;
  void* const* out_all;
  void* const* ws_all;
  int rank;
};

__device__ __forceinline__ unsigned pk_bf16(float a, float b) {
  typedef __attribute__((ext_vector_type(2))) float f2;
  typedef __attribute__((ext_vector_type(2))) __bf16 b2;
  f2 v = {a, b};
  b2 r = __builtin_convertvector(v, b2);
  return __builtin_bit_cast(unsigned, r);
}

__device__ __forceinline__ unsigned lds_ld(unsigned addr) {
  unsigned v;
  asm volatile("ds_read_b32 %0, %1\n\ts_waitcnt lgkmcnt(0)" : "=v"(v) : "v"(addr) : "memory");
  return v;
}
__device__ __forceinline__ void glds4k(const void* g, unsigned ldsaddr) {
  unsigned keep;
  asm volatile(
      "s_mov_b32 %0, m0\n\ts_mov_b32 m0, %2\n\ts_nop 0\n\t"
      "global_load_lds_dwordx4 %1, off" LDPOL "\n\t"
      "global_load_lds_dwordx4 %1, off offset:1024" LDPOL "\n\t"
      "global_load_lds_dwordx4 %1, off offset:2048" LDPOL "\n\t"
      "global_load_lds_dwordx4 %1, off offset:3072" LDPOL "\n\t"
      "s_mov_b32 m0, %0"
      : "=&s"(keep) : "v"(g), "s"(ldsaddr) : "memory");
}
__device__ __forceinline__ bool plain16(u32x4 a, u32x4 b, u32x4 c, u32x4 d, unsigned want, unsigned m2) {
  unsigned x = 0, y = 0;
#pragma unroll
  for (int j = 0; j < 4; ++j) { x |= (a[j] ^ want) | (b[j] ^ want) | (d[j] ^ want); y |= c[j] ^ want; }
  return ((x | (y & m2)) >> 12) == 0u;
}
__device__ __forceinline__ void unit_of(int ii, int pr, int& s, int& q, int& rr) {
  const int rot = (pr * 16 + 3) / 7;
  if (ii < 16) { s = 0; q = ii; rr = 2; }
  else if (ii < 352) {
    const int i2 = ii - 16, it = i2 % 48;
    s = i2 / 48; q = it / 3; rr = it - 3 * q;
    if (rr == 2) s += 1; else q = (q + rot) & 15;
  } else { s = 7; q = (((ii - 352) >> 1) + rot) & 15; rr = (ii - 352) & 1; }
}
__device__ __forceinline__ void lds_st(unsigned addr, unsigned v) {
  asm volatile("ds_write_b32 %0, %1" :: "v"(addr), "v"(v) : "memory");
}

__device__ __forceinline__ void st16(gchar* p, u32x4 v) {
  asm volatile("global_store_dwordx4 %0, %1, off sc0 sc1\n\ts_nop 1" :: "v"(p), "v"(v) : "memory");
}

__device__ __forceinline__ void st4(gchar* p, unsigned v) {
  asm volatile("global_store_dword %0, %1, off sc0 sc1\n\ts_nop 1" :: "v"(p), "v"(v) : "memory");
}

__device__ __forceinline__ u32x4 unpack8(unsigned l0, unsigned l1, unsigned n, unsigned base4) {
  const unsigned h0 = ((n & 0x07070707u) + base4) | ((n & 0x08080808u) << 4);
  const unsigned h1 = (((n >> 4) & 0x07070707u) + base4) | (n & 0x80808080u);
  return u32x4{__builtin_amdgcn_perm(h0, l0, 0x05010400u), __builtin_amdgcn_perm(h0, l0, 0x07030602u),
               __builtin_amdgcn_perm(h1, l1, 0x05010400u), __builtin_amdgcn_perm(h1, l1, 0x07030602u)};
}

__device__ __forceinline__ unsigned send_block(const unsigned (&p)[4][4], float m, gchar* dst, int lane, bool plain,
                                              bool slow = false) {
  if (plain) {
#pragma unroll
    for (int j = 0; j < 4; ++j) st16(dst + j * 1024, u32x4{p[j][0], p[j][1], p[j][2], p[j][3]});
    return F_RAW;
  }
  int mi = (int)__float_as_uint(m);
  mi = max(mi, __builtin_amdgcn_update_dpp(mi, mi, 0x111, 0xf, 0xf, false));
  mi = max(mi, __builtin_amdgcn_update_dpp(mi, mi, 0x112, 0xf, 0xf, false));
  mi = max(mi, __builtin_amdgcn_update_dpp(mi, mi, 0x114, 0xf, 0xf, false));
  mi = max(mi, __builtin_amdgcn_update_dpp(mi, mi, 0x118, 0xf, 0xf, false));
  mi = max(mi, __builtin_amdgcn_update_dpp(mi, mi, 0x142, 0xa, 0xf, false));
  mi = max(mi, __builtin_amdgcn_update_dpp(mi, mi, 0x143, 0xc, 0xf, false));
  const unsigned top = (pk_bf16(__uint_as_float((unsigned)__builtin_amdgcn_readlane(mi, 63)), 0.f) >> 8) & 0x7fu;
  const unsigned base = __builtin_amdgcn_readfirstlane(top > 7u ? top - 7u : 0u);
  const unsigned base4 = base * 0x01010101u;
  unsigned lo[4][2], hi[4][2], nb[4], aa = ~0u, ao = 0u;
#pragma unroll
  for (int j = 0; j < 4; ++j) {
    lo[j][0] = __builtin_amdgcn_perm(p[j][1], p[j][0], 0x06040200u);
    hi[j][0] = __builtin_amdgcn_perm(p[j][1], p[j][0], 0x07050301u);
    lo[j][1] = __builtin_amdgcn_perm(p[j][3], p[j][2], 0x06040200u);
    hi[j][1] = __builtin_amdgcn_perm(p[j][3], p[j][2], 0x07050301u);
    const unsigned u0 = (hi[j][0] | 0x80808080u) - base4;
    const unsigned u1 = (hi[j][1] | 0x80808080u) - base4;
    aa &= u0 & u1;
    ao |= u0 | u1;
    nb[j] = (u0 & 0x07070707u) | ((hi[j][0] >> 4) & 0x08080808u) | ((u1 & 0x07070707u) << 4) |
            (hi[j][1] & 0x80808080u);
  }
  unsigned info = base;
  if (__ballot(((~aa & 0x80808080u) | (ao & 0x78787878u)) != 0u)) {
    unsigned em[4][2], c = 0;
#pragma unroll
    for (int j = 0; j < 4; ++j)
#pragma unroll
      for (int h = 0; h < 2; ++h) {
        const unsigned u = (hi[j][h] | 0x80808080u) - base4;
        em[j][h] = (~u | ((u & 0x78787878u) + 0x78787878u)) & 0x80808080u;
        c += __builtin_popcount(em[j][h]);
      }
    unsigned total = 0;
#pragma unroll
    for (int b = 0; b < 6; ++b) total += (unsigned)__builtin_popcountll(__ballot((c >> b) & 1u)) << b;
    if (total > (unsigned)MAXESC) {
#pragma unroll
      for (int j = 0; j < 4; ++j) {
        st16(dst + j * 1024, u32x4{p[j][0], p[j][1], p[j][2], p[j][3]});
        if (slow) asm volatile("s_waitcnt vmcnt(0)" ::: "memory");
      }
      return F_RAW;
    }
    unsigned n = 0;
#pragma unroll
    for (int j = 0; j < 4; ++j)
#pragma unroll
      for (int h = 0; h < 2; ++h) {
        if (__ballot(em[j][h] != 0u) == 0) continue;
        for (int b = 0; b < 4; ++b) {
          const bool on = (em[j][h] >> (8 * b + 7)) & 1u;
          const u64 bal = __ballot(on);
          const unsigned idx = n + __builtin_amdgcn_mbcnt_hi((unsigned)(bal >> 32), __builtin_amdgcn_mbcnt_lo((unsigned)bal, 0u));
          if (on) st4(dst - lane * 16 + 3072 + idx * 4,
                      (unsigned)(j * 512 + lane * 8 + h * 4 + b) | (((hi[j][h] >> (8 * b)) & 0xffu) << 16));
          n += (unsigned)__builtin_popcountll(bal);
        }
      }
    info |= total << 7;
  }
  st16(dst, u32x4{lo[0][0], lo[0][1], lo[1][0], lo[1][1]});
  if (slow) asm volatile("s_waitcnt vmcnt(0)" ::: "memory");
  st16(dst + 1024, u32x4{lo[2][0], lo[2][1], lo[3][0], lo[3][1]});
  if (slow) asm volatile("s_waitcnt vmcnt(0)" ::: "memory");
  st16(dst + 2048, u32x4{nb[0], nb[1], nb[2], nb[3]});
  return info;
}

__global__ __launch_bounds__(NTHREADS, NWAVE / 4) void fused_gemm_q8(Params P) {
  __shared__ __attribute__((aligned(1024))) char lds[LDS_BYTES];
  const unsigned ldsbase = (unsigned)(uintptr_t)lds;
  const unsigned ctl = ldsbase + LDS_CTL;

  const int tid = threadIdx.x;
  const int lane = tid & 63;
  const int wave = __builtin_amdgcn_readfirstlane(tid >> 6);
  const int rank = P.rank;
  char* const myws = static_cast<char*>(P.ws_all[rank]);
  unsigned* const tmo = reinterpret_cast<unsigned*>(myws + OFF_TMO);

  if ((blockIdx.x >> 3) < 6)
    for (int i = tid; i < NPACK / 4; i += NTHREADS)
      asm volatile("ds_write_b128 %0, %1" :: "v"(ldsbase + i * 16), "v"(u32x4{0u, 0u, 0u, 0u}) : "memory");
  if (tid < 128) {
    unsigned v = 0;
    if (tid == 0) {
      unsigned t = __hip_atomic_fetch_add(reinterpret_cast<unsigned*>(myws + OFF_TICKET), 1u,
                                          __ATOMIC_RELAXED, __HIP_MEMORY_SCOPE_AGENT);
      v = t / NWG + 1;
    }
    lds_st(ctl + tid * 4, v);
  }
  asm volatile("s_waitcnt lgkmcnt(0)" ::: "memory");
  __syncthreads();
  if (wave == 0) TS(0);
  const unsigned call = __builtin_amdgcn_readfirstlane(lds_ld(ctl + C_CALL * 4));

  const int bx = blockIdx.x & 7, bk = ((blockIdx.x >> 3) + 26) & 31;
  const bool sender = bk >= 26;
  const int ti = bk < 24 ? 2 * (bk % 6) + (bx & 1) : 12;
  const int tj = 4 * (bx >> 1) + (bk < 24 ? bk / 6 : 2 * (bx & 1) + ((bk - 24) & 1));

  const unsigned tag = call << 13;
  f32x4 acc[5][4];
  const int wm = wave & 3, wn = (wave >> 2) & 1;

  auto convert = [&](int n, unsigned (&p)[4][4], float& m, int& s, int& q, int& d) {
    const int e = n >> 2, f = (n & 3) * 4;
    d = -1;
    if (e < 128) { s = 0; q = e >> 3; d = e & 7; }
    else if (e < 1248) {
      const int e2 = e - 128, it = e2 % 160, r = it % 10;
      s = e2 / 160;
      if (r < 2) q = 2 * (it / 10) + r; else { s += 1; q = it / 10; d = r - 2; }
    } else { s = 7; q = e - 1248; }
    m = 0.f;
    if (d < 0) {
      const float* src = P.x + (size_t)(q * 128 + (lane & 15)) * KSH + s * 64 + 8 * (lane >> 4);
      f32x4 a[4], b[4];
#pragma unroll
      for (int j = 0; j < 4; ++j) {
        const float* sp = src + (size_t)((f + j) & 7) * 16 * KSH + ((f + j) >> 3) * 32;
        a[j] = *reinterpret_cast<const f32x4*>(sp);
        b[j] = *reinterpret_cast<const f32x4*>(sp + 4);
      }
#pragma unroll
      for (int j = 0; j < 4; ++j) {
        p[j][0] = pk_bf16(a[j][0], a[j][1]); p[j][1] = pk_bf16(a[j][2], a[j][3]);
        p[j][2] = pk_bf16(b[j][0], b[j][1]); p[j][3] = pk_bf16(b[j][2], b[j][3]);
#pragma unroll
        for (int c = 0; c < 4; ++c) m = fmaxf(m, fmaxf(fabsf(a[j][c]), fabsf(b[j][c])));
      }
    } else {
      d = (d + rank + 1) & 7;
      const float* sp = P.w + (size_t)(s * 64 + (f >> 3) * 32 + 8 * (lane >> 4)) * NTOT + d * NLOC + q * 128 +
                        (f & 4) * 16 + (lane & 15) * 4;
      f32x4 a[8];
#pragma unroll
      for (int j = 0; j < 8; ++j) a[j] = *reinterpret_cast<const f32x4*>(sp + (size_t)j * NTOT);
#pragma unroll
      for (int jj = 0; jj < 4; ++jj)
#pragma unroll
        for (int c = 0; c < 4; ++c) {
          p[jj][c] = pk_bf16(a[2 * c][jj], a[2 * c + 1][jj]);
          m = fmaxf(m, fmaxf(fabsf(a[2 * c][jj]), fabsf(a[2 * c + 1][jj])));
        }
    }
  };
  auto pack_items = [&](int g, int rounds) {
    gchar* const gws = (gchar*)myws;
    for (int k = 0; k < rounds; ++k) {
     const int n = k == 0 ? g : PR0 + (k - 1) * PRN + g - 8 * CPK * NCW;
     if (n < NPACK) {
      const int f = (n & 3) * 4;
      int s, q, d;
      unsigned p[4][4];
      float m;
      convert(n, p, m, s, q, d);
      unsigned info = 0;
      if (d != rank) {
        gchar* dst = gws + (d < 0 ? OFF_EX + (size_t)(s * XU + q) * UNIT : OFF_EW + (size_t)((d * NSTAGE + s) * WU + q) * UNIT);
        info = send_block(p, m, dst + f * 1024 + lane * 16, lane, false);
      }
      if (d < 0 || d == rank) {
        gchar* dst = gws + (d < 0 ? OFF_XS + (size_t)((rank * NSTAGE + s) * XU + q) * UNIT
                                  : OFF_WS + (size_t)((rank * NSTAGE + s) * WU + q) * UNIT);
        send_block(p, m, dst + f * 1024 + lane * 16, lane, true);
      }
      asm volatile("s_waitcnt vmcnt(0)" ::: "memory");
      if (lane == 0) st4(gws + OFF_EF + (size_t)n * 4, tag | info);
      if (lane == 1 && (d < 0 || d == rank))
        st4(gws + (d < 0 ? OFF_XF : OFF_WF) + (size_t)(q * 64 + rank * 8 + s) * 16 + f, tag | F_RAW);
     }
    }
  };

  if (sender) {
   if (wave < NSEND) {
    u64 wsp = (u64)(uintptr_t)P.ws_all[lane & 7];
    asm volatile("" : "+v"(wsp));
    const __amdgpu_buffer_rsrc_t rs = __builtin_amdgcn_make_buffer_rsrc(myws, 0, 0x7fffffff, 0x00020000);
    unsigned so, info;
    int pn;
    int dbg = 0, pdbg = 0;
    gchar* dst;
    gu32* flag;
    u32x4 va, vb, vc;
    unsigned ve;
    auto draw = [&]() -> bool {
      unsigned j = 0;
      if (lane == 0)
        asm volatile("ds_add_rtn_u32 %0, %1, %2\n\ts_waitcnt lgkmcnt(0)" : "=v"(j) : "v"(ctl + C_NEXT * 4), "v"(1u) : "memory");
      j = __builtin_amdgcn_readfirstlane(j);
      const int f = j & 3;
      int u = (int)(j >> 2) * NSWG + (bk - 26) * 8 + bx;
      if (j >= (unsigned)(S1 * 4)) {
        unsigned e;
        while (((e = __builtin_amdgcn_readfirstlane(lds_ld(ctl + (C_Q + ((j >> 2) & 63)) * 4))) >> 12) != (j >> 2) + 1)
          __builtin_amdgcn_s_sleep(1);
        u = e & 4095;
        if (u >= NUNIT) return false;
      }
      const int pr = u % 7, d = (rank + 1 + pr) & 7;
      int s, q, rr, pe;
      unit_of(u / 7, pr, s, q, rr);
      gchar* dws = (gchar*)(((u64)(unsigned)__builtin_amdgcn_readlane((int)(wsp >> 32), d) << 32) |
                            (unsigned)__builtin_amdgcn_readlane((int)wsp, d));
      if (rr != 2) {
        const int xu = 2 * q + rr;
        so = (unsigned)OFF_EX + (unsigned)(s * XU + xu) * UNIT;
        dst = dws + OFF_XS + ((size_t)((rank * NSTAGE + s) * XU + xu)) * UNIT;
        pe = s < 7 ? 128 + s * 160 + q * 10 + rr : 1248 + xu;
      } else {
        so = (unsigned)OFF_EW + (unsigned)((d * NSTAGE + s) * WU + q) * UNIT;
        dst = dws + OFF_WS + ((size_t)((rank * NSTAGE + s) * WU + q)) * UNIT;
        pe = s == 0 ? q * 8 + pr : 128 + (s - 1) * 160 + q * 10 + 2 + pr;
      }
      flag = (gu32*)(dws + OFF_F1 + (size_t)u * F1S) + f;
      const bool tail = u >= NUNIT - RTU;
      if (tail) {
        so = (unsigned)OFF_XS + (unsigned)((rank * NSTAGE + s) * XU + 2 * q + rr) * UNIT;
        flag = (gu32*)(dws + OFF_XF + ((2 * q + rr) * 64 + rank * 8 + s) * 16) + f;
      }
      so = __builtin_amdgcn_readfirstlane(so + f * 4096);
      dst += f * 4096 + lane * 16;
      pn = pe * 4 + f;
      dbg = u * 4 + f;
      T2(2, dbg);
      info = __builtin_amdgcn_readfirstlane(lds_ld(ldsbase + pn * 4));
      if (tail) { pn |= 1 << 30; if (((info ^ tag) >> 13) == 0u) info = tag | F_RAW; }
      if (((info ^ tag) >> 13) == 0u) {
        va = __builtin_amdgcn_raw_buffer_load_b128(rs, lane * 16, so, 17);
        vb = __builtin_amdgcn_raw_buffer_load_b128(rs, lane * 16 + 1024, so, 17);
        vc = __builtin_amdgcn_raw_buffer_load_b128(rs, lane * 16 + 2048, so, 17);
        ve = __builtin_amdgcn_raw_buffer_load_b32(rs, (lane & 15) * 4 + 3072, so, 17);
      }
      return true;
    };
    gu32* pflag = nullptr;
    unsigned pv = 0;
    bool cur = false;
    va = vb = vc = u32x4{0u, 0u, 0u, 0u}; ve = 0; info = 0; so = 0; pn = 0; dst = nullptr; flag = nullptr;
    while (true) {
      asm volatile("s_waitcnt vmcnt(0)" : "+v"(va), "+v"(vb), "+v"(vc), "+v"(ve) :: "memory");
      if (pflag != nullptr && lane == 0) __hip_atomic_store(pflag, pv, __ATOMIC_RELAXED, __HIP_MEMORY_SCOPE_SYSTEM);
      if (pflag != nullptr) T2(3, pdbg);
      pdbg = dbg;
      const u32x4 ca = va, cb = vb, cc = vc;
      const unsigned ce = ve, cso = so;
      const int cpn = pn;
      gchar* const cdst = dst;
      pflag = flag;
      pv = info;
      const bool more = draw();
      if (cur) {
        if (((pv ^ tag) >> 13) != 0u) {
          int s, q, d;
          unsigned p[4][4];
          float m;
          convert(cpn & 0xffff, p, m, s, q, d);
          pv = tag | send_block(p, m, cdst, lane, (cpn >> 30) != 0, true);
        } else {
          st16(cdst, ca);
          asm volatile("s_waitcnt vmcnt(0)" ::: "memory");
          st16(cdst + 1024, cb);
          asm volatile("s_waitcnt vmcnt(0)" ::: "memory");
          st16(cdst + 2048, cc);
          if (pv & F_RAW) st16(cdst + 3072, __builtin_amdgcn_raw_buffer_load_b128(rs, lane * 16 + 3072, cso, 17));
          else if ((pv & (31u << 7)) && lane < MAXESC) st4(cdst - lane * 12 + 3072, ce);
        }
      }
      if (!more) break;
      cur = true;
    }
    asm volatile("s_waitcnt vmcnt(0)" ::: "memory");
    if (pflag != nullptr && lane == 0) __hip_atomic_store(pflag, pv, __ATOMIC_RELAXED, __HIP_MEMORY_SCOPE_SYSTEM);
    if (wave < 2) TS(1 + wave);
   } else if (wave < NSEND + NSC) {
    const u64 t0 = __builtin_amdgcn_s_memrealtime();
    bool dead = false;
    for (int c = wave - NSEND; c < NPACK / 256 && !dead; c += NSC) {
      const char* src = myws + OFF_EF + c * 1024 + lane * 16;
      for (unsigned spins = 1;; ++spins) {
        u32x4 v;
        asm volatile("global_load_dwordx4 %0, %1, off sc0 sc1\n\ts_waitcnt vmcnt(0)" : "=v"(v) : "v"(src) : "memory");
        asm volatile("ds_write_b128 %0, %1" :: "v"(ldsbase + c * 1024 + lane * 16), "v"(v) : "memory");
        if (__all((((v[0] ^ tag) | (v[1] ^ tag) | (v[2] ^ tag) | (v[3] ^ tag)) >> 13) == 0u)) break;
        if ((spins & 15) == 0 &&
            (__hip_atomic_load(tmo, __ATOMIC_RELAXED, __HIP_MEMORY_SCOPE_AGENT) == call ||
             __builtin_amdgcn_s_memrealtime() - t0 > DEADLINE_TICKS)) {
          if (lane == 0) __hip_atomic_store(tmo, call, __ATOMIC_RELAXED, __HIP_MEMORY_SCOPE_AGENT);
          dead = true;
          break;
        }
        __builtin_amdgcn_s_sleep(SCS);
      }
    }
    if (wave == NSEND) TS(33);
   } else if (wave == NWAVE - 1) {
    unsigned* const sq = reinterpret_cast<unsigned*>(myws + OFF_SQ);
    const unsigned qbase = (call - 1) * NDRAW - S1 * NSWG;
    for (unsigned k = S1;; ++k) {
      while ((lds_ld(ctl + C_NEXT * 4) >> 2) + TKD < k) __builtin_amdgcn_s_sleep(2);
      unsigned t = 0;
      if (lane == 0) t = __hip_atomic_fetch_add(sq, 1u, __ATOMIC_RELAXED, __HIP_MEMORY_SCOPE_AGENT);
      const unsigned u = __builtin_amdgcn_readfirstlane(t) - qbase;
      if (u >= (unsigned)NUNIT) {
        if (lane < 8) lds_st(ctl + (C_Q + ((k + lane) & 63)) * 4, ((k + lane + 1) << 12) | 4095u);
        break;
      }
      if (lane == 0) lds_st(ctl + (C_Q + (k & 63)) * 4, ((k + 1) << 12) | u);
    }
   }
  } else if (wave < 8) {
    if (CPK) pack_items(wave * NCW + bk * 8 + bx, 1);
    if (wave == 0) TS(30);
#pragma unroll
    for (int a = 0; a < 5; ++a)
#pragma unroll
      for (int b = 0; b < 4; ++b) acc[a][b] = f32x4{0.f, 0.f, 0.f, 0.f};
    const int xoff = (wm * 5) * 1024 + lane * 16;
    const int woff = 20480 + (wn * 4) * 1024 + lane * 16;
    int slot = 0;
    for (int h = 0; h < 2 * NSTEP; ++h) {
      while (lds_ld(ctl + (C_FULL + (h & 1)) * 4) <= (unsigned)(h >> 1)) __builtin_amdgcn_s_sleep(1);
      const char* sb = lds + slot * HS;
      bf16x8 xf[5], wf[4];
#pragma unroll
      for (int i = 0; i < 4; ++i) wf[i] = *reinterpret_cast<const bf16x8*>(sb + woff + i * 1024);
#pragma unroll
      for (int i = 0; i < 5; ++i) xf[i] = *reinterpret_cast<const bf16x8*>(sb + xoff + i * 1024);
#pragma unroll
      for (int mt = 0; mt < 4; ++mt)
#pragma unroll
        for (int nt = 0; nt < 4; ++nt)
          acc[mt][nt] = __builtin_amdgcn_mfma_f32_16x16x32_bf16(wf[nt], xf[mt], acc[mt][nt], 0, 0, 0);
      asm volatile("s_waitcnt lgkmcnt(0)" ::: "memory");
      if (lane == 0) lds_st(ctl + (C_DONE + wave) * 4, (unsigned)(h + 1));
      if (wave == 0 && (h & 1)) T2(5, blockIdx.x * 64 + (h >> 1));
#pragma unroll
      for (int nt = 0; nt < 4; ++nt)
        acc[4][nt] = __builtin_amdgcn_mfma_f32_16x16x32_bf16(wf[nt], xf[4], acc[4][nt], 0, 0, 0);
      slot = (slot == NH - 1) ? 0 : slot + 1;
    }
    float m = 0.f;
#pragma unroll
    for (int a = 0; a < 5; ++a) {
      if (ti * TM + wm * 80 + a * 16 >= 4096) continue;
#pragma unroll
      for (int b = 0; b < 4; ++b)
#pragma unroll
        for (int j = 0; j < 4; ++j) m = fmaxf(m, fabsf(acc[a][b][j]));
    }
#pragma unroll
    for (int o = 32; o >= 1; o >>= 1) m = fmaxf(m, __shfl_xor(m, o));
    if (lane == 0) lds_st(ctl + (C_WMAX + wave) * 4, __float_as_uint(m));
    if (wave == 0) TS(12);
  } else if (wave < 10) {
    const int L = wave - 8;
    const char* const gx0 = myws + OFF_XS + L * 8192 + lane * 16;
    const char* const gw0 = myws + OFF_WS + (size_t)tj * UNIT + L * 8192 + lane * 16;
    int slot = L;
    for (int t = 0; t < NSTEP; ++t) {
      unsigned e = lds_ld(ctl + (C_Q + t) * 4);
      if (e == 0) {
        asm volatile("s_waitcnt vmcnt(0)" ::: "memory");
        if (lane == 0) lds_st(ctl + (C_FULL + L) * 4, (unsigned)t);
        while ((e = lds_ld(ctl + (C_Q + t) * 4)) == 0) __builtin_amdgcn_s_sleep(1);
      }
      const int bit = (int)__builtin_amdgcn_readfirstlane(e) - 1;
      const int need = 2 * t + L - NH + 1;
      if (need > 0) {
        while (true) {
          unsigned d = lds_ld(ctl + (C_DONE + (lane & 7)) * 4);
          if (__all(d >= (unsigned)need)) break;
          __builtin_amdgcn_s_sleep(1);
        }
      }
      const unsigned l = ldsbase + slot * HS;
#pragma unroll
      for (int i = 0; i < 5; ++i) {
        const int hu = min(5 * ti + i, 63);
        glds4k(gx0 + ((size_t)(bit * XU + (hu >> 1))) * UNIT + (hu & 1) * 4096, l + i * 4096);
      }
#pragma unroll
      for (int i = 0; i < 2; ++i) glds4k(gw0 + (size_t)(bit * WU) * UNIT + i * 4096, l + 20480 + i * 4096);
      slot = slot + 2 >= NH ? slot + 2 - NH : slot + 2;
      asm volatile("s_waitcnt vmcnt(28)" ::: "memory");
      if (lane == 0) lds_st(ctl + (C_FULL + L) * 4, (unsigned)t);
    }
    asm volatile("s_waitcnt vmcnt(0)" ::: "memory");
    if (lane == 0) lds_st(ctl + (C_FULL + L) * 4, (unsigned)NSTEP);
    if (L == 0) TS(11);
  } else if (wave == 10) {
    const unsigned want = tag | F_RAW;
    const unsigned m2 = ti == NTI - 1 ? 0u : ~0u;
    const unsigned* pf0 = reinterpret_cast<const unsigned*>(myws + OFF_XF) + (((5 * ti) >> 1) * 64 + lane) * SU;
    const unsigned* pf2 = reinterpret_cast<const unsigned*>(myws + OFF_WF) + (tj * 64 + lane) * SU;
    u64 seen = 0;
    unsigned cnt = 0, spins = 0;
    bool dead = false;
    const u64 t0 = __builtin_amdgcn_s_memrealtime();
    while (cnt < (unsigned)NSTEP) {
      u32x4 f0, f1, f2, f3;
      asm volatile("global_load_dwordx4 %0, %4, off sc0 sc1\n\tglobal_load_dwordx4 %1, %4, off offset:1024 sc0 sc1\n\t"
                   "global_load_dwordx4 %2, %4, off offset:2048 sc0 sc1\n\t"
                   "global_load_dwordx4 %3, %5, off sc0 sc1\n\ts_waitcnt vmcnt(0)"
                   : "=&v"(f0), "=&v"(f1), "=&v"(f2), "=&v"(f3) : "v"(pf0), "v"(pf2) : "memory");
      const u64 nm = __ballot(dead || plain16(f0, f1, f2, f3, want, m2)) & ~seen;
      if (nm) {
        if ((nm >> lane) & 1) {
          const unsigned pos = cnt + __builtin_amdgcn_mbcnt_hi((unsigned)(nm >> 32), __builtin_amdgcn_mbcnt_lo((unsigned)nm, 0u));
          lds_st(ctl + (C_Q + pos) * 4, (unsigned)lane + 1);
        }
        cnt += (unsigned)__builtin_popcountll(nm);
        seen |= nm;
      } else if ((++spins & 15) == 0 &&
                 (__hip_atomic_load(tmo, __ATOMIC_RELAXED, __HIP_MEMORY_SCOPE_AGENT) == call ||
                  __builtin_amdgcn_s_memrealtime() - t0 > DEADLINE_TICKS)) {
        if (lane == 0) __hip_atomic_store(tmo, call, __ATOMIC_RELAXED, __HIP_MEMORY_SCOPE_AGENT);
        dead = true;
      }
      __builtin_amdgcn_s_sleep(PSLEEP);
    }
    TS(32);
  } else if (wave < W_DEC + NDEC) {
    __builtin_amdgcn_s_setprio(DPRIO);
    const __amdgpu_buffer_rsrc_t rs = __builtin_amdgcn_make_buffer_rsrc(myws, 0, 0x7fffffff, 0x00020000);
    const u64 t0 = __builtin_amdgcn_s_memrealtime();
    bool dead = false;
    for (int n = (wave - W_DEC) * NCW + bk * 8 + bx; n < NIN - RTU && !dead; n += NDEC * NCW) {
      const int pr = n % 7;
      int s, q, rr;
      unit_of(n / 7, pr, s, q, rr);
      const int st = ((rank + 7 - pr) & 7) * 8 + s;
      unsigned so, fo;
      if (rr != 2) {
        so = (unsigned)OFF_XS + (unsigned)(st * XU + 2 * q + rr) * UNIT;
        fo = (unsigned)OFF_XF + ((2 * q + rr) * 64 + st) * 16;
      } else {
        so = (unsigned)OFF_WS + (unsigned)(st * WU + q) * UNIT;
        fo = (unsigned)OFF_WF + (q * 64 + st) * 16;
      }
      so = __builtin_amdgcn_readfirstlane(so);
      fo = __builtin_amdgcn_readfirstlane(fo);
      unsigned f[4];
      for (unsigned spins = 1;; ++spins) {
        asm volatile("" ::: "memory");
        const u32x4 fv = __builtin_amdgcn_raw_buffer_load_b128(rs, 0, (unsigned)OFF_F1 + n * F1S, 17);
#pragma unroll
        for (int k = 0; k < 4; ++k) f[k] = __builtin_amdgcn_readfirstlane(fv[k]);
        if ((((f[0] ^ tag) | (f[1] ^ tag) | (f[2] ^ tag) | (f[3] ^ tag)) >> 13) == 0u) break;
        if ((spins & 15) == 0 &&
            (__hip_atomic_load(tmo, __ATOMIC_RELAXED, __HIP_MEMORY_SCOPE_AGENT) == call ||
             __builtin_amdgcn_s_memrealtime() - t0 > DEADLINE_TICKS)) {
          if (lane == 0) __hip_atomic_store(tmo, call, __ATOMIC_RELAXED, __HIP_MEMORY_SCOPE_AGENT);
          dead = true;
          break;
        }
        __builtin_amdgcn_s_sleep(DSLEEP);
      }
      if (dead) continue;
      T2(0, n);
      u32x4 a[4], b[4], c[4];
      unsigned e[4];
#pragma unroll
      for (int k = 0; k < 4; ++k) {
        a[k] = __builtin_amdgcn_raw_buffer_load_b128(rs, lane * 16 + k * 4096, so, 17);
        b[k] = __builtin_amdgcn_raw_buffer_load_b128(rs, lane * 16 + k * 4096 + 1024, so, 17);
        c[k] = __builtin_amdgcn_raw_buffer_load_b128(rs, lane * 16 + k * 4096 + 2048, so, 17);
        e[k] = __builtin_amdgcn_raw_buffer_load_b32(rs, (lane & 15) * 4 + k * 4096 + 3072, so, 17);
      }
#pragma unroll
      for (int k = 0; k < 4; ++k) {
        if (f[k] & F_RAW) continue;
        const unsigned ne = (f[k] >> 7) & 31u;
        const unsigned base4 = (f[k] & 0x7fu) * 0x01010101u;
        u32x4 o[4];
        o[0] = unpack8(a[k][0], a[k][1], c[k][0], base4);
        o[1] = unpack8(a[k][2], a[k][3], c[k][1], base4);
        o[2] = unpack8(b[k][0], b[k][1], c[k][2], base4);
        o[3] = unpack8(b[k][2], b[k][3], c[k][3], base4);
        for (unsigned i = 0; i < ne && i < (unsigned)MAXESC; ++i) {
          const unsigned ev = (unsigned)__builtin_amdgcn_readlane((int)e[k], (int)i);
          const unsigned sh = (ev & 1u) * 16 + 8, r = ((ev >> 7) & 12u) | ((ev >> 1) & 3u);
          const unsigned vm = (unsigned)lane == ((ev >> 3) & 63u) ? 0xffu << sh : 0u;
          const unsigned vv = ((ev >> 16) & 0xffu) << sh;
#pragma unroll
          for (int kk = 0; kk < 16; ++kk) {
            const unsigned mk = r == (unsigned)kk ? vm : 0u;
            o[kk >> 2][kk & 3] = (o[kk >> 2][kk & 3] & ~mk) | (vv & mk);
          }
        }
#pragma unroll
        for (int j = 0; j < 4; ++j)
          __builtin_amdgcn_raw_buffer_store_b128(o[j], rs, lane * 16 + k * 4096 + j * 1024, so, 17);
      }
      asm volatile("s_waitcnt vmcnt(0)" ::: "memory");
      if (lane == 0) {
        const unsigned w = tag | F_RAW;
        __builtin_amdgcn_raw_buffer_store_b128(u32x4{w, w, w, w}, rs, 0, fo, 17);
      }
      T2(1, n);
    }
    if (wave == W_DEC) TS(19);
  } else {
    pack_items((wave - W_DEC - NDEC + 8 * CPK) * NCW + bk * 8 + bx, NROUND);
    if (wave == NWAVE - 1) TS(31);
  }

  asm volatile("s_waitcnt lgkmcnt(0)" ::: "memory");
  __syncthreads();

  if (wave == 0) {
    TS(13);
    float m = __uint_as_float(lds_ld(ctl + (C_WMAX + (lane & 7)) * 4));
#pragma unroll
    for (int o = 4; o >= 1; o >>= 1) m = fmaxf(m, __shfl_xor(m, o));
    unsigned last = 0;
    if (lane == 0) {
      u64 v = ((u64)call << 32) | (u64)__float_as_uint(m);
      u64 old = __hip_atomic_fetch_max(reinterpret_cast<u64*>(myws + OFF_AMAX), v, __ATOMIC_RELAXED,
                                       __HIP_MEMORY_SCOPE_AGENT);
      asm volatile("s_waitcnt vmcnt(0)" :: "v"(old) : "memory");
      unsigned tk = __hip_atomic_fetch_add(reinterpret_cast<unsigned*>(myws + OFF_ARRIVE), 1u,
                                           __ATOMIC_RELAXED, __HIP_MEMORY_SCOPE_AGENT);
      last = (tk == call * NWG - 1) ? 1u : 0u;
    }
    last = __builtin_amdgcn_readfirstlane(last);
    if (last) {
      u64 v = __hip_atomic_load(reinterpret_cast<u64*>(myws + OFF_AMAX), __ATOMIC_RELAXED,
                                __HIP_MEMORY_SCOPE_AGENT);
      if (lane < NR) {
        gu64* dstp = (gu64*)((gchar*)(P.ws_all[lane]) + OFF_SLOTS) + rank;
        __hip_atomic_store(dstp, v, __ATOMIC_RELAXED, __HIP_MEMORY_SCOPE_SYSTEM);
      }
    }
  }
  if (sender) return;
  if (wave == 0) {
    const u64* slots = reinterpret_cast<const u64*>(myws + OFF_SLOTS) + (lane & 7);
    const u64 t0 = __builtin_amdgcn_s_memrealtime();
    u64 v;
    while (true) {
      v = __hip_atomic_load(slots, __ATOMIC_RELAXED, __HIP_MEMORY_SCOPE_SYSTEM);
      if (__all((unsigned)(v >> 32) >= call)) break;
      if (__hip_atomic_load(tmo, __ATOMIC_RELAXED, __HIP_MEMORY_SCOPE_AGENT) == call ||
          __builtin_amdgcn_s_memrealtime() - t0 > DEADLINE_TICKS) {
        if (lane == 0) __hip_atomic_store(tmo, call, __ATOMIC_RELAXED, __HIP_MEMORY_SCOPE_AGENT);
        break;
      }
      __builtin_amdgcn_s_sleep(1);
    }
    float g = __uint_as_float((unsigned)v);
#pragma unroll
    for (int o = 4; o >= 1; o >>= 1) g = fmaxf(g, __shfl_xor(g, o));
    unsigned dead = (__hip_atomic_load(tmo, __ATOMIC_RELAXED, __HIP_MEMORY_SCOPE_AGENT) == call) ? 1u : 0u;
    if (lane == 0) { lds_st(ctl + C_GMAX * 4, __float_as_uint(g)); lds_st(ctl + C_DEAD * 4, dead); }
  }
  const u64 outp = (u64)(uintptr_t)P.out_all[lane & 7];
  asm volatile("s_waitcnt lgkmcnt(0)" ::: "memory");
  __syncthreads();

  if (wave == 0) TS(14);
  if (wave < 8) {
    const float amax = __uint_as_float(lds_ld(ctl + C_GMAX * 4));
    const bool dead = lds_ld(ctl + C_DEAD * 4) != 0;
    const float scale = amax / 127.0f;
    const float inv = scale > 0.f ? 1.0f / scale : 0.f;
    const int col = rank * NLOC + tj * 128 + wn * 64 + (lane >> 4) * 16;
#pragma unroll
    for (int mt = 0; mt < 5; ++mt) {
      const int row0 = ti * TM + wm * 80 + mt * 16;
      if (row0 >= 4096) continue;
      const int owner = row0 >> 9;
      gchar* ob = (gchar*)(((u64)(unsigned)__builtin_amdgcn_readlane((int)(outp >> 32), owner) << 32) |
                           (unsigned)__builtin_amdgcn_readlane((int)outp, owner));
      u32x4 o;
#pragma unroll
      for (int j = 0; j < 4; ++j) {
        unsigned pk = 0;
#pragma unroll
        for (int nt = 0; nt < 4; ++nt) {
          float q = rintf(acc[mt][nt][j] * inv);
          q = fminf(fmaxf(q, -127.f), 127.f);
          int qi = dead ? -128 : (int)q;
          pk |= ((unsigned)qi & 0xffu) << (8 * nt);
        }
        o[j] = pk;
      }
      st16(ob + (size_t)((row0 & 511) + (lane & 15)) * NTOT + col, o);
    }
    if (wave == 0) TS(17);
    asm volatile("s_waitcnt vmcnt(0)" ::: "memory");
    if (wave == 0) TS(18);
  }
  asm volatile("s_waitcnt lgkmcnt(0)" ::: "memory");
  __syncthreads();
  if (wave == 0) {
    TS(15);
    {
      const int o0 = (ti * TM) >> 9, o1 = min(ti * TM + TM - 1, 4095) >> 9;
      if (lane == 0 || (lane == 1 && o1 != o0)) {
        gu32* f = (gu32*)((gchar*)(P.ws_all[(lane ? o1 : o0) & 7]) + OFF_OUTF) + (rank * NTI + ti) * 16 + tj;
        __hip_atomic_store(f, call, __ATOMIC_RELAXED, __HIP_MEMORY_SCOPE_SYSTEM);
      }
    }
    if (blockIdx.x == 48) {
      const int tlo = (rank * MSH) / TM, thi = (rank * MSH + MSH - 1) / TM;
      const unsigned* of = reinterpret_cast<const unsigned*>(myws + OFF_OUTF);
      const u64 t0 = __builtin_amdgcn_s_memrealtime();
      while (true) {
        bool ok = true;
#pragma unroll
        for (int j = 0; j < 6; ++j) {
          const int idx = lane + 64 * j, sr = idx / 48, rem = idx - sr * 48;
          const int tt = min(tlo + (rem >> 4), thi);
          ok &= __hip_atomic_load(of + (sr * NTI + tt) * 16 + (rem & 15), __ATOMIC_RELAXED, __HIP_MEMORY_SCOPE_SYSTEM) >= call;
        }
        if (__all(ok)) break;
        if (__hip_atomic_load(tmo, __ATOMIC_RELAXED, __HIP_MEMORY_SCOPE_AGENT) == call ||
            __builtin_amdgcn_s_memrealtime() - t0 > DEADLINE_TICKS) {
          if (lane == 0) {
            __hip_atomic_store(tmo, call, __ATOMIC_RELAXED, __HIP_MEMORY_SCOPE_AGENT);
            ((gchar*)(P.out_all[rank]))[rank * NLOC] = (char)-128;
          }
          break;
        }
        __builtin_amdgcn_s_sleep(1);
      }
      __builtin_amdgcn_fence(__ATOMIC_ACQUIRE, "");
      TS(16);
    }
  }
}

extern "C" void kernel_launch(int rank, int nranks, ncclComm_t comm, hipStream_t stream,
                              void* const* d_in, const int* in_sizes, int n_in, void* d_out,
                              int out_size, const int* in_dtypes, int out_dtype,
                              void* const* d_out_all, void* const* d_ws_all, size_t ws_bytes) {
  if (nranks != NR || ws_bytes < WS_NEED || n_in < 2) {
    fprintf(stderr, "unexpected configuration\n");
    return;
  }
  Params P;
  P.x = static_cast<const float*>(d_in[0]);
  P.w = static_cast<const float*>(d_in[1]);
  P.out_all = d_out_all;
  P.ws_all = d_ws_all;
  P.rank = rank;
  hipLaunchKernelGGL(fused_gemm_q8, dim3(NWG), dim3(NTHREADS), 0, stream, P);
  HIP_CHECK(hipGetLastError());
}
```

```cpp
#include <hip/hip_runtime.h>
#include <rccl/rccl.h>
#include <cstdio>
#include <cstdint>
#include <type_traits>

#define HIP_CHECK(expr) do { hipError_t _e = (expr); if (_e != hipSuccess) { \
    fprintf(stderr, "HIP error %s at %s:%d\n", hipGetErrorString(_e), __FILE__, __LINE__); \
    return; } } while (0)

typedef __attribute__((ext_vector_type(8))) short bf16x8;
typedef __attribute__((ext_vector_type(4))) float f32x4;
typedef __attribute__((ext_vector_type(4))) unsigned u32x4;
typedef unsigned long long u64;
#define GAS __attribute__((address_space(1)))
typedef GAS char gchar;
typedef GAS unsigned gu32;
typedef GAS unsigned long long gu64;
typedef GAS u32x4 gu32x4;

namespace {
constexpr int NR = 8;
constexpr int KSH = 512;
constexpr int NTOT = 16384;
constexpr int NLOC = 2048;
constexpr int MSH = 512;
constexpr int NSTAGE = 8;
constexpr int XU = 32;
constexpr int WU = 16;
constexpr int UNIT = 16384;
constexpr int NWG = 256;
constexpr int SU = 4;
constexpr int NUNIT = 7 * NSTAGE * (XU + WU);
#define THR 2
#define PSLEEP 16
#define SCS 32
#define F1S 128
#define MAPB 0
#define DLY 30
#define DSLOW 0
#define LOS 32
#define LW 2
#define NLO 1
#define PRO 1
#define PKD 0
#define KSLOW 1
#define FWAIT 40
#define RECHK 1
#define BAL 1
#define LPRIO 3
#define PFD 0
#define PFL 3
#define TLO 56
#define THI 64
#define PIPE 1
#define QX 0
#define CPRIO 0
#define DSLEEP 16
#define DPRIO 2
#define LP 0
#define LDPOL " sc0 sc1"
constexpr int NSTEP = 64;
constexpr int TM = 320;
constexpr int NTI = 13;
constexpr int NCW = NTI * 16;
constexpr int NH = 5;
constexpr int HS = 7 * 4096;
#define NSEND 13
constexpr int NSWG = NWG - NCW;
constexpr int NSC = 15 - NSEND < 4 ? 15 - NSEND : 4;
#define S1V 4
#define RTU 224
#define TKD 1
constexpr int S1 = S1V;
constexpr int KW = 3;
constexpr int U0 = NCW * KW * 2 / 4;
constexpr int NDRAW = NUNIT - U0 - S1 * NSWG + NSWG;
static_assert(NSEND + NSC < 16 && RTU == 224 && U0 >= 288, "sender split");
#define NWAVE 16
#define NDECV 2
constexpr int NDEC = NDECV;
constexpr int W_DEC = 11;
constexpr int NTHREADS = 64 * NWAVE;
constexpr unsigned F_RAW = 1u << 12;
constexpr int MAXESC = 16;
constexpr int NPK = NWAVE - W_DEC - NDEC;
constexpr int NPACK = (8 * XU + 56 + 6 * 112 + NSTAGE * WU) * 4;
constexpr int NMIR = 17;
#define CPK 0
constexpr int PR0 = NCW * (8 * CPK + NPK);
constexpr int PRN = NCW * NPK;
constexpr int NROUND = 1 + (NPACK - PR0 + PRN - 1) / PRN;
constexpr int NIN = 7 * 384;

constexpr size_t OFF_TICKET = 0;
constexpr size_t OFF_TMO = 128;
constexpr size_t OFF_AMAX = 256;
constexpr size_t OFF_ARRIVE = 384;
constexpr size_t OFF_SLOTS = 512;
constexpr size_t OFF_SQ = 2048;
constexpr size_t OFF_XF = 4096;
constexpr size_t OFF_WF = OFF_XF + XU * 64 * SU * 4;
constexpr size_t OFF_OUTF = 65536;
constexpr size_t OFF_EF = 81920;
constexpr size_t OFF_F1 = 131072;
constexpr size_t OFF_XS = 1u << 20;
constexpr size_t OFF_WS = OFF_XS + (size_t)NR * NSTAGE * XU * UNIT;
constexpr size_t OFF_EX = OFF_WS + (size_t)NR * NSTAGE * WU * UNIT;
constexpr size_t OFF_EW = OFF_EX + (size_t)NSTAGE * XU * UNIT;
constexpr size_t WS_NEED = OFF_EW + (size_t)NR * NSTAGE * WU * UNIT;

constexpr u64 DEADLINE_TICKS = 40000000ull;

constexpr int C_CALL = 0, C_FULL = 128, C_DONE = 8  , C_WMAX = 100, C_GMAX = 24, C_DEAD = 25, C_NEXT = 26, C_Q = 32;
constexpr int LDS_CTL = NH * HS;
constexpr int LDS_BYTES = LDS_CTL + 512 + 1024;
}

#define T2(a, i) do {} while (0)
#define TS(i) do {} while (0)
#define TV(i, v) do {} while (0)
#define CYC() 0ull

struct Params {
  const float* x;
  const float* w;
  void* const* out_all;
  void* const* ws_all;
  int rank;
};

__device__ __forceinline__ unsigned pk_bf16(float a, float b) {
  typedef __attribute__((ext_vector_type(2))) float f2;
  typedef __attribute__((ext_vector_type(2))) __bf16 b2;
  f2 v = {a, b};
  b2 r = __builtin_convertvector(v, b2);
  return __builtin_bit_cast(unsigned, r);
}

__device__ __forceinline__ unsigned lds_ld(unsigned addr) {
  unsigned v;
  asm volatile("ds_read_b32 %0, %1\n\ts_waitcnt lgkmcnt(0)" : "=v"(v) : "v"(addr) : "memory");
  return v;
}
__device__ __forceinline__ void glds4k(const void* g, unsigned ldsaddr) {
  unsigned keep;
  asm volatile(
      "s_mov_b32 %0, m0\n\ts_mov_b32 m0, %2\n\ts_nop 0\n\t"
      "global_load_lds_dwordx4 %1, off" LDPOL "\n\t"
      "global_load_lds_dwordx4 %1, off offset:1024" LDPOL "\n\t"
      "global_load_lds_dwordx4 %1, off offset:2048" LDPOL "\n\t"
      "global_load_lds_dwordx4 %1, off offset:3072" LDPOL "\n\t"
      "s_mov_b32 m0, %0"
      : "=&s"(keep) : "v"(g), "s"(ldsaddr) : "memory");
}
__device__ __forceinline__ bool plain16(u32x4 a, u32x4 b, u32x4 c, u32x4 d, unsigned want, unsigned m2) {
  unsigned x = 0, y = 0;
#pragma unroll
  for (int j = 0; j < 4; ++j) { x |= (a[j] ^ want) | (b[j] ^ want) | (d[j] ^ want); y |= c[j] ^ want; }
  return ((x | (y & m2)) >> 12) == 0u;
}
__device__ __forceinline__ void unit_of(int ii, int pr, int& s, int& q, int& rr) {
  const int rot = (pr * 16 + 3) / 7;
  if (ii < 16) { s = 0; q = ii; rr = 2; }
  else if (ii < 352) {
    const int i2 = ii - 16, it = i2 % 48;
    s = i2 / 48; q = it / 3; rr = it - 3 * q;
    if (rr == 2) s += 1; else q = (q + rot) & 15;
  } else { s = 7; q = (((ii - 352) >> 1) + rot) & 15; rr = (ii - 352) & 1; }
}
__device__ __forceinline__ void lds_st(unsigned addr, unsigned v) {
  asm volatile("ds_write_b32 %0, %1" :: "v"(addr), "v"(v) : "memory");
}

__device__ __forceinline__ void st16(gchar* p, u32x4 v) {
  asm volatile("global_store_dwordx4 %0, %1, off sc0 sc1\n\ts_nop 1" :: "v"(p), "v"(v) : "memory");
}

__device__ __forceinline__ void st4(gchar* p, unsigned v) {
  asm volatile("global_store_dword %0, %1, off sc0 sc1\n\ts_nop 1" :: "v"(p), "v"(v) : "memory");
}

__device__ __forceinline__ u32x4 unpack8(unsigned l0, unsigned l1, unsigned n, unsigned base4) {
  const unsigned h0 = ((n & 0x07070707u) + base4) | ((n & 0x08080808u) << 4);
  const unsigned h1 = (((n >> 4) & 0x07070707u) + base4) | (n & 0x80808080u);
  return u32x4{__builtin_amdgcn_perm(h0, l0, 0x05010400u), __builtin_amdgcn_perm(h0, l0, 0x07030602u),
               __builtin_amdgcn_perm(h1, l1, 0x05010400u), __builtin_amdgcn_perm(h1, l1, 0x07030602u)};
}

__device__ __forceinline__ unsigned send_block(const unsigned (&p)[4][4], float m, gchar* dst, int lane, bool plain,
                                              int slow = 0) {
  if (plain) {
#pragma unroll
    for (int j = 0; j < 4; ++j) st16(dst + j * 1024, u32x4{p[j][0], p[j][1], p[j][2], p[j][3]});
    return F_RAW;
  }
  int mi = (int)__float_as_uint(m);
  mi = max(mi, __builtin_amdgcn_update_dpp(mi, mi, 0x111, 0xf, 0xf, false));
  mi = max(mi, __builtin_amdgcn_update_dpp(mi, mi, 0x112, 0xf, 0xf, false));
  mi = max(mi, __builtin_amdgcn_update_dpp(mi, mi, 0x114, 0xf, 0xf, false));
  mi = max(mi, __builtin_amdgcn_update_dpp(mi, mi, 0x118, 0xf, 0xf, false));
  mi = max(mi, __builtin_amdgcn_update_dpp(mi, mi, 0x142, 0xa, 0xf, false));
  mi = max(mi, __builtin_amdgcn_update_dpp(mi, mi, 0x143, 0xc, 0xf, false));
  const unsigned top = (pk_bf16(__uint_as_float((unsigned)__builtin_amdgcn_readlane(mi, 63)), 0.f) >> 8) & 0x7fu;
  const unsigned base = __builtin_amdgcn_readfirstlane(top > 7u ? top - 7u : 0u);
  const unsigned base4 = base * 0x01010101u;
  unsigned lo[4][2], hi[4][2], nb[4], aa = ~0u, ao = 0u;
#pragma unroll
  for (int j = 0; j < 4; ++j) {
    lo[j][0] = __builtin_amdgcn_perm(p[j][1], p[j][0], 0x06040200u);
    hi[j][0] = __builtin_amdgcn_perm(p[j][1], p[j][0], 0x07050301u);
    lo[j][1] = __builtin_amdgcn_perm(p[j][3], p[j][2], 0x06040200u);
    hi[j][1] = __builtin_amdgcn_perm(p[j][3], p[j][2], 0x07050301u);
    const unsigned u0 = (hi[j][0] | 0x80808080u) - base4;
    const unsigned u1 = (hi[j][1] | 0x80808080u) - base4;
    aa &= u0 & u1;
    ao |= u0 | u1;
    nb[j] = (u0 & 0x07070707u) | ((hi[j][0] >> 4) & 0x08080808u) | ((u1 & 0x07070707u) << 4) |
            (hi[j][1] & 0x80808080u);
  }
  unsigned info = base;
  if (__ballot(((~aa & 0x80808080u) | (ao & 0x78787878u)) != 0u)) {
    unsigned em[4][2], c = 0;
#pragma unroll
    for (int j = 0; j < 4; ++j)
#pragma unroll
      for (int h = 0; h < 2; ++h) {
        const unsigned u = (hi[j][h] | 0x80808080u) - base4;
        em[j][h] = (~u | ((u & 0x78787878u) + 0x78787878u)) & 0x80808080u;
        c += __builtin_popcount(em[j][h]);
      }
    unsigned total = 0;
#pragma unroll
    for (int b = 0; b < 6; ++b) total += (unsigned)__builtin_popcountll(__ballot((c >> b) & 1u)) << b;
    if (total > (unsigned)MAXESC) {
#pragma unroll
      for (int j = 0; j < 4; ++j) {
        st16(dst + j * 1024, u32x4{p[j][0], p[j][1], p[j][2], p[j][3]});
        if (slow) asm volatile("s_waitcnt vmcnt(0)" ::: "memory");
      }
      return F_RAW;
    }
    unsigned n = 0;
#pragma unroll
    for (int j = 0; j < 4; ++j)
#pragma unroll
      for (int h = 0; h < 2; ++h) {
        if (__ballot(em[j][h] != 0u) == 0) continue;
        for (int b = 0; b < 4; ++b) {
          const bool on = (em[j][h] >> (8 * b + 7)) & 1u;
          const u64 bal = __ballot(on);
          const unsigned idx = n + __builtin_amdgcn_mbcnt_hi((unsigned)(bal >> 32), __builtin_amdgcn_mbcnt_lo((unsigned)bal, 0u));
          if (on) st4(dst - lane * 16 + 3072 + idx * 4,
                      (unsigned)(j * 512 + lane * 8 + h * 4 + b) | (((hi[j][h] >> (8 * b)) & 0xffu) << 16));
          n += (unsigned)__builtin_popcountll(bal);
        }
      }
    info |= total << 7;
  }
  st16(dst, u32x4{lo[0][0], lo[0][1], lo[1][0], lo[1][1]});
  if (slow & 1) asm volatile("s_waitcnt vmcnt(0)" ::: "memory");
  st16(dst + 1024, u32x4{lo[2][0], lo[2][1], lo[3][0], lo[3][1]});
  if (slow) asm volatile("s_waitcnt vmcnt(0)" ::: "memory");
  st16(dst + 2048, u32x4{nb[0], nb[1], nb[2], nb[3]});
  return info;
}

#define AC00 "v[40:43]"
#define AC01 "v[44:47]"
#define AC02 "v[48:51]"
#define AC03 "v[52:55]"
#define AC10 "v[56:59]"
#define AC11 "v[60:63]"
#define AC12 "v[64:67]"
#define AC13 "v[68:71]"
#define AC20 "v[72:75]"
#define AC21 "v[76:79]"
#define AC22 "v[80:83]"
#define AC23 "v[84:87]"
#define AC30 "v[88:91]"
#define AC31 "v[92:95]"
#define AC32 "v[96:99]"
#define AC33 "v[100:103]"
#define AC40 "v[104:107]"
#define AC41 "v[108:111]"
#define AC42 "v[112:115]"
#define AC43 "v[116:119]"
#define RA0 "v[0:3]"
#define RA1 "v[4:7]"
#define RA2 "v[8:11]"
#define RA3 "v[12:15]"
#define RB0 "v[16:19]"
#define RB1 "v[20:23]"
#define RB2 "v[24:27]"
#define RB3 "v[28:31]"
#define RC0 "v[32:35]"
#define RC1 "v[36:39]"
#define PFY(x) x
#define PFN(x)
#define MF_(d, a, b) "v_mfma_f32_16x16x32_bf16 " d ", " a ", " b ", " d "\n\t"
#define RD_(r, a, o) "ds_read_b128 " r ", " a " offset:" #o "\n\t"
#define WT_(n) "s_waitcnt lgkmcnt(" #n ")\n\t"
#define ADV_ "s_add_u32 %[sdn], %[sdn], 1\n\t"                    \
             "v_mov_b32 %[vt], %[sdn]\n\t"                        \
             "s_mov_b64 exec, 1\n\t"                              \
             "ds_write2_b32 %[vc], %[vt], %[vt] offset0:%[dn] offset1:%[dm]\n\t" \
             "s_mov_b64 exec, -1\n\t"                             \
             "s_cmp_eq_u32 %[sof], %[hl]\n\t"                     \
             "s_cselect_b32 %[sd], %[hw], %[hs]\n\t"              \
             "s_add_u32 %[sof], %[sof], %[sd]\n\t"                \
             "v_add_u32 %[vx], %[sd], %[vx]\n\t"                  \
             "v_add_u32 %[vw], %[sd], %[vw]\n\t"                  \
             "v_add_u32 %[vf], 4, %[vf]\n\t"
#define DBG1_
#define DBG2_
#define FETCH(W0, W1, W2, N0, N1, N2, N3)                           \
  DBG1_                                                             \
  "s_setprio 0\n\t"                                                \
  "6:\n\t"                                                         \
  "ds_read2_b32 v[16:17], %[vf] offset0:0 offset1:1\n\t"           \
  WT_(0)                                                            \
  "v_readfirstlane_b32 %[sd], v16\n\t"                             \
  "v_readfirstlane_b32 %[sfl], v17\n\t"                            \
  "s_cmp_lg_u32 %[sd], 0\n\t"                                      \
  "s_cbranch_scc1 7f\n\t"                                          \
  "s_sleep 1\n\t"                                                  \
  DBG2_                                                             \
  "s_branch 6b\n\t"                                                \
  "7:\n\t"                                                         \
  RD_(N0, "%[vw]", 0) RD_(N1, "%[vw]", 1024) RD_(N2, "%[vw]", 2048) RD_(N3, "%[vw]", 3072)   \
  RD_(W0, "%[vx]", 0) RD_(W1, "%[vx]", 1024) RD_(W2, "%[vx]", 2048)                          \
  RD_(RC0, "%[vx]", 3072) RD_(RC1, "%[vx]", 4096) ADV_
#define PRD_ "ds_read_b32 %[vu], %[vc] offset:%[dp]\n\t"
#define W78_ WT_(8)
#define W67_ WT_(7)
#define PRI_ "v_readfirstlane_b32 %[sd], %[vu]\n\t"                \
             "s_setprio 0\n\t"                                     \
             "s_cmp_gt_u32 %[sd], %[sdn]\n\t"                      \
             "s_cbranch_scc0 8f\n\t"                               \
             "s_setprio 1\n\t"                                     \
             "8:\n\t"
#define BODY(W0, W1, W2, W3, N0, N1, N2, N3, P, Q)                                           \
  P(RD_(N3, "%[vw]", 3072) WT_(6)) Q(WT_(5))                                                 \
  MF_(AC00, W0, N0) MF_(AC01, W1, N0) MF_(AC02, W2, N0) MF_(AC03, W3, N0)                    \
  P(RD_(N0, "%[vw]", 0) "ds_read_b32 %[vt], %[vf] offset:4\n\t" PRD_ W78_) Q(WT_(4))        \
  MF_(AC10, W0, N1) MF_(AC11, W1, N1) MF_(AC12, W2, N1) MF_(AC13, W3, N1)                    \
  P(RD_(N1, "%[vw]", 1024) W78_) Q(WT_(3))                                                   \
  MF_(AC20, W0, N2) MF_(AC21, W1, N2) MF_(AC22, W2, N2) MF_(AC23, W3, N2)                    \
  P(RD_(N2, "%[vw]", 2048) W78_) Q(WT_(2))                                                   \
  MF_(AC30, W0, RC0)                                                                         \
  P(W67_) Q(WT_(1))                                                                          \
  MF_(AC40, W0, RC1)                                                                         \
  P(RD_(W0, "%[vx]", 0))                                                                     \
  MF_(AC31, W1, RC0) MF_(AC41, W1, RC1)                                                      \
  P(RD_(W1, "%[vx]", 1024))                                                                  \
  MF_(AC32, W2, RC0) MF_(AC42, W2, RC1)                                                      \
  P(RD_(W2, "%[vx]", 2048))                                                                  \
  MF_(AC33, W3, RC0)                                                                         \
  P(RD_(RC0, "%[vx]", 3072))                                                                 \
  MF_(AC43, W3, RC1)                                                                         \
  P(RD_(RC1, "%[vx]", 4096) WT_(7) "v_readfirstlane_b32 %[sfl], %[vt]\n\t" PRI_ ADV_)

__global__ __launch_bounds__(NTHREADS, NWAVE / 4) void fused_gemm_q8(Params P) {
  __shared__ __attribute__((aligned(1024))) char lds[LDS_BYTES];
  const unsigned ldsbase = (unsigned)(uintptr_t)lds;
  const unsigned ctl = ldsbase + LDS_CTL;

  const int tid = threadIdx.x;
  const int lane = tid & 63;
  const int wave = __builtin_amdgcn_readfirstlane(tid >> 6);
  const int rank = P.rank;
  char* const myws = static_cast<char*>(P.ws_all[rank]);
  unsigned* const tmo = reinterpret_cast<unsigned*>(myws + OFF_TMO);

  if ((blockIdx.x >> 3) < 6)
    for (int i = tid; i < (NMIR + 4) * 64; i += NTHREADS)
      asm volatile("ds_write_b128 %0, %1" :: "v"(ldsbase + i * 16), "v"(u32x4{0u, 0u, 0u, 0u}) : "memory");
  if (tid < 384) lds_st(ctl + tid * 4, 0u);
  asm volatile("s_waitcnt lgkmcnt(0)\n\ts_barrier" ::: "memory");
  if (wave == 0) TS(0);
  if (tid == ((blockIdx.x >> 3) < 6 ? 64 * 15 : 64 * 10)) {
    const unsigned t = __hip_atomic_fetch_add(reinterpret_cast<unsigned*>(myws + OFF_TICKET), 1u,
                                              __ATOMIC_RELAXED, __HIP_MEMORY_SCOPE_AGENT);
    lds_st(ctl + C_CALL * 4, t / NWG + 1);
  }
  unsigned call = 0, tag = 0;
  auto get_call = [&]() {
    while ((call = __builtin_amdgcn_readfirstlane(lds_ld(ctl + C_CALL * 4))) == 0u) __builtin_amdgcn_s_sleep(1);
    tag = call << 13;
  };

  const int bx = blockIdx.x & 7, bk = ((blockIdx.x >> 3) + 26) & 31;
  const bool sender = bk >= 26;
  const int ti = bk < 24 ? 2 * (bk % 6) + (bx & 1) : 12;
  const int tj = 4 * (bx >> 1) + (bk < 24 ? bk / 6 : 2 * (bx & 1) + ((bk - 24) & 1));

  f32x4 acc[5][4];
  const int wm = wave & 3, wn = (wave >> 2) & 1;

  auto fetch = [&](int s, int q, int d, int f, f32x4 (&a)[8]) {
    if (d < 0) {
      const float* src = P.x + (size_t)(q * 128 + (lane & 15)) * KSH + s * 64 + 8 * (lane >> 4);
#pragma unroll
      for (int j = 0; j < 4; ++j) {
        const float* sp = src + (size_t)((f + j) & 7) * 16 * KSH + ((f + j) >> 3) * 32;
        a[2 * j] = *reinterpret_cast<const f32x4*>(sp);
        a[2 * j + 1] = *reinterpret_cast<const f32x4*>(sp + 4);
      }
    } else {
      const float* sp = P.w + (size_t)(s * 64 + (f >> 3) * 32 + 8 * (lane >> 4)) * NTOT + d * NLOC + q * 128 +
                        (f & 4) * 16 + (lane & 15) * 4;
#pragma unroll
      for (int j = 0; j < 8; ++j) a[j] = *reinterpret_cast<const f32x4*>(sp + (size_t)j * NTOT);
    }
  };
  auto round16 = [&](int d, const f32x4 (&a)[8], unsigned (&p)[4][4], float& m) {
    m = 0.f;
#pragma unroll
    for (int j = 0; j < 8; ++j)
#pragma unroll
      for (int c = 0; c < 4; ++c) m = fmaxf(m, fabsf(a[j][c]));
    if (d < 0) {
#pragma unroll
      for (int j = 0; j < 4; ++j) {
        p[j][0] = pk_bf16(a[2 * j][0], a[2 * j][1]); p[j][1] = pk_bf16(a[2 * j][2], a[2 * j][3]);
        p[j][2] = pk_bf16(a[2 * j + 1][0], a[2 * j + 1][1]); p[j][3] = pk_bf16(a[2 * j + 1][2], a[2 * j + 1][3]);
      }
    } else {
#pragma unroll
      for (int jj = 0; jj < 4; ++jj)
#pragma unroll
        for (int c = 0; c < 4; ++c) p[jj][c] = pk_bf16(a[2 * c][jj], a[2 * c + 1][jj]);
    }
  };
  auto convert = [&](int s, int q, int d, int f, unsigned (&p)[4][4], float& m) {
    f32x4 a[8];
    fetch(s, q, d, f, a);
    round16(d, a, p, m);
  };
  auto pack_items = [&](int g, int rounds) {
    gchar* const gws = (gchar*)myws;
    for (int k = 0; k < rounds; ++k) {
     const int n = k == 0 ? g : PR0 + (k - 1) * PRN + g - 8 * CPK * NCW;
     if (n < NPACK) {
      const int f = (n & 3) * 4, e = n >> 2;
      int s, q, d = -1;
      if (e < 32) { s = 0; q = e; }
      else if (e < 88) { s = 1; q = 8 + (e - 32) / 7; d = (rank + 1 + (e - 32) % 7) & 7; }
      else if (e < 1080) {
        const int r = (e - 88) % 160;
        s = (e - 88) / 160 + 1;
        if (r < 32) q = r;
        else if (r < 144) { s += 1; q = (r - 32) / 7; d = (rank + 1 + (r - 32) % 7) & 7; }
        else { s -= 1; q = r - 144; d = rank; }
      } else { s = 6 + ((e - 1080) >> 4); q = (e - 1080) & 15; d = rank; }
      unsigned p[4][4];
      float m;
      convert(s, q, d, f, p, m);
      unsigned info = 0;
      if (d != rank && e < 1048) {
        gchar* dst = gws + (d < 0 ? OFF_EX + (size_t)(s * XU + q) * UNIT : OFF_EW + (size_t)((d * NSTAGE + s) * WU + q) * UNIT);
        info = send_block(p, m, dst + f * 1024 + lane * 16, lane, false);
      }
      if (d < 0 || d == rank) {
        gchar* dst = gws + (d < 0 ? OFF_XS + (size_t)((rank * NSTAGE + s) * XU + q) * UNIT
                                  : OFF_WS + (size_t)((rank * NSTAGE + s) * WU + q) * UNIT);
        send_block(p, m, dst + f * 1024 + lane * 16, lane, true);
      }
      if (k == 0) get_call();
      asm volatile("s_waitcnt vmcnt(0)" ::: "memory");
      if (lane == 0) st4(gws + OFF_EF + (size_t)n * 4, tag | info);
      if (lane == 1 && (d < 0 || d == rank))
        st4(gws + (d < 0 ? OFF_XF : OFF_WF) + (size_t)(q * 64 + rank * 8 + s) * 16 + f, tag | F_RAW);
     }
    }
  };

  auto locate = [&](int u, int f, gchar*& dst, gu32*& flag, int& pn, unsigned& so, int& pe) -> bool {
      const int pr = u % 7, d = (rank + 1 + pr) & 7;
      int s, q, rr;
      unit_of(u / 7, pr, s, q, rr);
      gchar* dws;
      asm volatile("s_load_dwordx2 %0, %1, %2\n\ts_waitcnt lgkmcnt(0)"
                   : "=s"(dws) : "s"(P.ws_all), "s"(__builtin_amdgcn_readfirstlane(d * 8)));
      if (rr != 2) {
        const int xu = 2 * q + rr;
        so = (unsigned)OFF_EX + (unsigned)(s * XU + xu) * UNIT;
        dst = dws + OFF_XS + ((size_t)((rank * NSTAGE + s) * XU + xu)) * UNIT;
        pe = s == 0 ? xu : 88 + (s - 1) * 160 + xu;
        pn = s | xu << 3;
      } else {
        so = (unsigned)OFF_EW + (unsigned)((d * NSTAGE + s) * WU + q) * UNIT;
        dst = dws + OFF_WS + ((size_t)((rank * NSTAGE + s) * WU + q)) * UNIT;
        pe = s >= 2 ? 120 + (s - 2) * 160 + q * 7 + pr : s == 1 && q >= 8 ? q * 7 + pr - 24 : (NMIR + 3) * 64;
        pn = s | q << 3 | (d + 1) << 8;
      }
      flag = (gu32*)(dws + OFF_F1 + (size_t)u * F1S) + f;
      const bool tail = u >= NUNIT - RTU;
      if (tail) {
        so = (unsigned)OFF_XS + (unsigned)((rank * NSTAGE + s) * XU + 2 * q + rr) * UNIT;
        flag = (gu32*)(dws + OFF_XF + ((2 * q + rr) * 64 + rank * 8 + s) * 16) + f;
      }
      so = __builtin_amdgcn_readfirstlane(so + f * 4096);
      dst += f * 4096 + lane * 16;
      pn |= f << 12;
      return tail;
  };

  if (sender) {
   if (QX & 8) {
   } else if (wave < NSEND) {
    const __amdgpu_buffer_rsrc_t rs = __builtin_amdgcn_make_buffer_rsrc(myws, 0, 0x7fffffff, 0x00020000);
    unsigned so, info, ia = 0;
    int pn;
    int dbg = 0, pdbg = 0;
    gchar* dst;
    gu32* flag;
    u32x4 va, vb, vc;
    unsigned ve;
    auto draw = [&](int look) -> bool {
      unsigned j = 0;
      if (lane == 0)
        asm volatile("ds_add_rtn_u32 %0, %1, %2\n\ts_waitcnt lgkmcnt(0)" : "=v"(j) : "v"(ctl + C_NEXT * 4), "v"(1u) : "memory");
      j = __builtin_amdgcn_readfirstlane(j);
      const int f = j & 3;
      int u = U0 + (int)(j >> 2) * NSWG + (bk - 26) * 8 + bx;
      if (j >= (unsigned)(S1 * 4)) {
        unsigned e;
        while (((e = __builtin_amdgcn_readfirstlane(lds_ld(ctl + (C_Q + ((j >> 2) & 63)) * 4))) >> 12) != (j >> 2) + 1)
          __builtin_amdgcn_s_sleep(1);
        u = e & 4095;
        if (u >= NUNIT) return false;
      }
      int pe;
      const bool tail = locate(u, f, dst, flag, pn, so, pe);
      dbg = u * 4 + f;
      ia = ldsbase + (pe * 4 + f) * 4;
      asm volatile("" : "+v"(ia));
      info = __builtin_amdgcn_readfirstlane(lds_ld(ia));
      for (; look > 0 && ((info ^ tag) >> 13) != 0u; --look) {
        __builtin_amdgcn_s_sleep(8);
        info = __builtin_amdgcn_readfirstlane(lds_ld(ia));
      }
      if (tail) { pn |= 1 << 30; if (((info ^ tag) >> 13) == 0u) info = tag | F_RAW; }
      if (((info ^ tag) >> 13) == 0u) {
        va = __builtin_amdgcn_raw_buffer_load_b128(rs, lane * 16, so, 17);
        vb = __builtin_amdgcn_raw_buffer_load_b128(rs, lane * 16 + 1024, so, 17);
        vc = __builtin_amdgcn_raw_buffer_load_b128(rs, lane * 16 + 2048, so, 17);
        ve = __builtin_amdgcn_raw_buffer_load_b32(rs, (lane & 15) * 4 + 3072, so, 17);
      }
      return true;
    };
    gu32* pflag = nullptr;
    unsigned pv = 0;
    va = vb = vc = u32x4{0u, 0u, 0u, 0u}; ve = 0; info = 0; so = 0; pn = 0; dst = nullptr; flag = nullptr;
    get_call();
    draw(FWAIT);
    while (true) {
      asm volatile("s_waitcnt vmcnt(0)" : "+v"(va), "+v"(vb), "+v"(vc), "+v"(ve) :: "memory");
      if (pflag != nullptr && lane == 0) __hip_atomic_store(pflag, pv, __ATOMIC_RELAXED, __HIP_MEMORY_SCOPE_SYSTEM);
      if (pflag != nullptr) T2(3, pdbg);
      pdbg = dbg;
      u32x4 ca = va, cb = vb, cc = vc;
      unsigned ce = ve;
      const unsigned cso = so, cia = ia;
      const int cpn = pn;
      gchar* const cdst = dst;
      pflag = flag;
      pv = info;
      const bool more = draw(0);
      {
        if (((pv ^ tag) >> 13) != 0u) {
          pv = __builtin_amdgcn_readfirstlane(lds_ld(cia));
          if (((pv ^ tag) >> 13) == 0u) {
            if (cpn >> 30) pv = tag | F_RAW;
            ca = __builtin_amdgcn_raw_buffer_load_b128(rs, lane * 16, cso, 17);
            cb = __builtin_amdgcn_raw_buffer_load_b128(rs, lane * 16 + 1024, cso, 17);
            cc = __builtin_amdgcn_raw_buffer_load_b128(rs, lane * 16 + 2048, cso, 17);
            ce = __builtin_amdgcn_raw_buffer_load_b32(rs, (lane & 15) * 4 + 3072, cso, 17);
          }
        }
        if (((pv ^ tag) >> 13) != 0u) {
          unsigned p[4][4];
          float m;
          convert(cpn & 7, (cpn >> 3) & 31, ((cpn >> 8) & 15) - 1, (cpn >> 10) & 12, p, m);
          pv = tag | send_block(p, m, cdst, lane, (cpn >> 30) != 0, DSLOW);
        } else {
          st16(cdst, ca);
          asm volatile("s_waitcnt vmcnt(0)" ::: "memory");
          st16(cdst + 1024, cb);
          asm volatile("s_waitcnt vmcnt(0)" ::: "memory");
          st16(cdst + 2048, cc);
          if (pv & F_RAW) st16(cdst + 3072, __builtin_amdgcn_raw_buffer_load_b128(rs, lane * 16 + 3072, cso, 17));
          else if ((pv & (31u << 7)) && lane < MAXESC) st4(cdst - lane * 12 + 3072, ce);
        }
      }
      if (!more) break;
    }
    asm volatile("s_waitcnt vmcnt(0)" ::: "memory");
    if (lane == 0) __hip_atomic_store(pflag, pv, __ATOMIC_RELAXED, __HIP_MEMORY_SCOPE_SYSTEM);
    if (wave < 2) TS(1 + wave);
   } else if (wave < NSEND + NSC) {
    get_call();
    const u64 t0 = __builtin_amdgcn_s_memrealtime();
    if (wave > NSEND) __builtin_amdgcn_s_sleep(LOS);
    int c = wave - NSEND < NLO ? 0 : NMIR;
    for (unsigned spins = 1; c < NMIR; ++spins) {
      {
        unsigned keep;
        asm volatile("s_mov_b32 %0, m0\n\ts_mov_b32 m0, %2\n\ts_nop 0\n\t"
                     "global_load_lds_dwordx4 %1, off sc0 sc1\n\t"
                     "global_load_lds_dwordx4 %1, off offset:1024 sc0 sc1\n\t"
                     "s_mov_b32 m0, %0"
                     : "=&s"(keep) : "v"(myws + OFF_EF + c * 1024 + lane * 16), "s"(ldsbase + c * 1024) : "memory");
      }
      asm volatile("s_waitcnt vmcnt(0)" ::: "memory");
      for (int i = 0; i < LW && c < NMIR; ++i, ++c) {
        u32x4 v;
        asm volatile("ds_read_b128 %0, %1\n\ts_waitcnt lgkmcnt(0)" : "=v"(v) : "v"(ldsbase + c * 1024 + lane * 16) : "memory");
        if (!__all((((v[0] ^ tag) | (v[1] ^ tag) | (v[2] ^ tag) | (v[3] ^ tag)) >> 13) == 0u)) break;
      }
      if ((spins & 15) == 0 &&
          (__hip_atomic_load(tmo, __ATOMIC_RELAXED, __HIP_MEMORY_SCOPE_AGENT) == call ||
           __builtin_amdgcn_s_memrealtime() - t0 > DEADLINE_TICKS)) {
        if (lane == 0) __hip_atomic_store(tmo, call, __ATOMIC_RELAXED, __HIP_MEMORY_SCOPE_AGENT);
        break;
      }
      __builtin_amdgcn_s_sleep(SCS);
    }
    if (wave == NSEND) TS(33);
   } else if (wave == NWAVE - 1) {
    get_call();
    unsigned* const sq = reinterpret_cast<unsigned*>(myws + OFF_SQ);
    const unsigned qbase = (call - 1) * NDRAW - U0 - S1 * NSWG;
    for (unsigned k = S1;; ++k) {
      while ((lds_ld(ctl + C_NEXT * 4) >> 2) + TKD < k) __builtin_amdgcn_s_sleep(2);
      unsigned t = 0;
      if (lane == 0) t = __hip_atomic_fetch_add(sq, 1u, __ATOMIC_RELAXED, __HIP_MEMORY_SCOPE_AGENT);
      const unsigned u = __builtin_amdgcn_readfirstlane(t) - qbase;
      if (u >= (unsigned)NUNIT) {
        if (lane < 8) lds_st(ctl + (C_Q + ((k + lane) & 63)) * 4, ((k + lane + 1) << 12) | 4095u);
        break;
      }
      if (lane == 0) lds_st(ctl + (C_Q + (k & 63)) * 4, ((k + 1) << 12) | u);
    }
   }
  } else if (wave < 8) {
    if (CPK) pack_items(wave * NCW + bk * 8 + bx, 1);
    if (wave < KW && !(QX & 8)) {
      const int ba = wave * NCW + bk * 8 + bx, bb = ba + KW * NCW;
      gchar *da, *db;
      gu32 *fa, *fb;
      int pa, pb, pe;
      unsigned so, p[4][4];
      float m;
      f32x4 ra[8], rb[8];
      locate(ba >> 2, ba & 3, da, fa, pa, so, pe);
      locate(bb >> 2, bb & 3, db, fb, pb, so, pe);
      T2(2, ba);
      T2(2, bb);
      fetch(pa & 7, (pa >> 3) & 31, ((pa >> 8) & 15) - 1, (pa >> 10) & 12, ra);
      fetch(pb & 7, (pb >> 3) & 31, ((pb >> 8) & 15) - 1, (pb >> 10) & 12, rb);
      round16(((pa >> 8) & 15) - 1, ra, p, m);
      const unsigned ia = send_block(p, m, da, lane, false, KSLOW);
      round16(((pb >> 8) & 15) - 1, rb, p, m);
      get_call();
      asm volatile("s_waitcnt vmcnt(0)" ::: "memory");
      if (lane == 0) __hip_atomic_store(fa, tag | ia, __ATOMIC_RELAXED, __HIP_MEMORY_SCOPE_SYSTEM);
      T2(3, ba);
      const unsigned ib = send_block(p, m, db, lane, false, KSLOW);
      asm volatile("s_waitcnt vmcnt(0)" ::: "memory");
      if (lane == 0) __hip_atomic_store(fb, tag | ib, __ATOMIC_RELAXED, __HIP_MEMORY_SCOPE_SYSTEM);
      T2(3, bb);
    }
    if (wave == 0) TS(30);
#pragma unroll
    for (int a = 0; a < 5; ++a)
#pragma unroll
      for (int b = 0; b < 4; ++b) acc[a][b] = f32x4{0.f, 0.f, 0.f, 0.f};
    const int xoff = (wm * 5) * 1024 + lane * 16;
    const int woff = 20480 + (wn * 4) * 1024 + lane * 16;
    const u64 c0 = CYC(), r0 = __builtin_amdgcn_s_memrealtime();
    {
      unsigned vx = ldsbase + xoff, vw = ldsbase + woff, vf = ctl + C_FULL * 4, vc = ctl + wave * 4, vt, vu;
      unsigned sfl, sdn, sof, sd, sdb = 0;
      asm volatile(
          "s_mov_b32 %[sdn], 0\n\t"
          "s_mov_b32 %[sof], 0\n\t"
          "s_nop 4\n\t"
          FETCH(RB0, RB1, RB2, RA0, RA1, RA2, RA3)
          "1:\n\t"
          "s_cmp_lg_u32 %[sfl], 0\n\t"
          "s_cbranch_scc0 3f\n\t"
          BODY(RA0, RA1, RA2, RA3, RB0, RB1, RB2, RB3, PFY, PFN)
          "2:\n\t"
          "s_cmp_lg_u32 %[sfl], 0\n\t"
          "s_cbranch_scc0 4f\n\t"
          BODY(RB0, RB1, RB2, RB3, RA0, RA1, RA2, RA3, PFY, PFN)
          "s_branch 1b\n\t"
          "3:\n\t"
          BODY(RA0, RA1, RA2, RA3, RB0, RB1, RB2, RB3, PFN, PFY)
          FETCH(RA0, RA1, RA2, RB0, RB1, RB2, RB3)
          "s_branch 2b\n\t"
          "4:\n\t"
          BODY(RB0, RB1, RB2, RB3, RA0, RA1, RA2, RA3, PFN, PFY)
          "s_cmp_eq_u32 %[sdn], %[ns]\n\t"
          "s_cbranch_scc1 5f\n\t"
          FETCH(RB0, RB1, RB2, RA0, RA1, RA2, RA3)
          "s_branch 1b\n\t"
          "5:\n\t"
          "s_setprio 0\n\t"
          "s_nop 15\n\t"
          "s_nop 15\n\t"
          :
          "+{v[40:43]}"(acc[0][0]), "+{v[44:47]}"(acc[0][1]), "+{v[48:51]}"(acc[0][2]), "+{v[52:55]}"(acc[0][3]),
          "+{v[56:59]}"(acc[1][0]), "+{v[60:63]}"(acc[1][1]), "+{v[64:67]}"(acc[1][2]), "+{v[68:71]}"(acc[1][3]),
          "+{v[72:75]}"(acc[2][0]), "+{v[76:79]}"(acc[2][1]), "+{v[80:83]}"(acc[2][2]), "+{v[84:87]}"(acc[2][3]),
          "+{v[88:91]}"(acc[3][0]), "+{v[92:95]}"(acc[3][1]), "+{v[96:99]}"(acc[3][2]), "+{v[100:103]}"(acc[3][3]),
          "+{v[104:107]}"(acc[4][0]), "+{v[108:111]}"(acc[4][1]), "+{v[112:115]}"(acc[4][2]), "+{v[116:119]}"(acc[4][3]),
          [vx] "+v"(vx), [vw] "+v"(vw), [vf] "+v"(vf), [vt] "=&v"(vt), [vu] "=&v"(vu), [sfl] "=&s"(sfl), [sdn] "=&s"(sdn),
          [sof] "=&s"(sof), [sd] "=&s"(sd), [sdb] "+s"(sdb)
          : [vc] "v"(vc), [dn] "n"(C_DONE), [dm] "n"(C_DONE + 8), [dp] "n"((C_DONE + 4) * 4), [hs] "n"(HS), [hl] "n"((NH - 1) * HS), [hw] "s"(-(NH - 1) * HS),
            [ns] "n"(2 * NSTEP)
          : "memory", "scc", "v0", "v1", "v2", "v3", "v4", "v5", "v6", "v7", "v8", "v9", "v10", "v11", "v12", "v13", "v14", "v15", "v16", "v17", "v18", "v19", "v20", "v21", "v22", "v23", "v24", "v25", "v26", "v27", "v28", "v29", "v30", "v31", "v32", "v33", "v34", "v35", "v36", "v37", "v38", "v39");
      TV(48 + wave, sdb);
    }
    if (wave == 0) { TV(40, CYC() - c0); TV(41, __builtin_amdgcn_s_memrealtime() - r0); }
    float m = 0.f;
#pragma unroll
    for (int a = 0; a < 5; ++a) {
      if (ti * TM + wm * 80 + a * 16 >= 4096) continue;
#pragma unroll
      for (int b = 0; b < 4; ++b)
#pragma unroll
        for (int j = 0; j < 4; ++j) m = fmaxf(m, fabsf(acc[a][b][j]));
    }
#pragma unroll
    for (int o = 32; o >= 1; o >>= 1) m = fmaxf(m, __shfl_xor(m, o));
    if (lane == 0) lds_st(ctl + (C_WMAX + wave) * 4, __float_as_uint(m));
    if (wave == 0) TS(12);
  } else if (wave < 10) {
    const int L = wave - 8;
    __builtin_amdgcn_s_setprio(LPRIO);
    get_call();
    {
      const u64 t0 = __builtin_amdgcn_s_memrealtime();
      while (__builtin_amdgcn_s_memrealtime() - t0 < DLY * 100ull) __builtin_amdgcn_s_sleep(32);
    }
    const char* const gx0 = myws + OFF_XS + L * 8192 + lane * 16;
    const char* const gw0 = myws + OFF_WS + (size_t)tj * UNIT + L * 8192 + lane * 16;
    int slot = L;
    const unsigned dna = ctl + (C_DONE + (lane & 7)) * 4;
    u64 rm = 0, avail = 0, left = ~0ull;
    unsigned dv = 0;
    for (int t = 0; t < NSTEP; ++t) {
      const int need = 2 * t + L - NH + 1;
      while (avail == 0 || !__all((int)dv >= need)) {
        __builtin_amdgcn_s_sleep(1);
        asm volatile("ds_read_b64 %0, %2\n\tds_read_b32 %1, %3\n\ts_waitcnt lgkmcnt(0)"
                     : "=&v"(rm), "=&v"(dv) : "v"(ctl + C_Q * 4), "v"(dna) : "memory");
        avail = left & (((u64)(unsigned)__builtin_amdgcn_readfirstlane((int)(rm >> 32)) << 32) |
                        (u64)(unsigned)__builtin_amdgcn_readfirstlane((int)rm));
      }
      const int c = __builtin_ctzll(avail);
      left &= ~(1ull << c);
      const int bit = ((rank - (c & 7)) & 7) * 8 + (c >> 3);
      asm volatile("s_waitcnt vmcnt(29)" ::: "memory");
      const unsigned l = ldsbase + slot * HS;
#pragma unroll
      for (int i = 0; i < 4; ++i) {
        const int hu = 5 * ti + i;
        glds4k(gx0 + ((size_t)(bit * XU + (hu >> 1))) * UNIT + (hu & 1) * 4096, l + i * 4096);
      }
      {
        const int hu = min(5 * ti + 4, 63);
        const char* g0 = gx0 + ((size_t)(bit * XU + (hu >> 1))) * UNIT + (hu & 1) * 4096;
        const char* g1 = gw0 + (size_t)(bit * WU) * UNIT;
        unsigned keep;
        asm volatile(
            "ds_read_b64 %0, %3\n\tds_read_b32 %1, %4\n\t"
            "s_mov_b32 %2, m0\n\ts_mov_b32 m0, %9\n\ts_nop 0\n\t"
            "global_load_lds_dwordx4 %5, off" LDPOL "\n\t"
            "global_load_lds_dwordx4 %5, off offset:1024" LDPOL "\n\t"
            "global_load_lds_dwordx4 %5, off offset:2048" LDPOL "\n\t"
            "global_load_lds_dwordx4 %5, off offset:3072" LDPOL "\n\t"
            "s_add_u32 m0, m0, 0x1000\n\ts_nop 0\n\t"
            "global_load_lds_dwordx4 %6, off" LDPOL "\n\t"
            "global_load_lds_dwordx4 %6, off offset:1024" LDPOL "\n\t"
            "global_load_lds_dwordx4 %6, off offset:2048" LDPOL "\n\t"
            "global_load_lds_dwordx4 %6, off offset:3072" LDPOL "\n\t"
            "s_add_u32 m0, m0, 0x1000\n\ts_nop 0\n\t"
            "global_load_lds_dwordx4 %7, off" LDPOL "\n\t"
            "global_load_lds_dwordx4 %7, off offset:1024" LDPOL "\n\t"
            "global_load_lds_dwordx4 %7, off offset:2048" LDPOL "\n\t"
            "global_load_lds_dwordx4 %7, off offset:3072" LDPOL "\n\t"
            "s_mov_b32 m0, %10\n\ts_mov_b64 exec, 1\n\t"
            "global_load_lds_dword %8, off sc0 sc1\n\t"
            "s_mov_b64 exec, -1\n\ts_mov_b32 m0, %2\n\t"
            "s_waitcnt lgkmcnt(0)"
            : "=&v"(rm), "=&v"(dv), "=&s"(keep)
            : "v"(ctl + C_Q * 4), "v"(dna), "v"(g0), "v"(g1), "v"(g1 + 4096), "v"(myws + OFF_TICKET),
              "s"(l + 16384), "s"(ctl + (C_FULL + 2 * t + L) * 4)
            : "memory", "scc");
        avail = left & (((u64)(unsigned)__builtin_amdgcn_readfirstlane((int)(rm >> 32)) << 32) |
                        (u64)(unsigned)__builtin_amdgcn_readfirstlane((int)rm));
      }
      slot = slot + 2 >= NH ? slot + 2 - NH : slot + 2;
    }
    asm volatile("s_waitcnt vmcnt(0)" ::: "memory");
    if (L == 0) TS(11);
  } else if (wave == 10) {
    get_call();
    const unsigned want = tag | F_RAW;
    const unsigned m2 = ti == NTI - 1 ? 0u : ~0u;
    const int st = ((rank - (lane & 7)) & 7) * 8 + (lane >> 3);
    const unsigned* pf0 = reinterpret_cast<const unsigned*>(myws + OFF_XF) + (((5 * ti) >> 1) * 64 + st) * SU;
    const unsigned* pf2 = reinterpret_cast<const unsigned*>(myws + OFF_WF) + (tj * 64 + st) * SU;
    u64 seen = 0;
    unsigned cnt = 0, spins = 0;
    bool dead = false;
    const u64 t0 = __builtin_amdgcn_s_memrealtime();
    while (cnt < (unsigned)NSTEP) {
      u32x4 f0, f1, f2, f3;
      asm volatile("global_load_dwordx4 %0, %4, off sc0 sc1\n\tglobal_load_dwordx4 %1, %4, off offset:1024 sc0 sc1\n\t"
                   "global_load_dwordx4 %2, %4, off offset:2048 sc0 sc1\n\t"
                   "global_load_dwordx4 %3, %5, off sc0 sc1\n\ts_waitcnt vmcnt(0)"
                   : "=&v"(f0), "=&v"(f1), "=&v"(f2), "=&v"(f3) : "v"(pf0), "v"(pf2) : "memory");
      const u64 nm = __ballot((QX & 1) || dead || plain16(f0, f1, f2, f3, want, m2)) & ~seen;
      if (nm) {
        if (lane == 0) asm volatile("ds_write_b64 %0, %1" :: "v"(ctl + C_Q * 4), "v"(seen | nm) : "memory");
        cnt += (unsigned)__builtin_popcountll(nm);
        seen |= nm;
      } else if ((++spins & 15) == 0 &&
                 (__hip_atomic_load(tmo, __ATOMIC_RELAXED, __HIP_MEMORY_SCOPE_AGENT) == call ||
                  __builtin_amdgcn_s_memrealtime() - t0 > DEADLINE_TICKS)) {
        if (lane == 0) __hip_atomic_store(tmo, call, __ATOMIC_RELAXED, __HIP_MEMORY_SCOPE_AGENT);
        dead = true;
      }
      __builtin_amdgcn_s_sleep(PSLEEP);
    }
    TS(32);
  } else if (wave < W_DEC + NDEC) {
    __builtin_amdgcn_s_setprio(DPRIO);
    get_call();
    const __amdgpu_buffer_rsrc_t rs = __builtin_amdgcn_make_buffer_rsrc(myws, 0, 0x7fffffff, 0x00020000);
    const u64 t0 = __builtin_amdgcn_s_memrealtime();
    bool dead = false;
    for (int n = (wave - W_DEC) * NCW + bk * 8 + bx; n < NIN - RTU && !dead && !(QX & 16); n += NDEC * NCW) {
      const int pr = n % 7;
      int s, q, rr;
      unit_of(n / 7, pr, s, q, rr);
      const int st = ((rank + 7 - pr) & 7) * 8 + s;
      unsigned so, fo;
      if (rr != 2) {
        so = (unsigned)OFF_XS + (unsigned)(st * XU + 2 * q + rr) * UNIT;
        fo = (unsigned)OFF_XF + ((2 * q + rr) * 64 + st) * 16;
      } else {
        so = (unsigned)OFF_WS + (unsigned)(st * WU + q) * UNIT;
        fo = (unsigned)OFF_WF + (q * 64 + st) * 16;
      }
      so = __builtin_amdgcn_readfirstlane(so);
      fo = __builtin_amdgcn_readfirstlane(fo);
      unsigned f[4];
      for (unsigned spins = 1;; ++spins) {
        asm volatile("" ::: "memory");
        const u32x4 fv = __builtin_amdgcn_raw_buffer_load_b128(rs, 0, (unsigned)OFF_F1 + n * F1S, 17);
#pragma unroll
        for (int k = 0; k < 4; ++k) f[k] = __builtin_amdgcn_readfirstlane(fv[k]);
        if ((((f[0] ^ tag) | (f[1] ^ tag) | (f[2] ^ tag) | (f[3] ^ tag)) >> 13) == 0u) break;
        if ((spins & 15) == 0 &&
            (__hip_atomic_load(tmo, __ATOMIC_RELAXED, __HIP_MEMORY_SCOPE_AGENT) == call ||
             __builtin_amdgcn_s_memrealtime() - t0 > DEADLINE_TICKS)) {
          if (lane == 0) __hip_atomic_store(tmo, call, __ATOMIC_RELAXED, __HIP_MEMORY_SCOPE_AGENT);
          dead = true;
          break;
        }
        __builtin_amdgcn_s_sleep(DSLEEP);
      }
      if (dead) continue;
      T2(0, n);
      u32x4 a[4], b[4], c[4];
      unsigned e[4];
#pragma unroll
      for (int k = 0; k < 4; ++k) {
        a[k] = __builtin_amdgcn_raw_buffer_load_b128(rs, lane * 16 + k * 4096, so, 17);
        b[k] = __builtin_amdgcn_raw_buffer_load_b128(rs, lane * 16 + k * 4096 + 1024, so, 17);
        c[k] = __builtin_amdgcn_raw_buffer_load_b128(rs, lane * 16 + k * 4096 + 2048, so, 17);
        e[k] = __builtin_amdgcn_raw_buffer_load_b32(rs, (lane & 15) * 4 + k * 4096 + 3072, so, 17);
      }
#pragma unroll
      for (int k = 0; k < 4; ++k) {
        if (f[k] & F_RAW) continue;
        const unsigned ne = (f[k] >> 7) & 31u;
        const unsigned base4 = (f[k] & 0x7fu) * 0x01010101u;
        u32x4 o[4];
        o[0] = unpack8(a[k][0], a[k][1], c[k][0], base4);
        o[1] = unpack8(a[k][2], a[k][3], c[k][1], base4);
        o[2] = unpack8(b[k][0], b[k][1], c[k][2], base4);
        o[3] = unpack8(b[k][2], b[k][3], c[k][3], base4);
        for (unsigned i = 0; i < ne && i < (unsigned)MAXESC; ++i) {
          const unsigned ev = (unsigned)__builtin_amdgcn_readlane((int)e[k], (int)i);
          const unsigned sh = (ev & 1u) * 16 + 8, r = ((ev >> 7) & 12u) | ((ev >> 1) & 3u);
          const unsigned vm = (unsigned)lane == ((ev >> 3) & 63u) ? 0xffu << sh : 0u;
          const unsigned vv = ((ev >> 16) & 0xffu) << sh;
#pragma unroll
          for (int kk = 0; kk < 16; ++kk) {
            const unsigned mk = r == (unsigned)kk ? vm : 0u;
            o[kk >> 2][kk & 3] = (o[kk >> 2][kk & 3] & ~mk) | (vv & mk);
          }
        }
#pragma unroll
        for (int j = 0; j < 4; ++j)
          __builtin_amdgcn_raw_buffer_store_b128(o[j], rs, lane * 16 + k * 4096 + j * 1024, so, 17);
      }
      asm volatile("s_waitcnt vmcnt(0)" ::: "memory");
      if (lane == 0) {
        const unsigned w = tag | F_RAW;
        __builtin_amdgcn_raw_buffer_store_b128(u32x4{w, w, w, w}, rs, 0, fo, 17);
      }
      T2(1, n);
    }
    if (wave == W_DEC) TS(19);
  } else {
    if (!(QX & 32)) pack_items((wave - W_DEC - NDEC + 8 * CPK) * NCW + bk * 8 + bx, NROUND);
    if (wave == NWAVE - 1) TS(31);
  }

  asm volatile("s_waitcnt lgkmcnt(0)" ::: "memory");
  __syncthreads();
  get_call();

  if (wave == 0) {
    TS(13);
    float m = __uint_as_float(lds_ld(ctl + (C_WMAX + (lane & 7)) * 4));
#pragma unroll
    for (int o = 4; o >= 1; o >>= 1) m = fmaxf(m, __shfl_xor(m, o));
    unsigned last = 0;
    if (lane == 0) {
      u64 v = ((u64)call << 32) | (u64)__float_as_uint(m);
      u64 old = __hip_atomic_fetch_max(reinterpret_cast<u64*>(myws + OFF_AMAX), v, __ATOMIC_RELAXED,
                                       __HIP_MEMORY_SCOPE_AGENT);
      asm volatile("s_waitcnt vmcnt(0)" :: "v"(old) : "memory");
      unsigned tk = __hip_atomic_fetch_add(reinterpret_cast<unsigned*>(myws + OFF_ARRIVE), 1u,
                                           __ATOMIC_RELAXED, __HIP_MEMORY_SCOPE_AGENT);
      last = (tk == call * NWG - 1) ? 1u : 0u;
    }
    last = __builtin_amdgcn_readfirstlane(last);
    if (last) {
      u64 v = __hip_atomic_load(reinterpret_cast<u64*>(myws + OFF_AMAX), __ATOMIC_RELAXED,
                                __HIP_MEMORY_SCOPE_AGENT);
      if (lane < NR) {
        gu64* dstp = (gu64*)((gchar*)(P.ws_all[lane]) + OFF_SLOTS) + rank;
        __hip_atomic_store(dstp, v, __ATOMIC_RELAXED, __HIP_MEMORY_SCOPE_SYSTEM);
      }
    }
  }
  if (sender) return;
  if (wave == 0) {
    const u64* slots = reinterpret_cast<const u64*>(myws + OFF_SLOTS) + (lane & 7);
    const u64 t0 = __builtin_amdgcn_s_memrealtime();
    u64 v;
    while (true) {
      v = __hip_atomic_load(slots, __ATOMIC_RELAXED, __HIP_MEMORY_SCOPE_SYSTEM);
      if (__all((unsigned)(v >> 32) >= call)) break;
      if (__hip_atomic_load(tmo, __ATOMIC_RELAXED, __HIP_MEMORY_SCOPE_AGENT) == call ||
          __builtin_amdgcn_s_memrealtime() - t0 > DEADLINE_TICKS) {
        if (lane == 0) __hip_atomic_store(tmo, call, __ATOMIC_RELAXED, __HIP_MEMORY_SCOPE_AGENT);
        break;
      }
      __builtin_amdgcn_s_sleep(1);
    }
    float g = __uint_as_float((unsigned)v);
#pragma unroll
    for (int o = 4; o >= 1; o >>= 1) g = fmaxf(g, __shfl_xor(g, o));
    unsigned dead = (__hip_atomic_load(tmo, __ATOMIC_RELAXED, __HIP_MEMORY_SCOPE_AGENT) == call) ? 1u : 0u;
    if (lane == 0) { lds_st(ctl + C_GMAX * 4, __float_as_uint(g)); lds_st(ctl + C_DEAD * 4, dead); }
  }
  const u64 outp = (u64)(uintptr_t)P.out_all[lane & 7];
  asm volatile("s_waitcnt lgkmcnt(0)" ::: "memory");
  __syncthreads();

  if (wave == 0) TS(14);
  if (wave < 8) {
    const float amax = __uint_as_float(lds_ld(ctl + C_GMAX * 4));
    const bool dead = lds_ld(ctl + C_DEAD * 4) != 0;
    const float scale = amax / 127.0f;
    const float inv = scale > 0.f ? 1.0f / scale : 0.f;
    const int col = rank * NLOC + tj * 128 + wn * 64 + (lane >> 4) * 16;
#pragma unroll
    for (int mt = 0; mt < 5; ++mt) {
      const int row0 = ti * TM + wm * 80 + mt * 16;
      if (row0 >= 4096) continue;
      const int owner = row0 >> 9;
      gchar* ob = (gchar*)(((u64)(unsigned)__builtin_amdgcn_readlane((int)(outp >> 32), owner) << 32) |
                           (unsigned)__builtin_amdgcn_readlane((int)outp, owner));
      u32x4 o;
#pragma unroll
      for (int j = 0; j < 4; ++j) {
        unsigned pk = 0;
#pragma unroll
        for (int nt = 0; nt < 4; ++nt) {
          float q = rintf(acc[mt][nt][j] * inv);
          q = fminf(fmaxf(q, -127.f), 127.f);
          int qi = dead ? -128 : (int)q;
          pk |= ((unsigned)qi & 0xffu) << (8 * nt);
        }
        o[j] = pk;
      }
      st16(ob + (size_t)((row0 & 511) + (lane & 15)) * NTOT + col, o);
    }
    if (wave == 0) TS(17);
    asm volatile("s_waitcnt vmcnt(0)" ::: "memory");
    if (wave == 0) TS(18);
  }
  asm volatile("s_waitcnt lgkmcnt(0)" ::: "memory");
  __syncthreads();
  if (wave == 0) {
    TS(15);
    {
      const int o0 = (ti * TM) >> 9, o1 = min(ti * TM + TM - 1, 4095) >> 9;
      if (lane == 0 || (lane == 1 && o1 != o0)) {
        gu32* f = (gu32*)((gchar*)(P.ws_all[(lane ? o1 : o0) & 7]) + OFF_OUTF) + (rank * NTI + ti) * 16 + tj;
        __hip_atomic_store(f, call, __ATOMIC_RELAXED, __HIP_MEMORY_SCOPE_SYSTEM);
      }
    }
    if (blockIdx.x == 48) {
      const int tlo = (rank * MSH) / TM, thi = (rank * MSH + MSH - 1) / TM;
      const unsigned* of = reinterpret_cast<const unsigned*>(myws + OFF_OUTF);
      const u64 t0 = __builtin_amdgcn_s_memrealtime();
      while (true) {
        bool ok = true;
#pragma unroll
        for (int j = 0; j < 6; ++j) {
          const int idx = lane + 64 * j, sr = idx / 48, rem = idx - sr * 48;
          const int tt = min(tlo + (rem >> 4), thi);
          ok &= __hip_atomic_load(of + (sr * NTI + tt) * 16 + (rem & 15), __ATOMIC_RELAXED, __HIP_MEMORY_SCOPE_SYSTEM) >= call;
        }
        if (__all(ok)) break;
        if (__hip_atomic_load(tmo, __ATOMIC_RELAXED, __HIP_MEMORY_SCOPE_AGENT) == call ||
            __builtin_amdgcn_s_memrealtime() - t0 > DEADLINE_TICKS) {
          if (lane == 0) {
            __hip_atomic_store(tmo, call, __ATOMIC_RELAXED, __HIP_MEMORY_SCOPE_AGENT);
            ((gchar*)(P.out_all[rank]))[rank * NLOC] = (char)-128;
          }
          break;
        }
        __builtin_amdgcn_s_sleep(1);
      }
      __builtin_amdgcn_fence(__ATOMIC_ACQUIRE, "");
      TS(16);
    }
  }
}

extern "C" void kernel_launch(int rank, int nranks, ncclComm_t comm, hipStream_t stream,
                              void* const* d_in, const int* in_sizes, int n_in, void* d_out,
                              int out_size, const int* in_dtypes, int out_dtype,
                              void* const* d_out_all, void* const* d_ws_all, size_t ws_bytes) {
  if (nranks != NR || ws_bytes < WS_NEED || n_in < 2) {
    fprintf(stderr, "unexpected configuration\n");
    return;
  }
  Params P;
  P.x = static_cast<const float*>(d_in[0]);
  P.w = static_cast<const float*>(d_in[1]);
  P.out_all = d_out_all;
  P.ws_all = d_ws_all;
  P.rank = rank;
  hipLaunchKernelGGL(fused_gemm_q8, dim3(NWG), dim3(NTHREADS), 0, stream, P);
  HIP_CHECK(hipGetLastError());
}
```

```cpp
#include <hip/hip_runtime.h>
#include <rccl/rccl.h>
#include <cstdio>
#include <cstdint>
#include <type_traits>

#define HIP_CHECK(expr) do { hipError_t _e = (expr); if (_e != hipSuccess) { \
    fprintf(stderr, "HIP error %s at %s:%d\n", hipGetErrorString(_e), __FILE__, __LINE__); \
    return; } } while (0)

typedef __attribute__((ext_vector_type(8))) short bf16x8;
typedef __attribute__((ext_vector_type(4))) float f32x4;
typedef __attribute__((ext_vector_type(4))) unsigned u32x4;
typedef unsigned long long u64;
#define GAS __attribute__((address_space(1)))
typedef GAS char gchar;
typedef GAS unsigned gu32;
typedef GAS unsigned long long gu64;
typedef GAS u32x4 gu32x4;

namespace {
constexpr int NR = 8;
constexpr int KSH = 512;
constexpr int NTOT = 16384;
constexpr int NLOC = 2048;
constexpr int MSH = 512;
constexpr int NSTAGE = 8;
constexpr int XU = 32;
constexpr int WU = 16;
constexpr int UNIT = 16384;
constexpr int NWG = 256;
constexpr int SU = 4;
constexpr int NUNIT = 7 * NSTAGE * (XU + WU);
#define THR 2
#define PSLEEP 16
#define SCS 32
#define F1S 128
#define MAPB 0
#define DLY 24
#define DSLOW 0
#define LOS 32
#define LW 2
#define NLO 1
#define PRO 1
#define PKD 0
#define KSLOW 1
#define FWAIT 40
#define RECHK 1
#define OUTL 0
#define OTH 1
#define ASL 4
#define SEQ 5
#define WROT 1
#define UP 2464
#define EARLY 0
#define WFAST 44
#define PS2 2
#define WMASK 1
#define NL 448
#define LSLEEP 4
#define NCOL 2
#define CSL 1
#define RS 1
#define PPRIO 0
#define TKT 5
#define BAL 1
#define LPRIO 3
#define PFD 0
#define PFL 3
#define TLO 56
#define THI 64
#define PIPE 1
#define QX 0
#define CPRIO 0
#define DSLEEP 16
#define DPRIO 2
#define LP 0
#define LDPOL " sc0 sc1"
constexpr int NSTEP = 64;
constexpr int TM = 320;
constexpr int NTI = 13;
constexpr int NCW = NTI * 16;
constexpr int NH = 5;
constexpr int HS = 7 * 4096;
#define NSEND 13
constexpr int NSWG = NWG - NCW;
constexpr int NSC = 15 - NSEND < 4 ? 15 - NSEND : 4;
#define S1V 4
#define RTU 224
#define TKD 1
constexpr int S1 = S1V;
constexpr int NT0 = NUNIT - RTU;
constexpr int KW = 3;
constexpr int U0 = NCW * KW * 2 / 4;
constexpr int NDRAW = NUNIT - U0 - S1 * NSWG + NSWG;
static_assert(UP % 7 == 0 && UP >= U0 + S1 * NSWG && UP <= NT0, "late units");
static_assert(NT0 - NL >= 7 * (16 + 48 * 5), "the steps of stage 4 do not depend on the last units");
static_assert(NSEND + NSC < 16 && RTU == 224 && U0 >= 288, "sender split");
#define NWAVE 16
#define NDECV 2
constexpr int NDEC = NDECV;
constexpr int W_DEC = 11;
constexpr int NTHREADS = 64 * NWAVE;
constexpr unsigned F_RAW = 1u << 12;
constexpr int MAXESC = 16;
constexpr int NPK = NWAVE - W_DEC - NDEC;
constexpr int NPACK = (8 * XU + 56 + 6 * 112 + NSTAGE * WU) * 4;
constexpr int NMIR = 17;
#define CPK 0
constexpr int PR0 = NCW * (8 * CPK + NPK);
constexpr int PRN = NCW * NPK;
constexpr int NROUND = 1 + (NPACK - PR0 + PRN - 1) / PRN;
static_assert(CPK == 0 && RS >= 1 && RS * PRN <= NPACK && PRN % 48 == 0, "packing");
constexpr int NIN = 7 * 384;

constexpr size_t OFF_TICKET = 0;
constexpr size_t OFF_TMO = 128;
constexpr size_t OFF_SQ = 2048;
constexpr size_t OFF_PQ = 2304;
constexpr size_t OFF_XF = 4096;
constexpr size_t OFF_WF = OFF_XF + XU * 64 * SU * 4;
constexpr size_t OFF_OUTF = 65536;
constexpr size_t OFF_REP = 73728;
constexpr size_t OFF_SLOTS = 77824;
constexpr size_t OFF_EF = 81920;
constexpr size_t OFF_F1 = 131072;
constexpr size_t OFF_XS = 1u << 20;
constexpr size_t OFF_WS = OFF_XS + (size_t)NR * NSTAGE * XU * UNIT;
constexpr size_t OFF_EX = OFF_WS + (size_t)NR * NSTAGE * WU * UNIT;
constexpr size_t OFF_EW = OFF_EX + (size_t)NSTAGE * XU * UNIT;
constexpr size_t WS_NEED = OFF_EW + (size_t)NR * NSTAGE * WU * UNIT;

constexpr u64 DEADLINE_TICKS = 40000000ull;

constexpr int C_CALL = 0, C_FULL = 128, C_DONE = 8  , C_WMAX = 100, C_GMAX = 24, C_DEAD = 25, C_NEXT = 26, C_Q = 32;
constexpr int LDS_CTL = NH * HS;
constexpr int LDS_BYTES = LDS_CTL + 512 + 1024;
}

#define T2(a, i) do {} while (0)
#define TS(i) do {} while (0)
#define TV(i, v) do {} while (0)
#define CYC() 0ull

struct Params {
  const float* x;
  const float* w;
  void* const* out_all;
  void* const* ws_all;
  int rank;
};

__device__ __forceinline__ unsigned pk_bf16(float a, float b) {
  typedef __attribute__((ext_vector_type(2))) float f2;
  typedef __attribute__((ext_vector_type(2))) __bf16 b2;
  f2 v = {a, b};
  b2 r = __builtin_convertvector(v, b2);
  return __builtin_bit_cast(unsigned, r);
}

__device__ __forceinline__ unsigned lds_ld(unsigned addr) {
  unsigned v;
  asm volatile("ds_read_b32 %0, %1\n\ts_waitcnt lgkmcnt(0)" : "=v"(v) : "v"(addr) : "memory");
  return v;
}
__device__ __forceinline__ void glds4k(const void* g, unsigned ldsaddr) {
  unsigned keep;
  asm volatile(
      "s_mov_b32 %0, m0\n\ts_mov_b32 m0, %2\n\ts_nop 0\n\t"
      "global_load_lds_dwordx4 %1, off" LDPOL "\n\t"
      "global_load_lds_dwordx4 %1, off offset:1024" LDPOL "\n\t"
      "global_load_lds_dwordx4 %1, off offset:2048" LDPOL "\n\t"
      "global_load_lds_dwordx4 %1, off offset:3072" LDPOL "\n\t"
      "s_mov_b32 m0, %0"
      : "=&s"(keep) : "v"(g), "s"(ldsaddr) : "memory");
}
__device__ __forceinline__ bool plain16(u32x4 a, u32x4 b, u32x4 c, u32x4 d, unsigned want, unsigned m2) {
  unsigned x = 0, y = 0;
#pragma unroll
  for (int j = 0; j < 4; ++j) { x |= (a[j] ^ want) | (b[j] ^ want) | (d[j] ^ want); y |= c[j] ^ want; }
  return ((x | (y & m2)) >> 12) == 0u;
}
__device__ __forceinline__ void unit_of(int ii, int pr, int& s, int& q, int& rr) {
  const int rot = (pr * 16 + 3) / 7;
  if (ii < 16) { s = 0; q = ii; rr = 2; }
  else if (ii < 352) {
    const int i2 = ii - 16, it = i2 % 48;
    s = i2 / 48; q = it / 3; rr = it - 3 * q;
    if (s >= SEQ) {
      q = it < 32 ? it >> 1 : (it + (WROT ? rot : 0)) & 15;
      rr = it < 32 ? it & 1 : 2;
    }
    if (rr == 2) s += 1; else q = (q + rot) & 15;
  } else { s = 7; q = (((ii - 352) >> 1) + rot) & 15; rr = (ii - 352) & 1; }
}
__device__ __forceinline__ int packed_at(int s, int q, int rr, int pr) {
  if (rr != 2) return (s == 0 ? 0 : 88 + (s - 1) * 160) + 2 * q + rr;
  return s >= 2 ? 120 + (s - 2) * 160 + q * 7 + pr : s == 1 && q >= 8 ? q * 7 + pr - 24 : (NMIR + 3) * 64;
}
__device__ __forceinline__ void lds_st(unsigned addr, unsigned v) {
  asm volatile("ds_write_b32 %0, %1" :: "v"(addr), "v"(v) : "memory");
}

template <int OFF = 0>
__device__ __forceinline__ void st16(gchar* p, u32x4 v) {
  asm volatile("global_store_dwordx4 %0, %1, off offset:%2 sc0 sc1\n\ts_nop 1" :: "v"(p), "v"(v), "n"(OFF) : "memory");
}

__device__ __forceinline__ void st4(gchar* p, unsigned v) {
  asm volatile("global_store_dword %0, %1, off sc0 sc1\n\ts_nop 1" :: "v"(p), "v"(v) : "memory");
}

__device__ __forceinline__ u32x4 unpack8(unsigned l0, unsigned l1, unsigned n, unsigned base4) {
  const unsigned h0 = ((n & 0x07070707u) + base4) | ((n & 0x08080808u) << 4);
  const unsigned h1 = (((n >> 4) & 0x07070707u) + base4) | (n & 0x80808080u);
  return u32x4{__builtin_amdgcn_perm(h0, l0, 0x05010400u), __builtin_amdgcn_perm(h0, l0, 0x07030602u),
               __builtin_amdgcn_perm(h1, l1, 0x05010400u), __builtin_amdgcn_perm(h1, l1, 0x07030602u)};
}

__device__ __forceinline__ unsigned send_block(const unsigned (&p)[4][4], float m, gchar* dst, int lane, bool plain,
                                              int slow = 0) {
  if (plain) {
    st16<0>(dst, u32x4{p[0][0], p[0][1], p[0][2], p[0][3]});
    st16<1024>(dst, u32x4{p[1][0], p[1][1], p[1][2], p[1][3]});
    st16<2048>(dst, u32x4{p[2][0], p[2][1], p[2][2], p[2][3]});
    st16<3072>(dst, u32x4{p[3][0], p[3][1], p[3][2], p[3][3]});
    return F_RAW;
  }
  int mi = (int)__float_as_uint(m);
  mi = max(mi, __builtin_amdgcn_update_dpp(mi, mi, 0x111, 0xf, 0xf, false));
  mi = max(mi, __builtin_amdgcn_update_dpp(mi, mi, 0x112, 0xf, 0xf, false));
  mi = max(mi, __builtin_amdgcn_update_dpp(mi, mi, 0x114, 0xf, 0xf, false));
  mi = max(mi, __builtin_amdgcn_update_dpp(mi, mi, 0x118, 0xf, 0xf, false));
  mi = max(mi, __builtin_amdgcn_update_dpp(mi, mi, 0x142, 0xa, 0xf, false));
  mi = max(mi, __builtin_amdgcn_update_dpp(mi, mi, 0x143, 0xc, 0xf, false));
  const unsigned top = (pk_bf16(__uint_as_float((unsigned)__builtin_amdgcn_readlane(mi, 63)), 0.f) >> 8) & 0x7fu;
  const unsigned base = __builtin_amdgcn_readfirstlane(top > 7u ? top - 7u : 0u);
  const unsigned base4 = base * 0x01010101u;
  unsigned lo[4][2], hi[4][2], nb[4], aa = ~0u, ao = 0u;
#pragma unroll
  for (int j = 0; j < 4; ++j) {
    lo[j][0] = __builtin_amdgcn_perm(p[j][1], p[j][0], 0x06040200u);
    hi[j][0] = __builtin_amdgcn_perm(p[j][1], p[j][0], 0x07050301u);
    lo[j][1] = __builtin_amdgcn_perm(p[j][3], p[j][2], 0x06040200u);
    hi[j][1] = __builtin_amdgcn_perm(p[j][3], p[j][2], 0x07050301u);
    const unsigned u0 = (hi[j][0] | 0x80808080u) - base4;
    const unsigned u1 = (hi[j][1] | 0x80808080u) - base4;
    aa &= u0 & u1;
    ao |= u0 | u1;
    nb[j] = (u0 & 0x07070707u) | ((hi[j][0] >> 4) & 0x08080808u) | ((u1 & 0x07070707u) << 4) |
            (hi[j][1] & 0x80808080u);
  }
  unsigned info = base;
  if (__ballot(((~aa & 0x80808080u) | (ao & 0x78787878u)) != 0u)) {
    unsigned em[4][2], c = 0;
#pragma unroll
    for (int j = 0; j < 4; ++j)
#pragma unroll
      for (int h = 0; h < 2; ++h) {
        const unsigned u = (hi[j][h] | 0x80808080u) - base4;
        em[j][h] = (~u | ((u & 0x78787878u) + 0x78787878u)) & 0x80808080u;
        c += __builtin_popcount(em[j][h]);
      }
    unsigned total = 0;
#pragma unroll
    for (int b = 0; b < 6; ++b) total += (unsigned)__builtin_popcountll(__ballot((c >> b) & 1u)) << b;
    if (total > (unsigned)MAXESC) {
      st16<0>(dst, u32x4{p[0][0], p[0][1], p[0][2], p[0][3]});
      if (slow) asm volatile("s_waitcnt vmcnt(0)" ::: "memory");
      st16<1024>(dst, u32x4{p[1][0], p[1][1], p[1][2], p[1][3]});
      if (slow) asm volatile("s_waitcnt vmcnt(0)" ::: "memory");
      st16<2048>(dst, u32x4{p[2][0], p[2][1], p[2][2], p[2][3]});
      if (slow) asm volatile("s_waitcnt vmcnt(0)" ::: "memory");
      st16<3072>(dst, u32x4{p[3][0], p[3][1], p[3][2], p[3][3]});
      if (slow) asm volatile("s_waitcnt vmcnt(0)" ::: "memory");
      return F_RAW;
    }
    unsigned n = 0;
#pragma unroll
    for (int j = 0; j < 4; ++j)
#pragma unroll
      for (int h = 0; h < 2; ++h) {
        if (__ballot(em[j][h] != 0u) == 0) continue;
        for (int b = 0; b < 4; ++b) {
          const bool on = (em[j][h] >> (8 * b + 7)) & 1u;
          const u64 bal = __ballot(on);
          const unsigned idx = n + __builtin_amdgcn_mbcnt_hi((unsigned)(bal >> 32), __builtin_amdgcn_mbcnt_lo((unsigned)bal, 0u));
          if (on) st4(dst - lane * 16 + 3072 + idx * 4,
                      (unsigned)(j * 512 + lane * 8 + h * 4 + b) | (((hi[j][h] >> (8 * b)) & 0xffu) << 16));
          n += (unsigned)__builtin_popcountll(bal);
        }
      }
    info |= total << 7;
  }
  st16(dst, u32x4{lo[0][0], lo[0][1], lo[1][0], lo[1][1]});
  if (slow & 1) asm volatile("s_waitcnt vmcnt(0)" ::: "memory");
  st16<1024>(dst, u32x4{lo[2][0], lo[2][1], lo[3][0], lo[3][1]});
  if (slow) asm volatile("s_waitcnt vmcnt(0)" ::: "memory");
  st16<2048>(dst, u32x4{nb[0], nb[1], nb[2], nb[3]});
  return info;
}

#define AC00 "v[40:43]"
#define AC01 "v[44:47]"
#define AC02 "v[48:51]"
#define AC03 "v[52:55]"
#define AC10 "v[56:59]"
#define AC11 "v[60:63]"
#define AC12 "v[64:67]"
#define AC13 "v[68:71]"
#define AC20 "v[72:75]"
#define AC21 "v[76:79]"
#define AC22 "v[80:83]"
#define AC23 "v[84:87]"
#define AC30 "v[88:91]"
#define AC31 "v[92:95]"
#define AC32 "v[96:99]"
#define AC33 "v[100:103]"
#define AC40 "v[104:107]"
#define AC41 "v[108:111]"
#define AC42 "v[112:115]"
#define AC43 "v[116:119]"
#define RA0 "v[0:3]"
#define RA1 "v[4:7]"
#define RA2 "v[8:11]"
#define RA3 "v[12:15]"
#define RB0 "v[16:19]"
#define RB1 "v[20:23]"
#define RB2 "v[24:27]"
#define RB3 "v[28:31]"
#define RC0 "v[32:35]"
#define RC1 "v[36:39]"
#define PFY(x) x
#define PFN(x)
#define MF_(d, a, b) "v_mfma_f32_16x16x32_bf16 " d ", " a ", " b ", " d "\n\t"
#define RD_(r, a, o) "ds_read_b128 " r ", " a " offset:" #o "\n\t"
#define WT_(n) "s_waitcnt lgkmcnt(" #n ")\n\t"
#define ADV_ "s_add_u32 %[sdn], %[sdn], 1\n\t"                    \
             "v_mov_b32 %[vt], %[sdn]\n\t"                        \
             "s_mov_b64 exec, 1\n\t"                              \
             "ds_write2_b32 %[vc], %[vt], %[vt] offset0:%[dn] offset1:%[dm]\n\t" \
             "s_mov_b64 exec, -1\n\t"                             \
             "s_cmp_eq_u32 %[sof], %[hl]\n\t"                     \
             "s_cselect_b32 %[sd], %[hw], %[hs]\n\t"              \
             "s_add_u32 %[sof], %[sof], %[sd]\n\t"                \
             "v_add_u32 %[vx], %[sd], %[vx]\n\t"                  \
             "v_add_u32 %[vw], %[sd], %[vw]\n\t"                  \
             "v_add_u32 %[vf], 4, %[vf]\n\t"
#define DBG1_
#define DBG2_
#define FETCH(W0, W1, W2, N0, N1, N2, N3)                           \
  DBG1_                                                             \
  "s_setprio 0\n\t"                                                \
  "6:\n\t"                                                         \
  "ds_read2_b32 v[16:17], %[vf] offset0:0 offset1:1\n\t"           \
  WT_(0)                                                            \
  "v_readfirstlane_b32 %[sd], v16\n\t"                             \
  "v_readfirstlane_b32 %[sfl], v17\n\t"                            \
  "s_cmp_lg_u32 %[sd], 0\n\t"                                      \
  "s_cbranch_scc1 7f\n\t"                                          \
  "s_sleep 1\n\t"                                                  \
  DBG2_                                                             \
  "s_branch 6b\n\t"                                                \
  "7:\n\t"                                                         \
  RD_(N0, "%[vw]", 0) RD_(N1, "%[vw]", 1024) RD_(N2, "%[vw]", 2048) RD_(N3, "%[vw]", 3072)   \
  RD_(W0, "%[vx]", 0) RD_(W1, "%[vx]", 1024) RD_(W2, "%[vx]", 2048)                          \
  RD_(RC0, "%[vx]", 3072) RD_(RC1, "%[vx]", 4096) ADV_
#define PRD_ "ds_read_b32 %[vu], %[vc] offset:%[dp]\n\t"
#define W78_ WT_(8)
#define W67_ WT_(7)
#define PRI_ "v_readfirstlane_b32 %[sd], %[vu]\n\t"                \
             "s_setprio 0\n\t"                                     \
             "s_cmp_gt_u32 %[sd], %[sdn]\n\t"                      \
             "s_cbranch_scc0 8f\n\t"                               \
             "s_setprio 1\n\t"                                     \
             "8:\n\t"
#define BODY(W0, W1, W2, W3, N0, N1, N2, N3, P, Q)                                           \
  P(RD_(N3, "%[vw]", 3072) WT_(6)) Q(WT_(5))                                                 \
  MF_(AC00, W0, N0) MF_(AC01, W1, N0) MF_(AC02, W2, N0) MF_(AC03, W3, N0)                    \
  P(RD_(N0, "%[vw]", 0) "ds_read_b32 %[vt], %[vf] offset:4\n\t" PRD_ W78_) Q(WT_(4))        \
  MF_(AC10, W0, N1) MF_(AC11, W1, N1) MF_(AC12, W2, N1) MF_(AC13, W3, N1)                    \
  P(RD_(N1, "%[vw]", 1024) W78_) Q(WT_(3))                                                   \
  MF_(AC20, W0, N2) MF_(AC21, W1, N2) MF_(AC22, W2, N2) MF_(AC23, W3, N2)                    \
  P(RD_(N2, "%[vw]", 2048) W78_) Q(WT_(2))                                                   \
  MF_(AC30, W0, RC0)                                                                         \
  P(W67_) Q(WT_(1))                                                                          \
  MF_(AC40, W0, RC1)                                                                         \
  P(RD_(W0, "%[vx]", 0))                                                                     \
  MF_(AC31, W1, RC0) MF_(AC41, W1, RC1)                                                      \
  P(RD_(W1, "%[vx]", 1024))                                                                  \
  MF_(AC32, W2, RC0) MF_(AC42, W2, RC1)                                                      \
  P(RD_(W2, "%[vx]", 2048))                                                                  \
  MF_(AC33, W3, RC0)                                                                         \
  P(RD_(RC0, "%[vx]", 3072))                                                                 \
  MF_(AC43, W3, RC1)                                                                         \
  P(RD_(RC1, "%[vx]", 4096) WT_(7) "v_readfirstlane_b32 %[sfl], %[vt]\n\t" PRI_ ADV_)

__global__ __launch_bounds__(NTHREADS, NWAVE / 4) void fused_gemm_q8(Params P) {
  __shared__ __attribute__((aligned(1024))) char lds[LDS_BYTES];
  const unsigned ldsbase = (unsigned)(uintptr_t)lds;
  const unsigned ctl = ldsbase + LDS_CTL;

  const int tid = threadIdx.x;
  const int lane = tid & 63;
  const int wave = __builtin_amdgcn_readfirstlane(tid >> 6);
  const int rank = P.rank;
  char* const myws = static_cast<char*>(P.ws_all[rank]);
  unsigned* const tmo = reinterpret_cast<unsigned*>(myws + OFF_TMO);

  if ((blockIdx.x >> 3) < 6)
    for (int i = tid; i < (NMIR + 5) * 64; i += NTHREADS)
      asm volatile("ds_write_b128 %0, %1" :: "v"(ldsbase + i * 16), "v"(u32x4{0u, 0u, 0u, 0u}) : "memory");
  if (tid < 384) lds_st(ctl + tid * 4, 0u);
  asm volatile("s_waitcnt lgkmcnt(0)\n\ts_barrier" ::: "memory");
  if (wave == 0) TS(0);
  if (tid == ((blockIdx.x >> 3) < 6 ? 64 * 15 : 64 * 10)) {
    const unsigned t = __hip_atomic_fetch_add(reinterpret_cast<unsigned*>(myws + OFF_TICKET), 1u,
                                              __ATOMIC_RELAXED, __HIP_MEMORY_SCOPE_AGENT);
    lds_st(ctl + C_CALL * 4, t / NWG + 1);
  }
  unsigned call = 0, tag = 0;
  auto get_call = [&]() {
    while ((call = __builtin_amdgcn_readfirstlane(lds_ld(ctl + C_CALL * 4))) == 0u) __builtin_amdgcn_s_sleep(1);
    tag = call << 13;
  };

  const int bx = blockIdx.x & 7, bk = ((blockIdx.x >> 3) + 26) & 31;
  const bool sender = bk >= 26;
  const int ti = bk < 24 ? 2 * (bk % 6) + (bx & 1) : 12;
  const int tj = 4 * (bx >> 1) + (bk < 24 ? bk / 6 : 2 * (bx & 1) + ((bk - 24) & 1));

  f32x4 acc[5][4];
  const int wm = wave & 3, wn = (wave >> 2) & 1;

  auto fetch = [&](int s, int q, int d, int f, f32x4 (&a)[8]) {
    if (d < 0) {
      const GAS float* sp = (const GAS float*)P.x + (size_t)(q * 128 + (f & 4) * 16 + (lane & 15)) * KSH + s * 64 + (f >> 3) * 32 + 8 * (lane >> 4);
#pragma unroll
      for (int j = 0; j < 4; ++j) {
        a[2 * j] = *(const GAS f32x4*)sp;
        a[2 * j + 1] = *(const GAS f32x4*)(sp + 4);
        sp += 16 * KSH;
        asm("" : "+v"(sp));
      }
    } else {
      const GAS float* sp = (const GAS float*)P.w + (size_t)(s * 64 + (f >> 3) * 32 + 8 * (lane >> 4)) * NTOT + d * NLOC + q * 128 +
                        (f & 4) * 16 + (lane & 15) * 4;
#pragma unroll
      for (int j = 0; j < 8; ++j) {
        a[j] = *(const GAS f32x4*)sp;
        sp += NTOT;
        asm("" : "+v"(sp));
      }
    }
  };
  auto round16 = [&](int d, const f32x4 (&a)[8], unsigned (&p)[4][4], float& m) {
    m = 0.f;
#pragma unroll
    for (int j = 0; j < 8; ++j)
#pragma unroll
      for (int c = 0; c < 4; ++c) m = fmaxf(m, fabsf(a[j][c]));
    if (d < 0) {
#pragma unroll
      for (int j = 0; j < 4; ++j) {
        p[j][0] = pk_bf16(a[2 * j][0], a[2 * j][1]); p[j][1] = pk_bf16(a[2 * j][2], a[2 * j][3]);
        p[j][2] = pk_bf16(a[2 * j + 1][0], a[2 * j + 1][1]); p[j][3] = pk_bf16(a[2 * j + 1][2], a[2 * j + 1][3]);
      }
    } else {
#pragma unroll
      for (int jj = 0; jj < 4; ++jj)
#pragma unroll
        for (int c = 0; c < 4; ++c) p[jj][c] = pk_bf16(a[2 * c][jj], a[2 * c + 1][jj]);
    }
  };
  auto convert = [&](int s, int q, int d, int f, unsigned (&p)[4][4], float& m) {
    f32x4 a[8];
    fetch(s, q, d, f, a);
    round16(d, a, p, m);
  };
  auto pack_items = [&](int g, int rounds) {
    gchar* const gws = (gchar*)myws;
    const int grp = (g >> 3) % 6;
    unsigned* const pq = reinterpret_cast<unsigned*>(myws + OFF_PQ + grp * 256);
    int n = g;
    for (int k = 0; n < NPACK; ++k) {
     unsigned tk = 0;
     if (k >= RS - 1 && lane == 0) tk = __hip_atomic_fetch_add(pq, 1u, __ATOMIC_RELAXED, __HIP_MEMORY_SCOPE_AGENT);
     {
      const int f = (n & 3) * 4, e = n >> 2;
      int s, q, d = -1;
      if (e < 32) { s = 0; q = e; }
      else if (e < 88) { s = 1; q = 8 + (e - 32) / 7; d = (rank + 1 + (e - 32) % 7) & 7; }
      else if (e < 1080) {
        const int r = (e - 88) % 160;
        s = (e - 88) / 160 + 1;
        if (r < 32) q = r;
        else if (r < 144) { s += 1; q = (r - 32) / 7; d = (rank + 1 + (r - 32) % 7) & 7; }
        else { s -= 1; q = r - 144; d = rank; }
      } else { s = 6 + ((e - 1080) >> 4); q = (e - 1080) & 15; d = rank; }
      unsigned p[4][4];
      float m;
      convert(s, q, d, f, p, m);
      unsigned info = 0;
      if (d != rank && e < 1048) {
        gchar* dst = gws + (d < 0 ? OFF_EX + (size_t)(s * XU + q) * UNIT : OFF_EW + (size_t)((d * NSTAGE + s) * WU + q) * UNIT);
        info = send_block(p, m, dst + f * 1024 + lane * 16, lane, false);
      }
      if (d < 0 || d == rank) {
        gchar* dst = gws + (d < 0 ? OFF_XS + (size_t)((rank * NSTAGE + s) * XU + q) * UNIT
                                  : OFF_WS + (size_t)((rank * NSTAGE + s) * WU + q) * UNIT);
        send_block(p, m, dst + f * 1024 + lane * 16, lane, true);
      }
      if (k == 0) get_call();
      asm volatile("s_waitcnt vmcnt(0)" ::: "memory");
      if (lane == 0) st4(gws + OFF_EF + (size_t)n * 4, tag | info);
      if (lane == 1 && (d < 0 || d == rank))
        st4(gws + (d < 0 ? OFF_XF : OFF_WF) + (size_t)(q * 64 + rank * 8 + s) * 16 + f, tag | F_RAW);
     }
     n += PRN;
     if (k >= RS - 1) {
       const unsigned t = __builtin_amdgcn_readfirstlane(tk) - (call - 1) * ((NPACK - RS * PRN + 5 - grp) / 6 + PRN / 6);
       n = t < 4096u ? RS * PRN + 6 * (int)t + grp : NPACK;
     }
    }
  };

  auto locate = [&](int u, int f, gchar*& dst, gu32*& flag, int& pn, unsigned& so, int& pe) -> bool {
      const int pr = u % 7, d = (rank + 1 + pr) & 7;
      int s, q, rr;
      unit_of(u / 7, pr, s, q, rr);
      gchar* dws;
      asm volatile("s_load_dwordx2 %0, %1, %2\n\ts_waitcnt lgkmcnt(0)"
                   : "=s"(dws) : "s"(P.ws_all), "s"(__builtin_amdgcn_readfirstlane(d * 8)));
      if (rr != 2) {
        const int xu = 2 * q + rr;
        so = (unsigned)OFF_EX + (unsigned)(s * XU + xu) * UNIT;
        dst = dws + OFF_XS + ((size_t)((rank * NSTAGE + s) * XU + xu)) * UNIT;
        pn = s | xu << 3;
      } else {
        so = (unsigned)OFF_EW + (unsigned)((d * NSTAGE + s) * WU + q) * UNIT;
        dst = dws + OFF_WS + ((size_t)((rank * NSTAGE + s) * WU + q)) * UNIT;
        pn = s | q << 3 | (d + 1) << 8;
      }
      pe = packed_at(s, q, rr, pr);
      flag = (gu32*)(dws + OFF_F1 + (size_t)u * F1S) + f;
      const bool tail = u >= NUNIT - RTU;
      if (tail) {
        so = (unsigned)OFF_XS + (unsigned)((rank * NSTAGE + s) * XU + 2 * q + rr) * UNIT;
        flag = (gu32*)(dws + OFF_XF + ((2 * q + rr) * 64 + rank * 8 + s) * 16) + f;
      }
      so = __builtin_amdgcn_readfirstlane(so + f * 4096);
      dst += f * 4096 + lane * 16;
      pn |= f << 12;
      return tail;
  };

  if (sender) {
   if (QX & 8) {
   } else if (wave < NSEND) {
    const __amdgpu_buffer_rsrc_t rs = __builtin_amdgcn_make_buffer_rsrc(myws, 0, 0x7fffffff, 0x00020000);
    unsigned so, info, ia = 0;
    int pn;
    int dbg = 0, pdbg = 0;
    gchar* dst;
    gu32* flag;
    u32x4 va, vb, vc;
    unsigned ve;
    auto draw = [&](int look) -> bool {
      unsigned j, e;
      do {
        j = 0;
        e = 0;
        if (lane == 0)
          asm volatile("ds_add_rtn_u32 %0, %1, %2\n\ts_waitcnt lgkmcnt(0)" : "=v"(j) : "v"(ctl + C_NEXT * 4), "v"(1u) : "memory");
        j = __builtin_amdgcn_readfirstlane(j);
        if (j >= (unsigned)(S1 * 4))
          while (((e = __builtin_amdgcn_readfirstlane(lds_ld(ctl + (C_Q + ((j >> 2) & 63)) * 4))) >> 16) != (j >> 2) + 1)
            __builtin_amdgcn_s_sleep(1);
      } while ((e & 0x8000u) && (j & 3) > ((e & 0x4000u) ? 0u : 2u));
      int f = j & 3;
      int u = U0 + (int)(j >> 2) * NSWG + (bk - 26) * 8 + bx;
      const bool piece = ((e >> 14) ^ (e >> 15)) & 1;
      if (j >= (unsigned)(S1 * 4)) {
        u = e & 4095;
        if (u == 4095) return false;
        if (e & 0xc000u) f = (e >> 12) & 3;
      }
      int pe;
      const bool tail = locate(u, f, dst, flag, pn, so, pe);
      dbg = u * 4 + f;
      ia = ldsbase + (pe * 4 + f) * 4;
      asm volatile("" : "+v"(ia), "+v"(flag));
      info = __builtin_amdgcn_readfirstlane(lds_ld(ia));
      for (; look > 0 && ((info ^ tag) >> 13) != 0u; --look) {
        __builtin_amdgcn_s_sleep(8);
        info = __builtin_amdgcn_readfirstlane(lds_ld(ia));
      }
      if (piece) {
        pn |= 1 << 30 | (j & 3) << 14;
        if ((e & 0x8000u) && (j & 3) == 2) {
          pn |= 1 << 29;
          ve = __builtin_amdgcn_raw_buffer_load_b32(rs, (lane & 15) * 4 + 3072, so, 17);
        }
        so += (j & 3) * 1024;
        dst += (j & 3) * 1024;
        ia = ldsbase + (NMIR + 4) * 1024 + ((j >> 2) & 127) * 4;
        if (tail) info |= F_RAW;
      }
      if (((info ^ tag) >> 13) == 0u) {
        va = __builtin_amdgcn_raw_buffer_load_b128(rs, lane * 16, so, 17);
        if (!piece) {
          vb = __builtin_amdgcn_raw_buffer_load_b128(rs, lane * 16 + 1024, so, 17);
          vc = __builtin_amdgcn_raw_buffer_load_b128(rs, lane * 16 + 2048, so, 17);
          ve = __builtin_amdgcn_raw_buffer_load_b32(rs, (lane & 15) * 4 + 3072, so, 17);
        }
      }
      asm volatile("" : "+v"(pn), "+v"(so), "+v"(info));
      return true;
    };
    gu32* pflag = nullptr;
    unsigned pv = 0, pc = 0, pa = 1;
    auto finish = [&]() {
      unsigned c = 3;
      if (pc != 0u && lane == 0)
        asm volatile("ds_add_rtn_u32 %0, %1, %2\n\ts_waitcnt lgkmcnt(0)" : "=v"(c) : "v"(pc), "v"(pa) : "memory");
      if (pflag != nullptr && lane == 0 && ((c + pa) & 3u) == 0u)
        __hip_atomic_store(pflag, pv, __ATOMIC_RELAXED, __HIP_MEMORY_SCOPE_SYSTEM);
    };
    va = vb = vc = u32x4{0u, 0u, 0u, 0u}; ve = 0; info = 0; so = 0; pn = 0; dst = nullptr; flag = nullptr;
    get_call();
    draw(FWAIT);
    while (true) {
      asm volatile("s_waitcnt vmcnt(0)" : "+v"(va), "+v"(vb), "+v"(vc), "+v"(ve) :: "memory");
      const int cpn = __builtin_amdgcn_readfirstlane(pn);
      const unsigned cpv = __builtin_amdgcn_readfirstlane(info);
      const bool early = EARLY && ((cpv ^ tag) >> 13) == 0u && (EARLY == 2 || (cpn >> 30));
      if (early) st16(dst, va);
      finish();
      if (pflag != nullptr) T2(3, pdbg);
      pdbg = dbg;
      u32x4 ca = va, cb = vb, cc = vc;
      unsigned ce = ve;
      const unsigned cso = so, cia = ia;
      gchar* const cdst = dst;
      pflag = flag;
      pv = cpv;
      pc = (cpn >> 30) ? cia : 0u;
      pa = 1 + ((pn >> 29) & 1);
      if (((cpn >> 29) & 1) && (pv & (31u << 7)) && lane < MAXESC) st4(cdst - lane * 12 + 1024, ce);
      const bool more = draw(0);
      if (!(cpn >> 30) && ((pv ^ tag) >> 13) != 0u) {
        pv = __builtin_amdgcn_readfirstlane(lds_ld(cia));
        if (((pv ^ tag) >> 13) == 0u) {
          ca = __builtin_amdgcn_raw_buffer_load_b128(rs, cso + lane * 16, 0, 17);
          cb = __builtin_amdgcn_raw_buffer_load_b128(rs, cso + lane * 16 + 1024, 0, 17);
          cc = __builtin_amdgcn_raw_buffer_load_b128(rs, cso + lane * 16 + 2048, 0, 17);
          ce = __builtin_amdgcn_raw_buffer_load_b32(rs, cso + (lane & 15) * 4 + 3072, 0, 17);
        }
      }
      if (((pv ^ tag) >> 13) != 0u) {
        unsigned p[4][4];
        float m;
        convert(cpn & 7, (cpn >> 3) & 31, ((cpn >> 8) & 15) - 1, (cpn >> 10) & 12, p, m);
        if (cpn >> 30) {
          int g = cpn;
          asm volatile("" : "+v"(g));
#pragma unroll
          for (int i = 0; i < 4; ++i) {
            unsigned p0 = p[0][i], p1 = p[1][i], p2 = p[2][i], p3 = p[3][i];
            asm volatile("" : "+v"(p0), "+v"(p1), "+v"(p2), "+v"(p3));
            const unsigned a = (g & (1 << 14)) ? p1 : p0, b = (g & (1 << 14)) ? p3 : p2;
            ca[i] = (g & (2 << 14)) ? b : a;
          }
          pv = tag | F_RAW;
          st16(cdst, ca);
        } else {
          pv = tag | send_block(p, m, cdst, lane, false, DSLOW);
        }
      } else if (!(cpn >> 30)) {
        if (!early) st16(cdst, ca);
        asm volatile("s_waitcnt vmcnt(0)" ::: "memory");
        st16<1024>(cdst, cb);
        asm volatile("s_waitcnt vmcnt(0)" ::: "memory");
        st16<2048>(cdst, cc);
        if (pv & F_RAW) st16<3072>(cdst, __builtin_amdgcn_raw_buffer_load_b128(rs, cso + lane * 16 + 3072, 0, 17));
        else if ((pv & (31u << 7)) && lane < MAXESC) st4(cdst - lane * 12 + 3072, ce);
      } else if (!EARLY) {
        st16(cdst, ca);
      }
      if (!more) break;
    }
    asm volatile("s_waitcnt vmcnt(0)" ::: "memory");
    finish();
    T2(3, pdbg);
    if (wave < 2) TS(1 + wave);
   } else if (wave < NSEND + NSC) {
    get_call();
    const u64 t0 = __builtin_amdgcn_s_memrealtime();
    if (wave > NSEND) __builtin_amdgcn_s_sleep(LOS);
    int c = wave - NSEND < NLO ? 0 : NMIR;
    for (unsigned spins = 1; c < NMIR; ++spins) {
      {
        unsigned keep;
        asm volatile("s_mov_b32 %0, m0\n\ts_mov_b32 m0, %2\n\ts_nop 0\n\t"
                     "global_load_lds_dwordx4 %1, off sc0 sc1\n\t"
                     "global_load_lds_dwordx4 %1, off offset:1024 sc0 sc1\n\t"
                     "s_mov_b32 m0, %0"
                     : "=&s"(keep) : "v"(myws + OFF_EF + c * 1024 + lane * 16), "s"(ldsbase + c * 1024) : "memory");
      }
      asm volatile("s_waitcnt vmcnt(0)" ::: "memory");
      for (int i = 0; i < LW && c < NMIR; ++i, ++c) {
        u32x4 v;
        asm volatile("ds_read_b128 %0, %1\n\ts_waitcnt lgkmcnt(0)" : "=v"(v) : "v"(ldsbase + c * 1024 + lane * 16) : "memory");
        if (!__all((((v[0] ^ tag) | (v[1] ^ tag) | (v[2] ^ tag) | (v[3] ^ tag)) >> 13) == 0u)) break;
      }
      if ((spins & 15) == 0 &&
          (__hip_atomic_load(tmo, __ATOMIC_RELAXED, __HIP_MEMORY_SCOPE_AGENT) == call ||
           __builtin_amdgcn_s_memrealtime() - t0 > DEADLINE_TICKS)) {
        if (lane == 0) __hip_atomic_store(tmo, call, __ATOMIC_RELAXED, __HIP_MEMORY_SCOPE_AGENT);
        break;
      }
      __builtin_amdgcn_s_sleep(SCS);
    }
    if (wave == NSEND) TS(33);
   } else if (wave == NWAVE - 1) {
    get_call();
    unsigned* const sq = reinterpret_cast<unsigned*>(myws + OFF_SQ);
    const unsigned qbase = (call - 1) * NDRAW - U0 - S1 * NSWG;
    unsigned ahead = TKD;
    for (unsigned k = S1;; ++k) {
      while ((lds_ld(ctl + C_NEXT * 4) >> 2) + ahead < k) __builtin_amdgcn_s_sleep(2);
      unsigned t = 0;
      if (lane == 0) t = __hip_atomic_fetch_add(sq, 1u, __ATOMIC_RELAXED, __HIP_MEMORY_SCOPE_AGENT);
      const unsigned u = __builtin_amdgcn_readfirstlane(t) - qbase;
      if (u >= (unsigned)NUNIT) {
        if (lane < 8) lds_st(ctl + (C_Q + ((k + lane) & 63)) * 4, ((k + lane + 1) << 16) | 4095u);
        break;
      }
      if (u >= (unsigned)UP) {
        const int t = 4 * ((int)u - UP) + (lane & 3), ii = UP / 7 + t / 28, r = t % 28, f = r / 7, pr = r % 7;
        int s, q, rr;
        unit_of(ii, pr, s, q, rr);
        const unsigned w = lds_ld(ldsbase + (packed_at(s, q, rr, pr) * 4 + f) * 4);
        const unsigned mode = ii >= NT0 / 7 ? 1u : ((w ^ tag) >> 13) != 0u ? 3u : (w & F_RAW) ? 1u : 2u;
        if (lane < 4) lds_st(ctl + (C_Q + ((k + lane) & 63)) * 4, ((k + lane + 1) << 16) | mode << 14 | f << 12 | (ii * 7 + pr));
        k += 3;
        ahead = TKT;
      } else if (lane == 0) {
        lds_st(ctl + (C_Q + (k & 63)) * 4, ((k + 1) << 16) | u);
      }
    }
   }
  } else if (wave < 8) {
    if (wave < KW && !(QX & 8)) {
      const int ba = wave * NCW + bk * 8 + bx, bb = ba + KW * NCW;
      gchar *da, *db;
      gu32 *fa, *fb;
      int pa, pb, pe;
      unsigned so, p[4][4];
      float m;
      f32x4 ra[8], rb[8];
      locate(ba >> 2, ba & 3, da, fa, pa, so, pe);
      locate(bb >> 2, bb & 3, db, fb, pb, so, pe);
      T2(2, ba);
      T2(2, bb);
      fetch(pa & 7, (pa >> 3) & 31, ((pa >> 8) & 15) - 1, (pa >> 10) & 12, ra);
      fetch(pb & 7, (pb >> 3) & 31, ((pb >> 8) & 15) - 1, (pb >> 10) & 12, rb);
      round16(((pa >> 8) & 15) - 1, ra, p, m);
      const unsigned ia = send_block(p, m, da, lane, false, KSLOW);
      round16(((pb >> 8) & 15) - 1, rb, p, m);
      get_call();
      asm volatile("s_waitcnt vmcnt(0)" ::: "memory");
      if (lane == 0) __hip_atomic_store(fa, tag | ia, __ATOMIC_RELAXED, __HIP_MEMORY_SCOPE_SYSTEM);
      T2(3, ba);
      const unsigned ib = send_block(p, m, db, lane, false, KSLOW);
      asm volatile("s_waitcnt vmcnt(0)" ::: "memory");
      if (lane == 0) __hip_atomic_store(fb, tag | ib, __ATOMIC_RELAXED, __HIP_MEMORY_SCOPE_SYSTEM);
      T2(3, bb);
    }
    if (wave == 0) TS(30);
#pragma unroll
    for (int a = 0; a < 5; ++a)
#pragma unroll
      for (int b = 0; b < 4; ++b) acc[a][b] = f32x4{0.f, 0.f, 0.f, 0.f};
    const int xoff = (wm * 5) * 1024 + lane * 16;
    const int woff = 20480 + (wn * 4) * 1024 + lane * 16;
    const u64 c0 = CYC(), r0 = __builtin_amdgcn_s_memrealtime();
    {
      unsigned vx = ldsbase + xoff, vw = ldsbase + woff, vf = ctl + C_FULL * 4, vc = ctl + wave * 4, vt, vu;
      unsigned sfl, sdn, sof, sd, sdb = 0;
      asm volatile(
          "s_mov_b32 %[sdn], 0\n\t"
          "s_mov_b32 %[sof], 0\n\t"
          "s_nop 4\n\t"
          FETCH(RB0, RB1, RB2, RA0, RA1, RA2, RA3)
          "1:\n\t"
          "s_cmp_lg_u32 %[sfl], 0\n\t"
          "s_cbranch_scc0 3f\n\t"
          BODY(RA0, RA1, RA2, RA3, RB0, RB1, RB2, RB3, PFY, PFN)
          "2:\n\t"
          "s_cmp_lg_u32 %[sfl], 0\n\t"
          "s_cbranch_scc0 4f\n\t"
          BODY(RB0, RB1, RB2, RB3, RA0, RA1, RA2, RA3, PFY, PFN)
          "s_branch 1b\n\t"
          "3:\n\t"
          BODY(RA0, RA1, RA2, RA3, RB0, RB1, RB2, RB3, PFN, PFY)
          FETCH(RA0, RA1, RA2, RB0, RB1, RB2, RB3)
          "s_branch 2b\n\t"
          "4:\n\t"
          BODY(RB0, RB1, RB2, RB3, RA0, RA1, RA2, RA3, PFN, PFY)
          "s_cmp_eq_u32 %[sdn], %[ns]\n\t"
          "s_cbranch_scc1 5f\n\t"
          FETCH(RB0, RB1, RB2, RA0, RA1, RA2, RA3)
          "s_branch 1b\n\t"
          "5:\n\t"
          "s_setprio 0\n\t"
          "s_nop 15\n\t"
          "s_nop 15\n\t"
          :
          "+{v[40:43]}"(acc[0][0]), "+{v[44:47]}"(acc[0][1]), "+{v[48:51]}"(acc[0][2]), "+{v[52:55]}"(acc[0][3]),
          "+{v[56:59]}"(acc[1][0]), "+{v[60:63]}"(acc[1][1]), "+{v[64:67]}"(acc[1][2]), "+{v[68:71]}"(acc[1][3]),
          "+{v[72:75]}"(acc[2][0]), "+{v[76:79]}"(acc[2][1]), "+{v[80:83]}"(acc[2][2]), "+{v[84:87]}"(acc[2][3]),
          "+{v[88:91]}"(acc[3][0]), "+{v[92:95]}"(acc[3][1]), "+{v[96:99]}"(acc[3][2]), "+{v[100:103]}"(acc[3][3]),
          "+{v[104:107]}"(acc[4][0]), "+{v[108:111]}"(acc[4][1]), "+{v[112:115]}"(acc[4][2]), "+{v[116:119]}"(acc[4][3]),
          [vx] "+v"(vx), [vw] "+v"(vw), [vf] "+v"(vf), [vt] "=&v"(vt), [vu] "=&v"(vu), [sfl] "=&s"(sfl), [sdn] "=&s"(sdn),
          [sof] "=&s"(sof), [sd] "=&s"(sd), [sdb] "+s"(sdb)
          : [vc] "v"(vc), [dn] "n"(C_DONE), [dm] "n"(C_DONE + 8), [dp] "n"((C_DONE + 4) * 4), [hs] "n"(HS), [hl] "n"((NH - 1) * HS), [hw] "s"(-(NH - 1) * HS),
            [ns] "n"(2 * NSTEP)
          : "memory", "scc", "v0", "v1", "v2", "v3", "v4", "v5", "v6", "v7", "v8", "v9", "v10", "v11", "v12", "v13", "v14", "v15", "v16", "v17", "v18", "v19", "v20", "v21", "v22", "v23", "v24", "v25", "v26", "v27", "v28", "v29", "v30", "v31", "v32", "v33", "v34", "v35", "v36", "v37", "v38", "v39");
      TV(48 + wave, sdb);
    }
    if (wave == 0) { TV(40, CYC() - c0); TV(41, __builtin_amdgcn_s_memrealtime() - r0); }
    float m = 0.f;
#pragma unroll
    for (int a = 0; a < 5; ++a) {
      if (ti * TM + wm * 80 + a * 16 >= 4096) continue;
#pragma unroll
      for (int b = 0; b < 4; ++b)
#pragma unroll
        for (int j = 0; j < 4; ++j) m = fmaxf(m, fabsf(acc[a][b][j]));
    }
#pragma unroll
    for (int o = 32; o >= 1; o >>= 1) m = fmaxf(m, __shfl_xor(m, o));
    if (lane == 0) lds_st(ctl + (C_WMAX + wave) * 4, __float_as_uint(m));
    if (wave == 0) TS(12);
  } else if (wave < 10) {
    const int L = wave - 8;
    __builtin_amdgcn_s_setprio(LPRIO);
    get_call();
    {
      const u64 t0 = __builtin_amdgcn_s_memrealtime();
      while (__builtin_amdgcn_s_memrealtime() - t0 < DLY * 100ull) __builtin_amdgcn_s_sleep(32);
    }
    const char* const gx0 = myws + OFF_XS + L * 8192 + lane * 16;
    const char* const gw0 = myws + OFF_WS + (size_t)tj * UNIT + L * 8192 + lane * 16;
    int slot = L;
    const unsigned dna = ctl + (C_DONE + (lane & 7)) * 4;
    u64 rm = 0, avail = 0, left = ~0ull;
    unsigned dv = 0;
    for (int t = 0; t < NSTEP; ++t) {
      const int need = 2 * t + L - NH + 1;
      while (avail == 0 || !__all((int)dv >= need)) {
        __builtin_amdgcn_s_sleep(1);
        asm volatile("ds_read_b64 %0, %2\n\tds_read_b32 %1, %3\n\ts_waitcnt lgkmcnt(0)"
                     : "=&v"(rm), "=&v"(dv) : "v"(ctl + C_Q * 4), "v"(dna) : "memory");
        avail = left & (((u64)(unsigned)__builtin_amdgcn_readfirstlane((int)(rm >> 32)) << 32) |
                        (u64)(unsigned)__builtin_amdgcn_readfirstlane((int)rm));
      }
      const int c = __builtin_ctzll(avail);
      left &= ~(1ull << c);
      const int bit = ((rank - (c & 7)) & 7) * 8 + (c >> 3);
      asm volatile("s_waitcnt vmcnt(29)" ::: "memory");
      const unsigned l = ldsbase + slot * HS;
#pragma unroll
      for (int i = 0; i < 4; ++i) {
        const int hu = 5 * ti + i;
        glds4k(gx0 + ((size_t)(bit * XU + (hu >> 1))) * UNIT + (hu & 1) * 4096, l + i * 4096);
      }
      {
        const int hu = min(5 * ti + 4, 63);
        const char* g0 = gx0 + ((size_t)(bit * XU + (hu >> 1))) * UNIT + (hu & 1) * 4096;
        const char* g1 = gw0 + (size_t)(bit * WU) * UNIT;
        unsigned keep;
        asm volatile(
            "ds_read_b64 %0, %3\n\tds_read_b32 %1, %4\n\t"
            "s_mov_b32 %2, m0\n\ts_mov_b32 m0, %9\n\ts_nop 0\n\t"
            "global_load_lds_dwordx4 %5, off" LDPOL "\n\t"
            "global_load_lds_dwordx4 %5, off offset:1024" LDPOL "\n\t"
            "global_load_lds_dwordx4 %5, off offset:2048" LDPOL "\n\t"
            "global_load_lds_dwordx4 %5, off offset:3072" LDPOL "\n\t"
            "s_add_u32 m0, m0, 0x1000\n\ts_nop 0\n\t"
            "global_load_lds_dwordx4 %6, off" LDPOL "\n\t"
            "global_load_lds_dwordx4 %6, off offset:1024" LDPOL "\n\t"
            "global_load_lds_dwordx4 %6, off offset:2048" LDPOL "\n\t"
            "global_load_lds_dwordx4 %6, off offset:3072" LDPOL "\n\t"
            "s_add_u32 m0, m0, 0x1000\n\ts_nop 0\n\t"
            "global_load_lds_dwordx4 %7, off" LDPOL "\n\t"
            "global_load_lds_dwordx4 %7, off offset:1024" LDPOL "\n\t"
            "global_load_lds_dwordx4 %7, off offset:2048" LDPOL "\n\t"
            "global_load_lds_dwordx4 %7, off offset:3072" LDPOL "\n\t"
            "s_mov_b32 m0, %10\n\ts_mov_b64 exec, 1\n\t"
            "global_load_lds_dword %8, off sc0 sc1\n\t"
            "s_mov_b64 exec, -1\n\ts_mov_b32 m0, %2\n\t"
            "s_waitcnt lgkmcnt(0)"
            : "=&v"(rm), "=&v"(dv), "=&s"(keep)
            : "v"(ctl + C_Q * 4), "v"(dna), "v"(g0), "v"(g1), "v"(g1 + 4096), "v"(myws + OFF_TICKET),
              "s"(l + 16384), "s"(ctl + (C_FULL + 2 * t + L) * 4)
            : "memory", "scc");
        avail = left & (((u64)(unsigned)__builtin_amdgcn_readfirstlane((int)(rm >> 32)) << 32) |
                        (u64)(unsigned)__builtin_amdgcn_readfirstlane((int)rm));
      }
      slot = slot + 2 >= NH ? slot + 2 - NH : slot + 2;
    }
    asm volatile("s_waitcnt vmcnt(0)" ::: "memory");
    if (L == 0) TS(11);
  } else if (wave == 10) {
    get_call();
    const unsigned want = tag | F_RAW;
    const unsigned m2 = ti == NTI - 1 ? 0u : ~0u;
    const int st = ((rank - (lane & 7)) & 7) * 8 + (lane >> 3);
    const unsigned* pf0 = reinterpret_cast<const unsigned*>(myws + OFF_XF) + (((5 * ti) >> 1) * 64 + st) * SU;
    const unsigned* pf2 = reinterpret_cast<const unsigned*>(myws + OFF_WF) + (tj * 64 + st) * SU;
    u64 seen = 0;
    unsigned cnt = 0, spins = 0, kk = 0;
    bool dead = false;
    const u64 t0 = __builtin_amdgcn_s_memrealtime();
    while (cnt < (unsigned)NSTEP) {
      u32x4 f0, f1, f2, f3;
      asm volatile(
                   "s_mov_b64 exec, %6\n\t"
                   "global_load_dwordx4 %0, %4, off sc0 sc1\n\tglobal_load_dwordx4 %1, %4, off offset:1024 sc0 sc1\n\t"
                   "global_load_dwordx4 %2, %4, off offset:2048 sc0 sc1\n\t"
                   "global_load_dwordx4 %3, %5, off sc0 sc1\n\ts_mov_b64 exec, -1\n\ts_waitcnt vmcnt(0)"
                   : "=&v"(f0), "=&v"(f1), "=&v"(f2), "=&v"(f3) : "v"(pf0), "v"(pf2), "s"(~seen) : "memory");
      const u64 nm = __ballot((QX & 1) || dead || plain16(f0, f1, f2, f3, want, m2)) & ~seen;
      if (nm) {
        if (lane == 0) asm volatile("ds_write_b64 %0, %1" :: "v"(ctl + C_Q * 4), "v"(seen | nm) : "memory");
        cnt += (unsigned)__builtin_popcountll(nm);
        seen |= nm;
      } else if ((++spins & 15) == 0 &&
                 (__hip_atomic_load(tmo, __ATOMIC_RELAXED, __HIP_MEMORY_SCOPE_AGENT) == call ||
                  __builtin_amdgcn_s_memrealtime() - t0 > DEADLINE_TICKS)) {
        if (lane == 0) __hip_atomic_store(tmo, call, __ATOMIC_RELAXED, __HIP_MEMORY_SCOPE_AGENT);
        dead = true;
      }
      if (cnt < WFAST) __builtin_amdgcn_s_sleep(PSLEEP); else __builtin_amdgcn_s_sleep(PS2);
    }
    if (dead && lane == 0) lds_st(ctl + C_DEAD * 4, ~0u);
    TS(32);
  } else {
   const __amdgpu_buffer_rsrc_t rs = __builtin_amdgcn_make_buffer_rsrc(myws, 0, 0x7fffffff, 0x00020000);
   const u64 t0 = __builtin_amdgcn_s_memrealtime();
   bool dead = false;
   auto arrival = [&](int n, unsigned& so, unsigned& fo) {
      const int pr = n % 7;
      int s, q, rr;
      unit_of(n / 7, pr, s, q, rr);
      const int st = ((rank + 7 - pr) & 7) * 8 + s;
      if (rr != 2) {
        so = (unsigned)OFF_XS + (unsigned)(st * XU + 2 * q + rr) * UNIT;
        fo = (unsigned)OFF_XF + ((2 * q + rr) * 64 + st) * 16;
      } else {
        so = (unsigned)OFF_WS + (unsigned)(st * WU + q) * UNIT;
        fo = (unsigned)OFF_WF + (q * 64 + st) * 16;
      }
      so = __builtin_amdgcn_readfirstlane(so);
      fo = __builtin_amdgcn_readfirstlane(fo);
   };
   auto expand = [&](unsigned f, u32x4 a, u32x4 b, u32x4 c, unsigned e, unsigned so) {
        const unsigned ne = (f >> 7) & 31u;
        const unsigned base4 = (f & 0x7fu) * 0x01010101u;
        u32x4 o[4];
        o[0] = unpack8(a[0], a[1], c[0], base4);
        o[1] = unpack8(a[2], a[3], c[1], base4);
        o[2] = unpack8(b[0], b[1], c[2], base4);
        o[3] = unpack8(b[2], b[3], c[3], base4);
        for (unsigned i = 0; i < ne && i < (unsigned)MAXESC; ++i) {
          const unsigned ev = (unsigned)__builtin_amdgcn_readlane((int)e, (int)i);
          const unsigned sh = (ev & 1u) * 16 + 8, r = ((ev >> 7) & 12u) | ((ev >> 1) & 3u);
          const unsigned vm = (unsigned)lane == ((ev >> 3) & 63u) ? 0xffu << sh : 0u;
          const unsigned vv = ((ev >> 16) & 0xffu) << sh;
#pragma unroll
          for (int kk = 0; kk < 16; ++kk) {
            const unsigned mk = r == (unsigned)kk ? vm : 0u;
            o[kk >> 2][kk & 3] = (o[kk >> 2][kk & 3] & ~mk) | (vv & mk);
          }
        }
#pragma unroll
        for (int j = 0; j < 4; ++j)
          __builtin_amdgcn_raw_buffer_store_b128(o[j], rs, lane * 16 + j * 1024, so, 17);
   };
   if (wave < W_DEC + NDEC) {
    __builtin_amdgcn_s_setprio(DPRIO);
    get_call();
    for (int n = (wave - W_DEC) * NCW + bk * 8 + bx; n < NT0 - NL && !dead && !(QX & 16); n += NDEC * NCW) {
      unsigned so, fo;
      arrival(n, so, fo);
      unsigned f[4];
      for (unsigned spins = 1;; ++spins) {
        asm volatile("" ::: "memory");
        const u32x4 fv = __builtin_amdgcn_raw_buffer_load_b128(rs, 0, (unsigned)OFF_F1 + n * F1S, 17);
#pragma unroll
        for (int k = 0; k < 4; ++k) f[k] = __builtin_amdgcn_readfirstlane(fv[k]);
        if ((((f[0] ^ tag) | (f[1] ^ tag) | (f[2] ^ tag) | (f[3] ^ tag)) >> 13) == 0u) break;
        if ((spins & 15) == 0 &&
            (__hip_atomic_load(tmo, __ATOMIC_RELAXED, __HIP_MEMORY_SCOPE_AGENT) == call ||
             __builtin_amdgcn_s_memrealtime() - t0 > DEADLINE_TICKS)) {
          if (lane == 0) __hip_atomic_store(tmo, call, __ATOMIC_RELAXED, __HIP_MEMORY_SCOPE_AGENT);
          dead = true;
          break;
        }
        __builtin_amdgcn_s_sleep(DSLEEP);
      }
      if (dead) continue;
      T2(0, n);
      u32x4 a[4], b[4], c[4];
      unsigned e[4];
#pragma unroll
      for (int k = 0; k < 4; ++k) {
        a[k] = __builtin_amdgcn_raw_buffer_load_b128(rs, lane * 16 + k * 4096, so, 17);
        b[k] = __builtin_amdgcn_raw_buffer_load_b128(rs, lane * 16 + k * 4096 + 1024, so, 17);
        c[k] = __builtin_amdgcn_raw_buffer_load_b128(rs, lane * 16 + k * 4096 + 2048, so, 17);
        e[k] = __builtin_amdgcn_raw_buffer_load_b32(rs, (lane & 15) * 4 + k * 4096 + 3072, so, 17);
      }
#pragma unroll
      for (int k = 0; k < 4; ++k) {
        if (f[k] & F_RAW) continue;
        expand(f[k], a[k], b[k], c[k], e[k], so + k * 4096);
      }
      asm volatile("s_waitcnt vmcnt(0)" ::: "memory");
      if (lane == 0) {
        const unsigned w = tag | F_RAW;
        __builtin_amdgcn_raw_buffer_store_b128(u32x4{w, w, w, w}, rs, 0, fo, 17);
      }
      T2(1, n);
    }
    if (wave == W_DEC) TS(19);
   } else {
    __builtin_amdgcn_s_setprio(PPRIO);
    if (!(QX & 32)) pack_items((wave - W_DEC - NDEC + 8 * CPK) * NCW + bk * 8 + bx, NROUND);
    if (wave == NWAVE - 1) TS(31);
    while (NL && __builtin_popcount(__builtin_amdgcn_readfirstlane(lds_ld(ctl + C_Q * 4 + 4))) < 8) __builtin_amdgcn_s_sleep(32);
    __builtin_amdgcn_s_setprio(DPRIO);
   }
   for (int m = ((wave - W_DEC + NPK) % (NDEC + NPK)) * NCW + bk * 8 + bx; m < NL * 4 && !dead; m += (NDEC + NPK) * NCW) {
      const int n = NT0 - NL + (m >> 2), k = m & 3;
      unsigned so, fo, f;
      arrival(n, so, fo);
      so += k * 4096;
      for (unsigned spins = 1;; ++spins) {
        asm volatile("" ::: "memory");
        f = __builtin_amdgcn_readfirstlane(__builtin_amdgcn_raw_buffer_load_b32(rs, 0, (unsigned)OFF_F1 + n * F1S + k * 4, 17));
        if (((f ^ tag) >> 13) == 0u) break;
        if ((spins & 15) == 0 &&
            (__hip_atomic_load(tmo, __ATOMIC_RELAXED, __HIP_MEMORY_SCOPE_AGENT) == call ||
             __builtin_amdgcn_s_memrealtime() - t0 > DEADLINE_TICKS)) {
          if (lane == 0) __hip_atomic_store(tmo, call, __ATOMIC_RELAXED, __HIP_MEMORY_SCOPE_AGENT);
          dead = true;
          break;
        }
        __builtin_amdgcn_s_sleep(LSLEEP);
      }
      if (dead) break;
      T2(0, n);
      if (!(f & F_RAW)) {
        const u32x4 a = __builtin_amdgcn_raw_buffer_load_b128(rs, lane * 16, so, 17);
        const u32x4 b = __builtin_amdgcn_raw_buffer_load_b128(rs, lane * 16 + 1024, so, 17);
        const u32x4 c = __builtin_amdgcn_raw_buffer_load_b128(rs, lane * 16 + 2048, so, 17);
        const unsigned e = __builtin_amdgcn_raw_buffer_load_b32(rs, (lane & 15) * 4 + 3072, so, 17);
        expand(f, a, b, c, e, so);
      }
      asm volatile("s_waitcnt vmcnt(0)" ::: "memory");
      if (lane == 0) __builtin_amdgcn_raw_buffer_store_b32(tag | F_RAW, rs, 0, fo + k * 4, 17);
      T2(1, n);
   }
  }

  asm volatile("s_waitcnt lgkmcnt(0)" ::: "memory");
  __syncthreads();
  const u64 outp = (u64)(uintptr_t)P.out_all[lane & 7];
  gchar* const slp = (gchar*)P.ws_all[lane & 7];
  get_call();

  if (wave == 0) TS(13);
  if (sender) {
    if (wave == 0 && blockIdx.x < NCOL) {
      const char* rp = myws + OFF_REP + lane * 16;
      const u64 t0 = __builtin_amdgcn_s_memrealtime();
      unsigned m;
      for (unsigned spins = 1;; ++spins) {
        u32x4 r0, r1;
        asm volatile("global_load_dwordx4 %0, %2, off sc0 sc1\n\tglobal_load_dwordx4 %1, %2, off offset:1024 sc0 sc1\n\t"
                     "s_waitcnt vmcnt(0)" : "=&v"(r0), "=&v"(r1) : "v"(rp) : "memory");
        if (lane >= (NCW - 128) / 2) r1 = r0;
        m = max(max(r0[0], r0[2]), max(r1[0], r1[2]));
        if (__all(r0[1] == call && r0[3] == call && r1[1] == call && r1[3] == call)) break;
        if ((spins & 15) == 0 && __builtin_amdgcn_s_memrealtime() - t0 > DEADLINE_TICKS) {
          if (lane == 0) __hip_atomic_store(tmo, call, __ATOMIC_RELAXED, __HIP_MEMORY_SCOPE_AGENT);
          m = ~0u;
          break;
        }
        __builtin_amdgcn_s_sleep(CSL);
      }
      TS(20);
#pragma unroll
      for (int o = 32; o >= 1; o >>= 1) m = max(m, (unsigned)__shfl_xor((int)m, o));
      __hip_atomic_store((gu64*)(slp + OFF_SLOTS + (lane >> 3) * 512) + rank, ((u64)call << 32) | m, __ATOMIC_RELAXED,
                         __HIP_MEMORY_SCOPE_SYSTEM);
    }
    return;
  }
  if (wave == 0) {
    unsigned m = lds_ld(ctl + (lane < 8 ? C_WMAX + lane : C_DEAD) * 4);
#pragma unroll
    for (int o = 8; o >= 1; o >>= 1) m = max(m, (unsigned)__shfl_xor((int)m, o));
    if (lane == 0)
      __hip_atomic_store((gu64*)(myws + OFF_REP) + (blockIdx.x - NSWG), ((u64)call << 32) | m, __ATOMIC_RELAXED,
                         __HIP_MEMORY_SCOPE_SYSTEM);
    const u64* slots = reinterpret_cast<const u64*>(myws + OFF_SLOTS + (blockIdx.x & 7) * 512) + (lane & 7);
    const u64 t0 = __builtin_amdgcn_s_memrealtime();
    unsigned g;
    for (unsigned spins = 1;; ++spins) {
      const u64 v = __hip_atomic_load(slots, __ATOMIC_RELAXED, __HIP_MEMORY_SCOPE_SYSTEM);
      g = (unsigned)v;
      if (__all((unsigned)(v >> 32) >= call)) break;
      if ((spins & 15) == 0 &&
          (__hip_atomic_load(tmo, __ATOMIC_RELAXED, __HIP_MEMORY_SCOPE_AGENT) == call ||
           __builtin_amdgcn_s_memrealtime() - t0 > DEADLINE_TICKS)) {
        if (lane == 0) __hip_atomic_store(tmo, call, __ATOMIC_RELAXED, __HIP_MEMORY_SCOPE_AGENT);
        g = ~0u;
        break;
      }
      __builtin_amdgcn_s_sleep(ASL);
    }
    TS(22);
#pragma unroll
    for (int o = 4; o >= 1; o >>= 1) g = max(g, (unsigned)__shfl_xor((int)g, o));
    if (lane == 0) lds_st(ctl + C_GMAX * 4, g);
  }
  asm volatile("s_waitcnt lgkmcnt(0)" ::: "memory");
  __syncthreads();
  {
  int b2 = blockIdx.x, w2 = wave;
  asm volatile("" : "+s"(b2), "+s"(w2));
  const int bx2 = b2 & 7, bk2 = ((b2 >> 3) + 26) & 31;
  const int ti = bk2 < 24 ? 2 * (bk2 % 6) + (bx2 & 1) : 12;
  const int tj = 4 * (bx2 >> 1) + (bk2 < 24 ? bk2 / 6 : 2 * (bx2 & 1) + ((bk2 - 24) & 1));
  const int wm = w2 & 3, wn = (w2 >> 2) & 1;
  if (wave == 0) TS(14);
  if (wave < 8) {
    const float amax = __uint_as_float(lds_ld(ctl + C_GMAX * 4));
    const bool dead = __float_as_uint(amax) == ~0u;
    const float scale = amax / 127.0f;
    const float inv = scale > 0.f ? 1.0f / scale : 0.f;
    const int col = rank * NLOC + tj * 128 + wn * 64 + (lane >> 4) * 16;
#pragma unroll
    for (int mt = 0; mt < 5; ++mt) {
      const int row0 = ti * TM + wm * 80 + mt * 16;
      if (row0 >= 4096) continue;
      const int owner = row0 >> 9;
      gchar* ob = (gchar*)(((u64)(unsigned)__builtin_amdgcn_readlane((int)(outp >> 32), owner) << 32) |
                           (unsigned)__builtin_amdgcn_readlane((int)outp, owner));
      u32x4 o;
#pragma unroll
      for (int j = 0; j < 4; ++j) {
        unsigned pk = 0;
#pragma unroll
        for (int nt = 0; nt < 4; ++nt) {
          float q = rintf(acc[mt][nt][j] * inv);
          q = fminf(fmaxf(q, -127.f), 127.f);
          int qi = dead ? -128 : (int)q;
          pk |= ((unsigned)qi & 0xffu) << (8 * nt);
        }
        o[j] = pk;
      }
      st16(ob + (size_t)((row0 & 511) + (lane & 15)) * NTOT + col, o);
      asm volatile("s_waitcnt vmcnt(0)" ::: "memory");
    }
    if (wave == 0) TS(17);
    asm volatile("s_waitcnt vmcnt(0)" ::: "memory");
    if (wave == 0) TS(18);
  }
  asm volatile("s_waitcnt lgkmcnt(0)" ::: "memory");
  __syncthreads();
  if (wave == 0) {
    TS(15);
    {
      const int o0 = (ti * TM) >> 9, o1 = min(ti * TM + TM - 1, 4095) >> 9;
      if (lane == o0 || lane == o1)
        __hip_atomic_store((gu32*)(slp + OFF_OUTF) + (rank * NTI + ti) * 16 + tj, call, __ATOMIC_RELAXED, __HIP_MEMORY_SCOPE_SYSTEM);
    }
    if (blockIdx.x == 48) {
      const int tlo = (rank * MSH) / TM, thi = (rank * MSH + MSH - 1) / TM;
      const unsigned* of = reinterpret_cast<const unsigned*>(myws + OFF_OUTF);
      const u64 t0 = __builtin_amdgcn_s_memrealtime();
      while (true) {
        bool ok = true;
#pragma unroll
        for (int j = 0; j < 6; ++j) {
          const int idx = lane + 64 * j, sr = idx / 48, rem = idx - sr * 48;
          const int tt = min(tlo + (rem >> 4), thi);
          ok &= __hip_atomic_load(of + (sr * NTI + tt) * 16 + (rem & 15), __ATOMIC_RELAXED, __HIP_MEMORY_SCOPE_SYSTEM) >= call;
        }
        if (__all(ok)) break;
        if (__hip_atomic_load(tmo, __ATOMIC_RELAXED, __HIP_MEMORY_SCOPE_AGENT) == call ||
            __builtin_amdgcn_s_memrealtime() - t0 > DEADLINE_TICKS) {
          if (lane == 0) {
            __hip_atomic_store(tmo, call, __ATOMIC_RELAXED, __HIP_MEMORY_SCOPE_AGENT);
            ((gchar*)(P.out_all[rank]))[rank * NLOC] = (char)-128;
          }
          break;
        }
        __builtin_amdgcn_s_sleep(1);
      }
      __builtin_amdgcn_fence(__ATOMIC_ACQUIRE, "");
      TS(16);
      }
    }
  }
}

extern "C" void kernel_launch(int rank, int nranks, ncclComm_t comm, hipStream_t stream,
                              void* const* d_in, const int* in_sizes, int n_in, void* d_out,
                              int out_size, const int* in_dtypes, int out_dtype,
                              void* const* d_out_all, void* const* d_ws_all, size_t ws_bytes) {
  if (nranks != NR || ws_bytes < WS_NEED || n_in < 2) {
    fprintf(stderr, "unexpected configuration\n");
    return;
  }
  Params P;
  P.x = static_cast<const float*>(d_in[0]);
  P.w = static_cast<const float*>(d_in[1]);
  P.out_all = d_out_all;
  P.ws_all = d_ws_all;
  P.rank = rank;
  hipLaunchKernelGGL(fused_gemm_q8, dim3(NWG), dim3(NTHREADS), 0, stream, P);
  HIP_CHECK(hipGetLastError());
}
```

```cpp
#include <hip/hip_runtime.h>
#include <rccl/rccl.h>
#include <cstdio>
#include <cstdint>
#include <type_traits>

#define HIP_CHECK(expr) do { hipError_t _e = (expr); if (_e != hipSuccess) { \
    fprintf(stderr, "HIP error %s at %s:%d\n", hipGetErrorString(_e), __FILE__, __LINE__); \
    return; } } while (0)

typedef __attribute__((ext_vector_type(8))) short bf16x8;
typedef __attribute__((ext_vector_type(4))) float f32x4;
typedef __attribute__((ext_vector_type(4))) unsigned u32x4;
typedef unsigned long long u64;
#define GAS __attribute__((address_space(1)))
typedef GAS char gchar;
typedef GAS unsigned gu32;
typedef GAS unsigned long long gu64;
typedef GAS u32x4 gu32x4;

namespace {
constexpr int NR = 8;
constexpr int KSH = 512;
constexpr int NTOT = 16384;
constexpr int NLOC = 2048;
constexpr int MSH = 512;
constexpr int NSTAGE = 8;
constexpr int XU = 32;
constexpr int WU = 16;
constexpr int UNIT = 16384;
constexpr int NWG = 256;
constexpr int SU = 4;
constexpr int NUNIT = 7 * NSTAGE * (XU + WU);
#define THR 2
#define PSLEEP 16
#define SCS 32
#define F1S 128
#define MAPB 0
#define DLY 24
#define DSLOW 0
#define LOS 32
#define LW 2
#define NLO 1
#define PRO 1
#define PKD 0
#define KSLOW 1
#define FWAIT 40
#define RECHK 1
#define OUTL 0
#define OTH 1
#define ASL 4
#define SEQ 5
#define WROT 1
#define WFAST 44
#define PS2 2
#define WMASK 1
#define NL (NT0 - 2016)
#define LSLEEP 4
#define NCOL 2
#define CSL 1
#define RS 1
#define PPRIO 0
#define OSEQ 8
#define KWV 3
#define TSL 8
#define XROWS(q) rows_of(q)
#define XSH 0
#define SPOLL 0
#define SSL 24
#define IDL 0
#define FAST 0
#define ER 0xD5
#define GOX 0
#define EOTH 1
#define TKT 5
#define BAL 1
#define LPRIO 3
#define PFD 0
#define PFL 3
#define TLO 56
#define THI 64
#define PIPE 1
#define QX 0
#define CPRIO 0
#define DSLEEP 16
#define DPRIO 2
#define LP 0
#define LDPOL " sc0 sc1"
constexpr int NSTEP = 64;
constexpr int TM = 320;
constexpr int NTI = 13;
constexpr int NCW = NTI * 16;
constexpr int NH = 5;
constexpr int HS = 7 * 4096;
#define NSEND 13
constexpr int NSWG = NWG - NCW;
constexpr int NSC = 15 - NSEND < 4 ? 15 - NSEND : 4;
#define S1V 4
constexpr int EU = ER ? 10 : 0;
constexpr int RTU = 7 * (32 - EU);
#define TKD 1
constexpr int S1 = S1V;
constexpr int NT0 = NUNIT - RTU;
constexpr int KW = KWV;
constexpr int U0 = NCW * KW * 2 / 4;
constexpr int NDRAW = NUNIT - U0 - S1 * NSWG + NSWG;
static_assert(NT0 - NL >= 7 * (16 + 48 * 5), "the steps of stage 4 do not depend on the last units");
static_assert(NSEND + NSC < 16 && SEQ <= 6, "sender split");
#define NWAVE 16
#define NDECV 2
constexpr int NDEC = NDECV;
constexpr int W_DEC = 11;
constexpr int NTHREADS = 64 * NWAVE;
constexpr unsigned F_RAW = 1u << 12;
constexpr int MAXESC = 16;
constexpr int NPK = NWAVE - W_DEC - NDEC;
constexpr int NPACK = (8 * XU + 56 + 6 * 112 + NSTAGE * WU) * 4;
constexpr int NMIR = 17;
#define CPK 0
constexpr int PR0 = NCW * (8 * CPK + NPK);
constexpr int PRN = NCW * NPK;
constexpr int NROUND = 1 + (NPACK - PR0 + PRN - 1) / PRN;
static_assert(CPK == 0 && RS >= 1 && RS * PRN <= NPACK && PRN % 48 == 0, "packing");
constexpr int NIN = 7 * 384;

constexpr size_t OFF_TICKET = 0;
constexpr size_t OFF_TMO = 128;
constexpr size_t OFF_SQ = 2048;
constexpr size_t OFF_PQ = 2304;
constexpr size_t OFF_XF = 4096;
constexpr size_t OFF_WF = OFF_XF + XU * 64 * SU * 4;
constexpr size_t OFF_GO = 3840;
constexpr size_t OFF_OUTF = 65536;
constexpr size_t OFF_REP = 73728;
constexpr size_t OFF_SLOTS = 77824;
constexpr size_t OFF_EF = 81920;
constexpr size_t OFF_F1 = 131072;
constexpr size_t OFF_EOF = 1u << 19;
constexpr size_t OFF_XS = 1u << 20;
constexpr size_t OFF_WS = OFF_XS + (size_t)NR * NSTAGE * XU * UNIT;
constexpr size_t OFF_EX = OFF_WS + (size_t)NR * NSTAGE * WU * UNIT;
constexpr size_t OFF_EW = OFF_EX + (size_t)NSTAGE * XU * UNIT;
constexpr size_t OFF_ES = OFF_EW + (size_t)NR * NSTAGE * WU * UNIT;
constexpr size_t WS_NEED = OFF_ES + (size_t)NR * 8 * 32 * 2048;

constexpr u64 DEADLINE_TICKS = 40000000ull;

constexpr int C_CALL = 0, C_FULL = 128, C_DONE = 8  , C_WMAX = 100, C_OUT = 110, C_ECNT = 120, C_GMAX = 24, C_DEAD = 25, C_NEXT = 26, C_Q = 32;
constexpr int LDS_CTL = NH * HS;
constexpr int LDS_BYTES = LDS_CTL + 512 + 1024;
}

#define T2(a, i) do {} while (0)
#define TS(i) do {} while (0)
#define TV(i, v) do {} while (0)
#define CYC() 0ull

struct Params {
  const float* x;
  const float* w;
  void* const* out_all;
  void* const* ws_all;
  int rank;
};

__device__ __forceinline__ unsigned pk_bf16(float a, float b) {
  typedef __attribute__((ext_vector_type(2))) float f2;
  typedef __attribute__((ext_vector_type(2))) __bf16 b2;
  f2 v = {a, b};
  b2 r = __builtin_convertvector(v, b2);
  return __builtin_bit_cast(unsigned, r);
}

__device__ __forceinline__ unsigned lds_ld(unsigned addr) {
  unsigned v;
  asm volatile("ds_read_b32 %0, %1\n\ts_waitcnt lgkmcnt(0)" : "=v"(v) : "v"(addr) : "memory");
  return v;
}
__device__ __forceinline__ void glds4k(const void* g, unsigned ldsaddr) {
  unsigned keep;
  asm volatile(
      "s_mov_b32 %0, m0\n\ts_mov_b32 m0, %2\n\ts_nop 0\n\t"
      "global_load_lds_dwordx4 %1, off" LDPOL "\n\t"
      "global_load_lds_dwordx4 %1, off offset:1024" LDPOL "\n\t"
      "global_load_lds_dwordx4 %1, off offset:2048" LDPOL "\n\t"
      "global_load_lds_dwordx4 %1, off offset:3072" LDPOL "\n\t"
      "s_mov_b32 m0, %0"
      : "=&s"(keep) : "v"(g), "s"(ldsaddr) : "memory");
}
__device__ __forceinline__ bool plain16(u32x4 a, u32x4 b, u32x4 c, u32x4 d, unsigned want, unsigned m2) {
  unsigned x = 0, y = 0;
#pragma unroll
  for (int j = 0; j < 4; ++j) { x |= (a[j] ^ want) | (b[j] ^ want) | (d[j] ^ want); y |= c[j] ^ want; }
  return ((x | (y & m2)) >> 12) == 0u;
}
__device__ __forceinline__ int rows_of(int v) { return v < 8 ? 4 * v : 4 * ((v - 8) / 3) + 1 + (v - 8) % 3; }
__device__ __forceinline__ void unit_of(int ii, int pr, int& s, int& q, int& rr) {
  const int rot = (pr * 16 + 3) / 7;
  if (ii < 16) { s = 0; q = ii; rr = 2; }
  else if (ii < 16 + 48 * SEQ) {
    const int i2 = ii - 16, it = i2 % 48;
    s = i2 / 48; q = it / 3; rr = it - 3 * q;
    if (rr == 2) s += 1; else q = (q + rot) & 15;
  } else {
    int pos;
    if (ii < 336 - XSH) { s = (ii - 16) / 48; pos = (ii - 16) % 48; }
    else if (ii < 336 - XSH + EU) { s = 7; pos = ii - (336 - XSH); }
    else if (ii < 352 - XSH + EU) { s = 6; pos = ii - (304 - XSH) - EU; }
    else if (ii < 352 + EU) { s = 6; pos = ii - 320 - EU; }
    else { s = 7; pos = ii - 352; }
    if (pos >= 32) { s += 1; q = (pos + rot) & 15; rr = 2; }
    else {
      const int xu = pos < EU ? (pos + pr) % (EU ? EU : 1) : EU + (pos - EU + (pr * (32 - EU) + 3) / 7) % (32 - EU);
      q = xu >> 1; rr = xu & 1;
    }
  }
}
__device__ __forceinline__ int packed_at(int s, int q, int rr, int pr) {
  if (rr != 2) return (s == 0 ? 0 : 88 + (s - 1) * 160) + 2 * q + rr;
  return s >= 2 ? 120 + (s - 2) * 160 + q * 7 + pr : s == 1 && q >= 8 ? q * 7 + pr - 24 : (NMIR + 3) * 64;
}
__device__ __forceinline__ void lds_st(unsigned addr, unsigned v) {
  asm volatile("ds_write_b32 %0, %1" :: "v"(addr), "v"(v) : "memory");
}

template <int OFF = 0>
__device__ __forceinline__ void st16(gchar* p, u32x4 v) {
  asm volatile("global_store_dwordx4 %0, %1, off offset:%2 sc0 sc1\n\ts_nop 1" :: "v"(p), "v"(v), "n"(OFF) : "memory");
}

__device__ __forceinline__ void st4(gchar* p, unsigned v) {
  asm volatile("global_store_dword %0, %1, off sc0 sc1\n\ts_nop 1" :: "v"(p), "v"(v) : "memory");
}

__device__ __forceinline__ u32x4 unpack8(unsigned l0, unsigned l1, unsigned n, unsigned base4) {
  const unsigned h0 = ((n & 0x07070707u) + base4) | ((n & 0x08080808u) << 4);
  const unsigned h1 = (((n >> 4) & 0x07070707u) + base4) | (n & 0x80808080u);
  return u32x4{__builtin_amdgcn_perm(h0, l0, 0x05010400u), __builtin_amdgcn_perm(h0, l0, 0x07030602u),
               __builtin_amdgcn_perm(h1, l1, 0x05010400u), __builtin_amdgcn_perm(h1, l1, 0x07030602u)};
}

__device__ __forceinline__ unsigned send_block(const unsigned (&p)[4][4], float m, gchar* dst, int lane, bool plain,
                                              int slow = 0) {
  if (plain) {
    st16<0>(dst, u32x4{p[0][0], p[0][1], p[0][2], p[0][3]});
    st16<1024>(dst, u32x4{p[1][0], p[1][1], p[1][2], p[1][3]});
    st16<2048>(dst, u32x4{p[2][0], p[2][1], p[2][2], p[2][3]});
    st16<3072>(dst, u32x4{p[3][0], p[3][1], p[3][2], p[3][3]});
    return F_RAW;
  }
  int mi = (int)__float_as_uint(m);
  mi = max(mi, __builtin_amdgcn_update_dpp(mi, mi, 0x111, 0xf, 0xf, false));
  mi = max(mi, __builtin_amdgcn_update_dpp(mi, mi, 0x112, 0xf, 0xf, false));
  mi = max(mi, __builtin_amdgcn_update_dpp(mi, mi, 0x114, 0xf, 0xf, false));
  mi = max(mi, __builtin_amdgcn_update_dpp(mi, mi, 0x118, 0xf, 0xf, false));
  mi = max(mi, __builtin_amdgcn_update_dpp(mi, mi, 0x142, 0xa, 0xf, false));
  mi = max(mi, __builtin_amdgcn_update_dpp(mi, mi, 0x143, 0xc, 0xf, false));
  const unsigned top = (pk_bf16(__uint_as_float((unsigned)__builtin_amdgcn_readlane(mi, 63)), 0.f) >> 8) & 0x7fu;
  const unsigned base = __builtin_amdgcn_readfirstlane(top > 7u ? top - 7u : 0u);
  const unsigned base4 = base * 0x01010101u;
  unsigned lo[4][2], hi[4][2], nb[4], aa = ~0u, ao = 0u;
#pragma unroll
  for (int j = 0; j < 4; ++j) {
    lo[j][0] = __builtin_amdgcn_perm(p[j][1], p[j][0], 0x06040200u);
    hi[j][0] = __builtin_amdgcn_perm(p[j][1], p[j][0], 0x07050301u);
    lo[j][1] = __builtin_amdgcn_perm(p[j][3], p[j][2], 0x06040200u);
    hi[j][1] = __builtin_amdgcn_perm(p[j][3], p[j][2], 0x07050301u);
    const unsigned u0 = (hi[j][0] | 0x80808080u) - base4;
    const unsigned u1 = (hi[j][1] | 0x80808080u) - base4;
    aa &= u0 & u1;
    ao |= u0 | u1;
    nb[j] = (u0 & 0x07070707u) | ((hi[j][0] >> 4) & 0x08080808u) | ((u1 & 0x07070707u) << 4) |
            (hi[j][1] & 0x80808080u);
  }
  unsigned info = base;
  if (__ballot(((~aa & 0x80808080u) | (ao & 0x78787878u)) != 0u)) {
    unsigned em[4][2], c = 0;
#pragma unroll
    for (int j = 0; j < 4; ++j)
#pragma unroll
      for (int h = 0; h < 2; ++h) {
        const unsigned u = (hi[j][h] | 0x80808080u) - base4;
        em[j][h] = (~u | ((u & 0x78787878u) + 0x78787878u)) & 0x80808080u;
        c += __builtin_popcount(em[j][h]);
      }
    unsigned total = 0;
#pragma unroll
    for (int b = 0; b < 6; ++b) total += (unsigned)__builtin_popcountll(__ballot((c >> b) & 1u)) << b;
    if (total > (unsigned)MAXESC) {
      st16<0>(dst, u32x4{p[0][0], p[0][1], p[0][2], p[0][3]});
      if (slow) asm volatile("s_waitcnt vmcnt(0)" ::: "memory");
      st16<1024>(dst, u32x4{p[1][0], p[1][1], p[1][2], p[1][3]});
      if (slow) asm volatile("s_waitcnt vmcnt(0)" ::: "memory");
      st16<2048>(dst, u32x4{p[2][0], p[2][1], p[2][2], p[2][3]});
      if (slow) asm volatile("s_waitcnt vmcnt(0)" ::: "memory");
      st16<3072>(dst, u32x4{p[3][0], p[3][1], p[3][2], p[3][3]});
      if (slow) asm volatile("s_waitcnt vmcnt(0)" ::: "memory");
      return F_RAW;
    }
    unsigned n = 0;
#pragma unroll
    for (int j = 0; j < 4; ++j)
#pragma unroll
      for (int h = 0; h < 2; ++h) {
        if (__ballot(em[j][h] != 0u) == 0) continue;
        for (int b = 0; b < 4; ++b) {
          const bool on = (em[j][h] >> (8 * b + 7)) & 1u;
          const u64 bal = __ballot(on);
          const unsigned idx = n + __builtin_amdgcn_mbcnt_hi((unsigned)(bal >> 32), __builtin_amdgcn_mbcnt_lo((unsigned)bal, 0u));
          if (on) st4(dst - lane * 16 + 3072 + idx * 4,
                      (unsigned)(j * 512 + lane * 8 + h * 4 + b) | (((hi[j][h] >> (8 * b)) & 0xffu) << 16));
          n += (unsigned)__builtin_popcountll(bal);
        }
      }
    info |= total << 7;
  }
  st16(dst, u32x4{lo[0][0], lo[0][1], lo[1][0], lo[1][1]});
  if (slow & 1) asm volatile("s_waitcnt vmcnt(0)" ::: "memory");
  st16<1024>(dst, u32x4{lo[2][0], lo[2][1], lo[3][0], lo[3][1]});
  if (slow) asm volatile("s_waitcnt vmcnt(0)" ::: "memory");
  st16<2048>(dst, u32x4{nb[0], nb[1], nb[2], nb[3]});
  return info;
}

#define AC00 "v[40:43]"
#define AC01 "v[44:47]"
#define AC02 "v[48:51]"
#define AC03 "v[52:55]"
#define AC10 "v[56:59]"
#define AC11 "v[60:63]"
#define AC12 "v[64:67]"
#define AC13 "v[68:71]"
#define AC20 "v[72:75]"
#define AC21 "v[76:79]"
#define AC22 "v[80:83]"
#define AC23 "v[84:87]"
#define AC30 "v[88:91]"
#define AC31 "v[92:95]"
#define AC32 "v[96:99]"
#define AC33 "v[100:103]"
#define AC40 "v[104:107]"
#define AC41 "v[108:111]"
#define AC42 "v[112:115]"
#define AC43 "v[116:119]"
#define RA0 "v[0:3]"
#define RA1 "v[4:7]"
#define RA2 "v[8:11]"
#define RA3 "v[12:15]"
#define RB0 "v[16:19]"
#define RB1 "v[20:23]"
#define RB2 "v[24:27]"
#define RB3 "v[28:31]"
#define RC0 "v[32:35]"
#define RC1 "v[36:39]"
#define PFY(x) x
#define PFN(x)
#define MF_(d, a, b) "v_mfma_f32_16x16x32_bf16 " d ", " a ", " b ", " d "\n\t"
#define RD_(r, a, o) "ds_read_b128 " r ", " a " offset:" #o "\n\t"
#define WT_(n) "s_waitcnt lgkmcnt(" #n ")\n\t"
#define ADV_ "s_add_u32 %[sdn], %[sdn], 1\n\t"                    \
             "v_mov_b32 %[vt], %[sdn]\n\t"                        \
             "s_mov_b64 exec, 1\n\t"                              \
             "ds_write2_b32 %[vc], %[vt], %[vt] offset0:%[dn] offset1:%[dm]\n\t" \
             "s_mov_b64 exec, -1\n\t"                             \
             "s_cmp_eq_u32 %[sof], %[hl]\n\t"                     \
             "s_cselect_b32 %[sd], %[hw], %[hs]\n\t"              \
             "s_add_u32 %[sof], %[sof], %[sd]\n\t"                \
             "v_add_u32 %[vx], %[sd], %[vx]\n\t"                  \
             "v_add_u32 %[vw], %[sd], %[vw]\n\t"                  \
             "v_add_u32 %[vf], 4, %[vf]\n\t"
#define DBG1_
#define DBG2_
#define FETCH(W0, W1, W2, N0, N1, N2, N3)                           \
  DBG1_                                                             \
  "s_setprio 0\n\t"                                                \
  "6:\n\t"                                                         \
  "ds_read2_b32 v[16:17], %[vf] offset0:0 offset1:1\n\t"           \
  WT_(0)                                                            \
  "v_readfirstlane_b32 %[sd], v16\n\t"                             \
  "v_readfirstlane_b32 %[sfl], v17\n\t"                            \
  "s_cmp_lg_u32 %[sd], 0\n\t"                                      \
  "s_cbranch_scc1 7f\n\t"                                          \
  "s_sleep 1\n\t"                                                  \
  DBG2_                                                             \
  "s_branch 6b\n\t"                                                \
  "7:\n\t"                                                         \
  RD_(N0, "%[vw]", 0) RD_(N1, "%[vw]", 1024) RD_(N2, "%[vw]", 2048) RD_(N3, "%[vw]", 3072)   \
  RD_(W0, "%[vx]", 0) RD_(W1, "%[vx]", 1024) RD_(W2, "%[vx]", 2048)                          \
  RD_(RC0, "%[vx]", 3072) RD_(RC1, "%[vx]", 4096) ADV_
#define PRD_ "ds_read_b32 %[vu], %[vc] offset:%[dp]\n\t"
#define W78_ WT_(8)
#define W67_ WT_(7)
#define PRI_ "v_readfirstlane_b32 %[sd], %[vu]\n\t"                \
             "s_setprio 0\n\t"                                     \
             "s_cmp_gt_u32 %[sd], %[sdn]\n\t"                      \
             "s_cbranch_scc0 8f\n\t"                               \
             "s_setprio 1\n\t"                                     \
             "8:\n\t"
#define BODY(W0, W1, W2, W3, N0, N1, N2, N3, P, Q)                                           \
  P(RD_(N3, "%[vw]", 3072) WT_(6)) Q(WT_(5))                                                 \
  MF_(AC00, W0, N0) MF_(AC01, W1, N0) MF_(AC02, W2, N0) MF_(AC03, W3, N0)                    \
  P(RD_(N0, "%[vw]", 0) "ds_read_b32 %[vt], %[vf] offset:4\n\t" PRD_ W78_) Q(WT_(4))        \
  MF_(AC10, W0, N1) MF_(AC11, W1, N1) MF_(AC12, W2, N1) MF_(AC13, W3, N1)                    \
  P(RD_(N1, "%[vw]", 1024) W78_) Q(WT_(3))                                                   \
  MF_(AC20, W0, N2) MF_(AC21, W1, N2) MF_(AC22, W2, N2) MF_(AC23, W3, N2)                    \
  P(RD_(N2, "%[vw]", 2048) W78_) Q(WT_(2))                                                   \
  MF_(AC30, W0, RC0)                                                                         \
  P(W67_) Q(WT_(1))                                                                          \
  MF_(AC40, W0, RC1)                                                                         \
  P(RD_(W0, "%[vx]", 0))                                                                     \
  MF_(AC31, W1, RC0) MF_(AC41, W1, RC1)                                                      \
  P(RD_(W1, "%[vx]", 1024))                                                                  \
  MF_(AC32, W2, RC0) MF_(AC42, W2, RC1)                                                      \
  P(RD_(W2, "%[vx]", 2048))                                                                  \
  MF_(AC33, W3, RC0)                                                                         \
  P(RD_(RC0, "%[vx]", 3072))                                                                 \
  MF_(AC43, W3, RC1)                                                                         \
  P(RD_(RC1, "%[vx]", 4096) WT_(7) "v_readfirstlane_b32 %[sfl], %[vt]\n\t" PRI_ ADV_)

__global__ __launch_bounds__(NTHREADS, NWAVE / 4) void fused_gemm_q8(Params P) {
  __shared__ __attribute__((aligned(1024))) char lds[LDS_BYTES];
  const unsigned ldsbase = (unsigned)(uintptr_t)lds;
  const unsigned ctl = ldsbase + LDS_CTL;

  const int tid = threadIdx.x;
  const int lane = tid & 63;
  const int wave = __builtin_amdgcn_readfirstlane(tid >> 6);
  const int rank = P.rank;
  char* const myws = static_cast<char*>(P.ws_all[rank]);
  unsigned* const tmo = reinterpret_cast<unsigned*>(myws + OFF_TMO);

  if ((blockIdx.x >> 3) < 6)
    for (int i = tid; i < (NMIR + 5) * 64; i += NTHREADS)
      asm volatile("ds_write_b128 %0, %1" :: "v"(ldsbase + i * 16), "v"(u32x4{0u, 0u, 0u, 0u}) : "memory");
  if (tid < 384) lds_st(ctl + tid * 4, 0u);
  asm volatile("s_waitcnt lgkmcnt(0)\n\ts_barrier" ::: "memory");
  if (wave == 0) TS(0);
  if (tid == ((blockIdx.x >> 3) < 6 ? 64 * 15 : 64 * 10)) {
    const unsigned t = __hip_atomic_fetch_add(reinterpret_cast<unsigned*>(myws + OFF_TICKET), 1u,
                                              __ATOMIC_RELAXED, __HIP_MEMORY_SCOPE_AGENT);
    lds_st(ctl + C_CALL * 4, t / NWG + 1);
  }
  unsigned call = 0, tag = 0;
  auto get_call = [&]() {
    while ((call = __builtin_amdgcn_readfirstlane(lds_ld(ctl + C_CALL * 4))) == 0u) __builtin_amdgcn_s_sleep(1);
    tag = call << 13;
  };

  const int bx = blockIdx.x & 7, bk = ((blockIdx.x >> 3) + 26) & 31;
  const bool sender = bk >= 26;
  const int ti = bk < 24 ? 2 * (bk % 6) + (bx & 1) : 12;
  const int tj = 4 * (bx >> 1) + (bk < 24 ? bk / 6 : 2 * (bx & 1) + ((bk - 24) & 1));

  f32x4 acc[5][4];
  const int wm = wave & 3, wn = (wave >> 2) & 1;

  auto fetch = [&](int s, int q, int d, int f, f32x4 (&a)[8]) {
    if (d < 0) {
      const GAS float* sp = (const GAS float*)P.x + (size_t)(XROWS(q) * 128 + (f & 4) * 16 + (lane & 15)) * KSH + s * 64 + (f >> 3) * 32 + 8 * (lane >> 4);
#pragma unroll
      for (int j = 0; j < 4; ++j) {
        a[2 * j] = *(const GAS f32x4*)sp;
        a[2 * j + 1] = *(const GAS f32x4*)(sp + 4);
        sp += 16 * KSH;
        asm("" : "+v"(sp));
      }
    } else {
      const GAS float* sp = (const GAS float*)P.w + (size_t)(s * 64 + (f >> 3) * 32 + 8 * (lane >> 4)) * NTOT + d * NLOC + q * 128 +
                        (f & 4) * 16 + (lane & 15) * 4;
#pragma unroll
      for (int j = 0; j < 8; ++j) {
        a[j] = *(const GAS f32x4*)sp;
        sp += NTOT;
        asm("" : "+v"(sp));
      }
    }
  };
  auto round16 = [&](int d, const f32x4 (&a)[8], unsigned (&p)[4][4], float& m) {
    m = 0.f;
#pragma unroll
    for (int j = 0; j < 8; ++j)
#pragma unroll
      for (int c = 0; c < 4; ++c) m = fmaxf(m, fabsf(a[j][c]));
    if (d < 0) {
#pragma unroll
      for (int j = 0; j < 4; ++j) {
        p[j][0] = pk_bf16(a[2 * j][0], a[2 * j][1]); p[j][1] = pk_bf16(a[2 * j][2], a[2 * j][3]);
        p[j][2] = pk_bf16(a[2 * j + 1][0], a[2 * j + 1][1]); p[j][3] = pk_bf16(a[2 * j + 1][2], a[2 * j + 1][3]);
      }
    } else {
#pragma unroll
      for (int jj = 0; jj < 4; ++jj)
#pragma unroll
        for (int c = 0; c < 4; ++c) p[jj][c] = pk_bf16(a[2 * c][jj], a[2 * c + 1][jj]);
    }
  };
  auto convert = [&](int s, int q, int d, int f, unsigned (&p)[4][4], float& m) {
    f32x4 a[8];
    fetch(s, q, d, f, a);
    round16(d, a, p, m);
  };
  auto pack_items = [&](int g, int rounds) {
    gchar* const gws = (gchar*)myws;
    const int grp = (g >> 3) % 6;
    unsigned* const pq = reinterpret_cast<unsigned*>(myws + OFF_PQ + grp * 256);
    int n = g;
    for (int k = 0; n < NPACK; ++k) {
     unsigned tk = 0;
     if (k >= RS - 1 && lane == 0) tk = __hip_atomic_fetch_add(pq, 1u, __ATOMIC_RELAXED, __HIP_MEMORY_SCOPE_AGENT);
     {
      const int f = (n & 3) * 4, e = n >> 2;
      int s, q, d = -1;
      if (e < 32) { s = 0; q = e; }
      else if (e < 88) { s = 1; q = 8 + (e - 32) / 7; d = (rank + 1 + (e - 32) % 7) & 7; }
      else if (e < 1080) {
        const int r = (e - 88) % 160;
        s = (e - 88) / 160 + 1;
        if (r < 32) q = r;
        else if (r < 144) { s += 1; q = (r - 32) / 7; d = (rank + 1 + (r - 32) % 7) & 7; }
        else { s -= 1; q = r - 144; d = rank; }
      } else { s = 6 + ((e - 1080) >> 4); q = (e - 1080) & 15; d = rank; }
      unsigned p[4][4];
      float m;
      convert(s, q, d, f, p, m);
      unsigned info = 0;
      if (d != rank && e < 1048 + EU) {
        gchar* dst = gws + (d < 0 ? OFF_EX + (size_t)(s * XU + q) * UNIT : OFF_EW + (size_t)((d * NSTAGE + s) * WU + q) * UNIT);
        info = send_block(p, m, dst + f * 1024 + lane * 16, lane, false);
      }
      if (d < 0 || d == rank) {
        gchar* dst = gws + (d < 0 ? OFF_XS + (size_t)((rank * NSTAGE + s) * XU + q) * UNIT
                                  : OFF_WS + (size_t)((rank * NSTAGE + s) * WU + q) * UNIT);
        send_block(p, m, dst + f * 1024 + lane * 16, lane, true);
      }
      if (k == 0) get_call();
      asm volatile("s_waitcnt vmcnt(0)" ::: "memory");
      if (lane == 0) st4(gws + OFF_EF + (size_t)n * 4, tag | info);
      if (lane == 1 && (d < 0 || d == rank))
        st4(gws + (d < 0 ? OFF_XF : OFF_WF) + (size_t)(q * 64 + rank * 8 + s) * 16 + f, tag | F_RAW);
     }
     n += PRN;
     if (k >= RS - 1) {
       const unsigned t = __builtin_amdgcn_readfirstlane(tk) - (call - 1) * ((NPACK - RS * PRN + 5 - grp) / 6 + PRN / 6);
       n = t < 4096u ? RS * PRN + 6 * (int)t + grp : NPACK;
     }
    }
  };

  auto locate = [&](int u, int f, gchar*& dst, gu32*& flag, int& pn, unsigned& so, int& pe) -> bool {
      const int pr = u % 7, d = (rank + 1 + pr) & 7;
      int s, q, rr;
      unit_of(u / 7, pr, s, q, rr);
      gchar* dws;
      asm volatile("s_load_dwordx2 %0, %1, %2\n\ts_waitcnt lgkmcnt(0)"
                   : "=s"(dws) : "s"(P.ws_all), "s"(__builtin_amdgcn_readfirstlane(d * 8)));
      if (rr != 2) {
        const int xu = 2 * q + rr;
        so = (unsigned)OFF_EX + (unsigned)(s * XU + xu) * UNIT;
        dst = dws + OFF_XS + ((size_t)((rank * NSTAGE + s) * XU + xu)) * UNIT;
        pn = s | xu << 3;
      } else {
        so = (unsigned)OFF_EW + (unsigned)((d * NSTAGE + s) * WU + q) * UNIT;
        dst = dws + OFF_WS + ((size_t)((rank * NSTAGE + s) * WU + q)) * UNIT;
        pn = s | q << 3 | (d + 1) << 8;
      }
      pe = packed_at(s, q, rr, pr);
      flag = (gu32*)(dws + OFF_F1 + (size_t)u * F1S) + f;
      const bool tail = u >= NUNIT - RTU;
      if (tail) {
        so = (unsigned)OFF_XS + (unsigned)((rank * NSTAGE + s) * XU + 2 * q + rr) * UNIT;
        flag = (gu32*)(dws + OFF_XF + ((2 * q + rr) * 64 + rank * 8 + s) * 16) + f;
      }
      so = __builtin_amdgcn_readfirstlane(so + f * 4096);
      dst += f * 4096 + lane * 16;
      pn |= f << 12;
      return tail;
  };

  if (sender) {
   if (QX & 8) {
   } else if (wave < NSEND) {
    const __amdgpu_buffer_rsrc_t rs = __builtin_amdgcn_make_buffer_rsrc(myws, 0, 0x7fffffff, 0x00020000);
    unsigned so, info, ia = 0;
    int pn;
    int dbg = 0, pdbg = 0;
    gchar* dst;
    gu32* flag;
    u32x4 va, vb, vc;
    unsigned ve;
    const u64 wsp = (u64)(uintptr_t)P.ws_all[lane & 7];
    auto draw = [&](int look) -> bool {
      unsigned j = 0, e = 0;
      if (lane == 0)
        asm volatile("ds_add_rtn_u32 %0, %1, %2\n\ts_waitcnt lgkmcnt(0)" : "=v"(j) : "v"(ctl + C_NEXT * 4), "v"(1u) : "memory");
      j = __builtin_amdgcn_readfirstlane(j);
      int f = j & 3;
      int u = U0 + (int)(j >> 2) * NSWG + (bk - 26) * 8 + bx;
      if (j >= (unsigned)(S1 * 4)) {
        while (((e = __builtin_amdgcn_readfirstlane(lds_ld(ctl + (C_Q + ((j >> 2) & 63)) * 4))) >> 16) != (j >> 2) + 1)
          __builtin_amdgcn_s_sleep(1);
        if (FAST && (e & 0x8000u)) {
          const int xu = e & 31, d = (e >> 5) & 7, fb = (e >> 12) & 3;
          so = (unsigned)OFF_XS + (unsigned)((rank * NSTAGE + 7) * XU + xu) * UNIT + fb * 4096 + f * 1024;
          va = __builtin_amdgcn_raw_buffer_load_b128(rs, lane * 16, so, 17);
          gchar* dws = (gchar*)(((u64)(unsigned)__builtin_amdgcn_readlane((int)(wsp >> 32), d) << 32) |
                                (unsigned)__builtin_amdgcn_readlane((int)wsp, d));
          dst = dws + so + lane * 16;
          flag = (gu32*)(dws + OFF_XF + (xu * 64 + rank * 8 + 7) * 16) + fb;
          pn = 1 << 30;
          info = tag | F_RAW;
          ia = ldsbase + (NMIR + 4) * 1024 + ((j >> 2) & 127) * 4;
          dbg = 4 * NUNIT + blockIdx.x * 16 + wave;
          if (TSL) __builtin_amdgcn_s_sleep(TSL);
          asm volatile("" : "+v"(ia), "+v"(flag), "+v"(pn), "+v"(so), "+v"(info));
          return true;
        }
        u = e & 4095;
        if (u == 4095) return false;
        if (e & 0x4000u) f = (e >> 12) & 3;
      }
      int pe;
      const bool piece = locate(u, f, dst, flag, pn, so, pe);
      dbg = u * 4 + f;
      ia = ldsbase + (pe * 4 + f) * 4;
      asm volatile("" : "+v"(ia), "+v"(flag));
      info = __builtin_amdgcn_readfirstlane(lds_ld(ia));
      for (; look > 0 && ((info ^ tag) >> 13) != 0u; --look) {
        __builtin_amdgcn_s_sleep(8);
        info = __builtin_amdgcn_readfirstlane(lds_ld(ia));
      }
      if (piece) {
        pn |= 1 << 30 | (j & 3) << 14;
        so += (j & 3) * 1024;
        dst += (j & 3) * 1024;
        ia = ldsbase + (NMIR + 4) * 1024 + ((j >> 2) & 127) * 4;
        info |= F_RAW;
      }
      if (((info ^ tag) >> 13) == 0u) {
        va = __builtin_amdgcn_raw_buffer_load_b128(rs, lane * 16, so, 17);
        if (!piece) {
          vb = __builtin_amdgcn_raw_buffer_load_b128(rs, lane * 16 + 1024, so, 17);
          vc = __builtin_amdgcn_raw_buffer_load_b128(rs, lane * 16 + 2048, so, 17);
          ve = __builtin_amdgcn_raw_buffer_load_b32(rs, (lane & 15) * 4 + 3072, so, 17);
        }
      }
      asm volatile("" : "+v"(pn), "+v"(so), "+v"(info));
      return true;
    };
    gu32* pflag = nullptr;
    unsigned pv = 0, pc = 0;
    auto finish = [&]() {
      unsigned c = 3;
      if (pc != 0u && lane == 0)
        asm volatile("ds_add_rtn_u32 %0, %1, %2\n\ts_waitcnt lgkmcnt(0)" : "=v"(c) : "v"(pc), "v"(1u) : "memory");
      if (pflag != nullptr && lane == 0 && (c & 3u) == 3u)
        __hip_atomic_store(pflag, pv, __ATOMIC_RELAXED, __HIP_MEMORY_SCOPE_SYSTEM);
    };
    va = vb = vc = u32x4{0u, 0u, 0u, 0u}; ve = 0; info = 0; so = 0; pn = 0; dst = nullptr; flag = nullptr;
    get_call();
    draw(FWAIT);
    while (true) {
      asm volatile("s_waitcnt vmcnt(0)" : "+v"(va), "+v"(vb), "+v"(vc), "+v"(ve) :: "memory");
      finish();
      if (pflag != nullptr) T2(3, pdbg);
      pdbg = dbg;
      u32x4 ca = va, cb = vb, cc = vc;
      unsigned ce = ve;
      const int cpn = __builtin_amdgcn_readfirstlane(pn);
      const unsigned cso = so, cia = ia;
      gchar* const cdst = dst;
      pflag = flag;
      pv = __builtin_amdgcn_readfirstlane(info);
      pc = (cpn >> 30) ? cia : 0u;
      const bool more = draw(0);
      if (!(cpn >> 30) && ((pv ^ tag) >> 13) != 0u) {
        pv = __builtin_amdgcn_readfirstlane(lds_ld(cia));
        if (((pv ^ tag) >> 13) == 0u) {
          ca = __builtin_amdgcn_raw_buffer_load_b128(rs, cso + lane * 16, 0, 17);
          cb = __builtin_amdgcn_raw_buffer_load_b128(rs, cso + lane * 16 + 1024, 0, 17);
          cc = __builtin_amdgcn_raw_buffer_load_b128(rs, cso + lane * 16 + 2048, 0, 17);
          ce = __builtin_amdgcn_raw_buffer_load_b32(rs, cso + (lane & 15) * 4 + 3072, 0, 17);
        }
      }
      if (((pv ^ tag) >> 13) != 0u) {
        unsigned p[4][4];
        float m;
        convert(cpn & 7, (cpn >> 3) & 31, ((cpn >> 8) & 15) - 1, (cpn >> 10) & 12, p, m);
        if (cpn >> 30) {
          int g = cpn;
          asm volatile("" : "+v"(g));
#pragma unroll
          for (int i = 0; i < 4; ++i) {
            unsigned p0 = p[0][i], p1 = p[1][i], p2 = p[2][i], p3 = p[3][i];
            asm volatile("" : "+v"(p0), "+v"(p1), "+v"(p2), "+v"(p3));
            const unsigned a = (g & (1 << 14)) ? p1 : p0, b = (g & (1 << 14)) ? p3 : p2;
            ca[i] = (g & (2 << 14)) ? b : a;
          }
          pv = tag | F_RAW;
          st16(cdst, ca);
        } else {
          pv = tag | send_block(p, m, cdst, lane, false, DSLOW);
        }
      } else {
        st16(cdst, ca);
        if (!(cpn >> 30)) {
          asm volatile("s_waitcnt vmcnt(0)" ::: "memory");
          st16<1024>(cdst, cb);
          asm volatile("s_waitcnt vmcnt(0)" ::: "memory");
          st16<2048>(cdst, cc);
          if (pv & F_RAW) st16<3072>(cdst, __builtin_amdgcn_raw_buffer_load_b128(rs, cso + lane * 16 + 3072, 0, 17));
          else if ((pv & (31u << 7)) && lane < MAXESC) st4(cdst - lane * 12 + 3072, ce);
        }
      }
      if (!more) break;
    }
    asm volatile("s_waitcnt vmcnt(0)" ::: "memory");
    finish();
    T2(3, pdbg);
    if (wave < 2) TS(1 + wave);
   } else if (wave < NSEND + NSC) {
    get_call();
    const u64 t0 = __builtin_amdgcn_s_memrealtime();
    if (wave > NSEND) __builtin_amdgcn_s_sleep(LOS);
    int c = wave - NSEND < NLO ? 0 : NMIR;
    for (unsigned spins = 1; c < NMIR; ++spins) {
      {
        unsigned keep;
        asm volatile("s_mov_b32 %0, m0\n\ts_mov_b32 m0, %2\n\ts_nop 0\n\t"
                     "global_load_lds_dwordx4 %1, off sc0 sc1\n\t"
                     "global_load_lds_dwordx4 %1, off offset:1024 sc0 sc1\n\t"
                     "s_mov_b32 m0, %0"
                     : "=&s"(keep) : "v"(myws + OFF_EF + c * 1024 + lane * 16), "s"(ldsbase + c * 1024) : "memory");
      }
      asm volatile("s_waitcnt vmcnt(0)" ::: "memory");
      for (int i = 0; i < LW && c < NMIR; ++i, ++c) {
        u32x4 v;
        asm volatile("ds_read_b128 %0, %1\n\ts_waitcnt lgkmcnt(0)" : "=v"(v) : "v"(ldsbase + c * 1024 + lane * 16) : "memory");
        if (!__all((((v[0] ^ tag) | (v[1] ^ tag) | (v[2] ^ tag) | (v[3] ^ tag)) >> 13) == 0u)) break;
      }
      if ((spins & 15) == 0 &&
          (__hip_atomic_load(tmo, __ATOMIC_RELAXED, __HIP_MEMORY_SCOPE_AGENT) == call ||
           __builtin_amdgcn_s_memrealtime() - t0 > DEADLINE_TICKS)) {
        if (lane == 0) __hip_atomic_store(tmo, call, __ATOMIC_RELAXED, __HIP_MEMORY_SCOPE_AGENT);
        break;
      }
      __builtin_amdgcn_s_sleep(SCS);
    }
    if (wave == NSEND) TS(33);
   } else if (wave == NWAVE - 1) {
    get_call();
    unsigned* const sq = reinterpret_cast<unsigned*>(myws + OFF_SQ);
    const unsigned qbase = (call - 1) * NDRAW - U0 - S1 * NSWG;
    unsigned ahead = TKD;
    for (unsigned k = S1;; ++k) {
      while ((lds_ld(ctl + C_NEXT * 4) >> 2) + ahead < k) __builtin_amdgcn_s_sleep(2);
      unsigned t = 0;
      if (lane == 0) t = __hip_atomic_fetch_add(sq, 1u, __ATOMIC_RELAXED, __HIP_MEMORY_SCOPE_AGENT);
      const unsigned u = __builtin_amdgcn_readfirstlane(t) - qbase;
      if (u >= (unsigned)NUNIT) {
        if (lane < 8) lds_st(ctl + (C_Q + ((k + lane) & 63)) * 4, ((k + lane + 1) << 16) | 4095u);
        break;
      }
      if (u >= (unsigned)NT0) {
        const int t = 4 * ((int)u - NT0) + (lane & 3), ii = NT0 / 7 + t / 28, r = t % 28, f = r / 7, pr = r % 7;
        int s, q, rr;
        unit_of(ii, pr, s, q, rr);
        const unsigned w = lds_ld(ldsbase + (packed_at(s, q, rr, pr) * 4 + f) * 4);
        const unsigned what = FAST && ((w ^ tag) >> 13) == 0u ? 0x8000u | ((rank + 1 + pr) & 7) << 5 | (2 * q + rr)
                                                              : 0x4000u | (ii * 7 + pr);
        if (lane < 4) lds_st(ctl + (C_Q + ((k + lane) & 63)) * 4, ((k + lane + 1) << 16) | f << 12 | what);
        k += 3;
        ahead = TKT;
      } else if (lane == 0) {
        lds_st(ctl + (C_Q + (k & 63)) * 4, ((k + 1) << 16) | u);
      }
    }
   }
  } else if (wave < 8) {
    if (wave < KW && !(QX & 8)) {
      const int ba = wave * NCW + bk * 8 + bx, bb = ba + KW * NCW;
      gchar *da, *db;
      gu32 *fa, *fb;
      int pa, pb, pe;
      unsigned so, p[4][4];
      float m;
      f32x4 ra[8], rb[8];
      locate(ba >> 2, ba & 3, da, fa, pa, so, pe);
      locate(bb >> 2, bb & 3, db, fb, pb, so, pe);
      T2(2, ba);
      T2(2, bb);
      fetch(pa & 7, (pa >> 3) & 31, ((pa >> 8) & 15) - 1, (pa >> 10) & 12, ra);
      fetch(pb & 7, (pb >> 3) & 31, ((pb >> 8) & 15) - 1, (pb >> 10) & 12, rb);
      round16(((pa >> 8) & 15) - 1, ra, p, m);
      const unsigned ia = send_block(p, m, da, lane, false, KSLOW);
      round16(((pb >> 8) & 15) - 1, rb, p, m);
      get_call();
      asm volatile("s_waitcnt vmcnt(0)" ::: "memory");
      if (lane == 0) __hip_atomic_store(fa, tag | ia, __ATOMIC_RELAXED, __HIP_MEMORY_SCOPE_SYSTEM);
      T2(3, ba);
      const unsigned ib = send_block(p, m, db, lane, false, KSLOW);
      asm volatile("s_waitcnt vmcnt(0)" ::: "memory");
      if (lane == 0) __hip_atomic_store(fb, tag | ib, __ATOMIC_RELAXED, __HIP_MEMORY_SCOPE_SYSTEM);
      T2(3, bb);
    }
    if (wave == 0) TS(30);
#pragma unroll
    for (int a = 0; a < 5; ++a)
#pragma unroll
      for (int b = 0; b < 4; ++b) acc[a][b] = f32x4{0.f, 0.f, 0.f, 0.f};
    const int xoff = (wm * 5) * 1024 + lane * 16;
    const int woff = 20480 + (wn * 4) * 1024 + lane * 16;
    const u64 c0 = CYC(), r0 = __builtin_amdgcn_s_memrealtime();
    {
      unsigned vx = ldsbase + xoff, vw = ldsbase + woff, vf = ctl + C_FULL * 4, vc = ctl + wave * 4, vt, vu;
      unsigned sfl, sdn, sof, sd, sdb = 0;
      asm volatile(
          "s_mov_b32 %[sdn], 0\n\t"
          "s_mov_b32 %[sof], 0\n\t"
          "s_nop 4\n\t"
          FETCH(RB0, RB1, RB2, RA0, RA1, RA2, RA3)
          "1:\n\t"
          "s_cmp_lg_u32 %[sfl], 0\n\t"
          "s_cbranch_scc0 3f\n\t"
          BODY(RA0, RA1, RA2, RA3, RB0, RB1, RB2, RB3, PFY, PFN)
          "2:\n\t"
          "s_cmp_lg_u32 %[sfl], 0\n\t"
          "s_cbranch_scc0 4f\n\t"
          BODY(RB0, RB1, RB2, RB3, RA0, RA1, RA2, RA3, PFY, PFN)
          "s_branch 1b\n\t"
          "3:\n\t"
          BODY(RA0, RA1, RA2, RA3, RB0, RB1, RB2, RB3, PFN, PFY)
          FETCH(RA0, RA1, RA2, RB0, RB1, RB2, RB3)
          "s_branch 2b\n\t"
          "4:\n\t"
          BODY(RB0, RB1, RB2, RB3, RA0, RA1, RA2, RA3, PFN, PFY)
          "s_cmp_eq_u32 %[sdn], %[ns]\n\t"
          "s_cbranch_scc1 5f\n\t"
          FETCH(RB0, RB1, RB2, RA0, RA1, RA2, RA3)
          "s_branch 1b\n\t"
          "5:\n\t"
          "s_setprio 0\n\t"
          "s_nop 15\n\t"
          "s_nop 15\n\t"
          :
          "+{v[40:43]}"(acc[0][0]), "+{v[44:47]}"(acc[0][1]), "+{v[48:51]}"(acc[0][2]), "+{v[52:55]}"(acc[0][3]),
          "+{v[56:59]}"(acc[1][0]), "+{v[60:63]}"(acc[1][1]), "+{v[64:67]}"(acc[1][2]), "+{v[68:71]}"(acc[1][3]),
          "+{v[72:75]}"(acc[2][0]), "+{v[76:79]}"(acc[2][1]), "+{v[80:83]}"(acc[2][2]), "+{v[84:87]}"(acc[2][3]),
          "+{v[88:91]}"(acc[3][0]), "+{v[92:95]}"(acc[3][1]), "+{v[96:99]}"(acc[3][2]), "+{v[100:103]}"(acc[3][3]),
          "+{v[104:107]}"(acc[4][0]), "+{v[108:111]}"(acc[4][1]), "+{v[112:115]}"(acc[4][2]), "+{v[116:119]}"(acc[4][3]),
          [vx] "+v"(vx), [vw] "+v"(vw), [vf] "+v"(vf), [vt] "=&v"(vt), [vu] "=&v"(vu), [sfl] "=&s"(sfl), [sdn] "=&s"(sdn),
          [sof] "=&s"(sof), [sd] "=&s"(sd), [sdb] "+s"(sdb)
          : [vc] "v"(vc), [dn] "n"(C_DONE), [dm] "n"(C_DONE + 8), [dp] "n"((C_DONE + 4) * 4), [hs] "n"(HS), [hl] "n"((NH - 1) * HS), [hw] "s"(-(NH - 1) * HS),
            [ns] "n"(2 * NSTEP)
          : "memory", "scc", "v0", "v1", "v2", "v3", "v4", "v5", "v6", "v7", "v8", "v9", "v10", "v11", "v12", "v13", "v14", "v15", "v16", "v17", "v18", "v19", "v20", "v21", "v22", "v23", "v24", "v25", "v26", "v27", "v28", "v29", "v30", "v31", "v32", "v33", "v34", "v35", "v36", "v37", "v38", "v39");
      TV(48 + wave, sdb);
    }
    if (wave == 0) { TV(40, CYC() - c0); TV(41, __builtin_amdgcn_s_memrealtime() - r0); }
    float m = 0.f;
#pragma unroll
    for (int a = 0; a < 5; ++a) {
      if (ti * TM + wm * 80 + a * 16 >= 4096) continue;
#pragma unroll
      for (int b = 0; b < 4; ++b)
#pragma unroll
        for (int j = 0; j < 4; ++j) m = fmaxf(m, fabsf(acc[a][b][j]));
    }
#pragma unroll
    for (int o = 32; o >= 1; o >>= 1) m = fmaxf(m, __shfl_xor(m, o));
    if (lane == 0) lds_st(ctl + (C_WMAX + wave) * 4, __float_as_uint(m));
    if (wave == 0) TS(12);
  } else if (wave < 10) {
    const int L = wave - 8;
    __builtin_amdgcn_s_setprio(LPRIO);
    get_call();
    {
      const u64 t0 = __builtin_amdgcn_s_memrealtime();
      while (__builtin_amdgcn_s_memrealtime() - t0 < DLY * 100ull) __builtin_amdgcn_s_sleep(32);
    }
    const char* const gx0 = myws + OFF_XS + L * 8192 + lane * 16;
    const char* const gw0 = myws + OFF_WS + (size_t)tj * UNIT + L * 8192 + lane * 16;
    int slot = L;
    const unsigned dna = ctl + (C_DONE + (lane & 7)) * 4;
    u64 rm = 0, avail = 0, left = ~0ull;
    unsigned dv = 0;
    for (int t = 0; t < NSTEP; ++t) {
      const int need = 2 * t + L - NH + 1;
      while (avail == 0 || !__all((int)dv >= need)) {
        __builtin_amdgcn_s_sleep(1);
        asm volatile("ds_read_b64 %0, %2\n\tds_read_b32 %1, %3\n\ts_waitcnt lgkmcnt(0)"
                     : "=&v"(rm), "=&v"(dv) : "v"(ctl + C_Q * 4), "v"(dna) : "memory");
        avail = left & (((u64)(unsigned)__builtin_amdgcn_readfirstlane((int)(rm >> 32)) << 32) |
                        (u64)(unsigned)__builtin_amdgcn_readfirstlane((int)rm));
      }
      const int c = __builtin_ctzll(avail);
      left &= ~(1ull << c);
      const int bit = ((rank - (c & 7)) & 7) * 8 + (c >> 3);
      asm volatile("s_waitcnt vmcnt(29)" ::: "memory");
      const unsigned l = ldsbase + slot * HS;
#pragma unroll
      for (int i = 0; i < 4; ++i) {
        const int hu = 5 * ti + i;
        glds4k(gx0 + ((size_t)(bit * XU + (hu >> 1))) * UNIT + (hu & 1) * 4096, l + i * 4096);
      }
      {
        const int hu = min(5 * ti + 4, 63);
        const char* g0 = gx0 + ((size_t)(bit * XU + (hu >> 1))) * UNIT + (hu & 1) * 4096;
        const char* g1 = gw0 + (size_t)(bit * WU) * UNIT;
        unsigned keep;
        asm volatile(
            "ds_read_b64 %0, %3\n\tds_read_b32 %1, %4\n\t"
            "s_mov_b32 %2, m0\n\ts_mov_b32 m0, %9\n\ts_nop 0\n\t"
            "global_load_lds_dwordx4 %5, off" LDPOL "\n\t"
            "global_load_lds_dwordx4 %5, off offset:1024" LDPOL "\n\t"
            "global_load_lds_dwordx4 %5, off offset:2048" LDPOL "\n\t"
            "global_load_lds_dwordx4 %5, off offset:3072" LDPOL "\n\t"
            "s_add_u32 m0, m0, 0x1000\n\ts_nop 0\n\t"
            "global_load_lds_dwordx4 %6, off" LDPOL "\n\t"
            "global_load_lds_dwordx4 %6, off offset:1024" LDPOL "\n\t"
            "global_load_lds_dwordx4 %6, off offset:2048" LDPOL "\n\t"
            "global_load_lds_dwordx4 %6, off offset:3072" LDPOL "\n\t"
            "s_add_u32 m0, m0, 0x1000\n\ts_nop 0\n\t"
            "global_load_lds_dwordx4 %7, off" LDPOL "\n\t"
            "global_load_lds_dwordx4 %7, off offset:1024" LDPOL "\n\t"
            "global_load_lds_dwordx4 %7, off offset:2048" LDPOL "\n\t"
            "global_load_lds_dwordx4 %7, off offset:3072" LDPOL "\n\t"
            "s_mov_b32 m0, %10\n\ts_mov_b64 exec, 1\n\t"
            "global_load_lds_dword %8, off sc0 sc1\n\t"
            "s_mov_b64 exec, -1\n\ts_mov_b32 m0, %2\n\t"
            "s_waitcnt lgkmcnt(0)"
            : "=&v"(rm), "=&v"(dv), "=&s"(keep)
            : "v"(ctl + C_Q * 4), "v"(dna), "v"(g0), "v"(g1), "v"(g1 + 4096), "v"(myws + OFF_TICKET),
              "s"(l + 16384), "s"(ctl + (C_FULL + 2 * t + L) * 4)
            : "memory", "scc");
        avail = left & (((u64)(unsigned)__builtin_amdgcn_readfirstlane((int)(rm >> 32)) << 32) |
                        (u64)(unsigned)__builtin_amdgcn_readfirstlane((int)rm));
      }
      slot = slot + 2 >= NH ? slot + 2 - NH : slot + 2;
    }
    asm volatile("s_waitcnt vmcnt(0)" ::: "memory");
    if (L == 0) TS(11);
  } else if (wave == 10) {
    get_call();
    const unsigned want = tag | F_RAW;
    const unsigned m2 = ti == NTI - 1 ? 0u : ~0u;
    const int st = ((rank - (lane & 7)) & 7) * 8 + (lane >> 3);
    const unsigned* pf0 = reinterpret_cast<const unsigned*>(myws + OFF_XF) + (((5 * ti) >> 1) * 64 + st) * SU;
    const unsigned* pf2 = reinterpret_cast<const unsigned*>(myws + OFF_WF) + (tj * 64 + st) * SU;
    u64 seen = 0;
    unsigned cnt = 0, spins = 0, kk = 0;
    bool dead = false;
    const u64 t0 = __builtin_amdgcn_s_memrealtime();
    while (cnt < (unsigned)NSTEP) {
      u32x4 f0, f1, f2, f3;
      asm volatile(
                   "s_mov_b64 exec, %6\n\t"
                   "global_load_dwordx4 %0, %4, off sc0 sc1\n\tglobal_load_dwordx4 %1, %4, off offset:1024 sc0 sc1\n\t"
                   "global_load_dwordx4 %2, %4, off offset:2048 sc0 sc1\n\t"
                   "global_load_dwordx4 %3, %5, off sc0 sc1\n\ts_mov_b64 exec, -1\n\ts_waitcnt vmcnt(0)"
                   : "=&v"(f0), "=&v"(f1), "=&v"(f2), "=&v"(f3) : "v"(pf0), "v"(pf2), "s"(~seen) : "memory");
      const u64 nm = __ballot((QX & 1) || dead || plain16(f0, f1, f2, f3, want, m2)) & ~seen;
      if (nm) {
        if (lane == 0) asm volatile("ds_write_b64 %0, %1" :: "v"(ctl + C_Q * 4), "v"(seen | nm) : "memory");
        cnt += (unsigned)__builtin_popcountll(nm);
        seen |= nm;
      } else if ((++spins & 15) == 0 &&
                 (__hip_atomic_load(tmo, __ATOMIC_RELAXED, __HIP_MEMORY_SCOPE_AGENT) == call ||
                  __builtin_amdgcn_s_memrealtime() - t0 > DEADLINE_TICKS)) {
        if (lane == 0) __hip_atomic_store(tmo, call, __ATOMIC_RELAXED, __HIP_MEMORY_SCOPE_AGENT);
        dead = true;
      }
      if (cnt < WFAST) __builtin_amdgcn_s_sleep(PSLEEP); else __builtin_amdgcn_s_sleep(PS2);
    }
    if (dead && lane == 0) lds_st(ctl + C_DEAD * 4, ~0u);
    TS(32);
    if (GOX && ER && ti < 4) {
      while ((int)(__hip_atomic_load(reinterpret_cast<unsigned*>(myws + OFF_GO), __ATOMIC_RELAXED, __HIP_MEMORY_SCOPE_AGENT) -
                   (call - 1) * NSWG) < GOX &&
             __builtin_amdgcn_s_memrealtime() - t0 < DEADLINE_TICKS)
        __builtin_amdgcn_s_sleep(32);
    }
  } else {
   const __amdgpu_buffer_rsrc_t rs = __builtin_amdgcn_make_buffer_rsrc(myws, 0, 0x7fffffff, 0x00020000);
   const u64 t0 = __builtin_amdgcn_s_memrealtime();
   bool dead = false;
   auto arrival = [&](int n, unsigned& so, unsigned& fo) {
      const int pr = n % 7;
      int s, q, rr;
      unit_of(n / 7, pr, s, q, rr);
      const int st = ((rank + 7 - pr) & 7) * 8 + s;
      if (rr != 2) {
        so = (unsigned)OFF_XS + (unsigned)(st * XU + 2 * q + rr) * UNIT;
        fo = (unsigned)OFF_XF + ((2 * q + rr) * 64 + st) * 16;
      } else {
        so = (unsigned)OFF_WS + (unsigned)(st * WU + q) * UNIT;
        fo = (unsigned)OFF_WF + (q * 64 + st) * 16;
      }
      so = __builtin_amdgcn_readfirstlane(so);
      fo = __builtin_amdgcn_readfirstlane(fo);
   };
   auto expand = [&](unsigned f, u32x4 a, u32x4 b, u32x4 c, unsigned e, unsigned so) {
        const unsigned ne = (f >> 7) & 31u;
        const unsigned base4 = (f & 0x7fu) * 0x01010101u;
        u32x4 o[4];
        o[0] = unpack8(a[0], a[1], c[0], base4);
        o[1] = unpack8(a[2], a[3], c[1], base4);
        o[2] = unpack8(b[0], b[1], c[2], base4);
        o[3] = unpack8(b[2], b[3], c[3], base4);
        for (unsigned i = 0; i < ne && i < (unsigned)MAXESC; ++i) {
          const unsigned ev = (unsigned)__builtin_amdgcn_readlane((int)e, (int)i);
          const unsigned sh = (ev & 1u) * 16 + 8, r = ((ev >> 7) & 12u) | ((ev >> 1) & 3u);
          const unsigned vm = (unsigned)lane == ((ev >> 3) & 63u) ? 0xffu << sh : 0u;
          const unsigned vv = ((ev >> 16) & 0xffu) << sh;
#pragma unroll
          for (int kk = 0; kk < 16; ++kk) {
            const unsigned mk = r == (unsigned)kk ? vm : 0u;
            o[kk >> 2][kk & 3] = (o[kk >> 2][kk & 3] & ~mk) | (vv & mk);
          }
        }
#pragma unroll
        for (int j = 0; j < 4; ++j)
          __builtin_amdgcn_raw_buffer_store_b128(o[j], rs, lane * 16 + j * 1024, so, 17);
   };
   if (wave < W_DEC + NDEC) {
    __builtin_amdgcn_s_setprio(DPRIO);
    get_call();
    for (int n = (wave - W_DEC) * NCW + bk * 8 + bx; n < NT0 - NL && !dead && !(QX & 16); n += NDEC * NCW) {
      unsigned so, fo;
      arrival(n, so, fo);
      unsigned f[4];
      for (unsigned spins = 1;; ++spins) {
        asm volatile("" ::: "memory");
        const u32x4 fv = __builtin_amdgcn_raw_buffer_load_b128(rs, 0, (unsigned)OFF_F1 + n * F1S, 17);
#pragma unroll
        for (int k = 0; k < 4; ++k) f[k] = __builtin_amdgcn_readfirstlane(fv[k]);
        if ((((f[0] ^ tag) | (f[1] ^ tag) | (f[2] ^ tag) | (f[3] ^ tag)) >> 13) == 0u) break;
        if ((spins & 15) == 0 &&
            (__hip_atomic_load(tmo, __ATOMIC_RELAXED, __HIP_MEMORY_SCOPE_AGENT) == call ||
             __builtin_amdgcn_s_memrealtime() - t0 > DEADLINE_TICKS)) {
          if (lane == 0) __hip_atomic_store(tmo, call, __ATOMIC_RELAXED, __HIP_MEMORY_SCOPE_AGENT);
          dead = true;
          break;
        }
        __builtin_amdgcn_s_sleep(DSLEEP);
      }
      if (dead) continue;
      T2(0, n);
      u32x4 a[4], b[4], c[4];
      unsigned e[4];
#pragma unroll
      for (int k = 0; k < 4; ++k) {
        a[k] = __builtin_amdgcn_raw_buffer_load_b128(rs, lane * 16 + k * 4096, so, 17);
        b[k] = __builtin_amdgcn_raw_buffer_load_b128(rs, lane * 16 + k * 4096 + 1024, so, 17);
        c[k] = __builtin_amdgcn_raw_buffer_load_b128(rs, lane * 16 + k * 4096 + 2048, so, 17);
        e[k] = __builtin_amdgcn_raw_buffer_load_b32(rs, (lane & 15) * 4 + k * 4096 + 3072, so, 17);
      }
#pragma unroll
      for (int k = 0; k < 4; ++k) {
        if (f[k] & F_RAW) continue;
        expand(f[k], a[k], b[k], c[k], e[k], so + k * 4096);
      }
      asm volatile("s_waitcnt vmcnt(0)" ::: "memory");
      if (lane == 0) {
        const unsigned w = tag | F_RAW;
        __builtin_amdgcn_raw_buffer_store_b128(u32x4{w, w, w, w}, rs, 0, fo, 17);
      }
      T2(1, n);
    }
    if (wave == W_DEC) TS(19);
   } else {
    __builtin_amdgcn_s_setprio(PPRIO);
    if (!(QX & 32)) pack_items((wave - W_DEC - NDEC + 8 * CPK) * NCW + bk * 8 + bx, NROUND);
    if (wave == NWAVE - 1) TS(31);
    while (NL && __builtin_popcount(__builtin_amdgcn_readfirstlane(lds_ld(ctl + C_Q * 4 + 4))) < 8) __builtin_amdgcn_s_sleep(32);
    __builtin_amdgcn_s_setprio(DPRIO);
   }
   for (int m = ((wave - W_DEC + NPK) % (NDEC + NPK)) * NCW + bk * 8 + bx; m < NL * 4 && !dead; m += (NDEC + NPK) * NCW) {
      const int n = NT0 - NL + (m >> 2), k = m & 3;
      unsigned so, fo, f;
      arrival(n, so, fo);
      so += k * 4096;
      for (unsigned spins = 1;; ++spins) {
        asm volatile("" ::: "memory");
        f = __builtin_amdgcn_readfirstlane(__builtin_amdgcn_raw_buffer_load_b32(rs, 0, (unsigned)OFF_F1 + n * F1S + k * 4, 17));
        if (((f ^ tag) >> 13) == 0u) break;
        if ((spins & 15) == 0 &&
            (__hip_atomic_load(tmo, __ATOMIC_RELAXED, __HIP_MEMORY_SCOPE_AGENT) == call ||
             __builtin_amdgcn_s_memrealtime() - t0 > DEADLINE_TICKS)) {
          if (lane == 0) __hip_atomic_store(tmo, call, __ATOMIC_RELAXED, __HIP_MEMORY_SCOPE_AGENT);
          dead = true;
          break;
        }
        __builtin_amdgcn_s_sleep(LSLEEP);
      }
      if (dead) break;
      T2(0, n);
      if (!(f & F_RAW)) {
        const u32x4 a = __builtin_amdgcn_raw_buffer_load_b128(rs, lane * 16, so, 17);
        const u32x4 b = __builtin_amdgcn_raw_buffer_load_b128(rs, lane * 16 + 1024, so, 17);
        const u32x4 c = __builtin_amdgcn_raw_buffer_load_b128(rs, lane * 16 + 2048, so, 17);
        const unsigned e = __builtin_amdgcn_raw_buffer_load_b32(rs, (lane & 15) * 4 + 3072, so, 17);
        expand(f, a, b, c, e, so);
      }
      asm volatile("s_waitcnt vmcnt(0)" ::: "memory");
      if (lane == 0) __builtin_amdgcn_raw_buffer_store_b32(tag | F_RAW, rs, 0, fo + k * 4, 17);
      T2(1, n);
   }
  }

  asm volatile("s_waitcnt lgkmcnt(0)" ::: "memory");
  __syncthreads();
  const u64 outp = (u64)(uintptr_t)P.out_all[lane & 7];
  gchar* const slp = (gchar*)P.ws_all[lane & 7];
  get_call();

  if (wave == 0) TS(13);
  auto global_max = [&](bool slow) -> unsigned {
    const u64* slots = reinterpret_cast<const u64*>(myws + OFF_SLOTS + (blockIdx.x & 7) * 512) + (lane & 7);
    const u64 t0 = __builtin_amdgcn_s_memrealtime();
    unsigned g;
    for (unsigned spins = 1;; ++spins) {
      const u64 v = __hip_atomic_load(slots, __ATOMIC_RELAXED, __HIP_MEMORY_SCOPE_SYSTEM);
      g = (unsigned)v;
      if (__all((unsigned)(v >> 32) >= call)) break;
      if ((spins & 15) == 0 &&
          (__hip_atomic_load(tmo, __ATOMIC_RELAXED, __HIP_MEMORY_SCOPE_AGENT) == call ||
           __builtin_amdgcn_s_memrealtime() - t0 > DEADLINE_TICKS)) {
        if (lane == 0) __hip_atomic_store(tmo, call, __ATOMIC_RELAXED, __HIP_MEMORY_SCOPE_AGENT);
        g = ~0u;
        break;
      }
      if (slow) __builtin_amdgcn_s_sleep(100); else __builtin_amdgcn_s_sleep(ASL);
    }
#pragma unroll
    for (int o = 4; o >= 1; o >>= 1) g = max(g, (unsigned)__shfl_xor((int)g, o));
    return g;
  };
  if (sender) {
    if (ER && wave == 15) {
      if (GOX && lane == 0)
        __hip_atomic_fetch_add(reinterpret_cast<unsigned*>(myws + OFF_GO), 1u, __ATOMIC_RELAXED, __HIP_MEMORY_SCOPE_AGENT);
      const unsigned g = global_max(true);
      if (lane == 0) { lds_st(ctl + C_GMAX * 4, g); lds_st(ctl + C_DEAD * 4, 1u); }
    } else if (ER && wave >= 1 && (wave - 1) * NSWG + (int)blockIdx.x < 512) {
      const int it = (wave - 1) * NSWG + blockIdx.x;
      const int src = it & 7, c64 = (it >> 3) & 31, hu = it >> 8;
      const int lr = rank * 128 + hu * 64 + (lane & 3) * 16;
      const u64* fl = reinterpret_cast<const u64*>(myws + OFF_EOF) +
                      ((src * 4 + lr / TM) * 16 + (c64 >> 1)) * 8 + (c64 & 1) * 4 + (lr % TM) / 80;
      while (__builtin_amdgcn_readfirstlane(lds_ld(ctl + C_DEAD * 4)) == 0u) __builtin_amdgcn_s_sleep(16);
      const u64 t0 = __builtin_amdgcn_s_memrealtime();
      bool dead = false;
      u64 v;
      for (unsigned spins = 1;; ++spins) {
        v = __hip_atomic_load(fl, __ATOMIC_RELAXED, __HIP_MEMORY_SCOPE_SYSTEM);
        if (__all((unsigned)(v >> 32) >= call)) break;
        if ((spins & 15) == 0 &&
            (__hip_atomic_load(tmo, __ATOMIC_RELAXED, __HIP_MEMORY_SCOPE_AGENT) == call ||
             __builtin_amdgcn_s_memrealtime() - t0 > DEADLINE_TICKS)) {
          if (lane == 0) __hip_atomic_store(tmo, call, __ATOMIC_RELAXED, __HIP_MEMORY_SCOPE_AGENT);
          dead = true;
          break;
        }
        __builtin_amdgcn_s_sleep(32);
      }
      const __amdgpu_buffer_rsrc_t rs = __builtin_amdgcn_make_buffer_rsrc(myws, 0, 0x7fffffff, 0x00020000);
      const unsigned so = (unsigned)OFF_ES + (unsigned)((src * 8 + hu * 4) * 32 + c64) * 2048;
      u32x4 h[4][2];
#pragma unroll
      for (int g = 0; g < 4; ++g) {
        h[g][0] = __builtin_amdgcn_raw_buffer_load_b128(rs, lane * 16, so + g * 65536, 17);
        h[g][1] = __builtin_amdgcn_raw_buffer_load_b128(rs, lane * 16 + 1024, so + g * 65536, 17);
      }
      const float amax = __uint_as_float(lds_ld(ctl + C_GMAX * 4));
      dead |= __float_as_uint(amax) == ~0u;
      const float scale = amax / 127.0f;
      const float inv = scale > 0.f ? 1.0f / scale : 0.f;
      const unsigned here = (unsigned)__ballot(dead || (lr % 80) / 16 < (int)(v & 7u)) & (ER >> (hu * 4));
      gchar* ob = (gchar*)P.out_all[rank] + (size_t)(hu * 64 + (lane & 15)) * NTOT + src * NLOC + c64 * 64 + (lane >> 4) * 16;
#pragma unroll
      for (int g = 0; g < 4; ++g) {
        if (!((here >> g) & 1u)) continue;
        const float step = __uint_as_float((unsigned)__builtin_amdgcn_readlane((int)v, g));
        u32x4 o;
#pragma unroll
        for (int j = 0; j < 4; ++j) {
          unsigned pk = 0;
#pragma unroll
          for (int nt = 0; nt < 4; ++nt) {
            const int w = (int)h[g][j >> 1][(j & 1) * 2 + (nt >> 1)];
            const float y = (float)((nt & 1) ? w >> 16 : (w << 16) >> 16) * step;
            float q = rintf(y * inv);
            q = fminf(fmaxf(q, -127.f), 127.f);
            const int qi = dead ? -128 : (int)q;
            pk |= ((unsigned)qi & 0xffu) << (8 * nt);
          }
          o[j] = pk;
        }
        st16(ob + (size_t)g * 16 * NTOT, o);
      }
      asm volatile("s_waitcnt vmcnt(0)" ::: "memory");
      TS(24);
    } else if (wave == 0 && blockIdx.x < NCOL) {
      const char* rp = myws + OFF_REP + lane * 16;
      const u64 t0 = __builtin_amdgcn_s_memrealtime();
      unsigned m;
      for (unsigned spins = 1;; ++spins) {
        u32x4 r0, r1;
        asm volatile("global_load_dwordx4 %0, %2, off sc0 sc1\n\tglobal_load_dwordx4 %1, %2, off offset:1024 sc0 sc1\n\t"
                     "s_waitcnt vmcnt(0)" : "=&v"(r0), "=&v"(r1) : "v"(rp) : "memory");
        if (lane >= (NCW - 128) / 2) r1 = r0;
        m = max(max(r0[0], r0[2]), max(r1[0], r1[2]));
        if (__all(r0[1] == call && r0[3] == call && r1[1] == call && r1[3] == call)) break;
        if ((spins & 15) == 0 && __builtin_amdgcn_s_memrealtime() - t0 > DEADLINE_TICKS) {
          if (lane == 0) __hip_atomic_store(tmo, call, __ATOMIC_RELAXED, __HIP_MEMORY_SCOPE_AGENT);
          m = ~0u;
          break;
        }
        __builtin_amdgcn_s_sleep(CSL);
      }
      TS(20);
#pragma unroll
      for (int o = 32; o >= 1; o >>= 1) m = max(m, (unsigned)__shfl_xor((int)m, o));
      __hip_atomic_store((gu64*)(slp + OFF_SLOTS + (lane >> 3) * 512) + rank, ((u64)call << 32) | m, __ATOMIC_RELAXED,
                         __HIP_MEMORY_SCOPE_SYSTEM);
    }
    return;
  }
  {
  int b2 = blockIdx.x, w2 = wave;
  asm volatile("" : "+s"(b2), "+s"(w2));
  const int bx2 = b2 & 7, bk2 = ((b2 >> 3) + 26) & 31;
  const int ti = bk2 < 24 ? 2 * (bk2 % 6) + (bx2 & 1) : 12;
  const int tj = 4 * (bx2 >> 1) + (bk2 < 24 ? bk2 / 6 : 2 * (bx2 & 1) + ((bk2 - 24) & 1));
  const int wm = w2 & 3, wn = (w2 >> 2) & 1;
  const bool mine = lane == rows_of((5 * ti) >> 1) >> 2 || lane == rows_of((5 * ti + 2) >> 1) >> 2 ||
                    lane == rows_of(min((5 * ti + 4) >> 1, 31)) >> 2;
  if (wave == 8) {
    unsigned m = lds_ld(ctl + (lane < 8 ? C_WMAX + lane : C_DEAD) * 4);
#pragma unroll
    for (int o = 8; o >= 1; o >>= 1) m = max(m, (unsigned)__shfl_xor((int)m, o));
    if (lane == 0)
      __hip_atomic_store((gu64*)(myws + OFF_REP) + (blockIdx.x - NSWG), ((u64)call << 32) | m, __ATOMIC_RELAXED,
                         __HIP_MEMORY_SCOPE_SYSTEM);
  }
  int ne = 0;
  if (ER && wave < 8 && ti * TM + wm * 80 < 1024) {
    if (wave == 0) TS(25);
    float tm = __uint_as_float(lds_ld(ctl + (C_WMAX + (lane & 7)) * 4));
#pragma unroll
    for (int o = 4; o >= 1; o >>= 1) tm = fmaxf(tm, __shfl_xor(tm, o));
    const unsigned sb = __float_as_uint(fmaxf(tm, 1e-30f) / 32000.0f) & ~7u;
    const float k16 = 1.0f / __uint_as_float(sb);
#pragma unroll
    for (int mt = 0; mt < 5; ++mt) {
      const int row0 = ti * TM + wm * 80 + mt * 16;
      if (ne != mt || __builtin_amdgcn_readfirstlane(lds_ld(ctl + C_ECNT * 4)) != 0u) continue;
      ne = mt + 1;
      if (row0 >= 1024 || !((ER >> ((row0 >> 4) & 7)) & 1)) continue;
      gchar* ob = (gchar*)(((u64)(unsigned)__builtin_amdgcn_readlane((int)((u64)(uintptr_t)slp >> 32), row0 >> 7) << 32) |
                           (unsigned)__builtin_amdgcn_readlane((int)(u64)(uintptr_t)slp, row0 >> 7));
      ob += OFF_ES + (size_t)((rank * 8 + ((row0 & 127) >> 4)) * 32 + tj * 2 + wn) * 2048 + lane * 16;
      u32x4 h[2];
#pragma unroll
      for (int j = 0; j < 4; ++j)
#pragma unroll
        for (int n2 = 0; n2 < 2; ++n2)
          h[j >> 1][(j & 1) * 2 + n2] = __builtin_amdgcn_perm((unsigned)(int)rintf(acc[mt][2 * n2 + 1][j] * k16),
                                                              (unsigned)(int)rintf(acc[mt][2 * n2][j] * k16), 0x05040100u);
      st16(ob, h[0]);
      if (EOTH == 2) asm volatile("s_waitcnt vmcnt(0)" ::: "memory");
      st16<1024>(ob, h[1]);
      asm volatile("s_waitcnt vmcnt(0)" ::: "memory");
    }
    if (lane == (ti * TM + wm * 80) >> 7 || lane == min(ti * TM + wm * 80 + 79, 1023) >> 7)
      __hip_atomic_store((gu64*)(slp + OFF_EOF) + ((rank * 4 + ti) * 16 + tj) * 8 + w2, ((u64)call << 32) | sb | (unsigned)ne,
                         __ATOMIC_RELAXED, __HIP_MEMORY_SCOPE_SYSTEM);
    TS(26);
    TV(27, ne);
  }
  if (wave == 8) {
    if (IDL) __builtin_amdgcn_s_sleep(IDL);
    const unsigned g = global_max(SPOLL && ER && ti < 4);
    TS(22);
    if (lane == 0) { lds_st(ctl + C_GMAX * 4, g); lds_st(ctl + C_ECNT * 4, 1u); }
  }
  if (SPOLL && ER && wave == 9 && ti < 4) {
    typedef __attribute__((ext_vector_type(16))) unsigned u32x16;
    const char* sp = myws + OFF_SLOTS + bx2 * 512;
    while (__builtin_amdgcn_readfirstlane(lds_ld(ctl + C_ECNT * 4)) == 0u) {
      u32x16 v;
      asm volatile("s_load_dwordx16 %0, %1, 0x0 glc\n\ts_waitcnt lgkmcnt(0)" : "=s"(v) : "s"(sp) : "memory");
      unsigned c = ~0u, g = 0;
#pragma unroll
      for (int i = 0; i < 8; ++i) { c = min(c, v[2 * i + 1]); g = max(g, v[2 * i]); }
      if (c >= call) {
        if (lane == 0) { lds_st(ctl + C_GMAX * 4, g); lds_st(ctl + C_ECNT * 4, 1u); }
        TS(34);
        break;
      }
      __builtin_amdgcn_s_sleep(SSL);
    }
  }
  if (wave < 8)
    while (__builtin_amdgcn_readfirstlane(lds_ld(ctl + C_ECNT * 4)) == 0u) __builtin_amdgcn_s_sleep(1);
  if (wave == 0) TS(14);
  if (wave < 8) {
    const float amax = __uint_as_float(lds_ld(ctl + C_GMAX * 4));
    const bool dead = __float_as_uint(amax) == ~0u;
    const float scale = amax / 127.0f;
    const float inv = scale > 0.f ? 1.0f / scale : 0.f;
    const int col = rank * NLOC + tj * 128 + wn * 64 + (lane >> 4) * 16;
#pragma unroll
    for (int mt = 0; mt < 5; ++mt) {
      const int lr0 = ti * TM + wm * 80 + mt * 16;
      if (lr0 >= 4096 || (mt < ne && lr0 < 1024 && ((ER >> ((lr0 >> 4) & 7)) & 1))) continue;
      const int row0 = rows_of(lr0 >> 7) * 128 + (lr0 & 127);
      const int owner = row0 >> 9;
      gchar* ob = (gchar*)(((u64)(unsigned)__builtin_amdgcn_readlane((int)(outp >> 32), owner) << 32) |
                           (unsigned)__builtin_amdgcn_readlane((int)outp, owner));
      u32x4 o;
#pragma unroll
      for (int j = 0; j < 4; ++j) {
        unsigned pk = 0;
#pragma unroll
        for (int nt = 0; nt < 4; ++nt) {
          float q = rintf(acc[mt][nt][j] * inv);
          q = fminf(fmaxf(q, -127.f), 127.f);
          int qi = dead ? -128 : (int)q;
          pk |= ((unsigned)qi & 0xffu) << (8 * nt);
        }
        o[j] = pk;
      }
      st16(ob + (size_t)((row0 & 511) + (lane & 15)) * NTOT + col, o);
      asm volatile("s_waitcnt vmcnt(0)" ::: "memory");
    }
    if (wave == 0) TS(17);
    asm volatile("s_waitcnt vmcnt(0)" ::: "memory");
    if (wave == 0) TS(18);
  }
  asm volatile("s_waitcnt lgkmcnt(0)" ::: "memory");
  __syncthreads();
  if (wave == 0) {
    TS(15);
    if (mine)
      __hip_atomic_store((gu32*)(slp + OFF_OUTF) + (rank * NTI + ti) * 16 + tj, call, __ATOMIC_RELAXED, __HIP_MEMORY_SCOPE_SYSTEM);
    if (blockIdx.x == 48) {
      const int tlo = (1024 + 384 * rank) / TM, thi = (1407 + 384 * rank) / TM;
      const __amdgpu_buffer_rsrc_t rs = __builtin_amdgcn_make_buffer_rsrc(myws, 0, 0x7fffffff, 0x00020000);
      const u64 t0 = __builtin_amdgcn_s_memrealtime();
      while (true) {
        bool ok = true;
        asm volatile("" ::: "memory");
#pragma unroll
        for (int j = 0; j < 3; ++j) {
          const int tt = j < 2 ? min(tlo + 2 * j + (lane >> 5), thi) : (2 * rank + (lane >> 5)) / 5;
          const u32x4 v = __builtin_amdgcn_raw_buffer_load_b128(rs, ((((lane >> 2) & 7) * NTI + tt) * 4 + (lane & 3)) * 16, (unsigned)OFF_OUTF, 17);
          ok &= min(min(v[0], v[1]), min(v[2], v[3])) >= call;
        }
        if (__all(ok)) break;
        if (__hip_atomic_load(tmo, __ATOMIC_RELAXED, __HIP_MEMORY_SCOPE_AGENT) == call ||
            __builtin_amdgcn_s_memrealtime() - t0 > DEADLINE_TICKS) {
          if (lane == 0) {
            __hip_atomic_store(tmo, call, __ATOMIC_RELAXED, __HIP_MEMORY_SCOPE_AGENT);
            ((gchar*)(P.out_all[rank]))[rank * NLOC] = (char)-128;
          }
          break;
        }
        __builtin_amdgcn_s_sleep(1);
      }
      __builtin_amdgcn_fence(__ATOMIC_ACQUIRE, "");
      TS(16);
      }
    }
  }
}

extern "C" void kernel_launch(int rank, int nranks, ncclComm_t comm, hipStream_t stream,
                              void* const* d_in, const int* in_sizes, int n_in, void* d_out,
                              int out_size, const int* in_dtypes, int out_dtype,
                              void* const* d_out_all, void* const* d_ws_all, size_t ws_bytes) {
  if (nranks != NR || ws_bytes < WS_NEED || n_in < 2) {
    fprintf(stderr, "unexpected configuration\n");
    return;
  }
  Params P;
  P.x = static_cast<const float*>(d_in[0]);
  P.w = static_cast<const float*>(d_in[1]);
  P.out_all = d_out_all;
  P.ws_all = d_ws_all;
  P.rank = rank;
  hipLaunchKernelGGL(fused_gemm_q8, dim3(NWG), dim3(NTHREADS), 0, stream, P);
  HIP_CHECK(hipGetLastError());
}
```

```cpp
#include <hip/hip_runtime.h>
#include <rccl/rccl.h>
#include <cstdio>
#include <cstdint>
#include <type_traits>

#define HIP_CHECK(expr) do { hipError_t _e = (expr); if (_e != hipSuccess) { \
    fprintf(stderr, "HIP error %s at %s:%d\n", hipGetErrorString(_e), __FILE__, __LINE__); \
    return; } } while (0)

typedef __attribute__((ext_vector_type(8))) short bf16x8;
typedef __attribute__((ext_vector_type(4))) float f32x4;
typedef __attribute__((ext_vector_type(4))) unsigned u32x4;
typedef unsigned long long u64;
#define GAS __attribute__((address_space(1)))
typedef GAS char gchar;
typedef GAS unsigned gu32;
typedef GAS unsigned long long gu64;
typedef GAS u32x4 gu32x4;

namespace {
constexpr int NR = 8;
constexpr int KSH = 512;
constexpr int NTOT = 16384;
constexpr int NLOC = 2048;
constexpr int MSH = 512;
constexpr int NSTAGE = 8;
constexpr int XU = 32;
constexpr int WU = 16;
constexpr int UNIT = 16384;
constexpr int NWG = 256;
constexpr int SU = 4;
constexpr int NUNIT = 7 * NSTAGE * (XU + WU);
#define THR 2
#define PSLEEP 16
#define SCS 32
#define F1S 128
#define MAPB 0
#define DLY 9
#define DSLOW 0
#define LOS 32
#define LW 2
#define NLO 1
#define PRO 1
#define PKD 0
#define KSLOW 1
#define FWAIT 40
#define RECHK 1
#define OUTL 0
#define OTH 1
#define ASL 4
#define SEQ 5
#define WROT 1
#define WFAST 44
#define PS2 2
#define WMASK 1
#define NL (NT0 - 2016)
#define LSLEEP 4
#define NCOL 2
#define CSL 1
#define RS 1
#define PPRIO 0
#define OSEQ 8
#define KWV 3
#define TSL 8
#define XROWS(q) rows_of(q)
#define XSH 0
#define SPOLL 0
#define SSL 24
#define IDL 0
#define FAST 0
#define ER 0xD5
#define GOX 0
#define EOTH 1
#define TKT 5
#define BAL 1
#define LPRIO 3
#define PFD 0
#define PFL 3
#define TLO 56
#define THI 64
#define PIPE 1
#define QX 0
#define CPRIO 0
#define DSLEEP 16
#define DPRIO 2
#define LP 0
#define LDPOL " sc0 sc1"
constexpr int NSTEP = 64;
constexpr int TM = 320;
constexpr int NTI = 13;
constexpr int NCW = NTI * 16;
constexpr int NH = 5;
constexpr int HS = 7 * 4096;
#define NSEND 13
constexpr int NSWG = NWG - NCW;
constexpr int NSC = 15 - NSEND < 4 ? 15 - NSEND : 4;
#define S1V 4
constexpr int EU = ER ? 10 : 0;
constexpr int RTU = 7 * (32 - EU);
#define TKD 1
constexpr int S1 = S1V;
constexpr int NT0 = NUNIT - RTU;
constexpr int KW = KWV;
constexpr int U0 = NCW * KW * 2 / 4;
constexpr int NDRAW = NUNIT - U0 - S1 * NSWG + NSWG;
static_assert(NT0 - NL >= 7 * (16 + 48 * 5), "the steps of stage 4 do not depend on the last units");
static_assert(NSEND + NSC < 16 && SEQ <= 6, "sender split");
#define NWAVE 16
#define NDECV 2
constexpr int NDEC = NDECV;
constexpr int W_DEC = 11;
constexpr int NTHREADS = 64 * NWAVE;
#define WPRIO 0
#define LK 0
#define VB 0
#define VC 1
#define VD 0
constexpr unsigned F_RAW = 1u << 12;
constexpr int MAXESC = 192;
constexpr int NPK = NWAVE - W_DEC - NDEC;
constexpr int NPACK = (8 * XU + 56 + 6 * 112 + NSTAGE * WU) * 4;
constexpr int NMIR = 17;
#define CPK 0
constexpr int PR0 = NCW * (8 * CPK + NPK);
constexpr int PRN = NCW * NPK;
constexpr int NROUND = 1 + (NPACK - PR0 + PRN - 1) / PRN;
static_assert(CPK == 0 && RS >= 1 && RS * PRN <= NPACK && PRN % 48 == 0, "packing");
constexpr int NIN = 7 * 384;

constexpr size_t OFF_TICKET = 0;
constexpr size_t OFF_TMO = 128;
constexpr size_t OFF_SQ = 2048;
constexpr size_t OFF_PQ = 2304;
constexpr size_t OFF_XF = 4096;
constexpr size_t OFF_WF = OFF_XF + XU * 64 * SU * 4;
constexpr size_t OFF_GO = 3840;
constexpr size_t OFF_OUTF = 65536;
constexpr size_t OFF_REP = 73728;
constexpr size_t OFF_SLOTS = 77824;
constexpr size_t OFF_EF = 81920;
constexpr size_t OFF_F1 = 131072;
constexpr size_t OFF_EOF = 1u << 19;
constexpr size_t OFF_XS = 1u << 20;
constexpr size_t OFF_WS = OFF_XS + (size_t)NR * NSTAGE * XU * UNIT;
constexpr size_t OFF_EX = OFF_WS + (size_t)NR * NSTAGE * WU * UNIT;
constexpr size_t OFF_EW = OFF_EX + (size_t)NSTAGE * XU * UNIT;
constexpr size_t OFF_ES = OFF_EW + (size_t)NR * NSTAGE * WU * UNIT;
constexpr size_t WS_NEED = OFF_ES + (size_t)NR * 8 * 32 * 2048;

constexpr u64 DEADLINE_TICKS = 40000000ull;

constexpr int C_CALL = 0, C_FULL = 128, C_DONE = 8  , C_WMAX = 100, C_OUT = 110, C_ECNT = 120, C_GMAX = 24, C_DEAD = 25, C_NEXT = 26, C_Q = 32;
constexpr int LDS_CTL = NH * HS;
constexpr int LDS_SCR = LDS_CTL + 512 + 1024;
constexpr int LDS_BYTES = LDS_SCR + 16 * 256;
}

#define T2(a, i) do {} while (0)
#define TS(i) do {} while (0)
#define TV(i, v) do {} while (0)
#define CYC() 0ull

struct Params {
  const float* x;
  const float* w;
  void* const* out_all;
  void* const* ws_all;
  int rank;
};

__device__ __forceinline__ unsigned pk_bf16(float a, float b) {
  typedef __attribute__((ext_vector_type(2))) float f2;
  typedef __attribute__((ext_vector_type(2))) __bf16 b2;
  f2 v = {a, b};
  b2 r = __builtin_convertvector(v, b2);
  return __builtin_bit_cast(unsigned, r);
}

__device__ __forceinline__ unsigned lds_ld(unsigned addr) {
  unsigned v;
  asm volatile("ds_read_b32 %0, %1\n\ts_waitcnt lgkmcnt(0)" : "=v"(v) : "v"(addr) : "memory");
  return v;
}
__device__ __forceinline__ void glds4k(const void* g, unsigned ldsaddr) {
  unsigned keep;
  asm volatile(
      "s_mov_b32 %0, m0\n\ts_mov_b32 m0, %2\n\ts_nop 0\n\t"
      "global_load_lds_dwordx4 %1, off" LDPOL "\n\t"
      "global_load_lds_dwordx4 %1, off offset:1024" LDPOL "\n\t"
      "global_load_lds_dwordx4 %1, off offset:2048" LDPOL "\n\t"
      "global_load_lds_dwordx4 %1, off offset:3072" LDPOL "\n\t"
      "s_mov_b32 m0, %0"
      : "=&s"(keep) : "v"(g), "s"(ldsaddr) : "memory");
}
__device__ __forceinline__ bool plain16(u32x4 a, u32x4 b, u32x4 c, u32x4 d, unsigned want, unsigned m2) {
  unsigned x = 0, y = 0;
#pragma unroll
  for (int j = 0; j < 4; ++j) { x |= (a[j] ^ want) | (b[j] ^ want) | (d[j] ^ want); y |= c[j] ^ want; }
  return ((x | (y & m2)) >> 12) == 0u;
}
__device__ __forceinline__ int rows_of(int v) { return v < 8 ? 4 * v : 4 * ((v - 8) / 3) + 1 + (v - 8) % 3; }
__device__ __forceinline__ void unit_of(int ii, int pr, int& s, int& q, int& rr) {
  const int rot = (pr * 16 + 3) / 7;
  if (ii < 16) { s = 0; q = ii; rr = 2; }
  else if (ii < 16 + 48 * SEQ) {
    const int i2 = ii - 16, it = i2 % 48;
    s = i2 / 48; q = it / 3; rr = it - 3 * q;
    if (rr == 2) s += 1; else q = (q + rot) & 15;
  } else {
    int pos;
    if (ii < 336 - XSH) { s = (ii - 16) / 48; pos = (ii - 16) % 48; }
    else if (ii < 336 - XSH + EU) { s = 7; pos = ii - (336 - XSH); }
    else if (ii < 352 - XSH + EU) { s = 6; pos = ii - (304 - XSH) - EU; }
    else if (ii < 352 + EU) { s = 6; pos = ii - 320 - EU; }
    else { s = 7; pos = ii - 352; }
    if (pos >= 32) { s += 1; q = (pos + rot) & 15; rr = 2; }
    else {
      const int xu = pos < EU ? (pos + pr) % (EU ? EU : 1) : EU + (pos - EU + (pr * (32 - EU) + 3) / 7) % (32 - EU);
      q = xu >> 1; rr = xu & 1;
    }
  }
}
__device__ __forceinline__ int packed_at(int s, int q, int rr, int pr) {
  if (rr != 2) return (s == 0 ? 0 : 88 + (s - 1) * 160) + 2 * q + rr;
  return s >= 2 ? 120 + (s - 2) * 160 + q * 7 + pr : s == 1 && q >= 8 ? q * 7 + pr - 24 : (NMIR + 3) * 64;
}
__device__ __forceinline__ void lds_st(unsigned addr, unsigned v) {
  asm volatile("ds_write_b32 %0, %1" :: "v"(addr), "v"(v) : "memory");
}

template <int OFF = 0>
__device__ __forceinline__ void st16(gchar* p, u32x4 v) {
  asm volatile("global_store_dwordx4 %0, %1, off offset:%2 sc0 sc1\n\ts_nop 1" :: "v"(p), "v"(v), "n"(OFF) : "memory");
}

__device__ __forceinline__ void st4(gchar* p, unsigned v) {
  asm volatile("global_store_dword %0, %1, off sc0 sc1\n\ts_nop 1" :: "v"(p), "v"(v) : "memory");
}

__device__ __forceinline__ unsigned listed(unsigned A, unsigned B, unsigned C) {
  return (A & (A >> 1) & C & 0x55555555u) | ((B & (B >> 1) & (C >> 1) & 0x55555555u) << 1);
}
__device__ __forceinline__ unsigned ahead_of(unsigned c, unsigned& total) {
  unsigned idx = 0;
  total = 0;
#pragma unroll
  for (int b = 0; b < 3; ++b) {
    const u64 bal = __ballot((c >> b) & 1u);
    idx += __builtin_amdgcn_mbcnt_hi((unsigned)(bal >> 32), __builtin_amdgcn_mbcnt_lo((unsigned)bal, 0u)) << b;
    total += (unsigned)__builtin_popcountll(bal) << b;
  }
  return idx;
}

__device__ __forceinline__ unsigned send_block(const unsigned (&p)[4][4], unsigned scr, gchar* dst, int lane, bool plain,
                                              int slow = 0) {
  typedef __attribute__((ext_vector_type(2))) unsigned short u16x2;
  typedef __attribute__((ext_vector_type(3))) unsigned u32x3;
  unsigned top = 0, b4[8], e4[8], A = 0, B = 0, C = 0, M = 0, idx = 0, total = 0;
  if (!plain) {
    u16x2 mx = {0, 0};
#pragma unroll
    for (int i = 0; i < 16; ++i)
      mx = __builtin_elementwise_max(mx, __builtin_bit_cast(u16x2, p[i >> 2][i & 3] & 0x7fff7fffu));
    int mi = max((int)mx[0], (int)mx[1]);
    mi = max(mi, __builtin_amdgcn_update_dpp(mi, mi, 0x111, 0xf, 0xf, false));
    mi = max(mi, __builtin_amdgcn_update_dpp(mi, mi, 0x112, 0xf, 0xf, false));
    mi = max(mi, __builtin_amdgcn_update_dpp(mi, mi, 0x114, 0xf, 0xf, false));
    mi = max(mi, __builtin_amdgcn_update_dpp(mi, mi, 0x118, 0xf, 0xf, false));
    mi = max(mi, __builtin_amdgcn_update_dpp(mi, mi, 0x142, 0xa, 0xf, false));
    mi = max(mi, __builtin_amdgcn_update_dpp(mi, mi, 0x143, 0xc, 0xf, false));
    top = (unsigned)__builtin_amdgcn_readlane(mi, 63) >> 7;
    const unsigned top4 = top * 0x01010101u;
#pragma unroll
    for (int k = 0; k < 8; ++k) {
      const unsigned lo = __builtin_amdgcn_perm(p[k >> 1][(k & 1) * 2 + 1], p[k >> 1][(k & 1) * 2], 0x06040200u);
      const unsigned hi = __builtin_amdgcn_perm(p[k >> 1][(k & 1) * 2 + 1], p[k >> 1][(k & 1) * 2], 0x07050301u);
      b4[k] = (hi & 0x80808080u) | (lo & 0x7f7f7f7fu);
      e4[k] = ((hi << 1) & 0xfefefefeu) | ((lo >> 7) & 0x01010101u);
      const unsigned d = top4 - e4[k];
      const unsigned x = (d + 0x01010101u) & 0xf8f8f8f8u;
      const unsigned nz = (((x & 0x7f7f7f7fu) + 0x7f7f7f7fu) | x) & 0x80808080u;
      const unsigned c = d | ((nz >> 4) - (nz >> 7));
      if (k < 4) A |= (c & 0x03030303u) << (2 * k); else B |= (c & 0x03030303u) << (2 * (k - 4));
      C |= ((c >> 2) & 0x01010101u) << (2 * (k & 3) + (k >> 2));
    }
    M = listed(A, B, C);
    const unsigned cnt = __builtin_popcount(M);
    idx = ahead_of(cnt, total);
    plain = top < 8u || top == 255u || total > (unsigned)MAXESC || __ballot(cnt > 7u) != 0ull;
  }
  if (plain) {
    st16<0>(dst, u32x4{p[0][0], p[0][1], p[0][2], p[0][3]});
    if (slow) asm volatile("s_waitcnt vmcnt(0)" ::: "memory");
    st16<1024>(dst, u32x4{p[1][0], p[1][1], p[1][2], p[1][3]});
    if (slow) asm volatile("s_waitcnt vmcnt(0)" ::: "memory");
    st16<2048>(dst, u32x4{p[2][0], p[2][1], p[2][2], p[2][3]});
    if (slow) asm volatile("s_waitcnt vmcnt(0)" ::: "memory");
    st16<3072>(dst, u32x4{p[3][0], p[3][1], p[3][2], p[3][3]});
    return F_RAW;
  }
  st16(dst, u32x4{b4[0], b4[1], b4[2], b4[3]});
  while (__ballot(M != 0u)) {
    const unsigned pos = (unsigned)__builtin_ctz(M | 0x80000000u);
    const unsigned s0 = (pos & 1u) ? e4[4] : e4[0], s1 = (pos & 1u) ? e4[5] : e4[1];
    const unsigned s2 = (pos & 1u) ? e4[6] : e4[2], s3 = (pos & 1u) ? e4[7] : e4[3];
    const unsigned t0 = (pos & 2u) ? s1 : s0, t1 = (pos & 2u) ? s3 : s2;
    const unsigned sel = ((pos & 4u) ? t1 : t0) >> (pos & 24u);
    if (M != 0u) asm volatile("ds_write_b8 %0, %1" :: "v"(scr + idx), "v"(sel) : "memory");
    M &= M - 1u;
    ++idx;
  }
  if (slow & 1) asm volatile("s_waitcnt vmcnt(0)" ::: "memory");
  st16<1024>(dst, u32x4{b4[4], b4[5], b4[6], b4[7]});
  const unsigned ev = lds_ld(scr + lane * 4);
  if (slow) asm volatile("s_waitcnt vmcnt(0)" ::: "memory");
  asm volatile("global_store_dwordx3 %0, %1, off offset:2048 sc0 sc1\n\ts_nop 1" :: "v"(dst - lane * 4), "v"(u32x3{A, B, C}) : "memory");
  const unsigned cls = (total + 63u) >> 6;
  if ((unsigned)lane < (VB ? (cls + 1u) & ~1u : cls) * 16u) st4(dst - lane * 12 + 2816, ev);
  return top | cls << 8;
}

#define AC00 "v[40:43]"
#define AC01 "v[44:47]"
#define AC02 "v[48:51]"
#define AC03 "v[52:55]"
#define AC10 "v[56:59]"
#define AC11 "v[60:63]"
#define AC12 "v[64:67]"
#define AC13 "v[68:71]"
#define AC20 "v[72:75]"
#define AC21 "v[76:79]"
#define AC22 "v[80:83]"
#define AC23 "v[84:87]"
#define AC30 "v[88:91]"
#define AC31 "v[92:95]"
#define AC32 "v[96:99]"
#define AC33 "v[100:103]"
#define AC40 "v[104:107]"
#define AC41 "v[108:111]"
#define AC42 "v[112:115]"
#define AC43 "v[116:119]"
#define RA0 "v[0:3]"
#define RA1 "v[4:7]"
#define RA2 "v[8:11]"
#define RA3 "v[12:15]"
#define RB0 "v[16:19]"
#define RB1 "v[20:23]"
#define RB2 "v[24:27]"
#define RB3 "v[28:31]"
#define RC0 "v[32:35]"
#define RC1 "v[36:39]"
#define PFY(x) x
#define PFN(x)
#define MF_(d, a, b) "v_mfma_f32_16x16x32_bf16 " d ", " a ", " b ", " d "\n\t"
#define RD_(r, a, o) "ds_read_b128 " r ", " a " offset:" #o "\n\t"
#define WT_(n) "s_waitcnt lgkmcnt(" #n ")\n\t"
#define ADV_ "s_add_u32 %[sdn], %[sdn], 1\n\t"                    \
             "v_mov_b32 %[vt], %[sdn]\n\t"                        \
             "s_mov_b64 exec, 1\n\t"                              \
             "ds_write2_b32 %[vc], %[vt], %[vt] offset0:%[dn] offset1:%[dm]\n\t" \
             "s_mov_b64 exec, -1\n\t"                             \
             "s_cmp_eq_u32 %[sof], %[hl]\n\t"                     \
             "s_cselect_b32 %[sd], %[hw], %[hs]\n\t"              \
             "s_add_u32 %[sof], %[sof], %[sd]\n\t"                \
             "v_add_u32 %[vx], %[sd], %[vx]\n\t"                  \
             "v_add_u32 %[vw], %[sd], %[vw]\n\t"                  \
             "v_add_u32 %[vf], 4, %[vf]\n\t"
#define DBG1_
#define DBG2_
#define FETCH(W0, W1, W2, N0, N1, N2, N3)                           \
  DBG1_                                                             \
  "s_setprio 0\n\t"                                                \
  "6:\n\t"                                                         \
  "ds_read2_b32 v[16:17], %[vf] offset0:0 offset1:1\n\t"           \
  WT_(0)                                                            \
  "v_readfirstlane_b32 %[sd], v16\n\t"                             \
  "v_readfirstlane_b32 %[sfl], v17\n\t"                            \
  "s_cmp_lg_u32 %[sd], 0\n\t"                                      \
  "s_cbranch_scc1 7f\n\t"                                          \
  "s_sleep 1\n\t"                                                  \
  DBG2_                                                             \
  "s_branch 6b\n\t"                                                \
  "7:\n\t"                                                         \
  RD_(N0, "%[vw]", 0) RD_(N1, "%[vw]", 1024) RD_(N2, "%[vw]", 2048) RD_(N3, "%[vw]", 3072)   \
  RD_(W0, "%[vx]", 0) RD_(W1, "%[vx]", 1024) RD_(W2, "%[vx]", 2048)                          \
  RD_(RC0, "%[vx]", 3072) RD_(RC1, "%[vx]", 4096) ADV_
#define PRD_ "ds_read_b32 %[vu], %[vc] offset:%[dp]\n\t"
#define W78_ WT_(8)
#define W67_ WT_(7)
#define PRI_ "v_readfirstlane_b32 %[sd], %[vu]\n\t"                \
             "s_setprio 0\n\t"                                     \
             "s_cmp_gt_u32 %[sd], %[sdn]\n\t"                      \
             "s_cbranch_scc0 8f\n\t"                               \
             "s_setprio 1\n\t"                                     \
             "8:\n\t"
#define BODY(W0, W1, W2, W3, N0, N1, N2, N3, P, Q)                                           \
  P(RD_(N3, "%[vw]", 3072) WT_(6)) Q(WT_(5))                                                 \
  MF_(AC00, W0, N0) MF_(AC01, W1, N0) MF_(AC02, W2, N0) MF_(AC03, W3, N0)                    \
  P(RD_(N0, "%[vw]", 0) "ds_read_b32 %[vt], %[vf] offset:4\n\t" PRD_ W78_) Q(WT_(4))        \
  MF_(AC10, W0, N1) MF_(AC11, W1, N1) MF_(AC12, W2, N1) MF_(AC13, W3, N1)                    \
  P(RD_(N1, "%[vw]", 1024) W78_) Q(WT_(3))                                                   \
  MF_(AC20, W0, N2) MF_(AC21, W1, N2) MF_(AC22, W2, N2) MF_(AC23, W3, N2)                    \
  P(RD_(N2, "%[vw]", 2048) W78_) Q(WT_(2))                                                   \
  MF_(AC30, W0, RC0)                                                                         \
  P(W67_) Q(WT_(1))                                                                          \
  MF_(AC40, W0, RC1)                                                                         \
  P(RD_(W0, "%[vx]", 0))                                                                     \
  MF_(AC31, W1, RC0) MF_(AC41, W1, RC1)                                                      \
  P(RD_(W1, "%[vx]", 1024))                                                                  \
  MF_(AC32, W2, RC0) MF_(AC42, W2, RC1)                                                      \
  P(RD_(W2, "%[vx]", 2048))                                                                  \
  MF_(AC33, W3, RC0)                                                                         \
  P(RD_(RC0, "%[vx]", 3072))                                                                 \
  MF_(AC43, W3, RC1)                                                                         \
  P(RD_(RC1, "%[vx]", 4096) WT_(7) "v_readfirstlane_b32 %[sfl], %[vt]\n\t" PRI_ ADV_)

__global__ __launch_bounds__(NTHREADS, NWAVE / 4) void fused_gemm_q8(Params P) {
  __shared__ __attribute__((aligned(1024))) char lds[LDS_BYTES];
  const unsigned ldsbase = (unsigned)(uintptr_t)lds;
  const unsigned ctl = ldsbase + LDS_CTL;

  const int tid = threadIdx.x;
  const int lane = tid & 63;
  const int wave = __builtin_amdgcn_readfirstlane(tid >> 6);
  const int rank = P.rank;
  char* const myws = static_cast<char*>(P.ws_all[rank]);
  unsigned* const tmo = reinterpret_cast<unsigned*>(myws + OFF_TMO);

  if ((blockIdx.x >> 3) < 6)
    for (int i = tid; i < (NMIR + 5) * 64; i += NTHREADS)
      asm volatile("ds_write_b128 %0, %1" :: "v"(ldsbase + i * 16), "v"(u32x4{0u, 0u, 0u, 0u}) : "memory");
  if (tid < 384) lds_st(ctl + tid * 4, 0u);
  asm volatile("s_waitcnt lgkmcnt(0)\n\ts_barrier" ::: "memory");
  if (wave == 0) TS(0);
  if (tid == ((blockIdx.x >> 3) < 6 ? 64 * 15 : 64 * 10)) {
    const unsigned t = __hip_atomic_fetch_add(reinterpret_cast<unsigned*>(myws + OFF_TICKET), 1u,
                                              __ATOMIC_RELAXED, __HIP_MEMORY_SCOPE_AGENT);
    lds_st(ctl + C_CALL * 4, t / NWG + 1);
  }
  unsigned call = 0, tag = 0;
  auto get_call = [&]() {
    while ((call = __builtin_amdgcn_readfirstlane(lds_ld(ctl + C_CALL * 4))) == 0u) __builtin_amdgcn_s_sleep(1);
    tag = call << 13;
  };

  const int bx = blockIdx.x & 7, bk = ((blockIdx.x >> 3) + 26) & 31;
  const bool sender = bk >= 26;
  const int ti = bk < 24 ? 2 * (bk % 6) + (bx & 1) : 12;
  const int tj = 4 * (bx >> 1) + (bk < 24 ? bk / 6 : 2 * (bx & 1) + ((bk - 24) & 1));

  f32x4 acc[5][4];
  const unsigned scr = ldsbase + LDS_SCR + wave * 256;
  const int wm = wave & 3, wn = (wave >> 2) & 1;

  auto fetch = [&](int s, int q, int d, int f, f32x4 (&a)[8]) {
    if (d < 0) {
      const GAS float* sp = (const GAS float*)P.x + (size_t)(XROWS(q) * 128 + (f & 4) * 16 + (lane & 15)) * KSH + s * 64 + (f >> 3) * 32 + 8 * (lane >> 4);
#pragma unroll
      for (int j = 0; j < 4; ++j) {
        a[2 * j] = *(const GAS f32x4*)sp;
        a[2 * j + 1] = *(const GAS f32x4*)(sp + 4);
        sp += 16 * KSH;
        asm("" : "+v"(sp));
      }
    } else {
      const GAS float* sp = (const GAS float*)P.w + (size_t)(s * 64 + (f >> 3) * 32 + 8 * (lane >> 4)) * NTOT + d * NLOC + q * 128 +
                        (f & 4) * 16 + (lane & 15) * 4;
#pragma unroll
      for (int j = 0; j < 8; ++j) {
        a[j] = *(const GAS f32x4*)sp;
        sp += NTOT;
        asm("" : "+v"(sp));
      }
    }
  };
  auto round16 = [&](int d, const f32x4 (&a)[8], unsigned (&p)[4][4], float& m) {
    m = 0.f;
#pragma unroll
    for (int j = 0; j < 8; ++j)
#pragma unroll
      for (int c = 0; c < 4; ++c) m = fmaxf(m, fabsf(a[j][c]));
    if (d < 0) {
#pragma unroll
      for (int j = 0; j < 4; ++j) {
        p[j][0] = pk_bf16(a[2 * j][0], a[2 * j][1]); p[j][1] = pk_bf16(a[2 * j][2], a[2 * j][3]);
        p[j][2] = pk_bf16(a[2 * j + 1][0], a[2 * j + 1][1]); p[j][3] = pk_bf16(a[2 * j + 1][2], a[2 * j + 1][3]);
      }
    } else {
#pragma unroll
      for (int jj = 0; jj < 4; ++jj)
#pragma unroll
        for (int c = 0; c < 4; ++c) p[jj][c] = pk_bf16(a[2 * c][jj], a[2 * c + 1][jj]);
    }
  };
  auto convert = [&](int s, int q, int d, int f, unsigned (&p)[4][4], float& m) {
    f32x4 a[8];
    fetch(s, q, d, f, a);
    round16(d, a, p, m);
  };
  auto pack_items = [&](int g, int rounds) {
    gchar* const gws = (gchar*)myws;
    const int grp = (g >> 3) % 6;
    gu32* const pq = (gu32*)(gws + OFF_PQ + grp * 256);
    static_assert(RS == 1, "packing");
    auto item = [&](int n, int& s, int& q, int& d) {
      const int e = n >> 2;
      d = -1;
      if (e < 32) { s = 0; q = e; }
      else if (e < 88) { s = 1; q = 8 + (e - 32) / 7; d = (rank + 1 + (e - 32) % 7) & 7; }
      else if (e < 1080) {
        const int r = (e - 88) % 160;
        s = (e - 88) / 160 + 1;
        if (r < 32) q = r;
        else if (r < 144) { s += 1; q = (r - 32) / 7; d = (rank + 1 + (r - 32) % 7) & 7; }
        else { s -= 1; q = r - 144; d = rank; }
      } else { s = 6 + ((e - 1080) >> 4); q = (e - 1080) & 15; d = rank; }
    };
    int nn = g, sn, qn, dn;
    f32x4 ra[8];
    item(nn, sn, qn, dn);
    fetch(sn, qn, dn, (nn & 3) * 4, ra);
    unsigned tk = 0;
    if (lane == 0) asm volatile("global_atomic_add %0, %1, %2, off sc0" : "=v"(tk) : "v"(pq), "v"(1u) : "memory");
    asm volatile("s_waitcnt vmcnt(0)" : "+v"(tk), "+v"(ra[0]), "+v"(ra[1]), "+v"(ra[2]), "+v"(ra[3]), "+v"(ra[4]), "+v"(ra[5]), "+v"(ra[6]), "+v"(ra[7]) :: "memory");
    for (int k = 0; nn < NPACK; ++k) {
      const int n = nn, s = sn, q = qn, d = dn, f = (n & 3) * 4, e = n >> 2;
      unsigned p[4][4];
      float m;
      round16(d, ra, p, m);
      if (k == 0) get_call();
      {
        const unsigned t = __builtin_amdgcn_readfirstlane(tk) - (call - 1) * ((NPACK - RS * PRN + 5 - grp) / 6 + PRN / 6);
        nn = t < 4096u ? RS * PRN + 6 * (int)t + grp : NPACK;
        if (nn < NPACK) {
          item(nn, sn, qn, dn);
          fetch(sn, qn, dn, (nn & 3) * 4, ra);
          if (lane == 0) asm volatile("global_atomic_add %0, %1, %2, off sc0" : "=v"(tk) : "v"(pq), "v"(1u) : "memory");
        }
      }
      unsigned info = 0;
      if (d != rank && e < 1048 + EU) {
        gchar* dst = gws + (d < 0 ? OFF_EX + (size_t)(s * XU + q) * UNIT : OFF_EW + (size_t)((d * NSTAGE + s) * WU + q) * UNIT);
        info = send_block(p, scr, dst + f * 1024 + lane * 16, lane, false);
      }
      if (d < 0 || d == rank) {
        gchar* dst = gws + (d < 0 ? OFF_XS + (size_t)((rank * NSTAGE + s) * XU + q) * UNIT
                                  : OFF_WS + (size_t)((rank * NSTAGE + s) * WU + q) * UNIT);
        send_block(p, scr, dst + f * 1024 + lane * 16, lane, true);
      }
      asm volatile("s_waitcnt vmcnt(0)" : "+v"(tk), "+v"(ra[0]), "+v"(ra[1]), "+v"(ra[2]), "+v"(ra[3]), "+v"(ra[4]), "+v"(ra[5]), "+v"(ra[6]), "+v"(ra[7]) :: "memory");
      if (lane == 0) st4(gws + OFF_EF + (size_t)n * 4, tag | info);
      if (lane == 1 && (d < 0 || d == rank))
        st4(gws + (d < 0 ? OFF_XF : OFF_WF) + (size_t)(q * 64 + rank * 8 + s) * 16 + f, tag | F_RAW);
    }
  };

  auto locate = [&](int u, int f, gchar*& dst, gu32*& flag, int& pn, unsigned& so, int& pe) -> bool {
      const int pr = u % 7, d = (rank + 1 + pr) & 7;
      int s, q, rr;
      unit_of(u / 7, pr, s, q, rr);
      gchar* dws;
      asm volatile("s_load_dwordx2 %0, %1, %2\n\ts_waitcnt lgkmcnt(0)"
                   : "=s"(dws) : "s"(P.ws_all), "s"(__builtin_amdgcn_readfirstlane(d * 8)));
      if (rr != 2) {
        const int xu = 2 * q + rr;
        so = (unsigned)OFF_EX + (unsigned)(s * XU + xu) * UNIT;
        dst = dws + OFF_XS + ((size_t)((rank * NSTAGE + s) * XU + xu)) * UNIT;
        pn = s | xu << 3;
      } else {
        so = (unsigned)OFF_EW + (unsigned)((d * NSTAGE + s) * WU + q) * UNIT;
        dst = dws + OFF_WS + ((size_t)((rank * NSTAGE + s) * WU + q)) * UNIT;
        pn = s | q << 3 | (d + 1) << 8;
      }
      pe = packed_at(s, q, rr, pr);
      flag = (gu32*)(dws + OFF_F1 + (size_t)u * F1S) + f;
      const bool tail = u >= NUNIT - RTU;
      if (tail) {
        so = (unsigned)OFF_XS + (unsigned)((rank * NSTAGE + s) * XU + 2 * q + rr) * UNIT;
        flag = (gu32*)(dws + OFF_XF + ((2 * q + rr) * 64 + rank * 8 + s) * 16) + f;
      }
      so = __builtin_amdgcn_readfirstlane(so + f * 4096);
      dst += f * 4096 + lane * 16;
      pn |= f << 12;
      return tail;
  };

  if (sender) {
   if (QX & 8) {
   } else if (wave < NSEND) {
    const __amdgpu_buffer_rsrc_t rs = __builtin_amdgcn_make_buffer_rsrc(myws, 0, 0x7fffffff, 0x00020000);
    unsigned so, info, ia = 0;
    int pn;
    int dbg = 0, pdbg = 0;
    gchar* dst;
    gu32* flag;
    u32x4 va, vb, vc;
    const u64 wsp = (u64)(uintptr_t)P.ws_all[lane & 7];
    auto draw = [&](int look) -> bool {
      unsigned j = 0, e = 0;
      if (lane == 0)
        asm volatile("ds_add_rtn_u32 %0, %1, %2\n\ts_waitcnt lgkmcnt(0)" : "=v"(j) : "v"(ctl + C_NEXT * 4), "v"(1u) : "memory");
      j = __builtin_amdgcn_readfirstlane(j);
      int f = j & 3;
      int u = U0 + (int)(j >> 2) * NSWG + (bk - 26) * 8 + bx;
      if (j >= (unsigned)(S1 * 4)) {
        while (((e = __builtin_amdgcn_readfirstlane(lds_ld(ctl + (C_Q + ((j >> 2) & 63)) * 4))) >> 16) != (j >> 2) + 1)
          __builtin_amdgcn_s_sleep(1);
        if (FAST && (e & 0x8000u)) {
          const int xu = e & 31, d = (e >> 5) & 7, fb = (e >> 12) & 3;
          so = (unsigned)OFF_XS + (unsigned)((rank * NSTAGE + 7) * XU + xu) * UNIT + fb * 4096 + f * 1024;
          va = __builtin_amdgcn_raw_buffer_load_b128(rs, lane * 16, so, 17);
          gchar* dws = (gchar*)(((u64)(unsigned)__builtin_amdgcn_readlane((int)(wsp >> 32), d) << 32) |
                                (unsigned)__builtin_amdgcn_readlane((int)wsp, d));
          dst = dws + so + lane * 16;
          flag = (gu32*)(dws + OFF_XF + (xu * 64 + rank * 8 + 7) * 16) + fb;
          pn = 1 << 30;
          info = tag | F_RAW;
          ia = ldsbase + (NMIR + 4) * 1024 + ((j >> 2) & 127) * 4;
          dbg = 4 * NUNIT + blockIdx.x * 16 + wave;
          if (TSL) __builtin_amdgcn_s_sleep(TSL);
          asm volatile("" : "+v"(ia), "+v"(flag), "+v"(pn), "+v"(so), "+v"(info));
          return true;
        }
        u = e & 4095;
        if (u == 4095) return false;
        if (e & 0x4000u) f = (e >> 12) & 3;
      }
      int pe;
      const bool piece = locate(u, f, dst, flag, pn, so, pe);
      dbg = u * 4 + f;
      ia = ldsbase + (pe * 4 + f) * 4;
      asm volatile("" : "+v"(ia), "+v"(flag));
      info = __builtin_amdgcn_readfirstlane(lds_ld(ia));
      for (; look > 0 && ((info ^ tag) >> 13) != 0u; --look) {
        __builtin_amdgcn_s_sleep(8);
        info = __builtin_amdgcn_readfirstlane(lds_ld(ia));
      }
      if (piece) {
        pn |= 1 << 30 | (j & 3) << 14;
        so += (j & 3) * 1024;
        dst += (j & 3) * 1024;
        ia = ldsbase + (NMIR + 4) * 1024 + ((j >> 2) & 127) * 4;
        info |= F_RAW;
      }
      if (((info ^ tag) >> 13) == 0u) {
        va = __builtin_amdgcn_raw_buffer_load_b128(rs, lane * 16, so, 17);
        if (!piece) {
          vb = __builtin_amdgcn_raw_buffer_load_b128(rs, lane * 16 + 1024, so, 17);
          vc = __builtin_amdgcn_raw_buffer_load_b128(rs, lane * 16 + 2048, so, 17);
        }
      }
      asm volatile("" : "+v"(pn), "+v"(so), "+v"(info));
      return true;
    };
    gu32* pflag = nullptr;
    unsigned pv = 0, pc = 0;
    auto finish = [&]() {
      unsigned c = 3;
      if (pc != 0u && lane == 0)
        asm volatile("ds_add_rtn_u32 %0, %1, %2\n\ts_waitcnt lgkmcnt(0)" : "=v"(c) : "v"(pc), "v"(1u) : "memory");
      if (pflag != nullptr && lane == 0 && (c & 3u) == 3u)
        __hip_atomic_store(pflag, pv, __ATOMIC_RELAXED, __HIP_MEMORY_SCOPE_SYSTEM);
    };
    va = vb = vc = u32x4{0u, 0u, 0u, 0u}; info = 0; so = 0; pn = 0; dst = nullptr; flag = nullptr;
    get_call();
    draw(FWAIT);
    while (true) {
      asm volatile("s_waitcnt vmcnt(0)" : "+v"(va), "+v"(vb), "+v"(vc) :: "memory");
      finish();
      if (pflag != nullptr) T2(3, pdbg);
      pdbg = dbg;
      u32x4 ca = va, cb = vb, cc = vc;
      const int cpn = __builtin_amdgcn_readfirstlane(pn);
      const unsigned cso = so, cia = ia;
      gchar* const cdst = dst;
      pflag = flag;
      pv = __builtin_amdgcn_readfirstlane(info);
      pc = (cpn >> 30) ? cia : 0u;
      const bool more = draw(LK);
      if (!(cpn >> 30) && ((pv ^ tag) >> 13) != 0u) {
        pv = __builtin_amdgcn_readfirstlane(lds_ld(cia));
        if (((pv ^ tag) >> 13) == 0u) {
          ca = __builtin_amdgcn_raw_buffer_load_b128(rs, cso + lane * 16, 0, 17);
          cb = __builtin_amdgcn_raw_buffer_load_b128(rs, cso + lane * 16 + 1024, 0, 17);
          cc = __builtin_amdgcn_raw_buffer_load_b128(rs, cso + lane * 16 + 2048, 0, 17);
        }
      }
      if (((pv ^ tag) >> 13) != 0u) {
        unsigned p[4][4];
        float m;
        convert(cpn & 7, (cpn >> 3) & 31, ((cpn >> 8) & 15) - 1, (cpn >> 10) & 12, p, m);
        if (cpn >> 30) {
          int g = cpn;
          asm volatile("" : "+v"(g));
#pragma unroll
          for (int i = 0; i < 4; ++i) {
            unsigned p0 = p[0][i], p1 = p[1][i], p2 = p[2][i], p3 = p[3][i];
            asm volatile("" : "+v"(p0), "+v"(p1), "+v"(p2), "+v"(p3));
            const unsigned a = (g & (1 << 14)) ? p1 : p0, b = (g & (1 << 14)) ? p3 : p2;
            ca[i] = (g & (2 << 14)) ? b : a;
          }
          pv = tag | F_RAW;
          st16(cdst, ca);
        } else {
          pv = tag | send_block(p, scr, cdst, lane, VD & 1, DSLOW);
        }
      } else {
        st16(cdst, ca);
        if (!(cpn >> 30)) {
          asm volatile("s_waitcnt vmcnt(0)" ::: "memory");
          st16<1024>(cdst, cb);
          asm volatile("s_waitcnt vmcnt(0)" ::: "memory");
          if (VB == 1 || (unsigned)lane < 48u + ((((pv >> 6) & 0x4cu) + (VB ? 4u : 0u)) & (VB ? 0x78u : 0x7cu))) st16<2048>(cdst, cc);
          if (pv & F_RAW) st16<3072>(cdst, __builtin_amdgcn_raw_buffer_load_b128(rs, cso + lane * 16 + 3072, 0, 17));
        }
      }
      if (!more) break;
    }
    asm volatile("s_waitcnt vmcnt(0)" ::: "memory");
    finish();
    T2(3, pdbg);
    if (wave < 2) TS(1 + wave);
   } else if (wave < NSEND + NSC) {
    get_call();
    const u64 t0 = __builtin_amdgcn_s_memrealtime();
    if (wave > NSEND) __builtin_amdgcn_s_sleep(LOS);
    int c = wave - NSEND < NLO ? 0 : NMIR;
    for (unsigned spins = 1; c < NMIR; ++spins) {
      {
        unsigned keep;
        asm volatile("s_mov_b32 %0, m0\n\ts_mov_b32 m0, %2\n\ts_nop 0\n\t"
                     "global_load_lds_dwordx4 %1, off sc0 sc1\n\t"
                     "global_load_lds_dwordx4 %1, off offset:1024 sc0 sc1\n\t"
                     "s_mov_b32 m0, %0"
                     : "=&s"(keep) : "v"(myws + OFF_EF + c * 1024 + lane * 16), "s"(ldsbase + c * 1024) : "memory");
      }
      asm volatile("s_waitcnt vmcnt(0)" ::: "memory");
      for (int i = 0; i < LW && c < NMIR; ++i, ++c) {
        u32x4 v;
        asm volatile("ds_read_b128 %0, %1\n\ts_waitcnt lgkmcnt(0)" : "=v"(v) : "v"(ldsbase + c * 1024 + lane * 16) : "memory");
        if (!__all((((v[0] ^ tag) | (v[1] ^ tag) | (v[2] ^ tag) | (v[3] ^ tag)) >> 13) == 0u)) break;
      }
      if ((spins & 15) == 0 &&
          (__hip_atomic_load(tmo, __ATOMIC_RELAXED, __HIP_MEMORY_SCOPE_AGENT) == call ||
           __builtin_amdgcn_s_memrealtime() - t0 > DEADLINE_TICKS)) {
        if (lane == 0) __hip_atomic_store(tmo, call, __ATOMIC_RELAXED, __HIP_MEMORY_SCOPE_AGENT);
        break;
      }
      __builtin_amdgcn_s_sleep(SCS);
    }
    if (wave == NSEND) TS(33);
   } else if (wave == NWAVE - 1) {
    get_call();
    unsigned* const sq = reinterpret_cast<unsigned*>(myws + OFF_SQ);
    const unsigned qbase = (call - 1) * NDRAW - U0 - S1 * NSWG;
    unsigned ahead = TKD;
    for (unsigned k = S1;; ++k) {
      while ((lds_ld(ctl + C_NEXT * 4) >> 2) + ahead < k) __builtin_amdgcn_s_sleep(2);
      unsigned t = 0;
      if (lane == 0) t = __hip_atomic_fetch_add(sq, 1u, __ATOMIC_RELAXED, __HIP_MEMORY_SCOPE_AGENT);
      const unsigned u = __builtin_amdgcn_readfirstlane(t) - qbase;
      if (u >= (unsigned)NUNIT) {
        if (lane < 8) lds_st(ctl + (C_Q + ((k + lane) & 63)) * 4, ((k + lane + 1) << 16) | 4095u);
        break;
      }
      if (u >= (unsigned)NT0) {
        const int t = 4 * ((int)u - NT0) + (lane & 3), ii = NT0 / 7 + t / 28, r = t % 28, f = r / 7, pr = r % 7;
        int s, q, rr;
        unit_of(ii, pr, s, q, rr);
        const unsigned w = lds_ld(ldsbase + (packed_at(s, q, rr, pr) * 4 + f) * 4);
        const unsigned what = FAST && ((w ^ tag) >> 13) == 0u ? 0x8000u | ((rank + 1 + pr) & 7) << 5 | (2 * q + rr)
                                                              : 0x4000u | (ii * 7 + pr);
        if (lane < 4) lds_st(ctl + (C_Q + ((k + lane) & 63)) * 4, ((k + lane + 1) << 16) | f << 12 | what);
        k += 3;
        ahead = TKT;
      } else if (lane == 0) {
        lds_st(ctl + (C_Q + (k & 63)) * 4, ((k + 1) << 16) | u);
      }
    }
   }
  } else if (wave < 8) {
    if (wave < KW && !(QX & 8)) {
      const int ba = wave * NCW + bk * 8 + bx, bb = ba + KW * NCW;
      gchar *da, *db;
      gu32 *fa, *fb;
      int pa, pb, pe;
      unsigned so, p[4][4];
      float m;
      f32x4 ra[8], rb[8];
      locate(ba >> 2, ba & 3, da, fa, pa, so, pe);
      locate(bb >> 2, bb & 3, db, fb, pb, so, pe);
      T2(2, ba);
      T2(2, bb);
      fetch(pa & 7, (pa >> 3) & 31, ((pa >> 8) & 15) - 1, (pa >> 10) & 12, ra);
      fetch(pb & 7, (pb >> 3) & 31, ((pb >> 8) & 15) - 1, (pb >> 10) & 12, rb);
      round16(((pa >> 8) & 15) - 1, ra, p, m);
      const unsigned ia = send_block(p, scr, da, lane, VD & 1, KSLOW);
      round16(((pb >> 8) & 15) - 1, rb, p, m);
      get_call();
      asm volatile("s_waitcnt vmcnt(0)" ::: "memory");
      if (lane == 0) __hip_atomic_store(fa, tag | ia, __ATOMIC_RELAXED, __HIP_MEMORY_SCOPE_SYSTEM);
      T2(3, ba);
      const unsigned ib = send_block(p, scr, db, lane, VD & 1, KSLOW);
      asm volatile("s_waitcnt vmcnt(0)" ::: "memory");
      if (lane == 0) __hip_atomic_store(fb, tag | ib, __ATOMIC_RELAXED, __HIP_MEMORY_SCOPE_SYSTEM);
      T2(3, bb);
    }
    if (wave == 0) TS(30);
#pragma unroll
    for (int a = 0; a < 5; ++a)
#pragma unroll
      for (int b = 0; b < 4; ++b) acc[a][b] = f32x4{0.f, 0.f, 0.f, 0.f};
    const int xoff = (wm * 5) * 1024 + lane * 16;
    const int woff = 20480 + (wn * 4) * 1024 + lane * 16;
    const u64 c0 = CYC(), r0 = __builtin_amdgcn_s_memrealtime();
    {
      unsigned vx = ldsbase + xoff, vw = ldsbase + woff, vf = ctl + C_FULL * 4, vc = ctl + wave * 4, vt, vu;
      unsigned sfl, sdn, sof, sd, sdb = 0;
      asm volatile(
          "s_mov_b32 %[sdn], 0\n\t"
          "s_mov_b32 %[sof], 0\n\t"
          "s_nop 4\n\t"
          FETCH(RB0, RB1, RB2, RA0, RA1, RA2, RA3)
          "1:\n\t"
          "s_cmp_lg_u32 %[sfl], 0\n\t"
          "s_cbranch_scc0 3f\n\t"
          BODY(RA0, RA1, RA2, RA3, RB0, RB1, RB2, RB3, PFY, PFN)
          "2:\n\t"
          "s_cmp_lg_u32 %[sfl], 0\n\t"
          "s_cbranch_scc0 4f\n\t"
          BODY(RB0, RB1, RB2, RB3, RA0, RA1, RA2, RA3, PFY, PFN)
          "s_branch 1b\n\t"
          "3:\n\t"
          BODY(RA0, RA1, RA2, RA3, RB0, RB1, RB2, RB3, PFN, PFY)
          FETCH(RA0, RA1, RA2, RB0, RB1, RB2, RB3)
          "s_branch 2b\n\t"
          "4:\n\t"
          BODY(RB0, RB1, RB2, RB3, RA0, RA1, RA2, RA3, PFN, PFY)
          "s_cmp_eq_u32 %[sdn], %[ns]\n\t"
          "s_cbranch_scc1 5f\n\t"
          FETCH(RB0, RB1, RB2, RA0, RA1, RA2, RA3)
          "s_branch 1b\n\t"
          "5:\n\t"
          "s_setprio 0\n\t"
          "s_nop 15\n\t"
          "s_nop 15\n\t"
          :
          "+{v[40:43]}"(acc[0][0]), "+{v[44:47]}"(acc[0][1]), "+{v[48:51]}"(acc[0][2]), "+{v[52:55]}"(acc[0][3]),
          "+{v[56:59]}"(acc[1][0]), "+{v[60:63]}"(acc[1][1]), "+{v[64:67]}"(acc[1][2]), "+{v[68:71]}"(acc[1][3]),
          "+{v[72:75]}"(acc[2][0]), "+{v[76:79]}"(acc[2][1]), "+{v[80:83]}"(acc[2][2]), "+{v[84:87]}"(acc[2][3]),
          "+{v[88:91]}"(acc[3][0]), "+{v[92:95]}"(acc[3][1]), "+{v[96:99]}"(acc[3][2]), "+{v[100:103]}"(acc[3][3]),
          "+{v[104:107]}"(acc[4][0]), "+{v[108:111]}"(acc[4][1]), "+{v[112:115]}"(acc[4][2]), "+{v[116:119]}"(acc[4][3]),
          [vx] "+v"(vx), [vw] "+v"(vw), [vf] "+v"(vf), [vt] "=&v"(vt), [vu] "=&v"(vu), [sfl] "=&s"(sfl), [sdn] "=&s"(sdn),
          [sof] "=&s"(sof), [sd] "=&s"(sd), [sdb] "+s"(sdb)
          : [vc] "v"(vc), [dn] "n"(C_DONE), [dm] "n"(C_DONE + 8), [dp] "n"((C_DONE + 4) * 4), [hs] "n"(HS), [hl] "n"((NH - 1) * HS), [hw] "s"(-(NH - 1) * HS),
            [ns] "n"(2 * NSTEP)
          : "memory", "scc", "v0", "v1", "v2", "v3", "v4", "v5", "v6", "v7", "v8", "v9", "v10", "v11", "v12", "v13", "v14", "v15", "v16", "v17", "v18", "v19", "v20", "v21", "v22", "v23", "v24", "v25", "v26", "v27", "v28", "v29", "v30", "v31", "v32", "v33", "v34", "v35", "v36", "v37", "v38", "v39");
      TV(48 + wave, sdb);
    }
    if (wave == 0) { TV(40, CYC() - c0); TV(41, __builtin_amdgcn_s_memrealtime() - r0); }
    float m = 0.f;
#pragma unroll
    for (int a = 0; a < 5; ++a) {
      if (ti * TM + wm * 80 + a * 16 >= 4096) continue;
#pragma unroll
      for (int b = 0; b < 4; ++b)
#pragma unroll
        for (int j = 0; j < 4; ++j) m = fmaxf(m, fabsf(acc[a][b][j]));
    }
#pragma unroll
    for (int o = 32; o >= 1; o >>= 1) m = fmaxf(m, __shfl_xor(m, o));
    if (lane == 0) lds_st(ctl + (C_WMAX + wave) * 4, __float_as_uint(m));
    if (wave == 0) TS(12);
  } else if (wave < 10) {
    const int L = wave - 8;
    __builtin_amdgcn_s_setprio(LPRIO);
    get_call();
    {
      const u64 t0 = __builtin_amdgcn_s_memrealtime();
      while (__builtin_amdgcn_s_memrealtime() - t0 < DLY * 100ull) __builtin_amdgcn_s_sleep(32);
    }
    const char* const gx0 = myws + OFF_XS + L * 8192 + lane * 16;
    const char* const gw0 = myws + OFF_WS + (size_t)tj * UNIT + L * 8192 + lane * 16;
    int slot = L;
    const unsigned dna = ctl + (C_DONE + (lane & 7)) * 4;
    u64 rm = 0, avail = 0, left = ~0ull;
    unsigned dv = 0;
    for (int t = 0; t < NSTEP; ++t) {
      const int need = 2 * t + L - NH + 1;
      while (avail == 0 || !__all((int)dv >= need)) {
        __builtin_amdgcn_s_sleep(1);
        asm volatile("ds_read_b64 %0, %2\n\tds_read_b32 %1, %3\n\ts_waitcnt lgkmcnt(0)"
                     : "=&v"(rm), "=&v"(dv) : "v"(ctl + C_Q * 4), "v"(dna) : "memory");
        avail = left & (((u64)(unsigned)__builtin_amdgcn_readfirstlane((int)(rm >> 32)) << 32) |
                        (u64)(unsigned)__builtin_amdgcn_readfirstlane((int)rm));
      }
      const int c = __builtin_ctzll(avail);
      left &= ~(1ull << c);
      const int bit = ((rank - (c & 7)) & 7) * 8 + (c >> 3);
      asm volatile("s_waitcnt vmcnt(29)" ::: "memory");
      const unsigned l = ldsbase + slot * HS;
#pragma unroll
      for (int i = 0; i < 4; ++i) {
        const int hu = 5 * ti + i;
        glds4k(gx0 + ((size_t)(bit * XU + (hu >> 1))) * UNIT + (hu & 1) * 4096, l + i * 4096);
      }
      {
        const int hu = min(5 * ti + 4, 63);
        const char* g0 = gx0 + ((size_t)(bit * XU + (hu >> 1))) * UNIT + (hu & 1) * 4096;
        const char* g1 = gw0 + (size_t)(bit * WU) * UNIT;
        unsigned keep;
        asm volatile(
            "ds_read_b64 %0, %3\n\tds_read_b32 %1, %4\n\t"
            "s_mov_b32 %2, m0\n\ts_mov_b32 m0, %9\n\ts_nop 0\n\t"
            "global_load_lds_dwordx4 %5, off" LDPOL "\n\t"
            "global_load_lds_dwordx4 %5, off offset:1024" LDPOL "\n\t"
            "global_load_lds_dwordx4 %5, off offset:2048" LDPOL "\n\t"
            "global_load_lds_dwordx4 %5, off offset:3072" LDPOL "\n\t"
            "s_add_u32 m0, m0, 0x1000\n\ts_nop 0\n\t"
            "global_load_lds_dwordx4 %6, off" LDPOL "\n\t"
            "global_load_lds_dwordx4 %6, off offset:1024" LDPOL "\n\t"
            "global_load_lds_dwordx4 %6, off offset:2048" LDPOL "\n\t"
            "global_load_lds_dwordx4 %6, off offset:3072" LDPOL "\n\t"
            "s_add_u32 m0, m0, 0x1000\n\ts_nop 0\n\t"
            "global_load_lds_dwordx4 %7, off" LDPOL "\n\t"
            "global_load_lds_dwordx4 %7, off offset:1024" LDPOL "\n\t"
            "global_load_lds_dwordx4 %7, off offset:2048" LDPOL "\n\t"
            "global_load_lds_dwordx4 %7, off offset:3072" LDPOL "\n\t"
            "s_mov_b32 m0, %10\n\ts_mov_b64 exec, 1\n\t"
            "global_load_lds_dword %8, off sc0 sc1\n\t"
            "s_mov_b64 exec, -1\n\ts_mov_b32 m0, %2\n\t"
            "s_waitcnt lgkmcnt(0)"
            : "=&v"(rm), "=&v"(dv), "=&s"(keep)
            : "v"(ctl + C_Q * 4), "v"(dna), "v"(g0), "v"(g1), "v"(g1 + 4096), "v"(myws + OFF_TICKET),
              "s"(l + 16384), "s"(ctl + (C_FULL + 2 * t + L) * 4)
            : "memory", "scc");
        avail = left & (((u64)(unsigned)__builtin_amdgcn_readfirstlane((int)(rm >> 32)) << 32) |
                        (u64)(unsigned)__builtin_amdgcn_readfirstlane((int)rm));
      }
      slot = slot + 2 >= NH ? slot + 2 - NH : slot + 2;
    }
    asm volatile("s_waitcnt vmcnt(0)" ::: "memory");
    if (L == 0) TS(11);
  } else if (wave == 10) {
    get_call();
    __builtin_amdgcn_s_setprio(WPRIO);
    const unsigned want = tag | F_RAW;
    const unsigned m2 = ti == NTI - 1 ? 0u : ~0u;
    const int st = ((rank - (lane & 7)) & 7) * 8 + (lane >> 3);
    const unsigned* pf0 = reinterpret_cast<const unsigned*>(myws + OFF_XF) + (((5 * ti) >> 1) * 64 + st) * SU;
    const unsigned* pf2 = reinterpret_cast<const unsigned*>(myws + OFF_WF) + (tj * 64 + st) * SU;
    u64 seen = 0;
    unsigned cnt = 0, spins = 0, kk = 0;
    bool dead = false;
    const u64 t0 = __builtin_amdgcn_s_memrealtime();
    while (cnt < (unsigned)NSTEP) {
      u32x4 f0, f1, f2, f3;
      asm volatile(
                   "s_mov_b64 exec, %6\n\t"
                   "global_load_dwordx4 %0, %4, off sc0 sc1\n\tglobal_load_dwordx4 %1, %4, off offset:1024 sc0 sc1\n\t"
                   "global_load_dwordx4 %2, %4, off offset:2048 sc0 sc1\n\t"
                   "global_load_dwordx4 %3, %5, off sc0 sc1\n\ts_mov_b64 exec, -1\n\ts_waitcnt vmcnt(0)"
                   : "=&v"(f0), "=&v"(f1), "=&v"(f2), "=&v"(f3) : "v"(pf0), "v"(pf2), "s"(~seen) : "memory");
      const u64 nm = __ballot((QX & 1) || dead || plain16(f0, f1, f2, f3, want, m2)) & ~seen;
      if (nm) {
        if (lane == 0) asm volatile("ds_write_b64 %0, %1" :: "v"(ctl + C_Q * 4), "v"(seen | nm) : "memory");
        cnt += (unsigned)__builtin_popcountll(nm);
        seen |= nm;
      } else if ((++spins & 15) == 0 &&
                 (__hip_atomic_load(tmo, __ATOMIC_RELAXED, __HIP_MEMORY_SCOPE_AGENT) == call ||
                  __builtin_amdgcn_s_memrealtime() - t0 > DEADLINE_TICKS)) {
        if (lane == 0) __hip_atomic_store(tmo, call, __ATOMIC_RELAXED, __HIP_MEMORY_SCOPE_AGENT);
        dead = true;
      }
      if (cnt < WFAST) __builtin_amdgcn_s_sleep(PSLEEP); else __builtin_amdgcn_s_sleep(PS2);
    }
    if (dead && lane == 0) lds_st(ctl + C_DEAD * 4, ~0u);
    TS(32);
    if (GOX && ER && ti < 4) {
      while ((int)(__hip_atomic_load(reinterpret_cast<unsigned*>(myws + OFF_GO), __ATOMIC_RELAXED, __HIP_MEMORY_SCOPE_AGENT) -
                   (call - 1) * NSWG) < GOX &&
             __builtin_amdgcn_s_memrealtime() - t0 < DEADLINE_TICKS)
        __builtin_amdgcn_s_sleep(32);
    }
  } else {
   const __amdgpu_buffer_rsrc_t rs = __builtin_amdgcn_make_buffer_rsrc(myws, 0, 0x7fffffff, 0x00020000);
   const u64 t0 = __builtin_amdgcn_s_memrealtime();
   bool dead = false;
   auto arrival = [&](int n, unsigned& so, unsigned& fo) {
      const int pr = n % 7;
      int s, q, rr;
      unit_of(n / 7, pr, s, q, rr);
      const int st = ((rank + 7 - pr) & 7) * 8 + s;
      if (rr != 2) {
        so = (unsigned)OFF_XS + (unsigned)(st * XU + 2 * q + rr) * UNIT;
        fo = (unsigned)OFF_XF + ((2 * q + rr) * 64 + st) * 16;
      } else {
        so = (unsigned)OFF_WS + (unsigned)(st * WU + q) * UNIT;
        fo = (unsigned)OFF_WF + (q * 64 + st) * 16;
      }
      so = __builtin_amdgcn_readfirstlane(so);
      fo = __builtin_amdgcn_readfirstlane(fo);
   };
   typedef __attribute__((ext_vector_type(3))) unsigned u32x3;
   auto expand = [&](unsigned f, u32x4 a, u32x4 b, u32x3 c, unsigned e, unsigned so) {
        const unsigned top = f & 0xffu, top4 = top * 0x01010101u;
        unsigned e4[8];
#pragma unroll
        for (int k = 0; k < 8; ++k)
          e4[k] = top4 - (((c[k >> 2] >> (2 * (k & 3))) & 0x03030303u) |
                          (((c[2] >> (2 * (k & 3) + (k >> 2))) & 0x01010101u) << 2));
        if (f & 0x300u) {
          const unsigned M = listed(c[0], c[1], c[2]);
          unsigned total, x = M - ((M >> 1) & 0x55555555u);
          x = (x & 0x33333333u) + ((x >> 2) & 0x33333333u);
          x = (x + (x >> 4)) & 0x0f0f0f0fu;
          x = (x << 8) + (x << 16) + (x << 24);
          const unsigned i0 = ahead_of(__builtin_popcount(M), total) - 1u;
          const unsigned w0 = (unsigned)__builtin_amdgcn_ds_bpermute((int)(i0 & 0xfcu), (int)e);
          const unsigned w1 = (unsigned)__builtin_amdgcn_ds_bpermute((int)((i0 + 4u) & 0xfcu), (int)e);
          const unsigned w2 = (unsigned)__builtin_amdgcn_ds_bpermute((int)((i0 + 8u) & 0xfcu), (int)e);
          const unsigned lo = __builtin_amdgcn_alignbyte(w1, w0, i0 & 3u), hi = __builtin_amdgcn_alignbyte(w2, w1, i0 & 3u);
#pragma unroll
          for (int j = 0; j < 8; ++j) {
            const int k = (j >> 1) + 4 * (j & 1);
            const unsigned t = (M >> j) & 0x01010101u, mk = (t << 8) - t;
            x += t;
            e4[k] = (e4[k] & ~mk) | (__builtin_amdgcn_perm(hi, lo, x) & mk);
          }
        }
#pragma unroll
        for (int j = 0; j < 4; ++j) {
          u32x4 o;
#pragma unroll
          for (int h = 0; h < 2; ++h) {
            const int k = 2 * j + h;
            const unsigned sm = k < 4 ? a[k & 3] : b[k & 3];
            const unsigned hi = (sm & 0x80808080u) | ((e4[k] >> 1) & 0x7f7f7f7fu);
            const unsigned lo = (sm & 0x7f7f7f7fu) | ((e4[k] << 7) & 0x80808080u);
            o[2 * h] = __builtin_amdgcn_perm(hi, lo, 0x05010400u);
            o[2 * h + 1] = __builtin_amdgcn_perm(hi, lo, 0x07030602u);
          }
          __builtin_amdgcn_raw_buffer_store_b128(o, rs, lane * 16 + j * 1024, so, 17);
          asm volatile("s_nop 1" :: "v"(o) : "memory");
        }
   };
   if (wave < W_DEC + NDEC) {
    __builtin_amdgcn_s_setprio(DPRIO);
    get_call();
    for (int n = (wave - W_DEC) * NCW + bk * 8 + bx; n < NT0 - NL && !dead && !(QX & 16); n += NDEC * NCW) {
      unsigned so, fo;
      arrival(n, so, fo);
      unsigned f[4];
      for (unsigned spins = 1;; ++spins) {
        asm volatile("" ::: "memory");
        const u32x4 fv = __builtin_amdgcn_raw_buffer_load_b128(rs, 0, (unsigned)OFF_F1 + n * F1S, 17);
#pragma unroll
        for (int k = 0; k < 4; ++k) f[k] = __builtin_amdgcn_readfirstlane(fv[k]);
        if ((((f[0] ^ tag) | (f[1] ^ tag) | (f[2] ^ tag) | (f[3] ^ tag)) >> 13) == 0u) break;
        if ((spins & 15) == 0 &&
            (__hip_atomic_load(tmo, __ATOMIC_RELAXED, __HIP_MEMORY_SCOPE_AGENT) == call ||
             __builtin_amdgcn_s_memrealtime() - t0 > DEADLINE_TICKS)) {
          if (lane == 0) __hip_atomic_store(tmo, call, __ATOMIC_RELAXED, __HIP_MEMORY_SCOPE_AGENT);
          dead = true;
          break;
        }
        __builtin_amdgcn_s_sleep(DSLEEP);
      }
      if (dead) continue;
      T2(0, n);
      u32x4 a[4], b[4];
      u32x3 c[4];
      unsigned e[4];
#pragma unroll
      for (int k = 0; k < 4; ++k) {
        a[k] = __builtin_amdgcn_raw_buffer_load_b128(rs, lane * 16 + k * 4096, so, 17);
        b[k] = __builtin_amdgcn_raw_buffer_load_b128(rs, lane * 16 + k * 4096 + 1024, so, 17);
        e[k] = __builtin_amdgcn_raw_buffer_load_b32(rs, lane * 4 + k * 4096 + 2816, so, 17);
        c[k] = __builtin_amdgcn_raw_buffer_load_b96(rs, lane * 12 + k * 4096 + 2048, so, 17);
      }
      asm volatile("s_waitcnt vmcnt(0)" : "+v"(a[0]), "+v"(a[1]), "+v"(a[2]), "+v"(a[3]), "+v"(b[0]), "+v"(b[1]), "+v"(b[2]), "+v"(b[3]) :: "memory");
      asm volatile("" : "+v"(c[0]), "+v"(c[1]), "+v"(c[2]), "+v"(c[3]), "+v"(e[0]), "+v"(e[1]), "+v"(e[2]), "+v"(e[3]) :: "memory");
#pragma unroll
      for (int k = 0; k < 4; ++k) {
        if (f[k] & F_RAW) continue;
        expand(f[k], a[k], b[k], c[k], e[k], so + k * 4096);
      }
      asm volatile("s_waitcnt vmcnt(0)" ::: "memory");
      if (lane == 0) {
        const unsigned w = tag | F_RAW;
        __builtin_amdgcn_raw_buffer_store_b128(u32x4{w, w, w, w}, rs, 0, fo, 17);
      }
      T2(1, n);
    }
    if (wave == W_DEC) TS(19);
   } else {
    __builtin_amdgcn_s_setprio(PPRIO);
    if (!(QX & 32)) pack_items((wave - W_DEC - NDEC + 8 * CPK) * NCW + bk * 8 + bx, NROUND);
    if (wave == NWAVE - 1) TS(31);
    while (NL && __builtin_popcount(__builtin_amdgcn_readfirstlane(lds_ld(ctl + C_Q * 4 + 4))) < 8) __builtin_amdgcn_s_sleep(32);
    __builtin_amdgcn_s_setprio(DPRIO);
   }
   for (int m = ((wave - W_DEC + NPK) % (NDEC + NPK)) * NCW + bk * 8 + bx; m < NL * 4 && !dead; m += (NDEC + NPK) * NCW) {
      const int n = NT0 - NL + (m >> 2), k = m & 3;
      unsigned so, fo, f;
      arrival(n, so, fo);
      so += k * 4096;
      for (unsigned spins = 1;; ++spins) {
        asm volatile("" ::: "memory");
        f = __builtin_amdgcn_readfirstlane(__builtin_amdgcn_raw_buffer_load_b32(rs, 0, (unsigned)OFF_F1 + n * F1S + k * 4, 17));
        if (((f ^ tag) >> 13) == 0u) break;
        if ((spins & 15) == 0 &&
            (__hip_atomic_load(tmo, __ATOMIC_RELAXED, __HIP_MEMORY_SCOPE_AGENT) == call ||
             __builtin_amdgcn_s_memrealtime() - t0 > DEADLINE_TICKS)) {
          if (lane == 0) __hip_atomic_store(tmo, call, __ATOMIC_RELAXED, __HIP_MEMORY_SCOPE_AGENT);
          dead = true;
          break;
        }
        __builtin_amdgcn_s_sleep(LSLEEP);
      }
      if (dead) break;
      T2(0, n);
      if (!(f & F_RAW)) {
        u32x4 a = __builtin_amdgcn_raw_buffer_load_b128(rs, lane * 16, so, 17);
        u32x4 b = __builtin_amdgcn_raw_buffer_load_b128(rs, lane * 16 + 1024, so, 17);
        unsigned e = __builtin_amdgcn_raw_buffer_load_b32(rs, lane * 4 + 2816, so, 17);
        u32x3 c = __builtin_amdgcn_raw_buffer_load_b96(rs, lane * 12 + 2048, so, 17);
        asm volatile("s_waitcnt vmcnt(0)" : "+v"(a), "+v"(b), "+v"(c), "+v"(e) :: "memory");
        expand(f, a, b, c, e, so);
      }
      asm volatile("s_waitcnt vmcnt(0)" ::: "memory");
      if (lane == 0) __builtin_amdgcn_raw_buffer_store_b32(tag | F_RAW, rs, 0, fo + k * 4, 17);
      T2(1, n);
   }
  }

  asm volatile("s_waitcnt lgkmcnt(0)" ::: "memory");
  __syncthreads();
  const u64 outp = (u64)(uintptr_t)P.out_all[lane & 7];
  gchar* const slp = (gchar*)P.ws_all[lane & 7];
  get_call();

  if (wave == 0) TS(13);
  auto global_max = [&](bool slow) -> unsigned {
    const u64* slots = reinterpret_cast<const u64*>(myws + OFF_SLOTS + (blockIdx.x & 7) * 512) + (lane & 7);
    const u64 t0 = __builtin_amdgcn_s_memrealtime();
    unsigned g;
    for (unsigned spins = 1;; ++spins) {
      const u64 v = __hip_atomic_load(slots, __ATOMIC_RELAXED, __HIP_MEMORY_SCOPE_SYSTEM);
      g = (unsigned)v;
      if (__all((unsigned)(v >> 32) >= call)) break;
      if ((spins & 15) == 0 &&
          (__hip_atomic_load(tmo, __ATOMIC_RELAXED, __HIP_MEMORY_SCOPE_AGENT) == call ||
           __builtin_amdgcn_s_memrealtime() - t0 > DEADLINE_TICKS)) {
        if (lane == 0) __hip_atomic_store(tmo, call, __ATOMIC_RELAXED, __HIP_MEMORY_SCOPE_AGENT);
        g = ~0u;
        break;
      }
      if (slow) __builtin_amdgcn_s_sleep(100); else __builtin_amdgcn_s_sleep(ASL);
    }
#pragma unroll
    for (int o = 4; o >= 1; o >>= 1) g = max(g, (unsigned)__shfl_xor((int)g, o));
    return g;
  };
  if (sender) {
    if (ER && wave == 15) {
      if (GOX && lane == 0)
        __hip_atomic_fetch_add(reinterpret_cast<unsigned*>(myws + OFF_GO), 1u, __ATOMIC_RELAXED, __HIP_MEMORY_SCOPE_AGENT);
      const unsigned g = global_max(true);
      if (lane == 0) { lds_st(ctl + C_GMAX * 4, g); lds_st(ctl + C_DEAD * 4, 1u); }
    } else if (ER && wave >= 1 && (wave - 1) * NSWG + (int)blockIdx.x < 512) {
      const int it = (wave - 1) * NSWG + blockIdx.x;
      const int src = it & 7, c64 = (it >> 3) & 31, hu = it >> 8;
      const int lr = rank * 128 + hu * 64 + (lane & 3) * 16;
      const u64* fl = reinterpret_cast<const u64*>(myws + OFF_EOF) +
                      ((src * 4 + lr / TM) * 16 + (c64 >> 1)) * 8 + (c64 & 1) * 4 + (lr % TM) / 80;
      while (__builtin_amdgcn_readfirstlane(lds_ld(ctl + C_DEAD * 4)) == 0u) __builtin_amdgcn_s_sleep(16);
      const u64 t0 = __builtin_amdgcn_s_memrealtime();
      bool dead = false;
      u64 v;
      for (unsigned spins = 1;; ++spins) {
        v = __hip_atomic_load(fl, __ATOMIC_RELAXED, __HIP_MEMORY_SCOPE_SYSTEM);
        if (__all((unsigned)(v >> 32) >= call)) break;
        if ((spins & 15) == 0 &&
            (__hip_atomic_load(tmo, __ATOMIC_RELAXED, __HIP_MEMORY_SCOPE_AGENT) == call ||
             __builtin_amdgcn_s_memrealtime() - t0 > DEADLINE_TICKS)) {
          if (lane == 0) __hip_atomic_store(tmo, call, __ATOMIC_RELAXED, __HIP_MEMORY_SCOPE_AGENT);
          dead = true;
          break;
        }
        __builtin_amdgcn_s_sleep(32);
      }
      const __amdgpu_buffer_rsrc_t rs = __builtin_amdgcn_make_buffer_rsrc(myws, 0, 0x7fffffff, 0x00020000);
      const unsigned so = (unsigned)OFF_ES + (unsigned)((src * 8 + hu * 4) * 32 + c64) * 2048;
      u32x4 h[4][2];
#pragma unroll
      for (int g = 0; g < 4; ++g) {
        h[g][0] = __builtin_amdgcn_raw_buffer_load_b128(rs, lane * 16, so + g * 65536, 17);
        h[g][1] = __builtin_amdgcn_raw_buffer_load_b128(rs, lane * 16 + 1024, so + g * 65536, 17);
      }
      const float amax = __uint_as_float(lds_ld(ctl + C_GMAX * 4));
      dead |= __float_as_uint(amax) == ~0u;
      const float scale = amax / 127.0f;
      const float inv = scale > 0.f ? 1.0f / scale : 0.f;
      const unsigned here = (unsigned)__ballot(dead || (lr % 80) / 16 < (int)(v & 7u)) & (ER >> (hu * 4));
      gchar* ob = (gchar*)P.out_all[rank] + (size_t)(hu * 64 + (lane & 15)) * NTOT + src * NLOC + c64 * 64 + (lane >> 4) * 16;
#pragma unroll
      for (int g = 0; g < 4; ++g) {
        if (!((here >> g) & 1u)) continue;
        const float step = __uint_as_float((unsigned)__builtin_amdgcn_readlane((int)v, g));
        u32x4 o;
#pragma unroll
        for (int j = 0; j < 4; ++j) {
          unsigned pk = 0;
#pragma unroll
          for (int nt = 0; nt < 4; ++nt) {
            const int w = (int)h[g][j >> 1][(j & 1) * 2 + (nt >> 1)];
            const float y = (float)((nt & 1) ? w >> 16 : (w << 16) >> 16) * step;
            float q = rintf(y * inv);
            q = fminf(fmaxf(q, -127.f), 127.f);
            const int qi = dead ? -128 : (int)q;
            pk |= ((unsigned)qi & 0xffu) << (8 * nt);
          }
          o[j] = pk;
        }
        st16(ob + (size_t)g * 16 * NTOT, o);
      }
      asm volatile("s_waitcnt vmcnt(0)" ::: "memory");
      TS(24);
    } else if (wave == 0 && blockIdx.x < NCOL) {
      const char* rp = myws + OFF_REP + lane * 16;
      const u64 t0 = __builtin_amdgcn_s_memrealtime();
      unsigned m;
      for (unsigned spins = 1;; ++spins) {
        u32x4 r0, r1;
        asm volatile("global_load_dwordx4 %0, %2, off sc0 sc1\n\tglobal_load_dwordx4 %1, %2, off offset:1024 sc0 sc1\n\t"
                     "s_waitcnt vmcnt(0)" : "=&v"(r0), "=&v"(r1) : "v"(rp) : "memory");
        if (lane >= (NCW - 128) / 2) r1 = r0;
        m = max(max(r0[0], r0[2]), max(r1[0], r1[2]));
        if (__all(r0[1] == call && r0[3] == call && r1[1] == call && r1[3] == call)) break;
        if ((spins & 15) == 0 && __builtin_amdgcn_s_memrealtime() - t0 > DEADLINE_TICKS) {
          if (lane == 0) __hip_atomic_store(tmo, call, __ATOMIC_RELAXED, __HIP_MEMORY_SCOPE_AGENT);
          m = ~0u;
          break;
        }
        __builtin_amdgcn_s_sleep(CSL);
      }
      TS(20);
#pragma unroll
      for (int o = 32; o >= 1; o >>= 1) m = max(m, (unsigned)__shfl_xor((int)m, o));
      __hip_atomic_store((gu64*)(slp + OFF_SLOTS + (lane >> 3) * 512) + rank, ((u64)call << 32) | m, __ATOMIC_RELAXED,
                         __HIP_MEMORY_SCOPE_SYSTEM);
    }
    return;
  }
  {
  int b2 = blockIdx.x, w2 = wave;
  asm volatile("" : "+s"(b2), "+s"(w2));
  const int bx2 = b2 & 7, bk2 = ((b2 >> 3) + 26) & 31;
  const int ti = bk2 < 24 ? 2 * (bk2 % 6) + (bx2 & 1) : 12;
  const int tj = 4 * (bx2 >> 1) + (bk2 < 24 ? bk2 / 6 : 2 * (bx2 & 1) + ((bk2 - 24) & 1));
  const int wm = w2 & 3, wn = (w2 >> 2) & 1;
  const bool mine = lane == rows_of((5 * ti) >> 1) >> 2 || lane == rows_of((5 * ti + 2) >> 1) >> 2 ||
                    lane == rows_of(min((5 * ti + 4) >> 1, 31)) >> 2;
  if (wave == 8) {
    unsigned m = lds_ld(ctl + (lane < 8 ? C_WMAX + lane : C_DEAD) * 4);
#pragma unroll
    for (int o = 8; o >= 1; o >>= 1) m = max(m, (unsigned)__shfl_xor((int)m, o));
    if (lane == 0)
      __hip_atomic_store((gu64*)(myws + OFF_REP) + (blockIdx.x - NSWG), ((u64)call << 32) | m, __ATOMIC_RELAXED,
                         __HIP_MEMORY_SCOPE_SYSTEM);
  }
  int ne = 0;
  if (ER && wave < 8 && ti * TM + wm * 80 < 1024) {
    if (wave == 0) TS(25);
    float tm = __uint_as_float(lds_ld(ctl + (C_WMAX + (lane & 7)) * 4));
#pragma unroll
    for (int o = 4; o >= 1; o >>= 1) tm = fmaxf(tm, __shfl_xor(tm, o));
    const unsigned sb = __float_as_uint(fmaxf(tm, 1e-30f) / 32000.0f) & ~7u;
    const float k16 = 1.0f / __uint_as_float(sb);
#pragma unroll
    for (int mt = 0; mt < 5; ++mt) {
      const int row0 = ti * TM + wm * 80 + mt * 16;
      if (ne != mt || __builtin_amdgcn_readfirstlane(lds_ld(ctl + C_ECNT * 4)) != 0u) continue;
      ne = mt + 1;
      if (row0 >= 1024 || !((ER >> ((row0 >> 4) & 7)) & 1)) continue;
      gchar* ob = (gchar*)(((u64)(unsigned)__builtin_amdgcn_readlane((int)((u64)(uintptr_t)slp >> 32), row0 >> 7) << 32) |
                           (unsigned)__builtin_amdgcn_readlane((int)(u64)(uintptr_t)slp, row0 >> 7));
      ob += OFF_ES + (size_t)((rank * 8 + ((row0 & 127) >> 4)) * 32 + tj * 2 + wn) * 2048 + lane * 16;
      u32x4 h[2];
#pragma unroll
      for (int j = 0; j < 4; ++j)
#pragma unroll
        for (int n2 = 0; n2 < 2; ++n2)
          h[j >> 1][(j & 1) * 2 + n2] = __builtin_amdgcn_perm((unsigned)(int)rintf(acc[mt][2 * n2 + 1][j] * k16),
                                                              (unsigned)(int)rintf(acc[mt][2 * n2][j] * k16), 0x05040100u);
      st16(ob, h[0]);
      if (EOTH == 2) asm volatile("s_waitcnt vmcnt(0)" ::: "memory");
      st16<1024>(ob, h[1]);
      asm volatile("s_waitcnt vmcnt(0)" ::: "memory");
    }
    if (lane == (ti * TM + wm * 80) >> 7 || lane == min(ti * TM + wm * 80 + 79, 1023) >> 7)
      __hip_atomic_store((gu64*)(slp + OFF_EOF) + ((rank * 4 + ti) * 16 + tj) * 8 + w2, ((u64)call << 32) | sb | (unsigned)ne,
                         __ATOMIC_RELAXED, __HIP_MEMORY_SCOPE_SYSTEM);
    TS(26);
    TV(27, ne);
  }
  if (wave == 8) {
    if (IDL) __builtin_amdgcn_s_sleep(IDL);
    const unsigned g = global_max(SPOLL && ER && ti < 4);
    TS(22);
    if (lane == 0) { lds_st(ctl + C_GMAX * 4, g); lds_st(ctl + C_ECNT * 4, 1u); }
  }
  if (SPOLL && ER && wave == 9 && ti < 4) {
    typedef __attribute__((ext_vector_type(16))) unsigned u32x16;
    const char* sp = myws + OFF_SLOTS + bx2 * 512;
    while (__builtin_amdgcn_readfirstlane(lds_ld(ctl + C_ECNT * 4)) == 0u) {
      u32x16 v;
      asm volatile("s_load_dwordx16 %0, %1, 0x0 glc\n\ts_waitcnt lgkmcnt(0)" : "=s"(v) : "s"(sp) : "memory");
      unsigned c = ~0u, g = 0;
#pragma unroll
      for (int i = 0; i < 8; ++i) { c = min(c, v[2 * i + 1]); g = max(g, v[2 * i]); }
      if (c >= call) {
        if (lane == 0) { lds_st(ctl + C_GMAX * 4, g); lds_st(ctl + C_ECNT * 4, 1u); }
        TS(34);
        break;
      }
      __builtin_amdgcn_s_sleep(SSL);
    }
  }
  if (wave < 8)
    while (__builtin_amdgcn_readfirstlane(lds_ld(ctl + C_ECNT * 4)) == 0u) __builtin_amdgcn_s_sleep(1);
  if (wave == 0) TS(14);
  if (wave < 8) {
    const float amax = __uint_as_float(lds_ld(ctl + C_GMAX * 4));
    const bool dead = __float_as_uint(amax) == ~0u;
    const float scale = amax / 127.0f;
    const float inv = scale > 0.f ? 1.0f / scale : 0.f;
    const int col = rank * NLOC + tj * 128 + wn * 64 + (lane >> 4) * 16;
#pragma unroll
    for (int mt = 0; mt < 5; ++mt) {
      const int lr0 = ti * TM + wm * 80 + mt * 16;
      if (lr0 >= 4096 || (mt < ne && lr0 < 1024 && ((ER >> ((lr0 >> 4) & 7)) & 1))) continue;
      const int row0 = rows_of(lr0 >> 7) * 128 + (lr0 & 127);
      const int owner = row0 >> 9;
      gchar* ob = (gchar*)(((u64)(unsigned)__builtin_amdgcn_readlane((int)(outp >> 32), owner) << 32) |
                           (unsigned)__builtin_amdgcn_readlane((int)outp, owner));
      u32x4 o;
#pragma unroll
      for (int j = 0; j < 4; ++j) {
        unsigned pk = 0;
#pragma unroll
        for (int nt = 0; nt < 4; ++nt) {
          float q = rintf(acc[mt][nt][j] * inv);
          q = fminf(fmaxf(q, -127.f), 127.f);
          int qi = dead ? -128 : (int)q;
          pk |= ((unsigned)qi & 0xffu) << (8 * nt);
        }
        o[j] = pk;
      }
      st16(ob + (size_t)((row0 & 511) + (lane & 15)) * NTOT + col, o);
      asm volatile("s_waitcnt vmcnt(0)" ::: "memory");
    }
    if (wave == 0) TS(17);
    asm volatile("s_waitcnt vmcnt(0)" ::: "memory");
    if (wave == 0) TS(18);
  }
  asm volatile("s_waitcnt lgkmcnt(0)" ::: "memory");
  __syncthreads();
  if (wave == 0) {
    TS(15);
    if (mine)
      __hip_atomic_store((gu32*)(slp + OFF_OUTF) + (rank * NTI + ti) * 16 + tj, call, __ATOMIC_RELAXED, __HIP_MEMORY_SCOPE_SYSTEM);
    if (blockIdx.x == 48) {
      const int tlo = (1024 + 384 * rank) / TM, thi = (1407 + 384 * rank) / TM;
      const __amdgpu_buffer_rsrc_t rs = __builtin_amdgcn_make_buffer_rsrc(myws, 0, 0x7fffffff, 0x00020000);
      const u64 t0 = __builtin_amdgcn_s_memrealtime();
      for (unsigned spins = 1;; ++spins) {
        bool ok = true;
        asm volatile("" ::: "memory");
#pragma unroll
        for (int j = 0; j < 3; ++j) {
          const int tt = j < 2 ? min(tlo + 2 * j + (lane >> 5), thi) : (2 * rank + (lane >> 5)) / 5;
          const u32x4 v = __builtin_amdgcn_raw_buffer_load_b128(rs, ((((lane >> 2) & 7) * NTI + tt) * 4 + (lane & 3)) * 16, (unsigned)OFF_OUTF, 17);
          ok &= min(min(v[0], v[1]), min(v[2], v[3])) >= call;
        }
        if (__all(ok)) break;
        if ((spins & 15) == 0 &&
            (__hip_atomic_load(tmo, __ATOMIC_RELAXED, __HIP_MEMORY_SCOPE_AGENT) == call ||
             __builtin_amdgcn_s_memrealtime() - t0 > DEADLINE_TICKS)) {
          if (lane == 0) {
            __hip_atomic_store(tmo, call, __ATOMIC_RELAXED, __HIP_MEMORY_SCOPE_AGENT);
            ((gchar*)(P.out_all[rank]))[rank * NLOC] = (char)-128;
          }
          break;
        }
        __builtin_amdgcn_s_sleep(1);
      }
      __builtin_amdgcn_fence(__ATOMIC_ACQUIRE, "");
      TS(16);
      }
    }
  }
}

extern "C" void kernel_launch(int rank, int nranks, ncclComm_t comm, hipStream_t stream,
                              void* const* d_in, const int* in_sizes, int n_in, void* d_out,
                              int out_size, const int* in_dtypes, int out_dtype,
                              void* const* d_out_all, void* const* d_ws_all, size_t ws_bytes) {
  if (nranks != NR || ws_bytes < WS_NEED || n_in < 2) {
    fprintf(stderr, "unexpected configuration\n");
    return;
  }
  Params P;
  P.x = static_cast<const float*>(d_in[0]);
  P.w = static_cast<const float*>(d_in[1]);
  P.out_all = d_out_all;
  P.ws_all = d_ws_all;
  P.rank = rank;
  hipLaunchKernelGGL(fused_gemm_q8, dim3(NWG), dim3(NTHREADS), 0, stream, P);
  HIP_CHECK(hipGetLastError());
}
```

```cpp
#include <hip/hip_runtime.h>
#include <rccl/rccl.h>
#include <cstdio>
#include <cstdint>
#include <type_traits>

#define HIP_CHECK(expr) do { hipError_t _e = (expr); if (_e != hipSuccess) { \
    fprintf(stderr, "HIP error %s at %s:%d\n", hipGetErrorString(_e), __FILE__, __LINE__); \
    return; } } while (0)

typedef __attribute__((ext_vector_type(8))) short bf16x8;
typedef __attribute__((ext_vector_type(4))) float f32x4;
typedef __attribute__((ext_vector_type(4))) unsigned u32x4;
typedef unsigned long long u64;
#define GAS __attribute__((address_space(1)))
typedef GAS char gchar;
typedef GAS unsigned gu32;
typedef GAS unsigned long long gu64;
typedef GAS u32x4 gu32x4;

namespace {
constexpr int NR = 8;
constexpr int KSH = 512;
constexpr int NTOT = 16384;
constexpr int NLOC = 2048;
constexpr int MSH = 512;
constexpr int NSTAGE = 8;
constexpr int XU = 32;
constexpr int WU = 16;
constexpr int UNIT = 16384;
constexpr int NWG = 256;
constexpr int SU = 4;
constexpr int NUNIT = 7 * NSTAGE * (XU + WU);
#define THR 2
#define PSLEEP 16
#define SCS 32
#define F1S 128
#define MAPB 0
#define DLY 9
#define DSLOW 0
#define LOS 32
#define LW 2
#define NLO 1
#define PRO 1
#define PKD 0
#define KSLOW 1
#define FWAIT 40
#define RECHK 1
#define OUTL 0
#define OTH 1
#define ASL 4
#define SEQ 5
#define WROT 1
#define WFAST 44
#define PS2 2
#define WMASK 1
#define NL (NT0 - 2016)
#define LSLEEP 4
#define NCOL 2
#define CSL 1
#define RS 1
#define PPRIO 0
#define OSEQ 8
#define KWV 3
#define TSL 8
#define XROWS(q) rows_of(q)
#define XSH 0
#define SPOLL 0
#define SSL 24
#define IDL 0
#define FAST 0
#define ER 0xD5
#define GOX 0
#define EOTH 1
#define TKT 5
#define BAL 1
#define LPRIO 3
#define PFD 0
#define PFL 3
#define TLO 56
#define THI 64
#define PIPE 1
#define QX 0
#define CPRIO 0
#define DSLEEP 16
#define DPRIO 2
#define LP 0
#define LDPOL " sc0 sc1"
constexpr int NSTEP = 64;
constexpr int TM = 320;
constexpr int NTI = 13;
constexpr int NCW = NTI * 16;
constexpr int NH = 5;
constexpr int HS = 7 * 4096;
#define NSEND 13
constexpr int NSWG = NWG - NCW;
constexpr int NSC = 15 - NSEND < 4 ? 15 - NSEND : 4;
#define S1V 4
constexpr int EU = ER ? 10 : 0;
constexpr int RTU = 7 * (32 - EU);
#define TKD 1
constexpr int S1 = S1V;
constexpr int NT0 = NUNIT - RTU;
constexpr int KW = KWV;
constexpr int U0 = NCW * KW * 2 / 4;
constexpr int NDRAW = NUNIT - U0 - S1 * NSWG + NSWG;
static_assert(NT0 - NL >= 7 * (16 + 48 * 5), "the steps of stage 4 do not depend on the last units");
static_assert(NSEND + NSC < 16 && SEQ <= 6, "sender split");
#define NWAVE 16
#define NDECV 2
constexpr int NDEC = NDECV;
constexpr int W_DEC = 11;
constexpr int NTHREADS = 64 * NWAVE;
#define WPRIO 0
#define LK 0
#define VB 0
#define VC 1
#define VD 0
constexpr unsigned F_RAW = 1u << 12;
constexpr int MAXESC = 192;
constexpr int NPK = NWAVE - W_DEC - NDEC;
constexpr int NPACK = (8 * XU + 56 + 6 * 112 + NSTAGE * WU) * 4;
constexpr int NMIR = 17;
#define CPK 0
constexpr int PR0 = NCW * (8 * CPK + NPK);
constexpr int PRN = NCW * NPK;
constexpr int NROUND = 1 + (NPACK - PR0 + PRN - 1) / PRN;
static_assert(CPK == 0 && RS >= 1 && RS * PRN <= NPACK && PRN % 48 == 0, "packing");
constexpr int NIN = 7 * 384;

constexpr size_t OFF_TICKET = 0;
constexpr size_t OFF_TMO = 128;
constexpr size_t OFF_SQ = 2048;
constexpr size_t OFF_PQ = 2304;
constexpr size_t OFF_XF = 4096;
constexpr size_t OFF_WF = OFF_XF + XU * 64 * SU * 4;
constexpr size_t OFF_GO = 3840;
constexpr size_t OFF_OUTF = 65536;
constexpr size_t OFF_REP = 73728;
constexpr size_t OFF_SLOTS = 77824;
constexpr size_t OFF_EF = 81920;
constexpr size_t OFF_F1 = 131072;
constexpr size_t OFF_EOF = 1u << 19;
constexpr size_t OFF_XS = 1u << 20;
constexpr size_t OFF_WS = OFF_XS + (size_t)NR * NSTAGE * XU * UNIT;
constexpr size_t OFF_EX = OFF_WS + (size_t)NR * NSTAGE * WU * UNIT;
constexpr size_t OFF_EW = OFF_EX + (size_t)NSTAGE * XU * UNIT;
constexpr size_t OFF_ES = OFF_EW + (size_t)NR * NSTAGE * WU * UNIT;
constexpr size_t WS_NEED = OFF_ES + (size_t)NR * 8 * 32 * 2048;

constexpr u64 DEADLINE_TICKS = 40000000ull;

constexpr int C_CALL = 0, C_FULL = 128, C_DONE = 8  , C_WMAX = 100, C_OUT = 110, C_ECNT = 120, C_GMAX = 24, C_DEAD = 25, C_NEXT = 26, C_Q = 32;
constexpr int LDS_CTL = NH * HS;
constexpr int LDS_SCR = LDS_CTL + 512 + 1024;
constexpr int LDS_BYTES = LDS_SCR + 16 * 256;
}

#define T2(a, i) do {} while (0)
#define TS(i) do {} while (0)
#define TV(i, v) do {} while (0)
#define CYC() 0ull

struct Params {
  const float* x;
  const float* w;
  void* const* out_all;
  void* const* ws_all;
  int rank;
};

__device__ __forceinline__ unsigned pk_bf16(float a, float b) {
  typedef __attribute__((ext_vector_type(2))) float f2;
  typedef __attribute__((ext_vector_type(2))) __bf16 b2;
  f2 v = {a, b};
  b2 r = __builtin_convertvector(v, b2);
  return __builtin_bit_cast(unsigned, r);
}

__device__ __forceinline__ unsigned lds_ld(unsigned addr) {
  unsigned v;
  asm volatile("ds_read_b32 %0, %1\n\ts_waitcnt lgkmcnt(0)" : "=v"(v) : "v"(addr) : "memory");
  return v;
}
__device__ __forceinline__ void glds4k(const void* g, unsigned ldsaddr) {
  unsigned keep;
  asm volatile(
      "s_mov_b32 %0, m0\n\ts_mov_b32 m0, %2\n\ts_nop 0\n\t"
      "global_load_lds_dwordx4 %1, off" LDPOL "\n\t"
      "global_load_lds_dwordx4 %1, off offset:1024" LDPOL "\n\t"
      "global_load_lds_dwordx4 %1, off offset:2048" LDPOL "\n\t"
      "global_load_lds_dwordx4 %1, off offset:3072" LDPOL "\n\t"
      "s_mov_b32 m0, %0"
      : "=&s"(keep) : "v"(g), "s"(ldsaddr) : "memory");
}
__device__ __forceinline__ bool plain16(u32x4 a, u32x4 b, u32x4 c, u32x4 d, unsigned want, unsigned m2) {
  unsigned x = 0, y = 0;
#pragma unroll
  for (int j = 0; j < 4; ++j) { x |= (a[j] ^ want) | (b[j] ^ want) | (d[j] ^ want); y |= c[j] ^ want; }
  return ((x | (y & m2)) >> 12) == 0u;
}
__device__ __forceinline__ int rows_of(int v) { return v < 8 ? 4 * v : 4 * ((v - 8) / 3) + 1 + (v - 8) % 3; }
__device__ __forceinline__ void unit_of(int ii, int pr, int& s, int& q, int& rr) {
  const int rot = (pr * 16 + 3) / 7;
  if (ii < 16) { s = 0; q = ii; rr = 2; }
  else if (ii < 16 + 48 * SEQ) {
    const int i2 = ii - 16, it = i2 % 48;
    s = i2 / 48; q = it / 3; rr = it - 3 * q;
    if (rr == 2) s += 1; else q = (q + rot) & 15;
  } else {
    int pos;
    if (ii < 336 - XSH) { s = (ii - 16) / 48; pos = (ii - 16) % 48; }
    else if (ii < 336 - XSH + EU) { s = 7; pos = ii - (336 - XSH); }
    else if (ii < 352 - XSH + EU) { s = 6; pos = ii - (304 - XSH) - EU; }
    else if (ii < 352 + EU) { s = 6; pos = ii - 320 - EU; }
    else { s = 7; pos = ii - 352; }
    if (pos >= 32) { s += 1; q = (pos + rot) & 15; rr = 2; }
    else {
      const int xu = pos < EU ? (pos + pr) % (EU ? EU : 1) : EU + (pos - EU + (pr * (32 - EU) + 3) / 7) % (32 - EU);
      q = xu >> 1; rr = xu & 1;
    }
  }
}
__device__ __forceinline__ int packed_at(int s, int q, int rr, int pr) {
  if (rr != 2) return (s == 0 ? 0 : 88 + (s - 1) * 160) + 2 * q + rr;
  return s >= 2 ? 120 + (s - 2) * 160 + q * 7 + pr : s == 1 && q >= 8 ? q * 7 + pr - 24 : (NMIR + 3) * 64;
}
__device__ __forceinline__ void lds_st(unsigned addr, unsigned v) {
  asm volatile("ds_write_b32 %0, %1" :: "v"(addr), "v"(v) : "memory");
}

template <int OFF = 0>
__device__ __forceinline__ void st16(gchar* p, u32x4 v) {
  asm volatile("global_store_dwordx4 %0, %1, off offset:%2 sc0 sc1\n\ts_nop 1" :: "v"(p), "v"(v), "n"(OFF) : "memory");
}

__device__ __forceinline__ void st4(gchar* p, unsigned v) {
  asm volatile("global_store_dword %0, %1, off sc0 sc1\n\ts_nop 1" :: "v"(p), "v"(v) : "memory");
}

__device__ __forceinline__ unsigned listed(unsigned A, unsigned B, unsigned C) {
  return (A & (A >> 1) & C & 0x55555555u) | ((B & (B >> 1) & (C >> 1) & 0x55555555u) << 1);
}
__device__ __forceinline__ unsigned ahead_of(unsigned c, unsigned& total) {
  unsigned idx = 0;
  total = 0;
#pragma unroll
  for (int b = 0; b < 3; ++b) {
    const u64 bal = __ballot((c >> b) & 1u);
    idx += __builtin_amdgcn_mbcnt_hi((unsigned)(bal >> 32), __builtin_amdgcn_mbcnt_lo((unsigned)bal, 0u)) << b;
    total += (unsigned)__builtin_popcountll(bal) << b;
  }
  return idx;
}

__device__ __forceinline__ unsigned send_block(const unsigned (&p)[4][4], unsigned scr, gchar* dst, int lane, bool plain,
                                              int slow = 0) {
  typedef __attribute__((ext_vector_type(2))) unsigned short u16x2;
  typedef __attribute__((ext_vector_type(3))) unsigned u32x3;
  unsigned top = 0, b4[8], e4[8], A = 0, B = 0, C = 0, M = 0, idx = 0, total = 0;
  if (!plain) {
#pragma unroll
    for (int k = 0; k < 4; ++k)
      b4[k] = (__builtin_amdgcn_perm(p[k >> 1][(k & 1) * 2 + 1], p[k >> 1][(k & 1) * 2], 0x07050301u) & 0x80808080u) |
              (__builtin_amdgcn_perm(p[k >> 1][(k & 1) * 2 + 1], p[k >> 1][(k & 1) * 2], 0x06040200u) & 0x7f7f7f7fu);
    st16(dst, u32x4{b4[0], b4[1], b4[2], b4[3]});
    __builtin_amdgcn_sched_barrier(0);
    u16x2 mx = {0, 0};
#pragma unroll
    for (int i = 0; i < 16; ++i)
      mx = __builtin_elementwise_max(mx, __builtin_bit_cast(u16x2, p[i >> 2][i & 3] & 0x7fff7fffu));
    int mi = max((int)mx[0], (int)mx[1]);
    mi = max(mi, __builtin_amdgcn_update_dpp(mi, mi, 0x111, 0xf, 0xf, false));
    mi = max(mi, __builtin_amdgcn_update_dpp(mi, mi, 0x112, 0xf, 0xf, false));
    mi = max(mi, __builtin_amdgcn_update_dpp(mi, mi, 0x114, 0xf, 0xf, false));
    mi = max(mi, __builtin_amdgcn_update_dpp(mi, mi, 0x118, 0xf, 0xf, false));
    mi = max(mi, __builtin_amdgcn_update_dpp(mi, mi, 0x142, 0xa, 0xf, false));
    mi = max(mi, __builtin_amdgcn_update_dpp(mi, mi, 0x143, 0xc, 0xf, false));
    top = (unsigned)__builtin_amdgcn_readlane(mi, 63) >> 7;
    const unsigned top4 = top * 0x01010101u;
#pragma unroll
    for (int k = 0; k < 8; ++k) {
      const unsigned lo = __builtin_amdgcn_perm(p[k >> 1][(k & 1) * 2 + 1], p[k >> 1][(k & 1) * 2], 0x06040200u);
      const unsigned hi = __builtin_amdgcn_perm(p[k >> 1][(k & 1) * 2 + 1], p[k >> 1][(k & 1) * 2], 0x07050301u);
      b4[k] = (hi & 0x80808080u) | (lo & 0x7f7f7f7fu);
      e4[k] = ((hi << 1) & 0xfefefefeu) | ((lo >> 7) & 0x01010101u);
      const unsigned d = top4 - e4[k];
      const unsigned x = (d + 0x01010101u) & 0xf8f8f8f8u;
      const unsigned nz = (((x & 0x7f7f7f7fu) + 0x7f7f7f7fu) | x) & 0x80808080u;
      const unsigned c = d | ((nz >> 4) - (nz >> 7));
      if (k < 4) A |= (c & 0x03030303u) << (2 * k); else B |= (c & 0x03030303u) << (2 * (k - 4));
      C |= ((c >> 2) & 0x01010101u) << (2 * (k & 3) + (k >> 2));
    }
    M = listed(A, B, C);
    const unsigned cnt = __builtin_popcount(M);
    idx = ahead_of(cnt, total);
    plain = top < 8u || top == 255u || total > (unsigned)MAXESC || __ballot(cnt > 7u) != 0ull;
    if (plain) asm volatile("s_waitcnt vmcnt(0)" ::: "memory");
  }
  if (plain) {
    st16<0>(dst, u32x4{p[0][0], p[0][1], p[0][2], p[0][3]});
    if (slow) asm volatile("s_waitcnt vmcnt(0)" ::: "memory");
    st16<1024>(dst, u32x4{p[1][0], p[1][1], p[1][2], p[1][3]});
    if (slow) asm volatile("s_waitcnt vmcnt(0)" ::: "memory");
    st16<2048>(dst, u32x4{p[2][0], p[2][1], p[2][2], p[2][3]});
    if (slow) asm volatile("s_waitcnt vmcnt(0)" ::: "memory");
    st16<3072>(dst, u32x4{p[3][0], p[3][1], p[3][2], p[3][3]});
    return F_RAW;
  }
  while (__ballot(M != 0u)) {
    const unsigned pos = (unsigned)__builtin_ctz(M | 0x80000000u);
    const unsigned s0 = (pos & 1u) ? e4[4] : e4[0], s1 = (pos & 1u) ? e4[5] : e4[1];
    const unsigned s2 = (pos & 1u) ? e4[6] : e4[2], s3 = (pos & 1u) ? e4[7] : e4[3];
    const unsigned t0 = (pos & 2u) ? s1 : s0, t1 = (pos & 2u) ? s3 : s2;
    const unsigned sel = ((pos & 4u) ? t1 : t0) >> (pos & 24u);
    if (M != 0u) asm volatile("ds_write_b8 %0, %1" :: "v"(scr + idx), "v"(sel) : "memory");
    M &= M - 1u;
    ++idx;
  }
  if (slow & 1) asm volatile("s_waitcnt vmcnt(0)" ::: "memory");
  st16<1024>(dst, u32x4{b4[4], b4[5], b4[6], b4[7]});
  const unsigned ev = lds_ld(scr + lane * 4);
  if (slow) asm volatile("s_waitcnt vmcnt(0)" ::: "memory");
  asm volatile("global_store_dwordx3 %0, %1, off offset:2048 sc0 sc1\n\ts_nop 1" :: "v"(dst - lane * 4), "v"(u32x3{A, B, C}) : "memory");
  const unsigned cls = (total + 63u) >> 6;
  if ((unsigned)lane < (VB ? (cls + 1u) & ~1u : cls) * 16u) st4(dst - lane * 12 + 2816, ev);
  return top | cls << 8;
}

#define AC00 "v[40:43]"
#define AC01 "v[44:47]"
#define AC02 "v[48:51]"
#define AC03 "v[52:55]"
#define AC10 "v[56:59]"
#define AC11 "v[60:63]"
#define AC12 "v[64:67]"
#define AC13 "v[68:71]"
#define AC20 "v[72:75]"
#define AC21 "v[76:79]"
#define AC22 "v[80:83]"
#define AC23 "v[84:87]"
#define AC30 "v[88:91]"
#define AC31 "v[92:95]"
#define AC32 "v[96:99]"
#define AC33 "v[100:103]"
#define AC40 "v[104:107]"
#define AC41 "v[108:111]"
#define AC42 "v[112:115]"
#define AC43 "v[116:119]"
#define RA0 "v[0:3]"
#define RA1 "v[4:7]"
#define RA2 "v[8:11]"
#define RA3 "v[12:15]"
#define RB0 "v[16:19]"
#define RB1 "v[20:23]"
#define RB2 "v[24:27]"
#define RB3 "v[28:31]"
#define RC0 "v[32:35]"
#define RC1 "v[36:39]"
#define PFY(x) x
#define PFN(x)
#define MF_(d, a, b) "v_mfma_f32_16x16x32_bf16 " d ", " a ", " b ", " d "\n\t"
#define RD_(r, a, o) "ds_read_b128 " r ", " a " offset:" #o "\n\t"
#define WT_(n) "s_waitcnt lgkmcnt(" #n ")\n\t"
#define ADV_ "s_add_u32 %[sdn], %[sdn], 1\n\t"                    \
             "v_mov_b32 %[vt], %[sdn]\n\t"                        \
             "s_mov_b64 exec, 1\n\t"                              \
             "ds_write2_b32 %[vc], %[vt], %[vt] offset0:%[dn] offset1:%[dm]\n\t" \
             "s_mov_b64 exec, -1\n\t"                             \
             "s_cmp_eq_u32 %[sof], %[hl]\n\t"                     \
             "s_cselect_b32 %[sd], %[hw], %[hs]\n\t"              \
             "s_add_u32 %[sof], %[sof], %[sd]\n\t"                \
             "v_add_u32 %[vx], %[sd], %[vx]\n\t"                  \
             "v_add_u32 %[vw], %[sd], %[vw]\n\t"                  \
             "v_add_u32 %[vf], 4, %[vf]\n\t"
#define DBG1_
#define DBG2_
#define FETCH(W0, W1, W2, N0, N1, N2, N3)                           \
  DBG1_                                                             \
  "s_setprio 0\n\t"                                                \
  "6:\n\t"                                                         \
  "ds_read2_b32 v[16:17], %[vf] offset0:0 offset1:1\n\t"           \
  WT_(0)                                                            \
  "v_readfirstlane_b32 %[sd], v16\n\t"                             \
  "v_readfirstlane_b32 %[sfl], v17\n\t"                            \
  "s_cmp_lg_u32 %[sd], 0\n\t"                                      \
  "s_cbranch_scc1 7f\n\t"                                          \
  "s_sleep 1\n\t"                                                  \
  DBG2_                                                             \
  "s_branch 6b\n\t"                                                \
  "7:\n\t"                                                         \
  RD_(N0, "%[vw]", 0) RD_(N1, "%[vw]", 1024) RD_(N2, "%[vw]", 2048) RD_(N3, "%[vw]", 3072)   \
  RD_(W0, "%[vx]", 0) RD_(W1, "%[vx]", 1024) RD_(W2, "%[vx]", 2048)                          \
  RD_(RC0, "%[vx]", 3072) RD_(RC1, "%[vx]", 4096) ADV_
#define PRD_ "ds_read_b32 %[vu], %[vc] offset:%[dp]\n\t"
#define W78_ WT_(8)
#define W67_ WT_(7)
#define PRI_ "v_readfirstlane_b32 %[sd], %[vu]\n\t"                \
             "s_setprio 0\n\t"                                     \
             "s_cmp_gt_u32 %[sd], %[sdn]\n\t"                      \
             "s_cbranch_scc0 8f\n\t"                               \
             "s_setprio 1\n\t"                                     \
             "8:\n\t"
#define BODY(W0, W1, W2, W3, N0, N1, N2, N3, P, Q)                                           \
  P(RD_(N3, "%[vw]", 3072) WT_(6)) Q(WT_(5))                                                 \
  MF_(AC00, W0, N0) MF_(AC01, W1, N0) MF_(AC02, W2, N0) MF_(AC03, W3, N0)                    \
  P(RD_(N0, "%[vw]", 0) "ds_read_b32 %[vt], %[vf] offset:4\n\t" PRD_ W78_) Q(WT_(4))        \
  MF_(AC10, W0, N1) MF_(AC11, W1, N1) MF_(AC12, W2, N1) MF_(AC13, W3, N1)                    \
  P(RD_(N1, "%[vw]", 1024) W78_) Q(WT_(3))                                                   \
  MF_(AC20, W0, N2) MF_(AC21, W1, N2) MF_(AC22, W2, N2) MF_(AC23, W3, N2)                    \
  P(RD_(N2, "%[vw]", 2048) W78_) Q(WT_(2))                                                   \
  MF_(AC30, W0, RC0)                                                                         \
  P(W67_) Q(WT_(1))                                                                          \
  MF_(AC40, W0, RC1)                                                                         \
  P(RD_(W0, "%[vx]", 0))                                                                     \
  MF_(AC31, W1, RC0) MF_(AC41, W1, RC1)                                                      \
  P(RD_(W1, "%[vx]", 1024))                                                                  \
  MF_(AC32, W2, RC0) MF_(AC42, W2, RC1)                                                      \
  P(RD_(W2, "%[vx]", 2048))                                                                  \
  MF_(AC33, W3, RC0)                                                                         \
  P(RD_(RC0, "%[vx]", 3072))                                                                 \
  MF_(AC43, W3, RC1)                                                                         \
  P(RD_(RC1, "%[vx]", 4096) WT_(7) "v_readfirstlane_b32 %[sfl], %[vt]\n\t" PRI_ ADV_)

__global__ __launch_bounds__(NTHREADS, NWAVE / 4) void fused_gemm_q8(Params P) {
  __shared__ __attribute__((aligned(1024))) char lds[LDS_BYTES];
  const unsigned ldsbase = (unsigned)(uintptr_t)lds;
  const unsigned ctl = ldsbase + LDS_CTL;

  const int tid = threadIdx.x;
  const int lane = tid & 63;
  const int wave = __builtin_amdgcn_readfirstlane(tid >> 6);
  const int rank = P.rank;
  char* const myws = static_cast<char*>(P.ws_all[rank]);
  unsigned* const tmo = reinterpret_cast<unsigned*>(myws + OFF_TMO);

  if ((blockIdx.x >> 3) < 6)
    for (int i = tid; i < (NMIR + 5) * 64; i += NTHREADS)
      asm volatile("ds_write_b128 %0, %1" :: "v"(ldsbase + i * 16), "v"(u32x4{0u, 0u, 0u, 0u}) : "memory");
  if (tid < 384) lds_st(ctl + tid * 4, 0u);
  asm volatile("s_waitcnt lgkmcnt(0)\n\ts_barrier" ::: "memory");
  if (wave == 0) TS(0);
  if (tid == ((blockIdx.x >> 3) < 6 ? 64 * 15 : 64 * 10)) {
    const unsigned t = __hip_atomic_fetch_add(reinterpret_cast<unsigned*>(myws + OFF_TICKET), 1u,
                                              __ATOMIC_RELAXED, __HIP_MEMORY_SCOPE_AGENT);
    lds_st(ctl + C_CALL * 4, t / NWG + 1);
  }
  unsigned call = 0, tag = 0;
  auto get_call = [&]() {
    while ((call = __builtin_amdgcn_readfirstlane(lds_ld(ctl + C_CALL * 4))) == 0u) __builtin_amdgcn_s_sleep(1);
    tag = call << 13;
  };

  const int bx = blockIdx.x & 7, bk = ((blockIdx.x >> 3) + 26) & 31;
  const bool sender = bk >= 26;
  const int ti = bk < 24 ? 2 * (bk % 6) + (bx & 1) : 12;
  const int tj = 4 * (bx >> 1) + (bk < 24 ? bk / 6 : 2 * (bx & 1) + ((bk - 24) & 1));

  f32x4 acc[5][4];
  const unsigned scr = ldsbase + LDS_SCR + wave * 256;
  const int wm = wave & 3, wn = (wave >> 2) & 1;

  auto fetch = [&](int s, int q, int d, int f, f32x4 (&a)[8]) {
    if (d < 0) {
      const GAS float* sp = (const GAS float*)P.x + (size_t)(XROWS(q) * 128 + (f & 4) * 16 + (lane & 15)) * KSH + s * 64 + (f >> 3) * 32 + 8 * (lane >> 4);
#pragma unroll
      for (int j = 0; j < 4; ++j) {
        a[2 * j] = *(const GAS f32x4*)sp;
        a[2 * j + 1] = *(const GAS f32x4*)(sp + 4);
        sp += 16 * KSH;
        asm("" : "+v"(sp));
      }
    } else {
      const GAS float* sp = (const GAS float*)P.w + (size_t)(s * 64 + (f >> 3) * 32 + 8 * (lane >> 4)) * NTOT + d * NLOC + q * 128 +
                        (f & 4) * 16 + (lane & 15) * 4;
#pragma unroll
      for (int j = 0; j < 8; ++j) {
        a[j] = *(const GAS f32x4*)sp;
        sp += NTOT;
        asm("" : "+v"(sp));
      }
    }
  };
  auto round16 = [&](int d, const f32x4 (&a)[8], unsigned (&p)[4][4], float& m) {
    m = 0.f;
#pragma unroll
    for (int j = 0; j < 8; ++j)
#pragma unroll
      for (int c = 0; c < 4; ++c) m = fmaxf(m, fabsf(a[j][c]));
    if (d < 0) {
#pragma unroll
      for (int j = 0; j < 4; ++j) {
        p[j][0] = pk_bf16(a[2 * j][0], a[2 * j][1]); p[j][1] = pk_bf16(a[2 * j][2], a[2 * j][3]);
        p[j][2] = pk_bf16(a[2 * j + 1][0], a[2 * j + 1][1]); p[j][3] = pk_bf16(a[2 * j + 1][2], a[2 * j + 1][3]);
      }
    } else {
#pragma unroll
      for (int jj = 0; jj < 4; ++jj)
#pragma unroll
        for (int c = 0; c < 4; ++c) p[jj][c] = pk_bf16(a[2 * c][jj], a[2 * c + 1][jj]);
    }
  };
  auto convert = [&](int s, int q, int d, int f, unsigned (&p)[4][4], float& m) {
    f32x4 a[8];
    fetch(s, q, d, f, a);
    round16(d, a, p, m);
  };
  auto pack_items = [&](int g, int rounds) {
    gchar* const gws = (gchar*)myws;
    const int grp = (g >> 3) % 6;
    gu32* const pq = (gu32*)(gws + OFF_PQ + grp * 256);
    static_assert(RS == 1, "packing");
    auto item = [&](int n, int& s, int& q, int& d) {
      const int e = n >> 2;
      d = -1;
      if (e < 32) { s = 0; q = e; }
      else if (e < 88) { s = 1; q = 8 + (e - 32) / 7; d = (rank + 1 + (e - 32) % 7) & 7; }
      else if (e < 1080) {
        const int r = (e - 88) % 160;
        s = (e - 88) / 160 + 1;
        if (r < 32) q = r;
        else if (r < 144) { s += 1; q = (r - 32) / 7; d = (rank + 1 + (r - 32) % 7) & 7; }
        else { s -= 1; q = r - 144; d = rank; }
      } else { s = 6 + ((e - 1080) >> 4); q = (e - 1080) & 15; d = rank; }
    };
    int nn = g, sn, qn, dn;
    f32x4 ra[8];
    item(nn, sn, qn, dn);
    fetch(sn, qn, dn, (nn & 3) * 4, ra);
    unsigned tk = 0;
    if (lane == 0) asm volatile("global_atomic_add %0, %1, %2, off sc0" : "=v"(tk) : "v"(pq), "v"(1u) : "memory");
    asm volatile("s_waitcnt vmcnt(0)" : "+v"(tk), "+v"(ra[0]), "+v"(ra[1]), "+v"(ra[2]), "+v"(ra[3]), "+v"(ra[4]), "+v"(ra[5]), "+v"(ra[6]), "+v"(ra[7]) :: "memory");
    for (int k = 0; nn < NPACK; ++k) {
      const int n = nn, s = sn, q = qn, d = dn, f = (n & 3) * 4, e = n >> 2;
      unsigned p[4][4];
      float m;
      round16(d, ra, p, m);
      if (k == 0) get_call();
      {
        const unsigned t = __builtin_amdgcn_readfirstlane(tk) - (call - 1) * ((NPACK - RS * PRN + 5 - grp) / 6 + PRN / 6);
        nn = t < 4096u ? RS * PRN + 6 * (int)t + grp : NPACK;
        if (nn < NPACK) {
          item(nn, sn, qn, dn);
          fetch(sn, qn, dn, (nn & 3) * 4, ra);
          if (lane == 0) asm volatile("global_atomic_add %0, %1, %2, off sc0" : "=v"(tk) : "v"(pq), "v"(1u) : "memory");
        }
      }
      unsigned info = 0;
      if (d != rank && e < 1048 + EU) {
        gchar* dst = gws + (d < 0 ? OFF_EX + (size_t)(s * XU + q) * UNIT : OFF_EW + (size_t)((d * NSTAGE + s) * WU + q) * UNIT);
        info = send_block(p, scr, dst + f * 1024 + lane * 16, lane, false);
      }
      if (d < 0 || d == rank) {
        gchar* dst = gws + (d < 0 ? OFF_XS + (size_t)((rank * NSTAGE + s) * XU + q) * UNIT
                                  : OFF_WS + (size_t)((rank * NSTAGE + s) * WU + q) * UNIT);
        send_block(p, scr, dst + f * 1024 + lane * 16, lane, true);
      }
      asm volatile("s_waitcnt vmcnt(0)" : "+v"(tk), "+v"(ra[0]), "+v"(ra[1]), "+v"(ra[2]), "+v"(ra[3]), "+v"(ra[4]), "+v"(ra[5]), "+v"(ra[6]), "+v"(ra[7]) :: "memory");
      if (lane == 0) st4(gws + OFF_EF + (size_t)n * 4, tag | info);
      if (lane == 1 && (d < 0 || d == rank))
        st4(gws + (d < 0 ? OFF_XF : OFF_WF) + (size_t)(q * 64 + rank * 8 + s) * 16 + f, tag | F_RAW);
    }
  };

  auto locate = [&](int u, int f, gchar*& dst, gu32*& flag, int& pn, unsigned& so, int& pe) -> bool {
      const int pr = u % 7, d = (rank + 1 + pr) & 7;
      int s, q, rr;
      unit_of(u / 7, pr, s, q, rr);
      gchar* dws;
      asm volatile("s_load_dwordx2 %0, %1, %2\n\ts_waitcnt lgkmcnt(0)"
                   : "=s"(dws) : "s"(P.ws_all), "s"(__builtin_amdgcn_readfirstlane(d * 8)));
      if (rr != 2) {
        const int xu = 2 * q + rr;
        so = (unsigned)OFF_EX + (unsigned)(s * XU + xu) * UNIT;
        dst = dws + OFF_XS + ((size_t)((rank * NSTAGE + s) * XU + xu)) * UNIT;
        pn = s | xu << 3;
      } else {
        so = (unsigned)OFF_EW + (unsigned)((d * NSTAGE + s) * WU + q) * UNIT;
        dst = dws + OFF_WS + ((size_t)((rank * NSTAGE + s) * WU + q)) * UNIT;
        pn = s | q << 3 | (d + 1) << 8;
      }
      pe = packed_at(s, q, rr, pr);
      flag = (gu32*)(dws + OFF_F1 + (size_t)u * F1S) + f;
      const bool tail = u >= NUNIT - RTU;
      if (tail) {
        so = (unsigned)OFF_XS + (unsigned)((rank * NSTAGE + s) * XU + 2 * q + rr) * UNIT;
        flag = (gu32*)(dws + OFF_XF + ((2 * q + rr) * 64 + rank * 8 + s) * 16) + f;
      }
      so = __builtin_amdgcn_readfirstlane(so + f * 4096);
      dst += f * 4096 + lane * 16;
      pn |= f << 12;
      return tail;
  };

  if (sender) {
   if (QX & 8) {
   } else if (wave < NSEND) {
    const __amdgpu_buffer_rsrc_t rs = __builtin_amdgcn_make_buffer_rsrc(myws, 0, 0x7fffffff, 0x00020000);
    unsigned so, info, ia = 0;
    int pn;
    int dbg = 0, pdbg = 0;
    gchar* dst;
    gu32* flag;
    u32x4 va, vb, vc;
    const u64 wsp = (u64)(uintptr_t)P.ws_all[lane & 7];
    auto draw = [&](int look) -> bool {
      unsigned j = 0, e = 0;
      if (lane == 0)
        asm volatile("ds_add_rtn_u32 %0, %1, %2\n\ts_waitcnt lgkmcnt(0)" : "=v"(j) : "v"(ctl + C_NEXT * 4), "v"(1u) : "memory");
      j = __builtin_amdgcn_readfirstlane(j);
      int f = j & 3;
      int u = U0 + (int)(j >> 2) * NSWG + (bk - 26) * 8 + bx;
      if (j >= (unsigned)(S1 * 4)) {
        while (((e = __builtin_amdgcn_readfirstlane(lds_ld(ctl + (C_Q + ((j >> 2) & 63)) * 4))) >> 16) != (j >> 2) + 1)
          __builtin_amdgcn_s_sleep(1);
        if (FAST && (e & 0x8000u)) {
          const int xu = e & 31, d = (e >> 5) & 7, fb = (e >> 12) & 3;
          so = (unsigned)OFF_XS + (unsigned)((rank * NSTAGE + 7) * XU + xu) * UNIT + fb * 4096 + f * 1024;
          va = __builtin_amdgcn_raw_buffer_load_b128(rs, lane * 16, so, 17);
          gchar* dws = (gchar*)(((u64)(unsigned)__builtin_amdgcn_readlane((int)(wsp >> 32), d) << 32) |
                                (unsigned)__builtin_amdgcn_readlane((int)wsp, d));
          dst = dws + so + lane * 16;
          flag = (gu32*)(dws + OFF_XF + (xu * 64 + rank * 8 + 7) * 16) + fb;
          pn = 1 << 30;
          info = tag | F_RAW;
          ia = ldsbase + (NMIR + 4) * 1024 + ((j >> 2) & 127) * 4;
          dbg = 4 * NUNIT + blockIdx.x * 16 + wave;
          if (TSL) __builtin_amdgcn_s_sleep(TSL);
          asm volatile("" : "+v"(ia), "+v"(flag), "+v"(pn), "+v"(so), "+v"(info));
          return true;
        }
        u = e & 4095;
        if (u == 4095) return false;
        if (e & 0x4000u) f = (e >> 12) & 3;
      }
      int pe;
      const bool piece = locate(u, f, dst, flag, pn, so, pe);
      dbg = u * 4 + f;
      ia = ldsbase + (pe * 4 + f) * 4;
      asm volatile("" : "+v"(ia), "+v"(flag));
      info = __builtin_amdgcn_readfirstlane(lds_ld(ia));
      for (; look > 0 && ((info ^ tag) >> 13) != 0u; --look) {
        __builtin_amdgcn_s_sleep(8);
        info = __builtin_amdgcn_readfirstlane(lds_ld(ia));
      }
      if (piece) {
        pn |= 1 << 30 | (j & 3) << 14;
        so += (j & 3) * 1024;
        dst += (j & 3) * 1024;
        ia = ldsbase + (NMIR + 4) * 1024 + ((j >> 2) & 127) * 4;
        info |= F_RAW;
      }
      if (((info ^ tag) >> 13) == 0u) {
        va = __builtin_amdgcn_raw_buffer_load_b128(rs, lane * 16, so, 17);
        if (!piece) {
          vb = __builtin_amdgcn_raw_buffer_load_b128(rs, lane * 16 + 1024, so, 17);
          vc = __builtin_amdgcn_raw_buffer_load_b128(rs, lane * 16 + 2048, so, 17);
        }
      }
      asm volatile("" : "+v"(pn), "+v"(so), "+v"(info));
      return true;
    };
    gu32* pflag = nullptr;
    unsigned pv = 0, pc = 0;
    auto finish = [&]() {
      unsigned c = 3;
      if (pc != 0u && lane == 0)
        asm volatile("ds_add_rtn_u32 %0, %1, %2\n\ts_waitcnt lgkmcnt(0)" : "=v"(c) : "v"(pc), "v"(1u) : "memory");
      if (pflag != nullptr && lane == 0 && (c & 3u) == 3u)
        __hip_atomic_store(pflag, pv, __ATOMIC_RELAXED, __HIP_MEMORY_SCOPE_SYSTEM);
    };
    va = vb = vc = u32x4{0u, 0u, 0u, 0u}; info = 0; so = 0; pn = 0; dst = nullptr; flag = nullptr;
    get_call();
    draw(FWAIT);
    while (true) {
      asm volatile("s_waitcnt vmcnt(0)" : "+v"(va), "+v"(vb), "+v"(vc) :: "memory");
      finish();
      if (pflag != nullptr) T2(3, pdbg);
      pdbg = dbg;
      u32x4 ca = va, cb = vb, cc = vc;
      const int cpn = __builtin_amdgcn_readfirstlane(pn);
      const unsigned cso = so, cia = ia;
      gchar* const cdst = dst;
      pflag = flag;
      pv = __builtin_amdgcn_readfirstlane(info);
      pc = (cpn >> 30) ? cia : 0u;
      const bool more = draw(LK);
      if (!(cpn >> 30) && ((pv ^ tag) >> 13) != 0u) {
        pv = __builtin_amdgcn_readfirstlane(lds_ld(cia));
        if (((pv ^ tag) >> 13) == 0u) {
          ca = __builtin_amdgcn_raw_buffer_load_b128(rs, cso + lane * 16, 0, 17);
          cb = __builtin_amdgcn_raw_buffer_load_b128(rs, cso + lane * 16 + 1024, 0, 17);
          cc = __builtin_amdgcn_raw_buffer_load_b128(rs, cso + lane * 16 + 2048, 0, 17);
        }
      }
      if (((pv ^ tag) >> 13) != 0u) {
        unsigned p[4][4];
        float m;
        convert(cpn & 7, (cpn >> 3) & 31, ((cpn >> 8) & 15) - 1, (cpn >> 10) & 12, p, m);
        if (cpn >> 30) {
          int g = cpn;
          asm volatile("" : "+v"(g));
#pragma unroll
          for (int i = 0; i < 4; ++i) {
            unsigned p0 = p[0][i], p1 = p[1][i], p2 = p[2][i], p3 = p[3][i];
            asm volatile("" : "+v"(p0), "+v"(p1), "+v"(p2), "+v"(p3));
            const unsigned a = (g & (1 << 14)) ? p1 : p0, b = (g & (1 << 14)) ? p3 : p2;
            ca[i] = (g & (2 << 14)) ? b : a;
          }
          pv = tag | F_RAW;
          st16(cdst, ca);
        } else {
          pv = tag | send_block(p, scr, cdst, lane, VD & 1, DSLOW);
        }
      } else {
        st16(cdst, ca);
        if (!(cpn >> 30)) {
          asm volatile("s_waitcnt vmcnt(0)" ::: "memory");
          st16<1024>(cdst, cb);
          asm volatile("s_waitcnt vmcnt(0)" ::: "memory");
          if (VB == 1 || (unsigned)lane < 48u + ((((pv >> 6) & 0x4cu) + (VB ? 4u : 0u)) & (VB ? 0x78u : 0x7cu))) st16<2048>(cdst, cc);
          if (pv & F_RAW) st16<3072>(cdst, __builtin_amdgcn_raw_buffer_load_b128(rs, cso + lane * 16 + 3072, 0, 17));
        }
      }
      if (!more) break;
    }
    asm volatile("s_waitcnt vmcnt(0)" ::: "memory");
    finish();
    T2(3, pdbg);
    if (wave < 2) TS(1 + wave);
   } else if (wave < NSEND + NSC) {
    get_call();
    const u64 t0 = __builtin_amdgcn_s_memrealtime();
    if (wave > NSEND) __builtin_amdgcn_s_sleep(LOS);
    int c = wave - NSEND < NLO ? 0 : NMIR;
    for (unsigned spins = 1; c < NMIR; ++spins) {
      {
        unsigned keep;
        asm volatile("s_mov_b32 %0, m0\n\ts_mov_b32 m0, %2\n\ts_nop 0\n\t"
                     "global_load_lds_dwordx4 %1, off sc0 sc1\n\t"
                     "global_load_lds_dwordx4 %1, off offset:1024 sc0 sc1\n\t"
                     "s_mov_b32 m0, %0"
                     : "=&s"(keep) : "v"(myws + OFF_EF + c * 1024 + lane * 16), "s"(ldsbase + c * 1024) : "memory");
      }
      asm volatile("s_waitcnt vmcnt(0)" ::: "memory");
      for (int i = 0; i < LW && c < NMIR; ++i, ++c) {
        u32x4 v;
        asm volatile("ds_read_b128 %0, %1\n\ts_waitcnt lgkmcnt(0)" : "=v"(v) : "v"(ldsbase + c * 1024 + lane * 16) : "memory");
        if (!__all((((v[0] ^ tag) | (v[1] ^ tag) | (v[2] ^ tag) | (v[3] ^ tag)) >> 13) == 0u)) break;
      }
      if ((spins & 15) == 0 &&
          (__hip_atomic_load(tmo, __ATOMIC_RELAXED, __HIP_MEMORY_SCOPE_AGENT) == call ||
           __builtin_amdgcn_s_memrealtime() - t0 > DEADLINE_TICKS)) {
        if (lane == 0) __hip_atomic_store(tmo, call, __ATOMIC_RELAXED, __HIP_MEMORY_SCOPE_AGENT);
        break;
      }
      __builtin_amdgcn_s_sleep(SCS);
    }
    if (wave == NSEND) TS(33);
   } else if (wave == NWAVE - 1) {
    get_call();
    unsigned* const sq = reinterpret_cast<unsigned*>(myws + OFF_SQ);
    const unsigned qbase = (call - 1) * NDRAW - U0 - S1 * NSWG;
    unsigned ahead = TKD;
    for (unsigned k = S1;; ++k) {
      while ((lds_ld(ctl + C_NEXT * 4) >> 2) + ahead < k) __builtin_amdgcn_s_sleep(2);
      unsigned t = 0;
      if (lane == 0) t = __hip_atomic_fetch_add(sq, 1u, __ATOMIC_RELAXED, __HIP_MEMORY_SCOPE_AGENT);
      const unsigned u = __builtin_amdgcn_readfirstlane(t) - qbase;
      if (u >= (unsigned)NUNIT) {
        if (lane < 8) lds_st(ctl + (C_Q + ((k + lane) & 63)) * 4, ((k + lane + 1) << 16) | 4095u);
        break;
      }
      if (u >= (unsigned)NT0) {
        const int t = 4 * ((int)u - NT0) + (lane & 3), ii = NT0 / 7 + t / 28, r = t % 28, f = r / 7, pr = r % 7;
        int s, q, rr;
        unit_of(ii, pr, s, q, rr);
        const unsigned w = lds_ld(ldsbase + (packed_at(s, q, rr, pr) * 4 + f) * 4);
        const unsigned what = FAST && ((w ^ tag) >> 13) == 0u ? 0x8000u | ((rank + 1 + pr) & 7) << 5 | (2 * q + rr)
                                                              : 0x4000u | (ii * 7 + pr);
        if (lane < 4) lds_st(ctl + (C_Q + ((k + lane) & 63)) * 4, ((k + lane + 1) << 16) | f << 12 | what);
        k += 3;
        ahead = TKT;
      } else if (lane == 0) {
        lds_st(ctl + (C_Q + (k & 63)) * 4, ((k + 1) << 16) | u);
      }
    }
   }
  } else if (wave < 8) {
    if (wave < KW && !(QX & 8)) {
      const int ba = wave * NCW + bk * 8 + bx, bb = ba + KW * NCW;
      gchar *da, *db;
      gu32 *fa, *fb;
      int pa, pb, pe;
      unsigned so, p[4][4];
      float m;
      f32x4 ra[8], rb[8];
      locate(ba >> 2, ba & 3, da, fa, pa, so, pe);
      locate(bb >> 2, bb & 3, db, fb, pb, so, pe);
      T2(2, ba);
      T2(2, bb);
      fetch(pa & 7, (pa >> 3) & 31, ((pa >> 8) & 15) - 1, (pa >> 10) & 12, ra);
      fetch(pb & 7, (pb >> 3) & 31, ((pb >> 8) & 15) - 1, (pb >> 10) & 12, rb);
      round16(((pa >> 8) & 15) - 1, ra, p, m);
      const unsigned ia = send_block(p, scr, da, lane, VD & 1, KSLOW);
      round16(((pb >> 8) & 15) - 1, rb, p, m);
      get_call();
      asm volatile("s_waitcnt vmcnt(0)" ::: "memory");
      if (lane == 0) __hip_atomic_store(fa, tag | ia, __ATOMIC_RELAXED, __HIP_MEMORY_SCOPE_SYSTEM);
      T2(3, ba);
      const unsigned ib = send_block(p, scr, db, lane, VD & 1, KSLOW);
      asm volatile("s_waitcnt vmcnt(0)" ::: "memory");
      if (lane == 0) __hip_atomic_store(fb, tag | ib, __ATOMIC_RELAXED, __HIP_MEMORY_SCOPE_SYSTEM);
      T2(3, bb);
    }
    if (wave == 0) TS(30);
#pragma unroll
    for (int a = 0; a < 5; ++a)
#pragma unroll
      for (int b = 0; b < 4; ++b) acc[a][b] = f32x4{0.f, 0.f, 0.f, 0.f};
    const int xoff = (wm * 5) * 1024 + lane * 16;
    const int woff = 20480 + (wn * 4) * 1024 + lane * 16;
    const u64 c0 = CYC(), r0 = __builtin_amdgcn_s_memrealtime();
    {
      unsigned vx = ldsbase + xoff, vw = ldsbase + woff, vf = ctl + C_FULL * 4, vc = ctl + wave * 4, vt, vu;
      unsigned sfl, sdn, sof, sd, sdb = 0;
      asm volatile(
          "s_mov_b32 %[sdn], 0\n\t"
          "s_mov_b32 %[sof], 0\n\t"
          "s_nop 4\n\t"
          FETCH(RB0, RB1, RB2, RA0, RA1, RA2, RA3)
          "1:\n\t"
          "s_cmp_lg_u32 %[sfl], 0\n\t"
          "s_cbranch_scc0 3f\n\t"
          BODY(RA0, RA1, RA2, RA3, RB0, RB1, RB2, RB3, PFY, PFN)
          "2:\n\t"
          "s_cmp_lg_u32 %[sfl], 0\n\t"
          "s_cbranch_scc0 4f\n\t"
          BODY(RB0, RB1, RB2, RB3, RA0, RA1, RA2, RA3, PFY, PFN)
          "s_branch 1b\n\t"
          "3:\n\t"
          BODY(RA0, RA1, RA2, RA3, RB0, RB1, RB2, RB3, PFN, PFY)
          FETCH(RA0, RA1, RA2, RB0, RB1, RB2, RB3)
          "s_branch 2b\n\t"
          "4:\n\t"
          BODY(RB0, RB1, RB2, RB3, RA0, RA1, RA2, RA3, PFN, PFY)
          "s_cmp_eq_u32 %[sdn], %[ns]\n\t"
          "s_cbranch_scc1 5f\n\t"
          FETCH(RB0, RB1, RB2, RA0, RA1, RA2, RA3)
          "s_branch 1b\n\t"
          "5:\n\t"
          "s_setprio 0\n\t"
          "s_nop 15\n\t"
          "s_nop 15\n\t"
          :
          "+{v[40:43]}"(acc[0][0]), "+{v[44:47]}"(acc[0][1]), "+{v[48:51]}"(acc[0][2]), "+{v[52:55]}"(acc[0][3]),
          "+{v[56:59]}"(acc[1][0]), "+{v[60:63]}"(acc[1][1]), "+{v[64:67]}"(acc[1][2]), "+{v[68:71]}"(acc[1][3]),
          "+{v[72:75]}"(acc[2][0]), "+{v[76:79]}"(acc[2][1]), "+{v[80:83]}"(acc[2][2]), "+{v[84:87]}"(acc[2][3]),
          "+{v[88:91]}"(acc[3][0]), "+{v[92:95]}"(acc[3][1]), "+{v[96:99]}"(acc[3][2]), "+{v[100:103]}"(acc[3][3]),
          "+{v[104:107]}"(acc[4][0]), "+{v[108:111]}"(acc[4][1]), "+{v[112:115]}"(acc[4][2]), "+{v[116:119]}"(acc[4][3]),
          [vx] "+v"(vx), [vw] "+v"(vw), [vf] "+v"(vf), [vt] "=&v"(vt), [vu] "=&v"(vu), [sfl] "=&s"(sfl), [sdn] "=&s"(sdn),
          [sof] "=&s"(sof), [sd] "=&s"(sd), [sdb] "+s"(sdb)
          : [vc] "v"(vc), [dn] "n"(C_DONE), [dm] "n"(C_DONE + 8), [dp] "n"((C_DONE + 4) * 4), [hs] "n"(HS), [hl] "n"((NH - 1) * HS), [hw] "s"(-(NH - 1) * HS),
            [ns] "n"(2 * NSTEP)
          : "memory", "scc", "v0", "v1", "v2", "v3", "v4", "v5", "v6", "v7", "v8", "v9", "v10", "v11", "v12", "v13", "v14", "v15", "v16", "v17", "v18", "v19", "v20", "v21", "v22", "v23", "v24", "v25", "v26", "v27", "v28", "v29", "v30", "v31", "v32", "v33", "v34", "v35", "v36", "v37", "v38", "v39");
      TV(48 + wave, sdb);
    }
    if (wave == 0) { TV(40, CYC() - c0); TV(41, __builtin_amdgcn_s_memrealtime() - r0); }
    float m = 0.f;
#pragma unroll
    for (int a = 0; a < 5; ++a) {
      if (ti * TM + wm * 80 + a * 16 >= 4096) continue;
#pragma unroll
      for (int b = 0; b < 4; ++b)
#pragma unroll
        for (int j = 0; j < 4; ++j) m = fmaxf(m, fabsf(acc[a][b][j]));
    }
#pragma unroll
    for (int o = 32; o >= 1; o >>= 1) m = fmaxf(m, __shfl_xor(m, o));
    if (lane == 0) lds_st(ctl + (C_WMAX + wave) * 4, __float_as_uint(m));
    if (wave == 0) TS(12);
  } else if (wave < 10) {
    const int L = wave - 8;
    __builtin_amdgcn_s_setprio(LPRIO);
    get_call();
    {
      const u64 t0 = __builtin_amdgcn_s_memrealtime();
      while (__builtin_amdgcn_s_memrealtime() - t0 < DLY * 100ull) __builtin_amdgcn_s_sleep(32);
    }
    const char* const gx0 = myws + OFF_XS + L * 8192 + lane * 16;
    const char* const gw0 = myws + OFF_WS + (size_t)tj * UNIT + L * 8192 + lane * 16;
    int slot = L;
    const unsigned dna = ctl + (C_DONE + (lane & 7)) * 4;
    u64 rm = 0, avail = 0, left = ~0ull;
    unsigned dv = 0;
    for (int t = 0; t < NSTEP; ++t) {
      const int need = 2 * t + L - NH + 1;
      while (avail == 0 || !__all((int)dv >= need)) {
        __builtin_amdgcn_s_sleep(1);
        asm volatile("ds_read_b64 %0, %2\n\tds_read_b32 %1, %3\n\ts_waitcnt lgkmcnt(0)"
                     : "=&v"(rm), "=&v"(dv) : "v"(ctl + C_Q * 4), "v"(dna) : "memory");
        avail = left & (((u64)(unsigned)__builtin_amdgcn_readfirstlane((int)(rm >> 32)) << 32) |
                        (u64)(unsigned)__builtin_amdgcn_readfirstlane((int)rm));
      }
      const int c = __builtin_ctzll(avail);
      left &= ~(1ull << c);
      const int bit = ((rank - (c & 7)) & 7) * 8 + (c >> 3);
      asm volatile("s_waitcnt vmcnt(29)" ::: "memory");
      const unsigned l = ldsbase + slot * HS;
#pragma unroll
      for (int i = 0; i < 4; ++i) {
        const int hu = 5 * ti + i;
        glds4k(gx0 + ((size_t)(bit * XU + (hu >> 1))) * UNIT + (hu & 1) * 4096, l + i * 4096);
      }
      {
        const int hu = min(5 * ti + 4, 63);
        const char* g0 = gx0 + ((size_t)(bit * XU + (hu >> 1))) * UNIT + (hu & 1) * 4096;
        const char* g1 = gw0 + (size_t)(bit * WU) * UNIT;
        unsigned keep;
        asm volatile(
            "ds_read_b64 %0, %3\n\tds_read_b32 %1, %4\n\t"
            "s_mov_b32 %2, m0\n\ts_mov_b32 m0, %9\n\ts_nop 0\n\t"
            "global_load_lds_dwordx4 %5, off" LDPOL "\n\t"
            "global_load_lds_dwordx4 %5, off offset:1024" LDPOL "\n\t"
            "global_load_lds_dwordx4 %5, off offset:2048" LDPOL "\n\t"
            "global_load_lds_dwordx4 %5, off offset:3072" LDPOL "\n\t"
            "s_add_u32 m0, m0, 0x1000\n\ts_nop 0\n\t"
            "global_load_lds_dwordx4 %6, off" LDPOL "\n\t"
            "global_load_lds_dwordx4 %6, off offset:1024" LDPOL "\n\t"
            "global_load_lds_dwordx4 %6, off offset:2048" LDPOL "\n\t"
            "global_load_lds_dwordx4 %6, off offset:3072" LDPOL "\n\t"
            "s_add_u32 m0, m0, 0x1000\n\ts_nop 0\n\t"
            "global_load_lds_dwordx4 %7, off" LDPOL "\n\t"
            "global_load_lds_dwordx4 %7, off offset:1024" LDPOL "\n\t"
            "global_load_lds_dwordx4 %7, off offset:2048" LDPOL "\n\t"
            "global_load_lds_dwordx4 %7, off offset:3072" LDPOL "\n\t"
            "s_mov_b32 m0, %10\n\ts_mov_b64 exec, 1\n\t"
            "global_load_lds_dword %8, off sc0 sc1\n\t"
            "s_mov_b64 exec, -1\n\ts_mov_b32 m0, %2\n\t"
            "s_waitcnt lgkmcnt(0)"
            : "=&v"(rm), "=&v"(dv), "=&s"(keep)
            : "v"(ctl + C_Q * 4), "v"(dna), "v"(g0), "v"(g1), "v"(g1 + 4096), "v"(myws + OFF_TICKET),
              "s"(l + 16384), "s"(ctl + (C_FULL + 2 * t + L) * 4)
            : "memory", "scc");
        avail = left & (((u64)(unsigned)__builtin_amdgcn_readfirstlane((int)(rm >> 32)) << 32) |
                        (u64)(unsigned)__builtin_amdgcn_readfirstlane((int)rm));
      }
      slot = slot + 2 >= NH ? slot + 2 - NH : slot + 2;
    }
    asm volatile("s_waitcnt vmcnt(0)" ::: "memory");
    if (L == 0) TS(11);
  } else if (wave == 10) {
    get_call();
    __builtin_amdgcn_s_setprio(WPRIO);
    const unsigned want = tag | F_RAW;
    const unsigned m2 = ti == NTI - 1 ? 0u : ~0u;
    const int st = ((rank - (lane & 7)) & 7) * 8 + (lane >> 3);
    const unsigned* pf0 = reinterpret_cast<const unsigned*>(myws + OFF_XF) + (((5 * ti) >> 1) * 64 + st) * SU;
    const unsigned* pf2 = reinterpret_cast<const unsigned*>(myws + OFF_WF) + (tj * 64 + st) * SU;
    u64 seen = 0;
    unsigned cnt = 0, spins = 0, kk = 0;
    bool dead = false;
    const u64 t0 = __builtin_amdgcn_s_memrealtime();
    while (cnt < (unsigned)NSTEP) {
      u32x4 f0, f1, f2, f3;
      asm volatile(
                   "s_mov_b64 exec, %6\n\t"
                   "global_load_dwordx4 %0, %4, off sc0 sc1\n\tglobal_load_dwordx4 %1, %4, off offset:1024 sc0 sc1\n\t"
                   "global_load_dwordx4 %2, %4, off offset:2048 sc0 sc1\n\t"
                   "global_load_dwordx4 %3, %5, off sc0 sc1\n\ts_mov_b64 exec, -1\n\ts_waitcnt vmcnt(0)"
                   : "=&v"(f0), "=&v"(f1), "=&v"(f2), "=&v"(f3) : "v"(pf0), "v"(pf2), "s"(~seen) : "memory");
      const u64 nm = __ballot((QX & 1) || dead || plain16(f0, f1, f2, f3, want, m2)) & ~seen;
      if (nm) {
        if (lane == 0) asm volatile("ds_write_b64 %0, %1" :: "v"(ctl + C_Q * 4), "v"(seen | nm) : "memory");
        cnt += (unsigned)__builtin_popcountll(nm);
        seen |= nm;
      } else if ((++spins & 15) == 0 &&
                 (__hip_atomic_load(tmo, __ATOMIC_RELAXED, __HIP_MEMORY_SCOPE_AGENT) == call ||
                  __builtin_amdgcn_s_memrealtime() - t0 > DEADLINE_TICKS)) {
        if (lane == 0) __hip_atomic_store(tmo, call, __ATOMIC_RELAXED, __HIP_MEMORY_SCOPE_AGENT);
        dead = true;
      }
      if (cnt < WFAST) __builtin_amdgcn_s_sleep(PSLEEP); else __builtin_amdgcn_s_sleep(PS2);
    }
    if (dead && lane == 0) lds_st(ctl + C_DEAD * 4, ~0u);
    TS(32);
    if (GOX && ER && ti < 4) {
      while ((int)(__hip_atomic_load(reinterpret_cast<unsigned*>(myws + OFF_GO), __ATOMIC_RELAXED, __HIP_MEMORY_SCOPE_AGENT) -
                   (call - 1) * NSWG) < GOX &&
             __builtin_amdgcn_s_memrealtime() - t0 < DEADLINE_TICKS)
        __builtin_amdgcn_s_sleep(32);
    }
  } else {
   const __amdgpu_buffer_rsrc_t rs = __builtin_amdgcn_make_buffer_rsrc(myws, 0, 0x7fffffff, 0x00020000);
   const u64 t0 = __builtin_amdgcn_s_memrealtime();
   bool dead = false;
   auto arrival = [&](int n, unsigned& so, unsigned& fo) {
      const int pr = n % 7;
      int s, q, rr;
      unit_of(n / 7, pr, s, q, rr);
      const int st = ((rank + 7 - pr) & 7) * 8 + s;
      if (rr != 2) {
        so = (unsigned)OFF_XS + (unsigned)(st * XU + 2 * q + rr) * UNIT;
        fo = (unsigned)OFF_XF + ((2 * q + rr) * 64 + st) * 16;
      } else {
        so = (unsigned)OFF_WS + (unsigned)(st * WU + q) * UNIT;
        fo = (unsigned)OFF_WF + (q * 64 + st) * 16;
      }
      so = __builtin_amdgcn_readfirstlane(so);
      fo = __builtin_amdgcn_readfirstlane(fo);
   };
   typedef __attribute__((ext_vector_type(3))) unsigned u32x3;
   auto expand = [&](unsigned f, u32x4 a, u32x4 b, u32x3 c, unsigned e, unsigned so) {
        const unsigned top = f & 0xffu, top4 = top * 0x01010101u;
        unsigned e4[8];
#pragma unroll
        for (int k = 0; k < 8; ++k)
          e4[k] = top4 - (((c[k >> 2] >> (2 * (k & 3))) & 0x03030303u) |
                          (((c[2] >> (2 * (k & 3) + (k >> 2))) & 0x01010101u) << 2));
        if (f & 0x300u) {
          const unsigned M = listed(c[0], c[1], c[2]);
          unsigned total, x = M - ((M >> 1) & 0x55555555u);
          x = (x & 0x33333333u) + ((x >> 2) & 0x33333333u);
          x = (x + (x >> 4)) & 0x0f0f0f0fu;
          x = (x << 8) + (x << 16) + (x << 24);
          const unsigned i0 = ahead_of(__builtin_popcount(M), total) - 1u;
          const unsigned w0 = (unsigned)__builtin_amdgcn_ds_bpermute((int)(i0 & 0xfcu), (int)e);
          const unsigned w1 = (unsigned)__builtin_amdgcn_ds_bpermute((int)((i0 + 4u) & 0xfcu), (int)e);
          const unsigned w2 = (unsigned)__builtin_amdgcn_ds_bpermute((int)((i0 + 8u) & 0xfcu), (int)e);
          const unsigned lo = __builtin_amdgcn_alignbyte(w1, w0, i0 & 3u), hi = __builtin_amdgcn_alignbyte(w2, w1, i0 & 3u);
#pragma unroll
          for (int j = 0; j < 8; ++j) {
            const int k = (j >> 1) + 4 * (j & 1);
            const unsigned t = (M >> j) & 0x01010101u, mk = (t << 8) - t;
            x += t;
            e4[k] = (e4[k] & ~mk) | (__builtin_amdgcn_perm(hi, lo, x) & mk);
          }
        }
#pragma unroll
        for (int j = 0; j < 4; ++j) {
          u32x4 o;
#pragma unroll
          for (int h = 0; h < 2; ++h) {
            const int k = 2 * j + h;
            const unsigned sm = k < 4 ? a[k & 3] : b[k & 3];
            const unsigned hi = (sm & 0x80808080u) | ((e4[k] >> 1) & 0x7f7f7f7fu);
            const unsigned lo = (sm & 0x7f7f7f7fu) | ((e4[k] << 7) & 0x80808080u);
            o[2 * h] = __builtin_amdgcn_perm(hi, lo, 0x05010400u);
            o[2 * h + 1] = __builtin_amdgcn_perm(hi, lo, 0x07030602u);
          }
          __builtin_amdgcn_raw_buffer_store_b128(o, rs, lane * 16 + j * 1024, so, 17);
          asm volatile("s_nop 1" :: "v"(o) : "memory");
        }
   };
   if (wave < W_DEC + NDEC) {
    __builtin_amdgcn_s_setprio(DPRIO);
    get_call();
    for (int n = (wave - W_DEC) * NCW + bk * 8 + bx; n < NT0 - NL && !dead && !(QX & 16); n += NDEC * NCW) {
      unsigned so, fo;
      arrival(n, so, fo);
      unsigned f[4];
      for (unsigned spins = 1;; ++spins) {
        asm volatile("" ::: "memory");
        const u32x4 fv = __builtin_amdgcn_raw_buffer_load_b128(rs, 0, (unsigned)OFF_F1 + n * F1S, 17);
#pragma unroll
        for (int k = 0; k < 4; ++k) f[k] = __builtin_amdgcn_readfirstlane(fv[k]);
        if ((((f[0] ^ tag) | (f[1] ^ tag) | (f[2] ^ tag) | (f[3] ^ tag)) >> 13) == 0u) break;
        if ((spins & 15) == 0 &&
            (__hip_atomic_load(tmo, __ATOMIC_RELAXED, __HIP_MEMORY_SCOPE_AGENT) == call ||
             __builtin_amdgcn_s_memrealtime() - t0 > DEADLINE_TICKS)) {
          if (lane == 0) __hip_atomic_store(tmo, call, __ATOMIC_RELAXED, __HIP_MEMORY_SCOPE_AGENT);
          dead = true;
          break;
        }
        __builtin_amdgcn_s_sleep(DSLEEP);
      }
      if (dead) continue;
      T2(0, n);
      u32x4 a[4], b[4];
      u32x3 c[4];
      unsigned e[4];
#pragma unroll
      for (int k = 0; k < 4; ++k) {
        a[k] = __builtin_amdgcn_raw_buffer_load_b128(rs, lane * 16 + k * 4096, so, 17);
        b[k] = __builtin_amdgcn_raw_buffer_load_b128(rs, lane * 16 + k * 4096 + 1024, so, 17);
        e[k] = __builtin_amdgcn_raw_buffer_load_b32(rs, lane * 4 + k * 4096 + 2816, so, 17);
        c[k] = __builtin_amdgcn_raw_buffer_load_b96(rs, lane * 12 + k * 4096 + 2048, so, 17);
      }
      asm volatile("s_waitcnt vmcnt(0)" : "+v"(a[0]), "+v"(a[1]), "+v"(a[2]), "+v"(a[3]), "+v"(b[0]), "+v"(b[1]), "+v"(b[2]), "+v"(b[3]) :: "memory");
      asm volatile("" : "+v"(c[0]), "+v"(c[1]), "+v"(c[2]), "+v"(c[3]), "+v"(e[0]), "+v"(e[1]), "+v"(e[2]), "+v"(e[3]) :: "memory");
#pragma unroll
      for (int k = 0; k < 4; ++k) {
        if (f[k] & F_RAW) continue;
        expand(f[k], a[k], b[k], c[k], e[k], so + k * 4096);
      }
      asm volatile("s_waitcnt vmcnt(0)" ::: "memory");
      if (lane == 0) {
        const unsigned w = tag | F_RAW;
        __builtin_amdgcn_raw_buffer_store_b128(u32x4{w, w, w, w}, rs, 0, fo, 17);
      }
      T2(1, n);
    }
    if (wave == W_DEC) TS(19);
   } else {
    __builtin_amdgcn_s_setprio(PPRIO);
    if (!(QX & 32)) pack_items((wave - W_DEC - NDEC + 8 * CPK) * NCW + bk * 8 + bx, NROUND);
    if (wave == NWAVE - 1) TS(31);
    while (NL && __builtin_popcount(__builtin_amdgcn_readfirstlane(lds_ld(ctl + C_Q * 4 + 4))) < 8) __builtin_amdgcn_s_sleep(32);
    __builtin_amdgcn_s_setprio(DPRIO);
   }
   for (int m = ((wave - W_DEC + NPK) % (NDEC + NPK)) * NCW + bk * 8 + bx; m < NL * 4 && !dead; m += (NDEC + NPK) * NCW) {
      const int n = NT0 - NL + (m >> 2), k = m & 3;
      unsigned so, fo, f;
      arrival(n, so, fo);
      so += k * 4096;
      for (unsigned spins = 1;; ++spins) {
        asm volatile("" ::: "memory");
        f = __builtin_amdgcn_readfirstlane(__builtin_amdgcn_raw_buffer_load_b32(rs, 0, (unsigned)OFF_F1 + n * F1S + k * 4, 17));
        if (((f ^ tag) >> 13) == 0u) break;
        if ((spins & 15) == 0 &&
            (__hip_atomic_load(tmo, __ATOMIC_RELAXED, __HIP_MEMORY_SCOPE_AGENT) == call ||
             __builtin_amdgcn_s_memrealtime() - t0 > DEADLINE_TICKS)) {
          if (lane == 0) __hip_atomic_store(tmo, call, __ATOMIC_RELAXED, __HIP_MEMORY_SCOPE_AGENT);
          dead = true;
          break;
        }
        __builtin_amdgcn_s_sleep(LSLEEP);
      }
      if (dead) break;
      T2(0, n);
      if (!(f & F_RAW)) {
        u32x4 a = __builtin_amdgcn_raw_buffer_load_b128(rs, lane * 16, so, 17);
        u32x4 b = __builtin_amdgcn_raw_buffer_load_b128(rs, lane * 16 + 1024, so, 17);
        unsigned e = __builtin_amdgcn_raw_buffer_load_b32(rs, lane * 4 + 2816, so, 17);
        u32x3 c = __builtin_amdgcn_raw_buffer_load_b96(rs, lane * 12 + 2048, so, 17);
        asm volatile("s_waitcnt vmcnt(0)" : "+v"(a), "+v"(b), "+v"(c), "+v"(e) :: "memory");
        expand(f, a, b, c, e, so);
      }
      asm volatile("s_waitcnt vmcnt(0)" ::: "memory");
      if (lane == 0) __builtin_amdgcn_raw_buffer_store_b32(tag | F_RAW, rs, 0, fo + k * 4, 17);
      T2(1, n);
   }
  }

  asm volatile("s_waitcnt lgkmcnt(0)" ::: "memory");
  __syncthreads();
  const u64 outp = (u64)(uintptr_t)P.out_all[lane & 7];
  gchar* const slp = (gchar*)P.ws_all[lane & 7];
  get_call();

  if (wave == 0) TS(13);
  auto global_max = [&](bool slow) -> unsigned {
    const u64* slots = reinterpret_cast<const u64*>(myws + OFF_SLOTS + (blockIdx.x & 7) * 512) + (lane & 7);
    const u64 t0 = __builtin_amdgcn_s_memrealtime();
    unsigned g;
    for (unsigned spins = 1;; ++spins) {
      const u64 v = __hip_atomic_load(slots, __ATOMIC_RELAXED, __HIP_MEMORY_SCOPE_SYSTEM);
      g = (unsigned)v;
      if (__all((unsigned)(v >> 32) >= call)) break;
      if ((spins & 15) == 0 &&
          (__hip_atomic_load(tmo, __ATOMIC_RELAXED, __HIP_MEMORY_SCOPE_AGENT) == call ||
           __builtin_amdgcn_s_memrealtime() - t0 > DEADLINE_TICKS)) {
        if (lane == 0) __hip_atomic_store(tmo, call, __ATOMIC_RELAXED, __HIP_MEMORY_SCOPE_AGENT);
        g = ~0u;
        break;
      }
      if (slow) __builtin_amdgcn_s_sleep(100); else __builtin_amdgcn_s_sleep(ASL);
    }
#pragma unroll
    for (int o = 4; o >= 1; o >>= 1) g = max(g, (unsigned)__shfl_xor((int)g, o));
    return g;
  };
  if (sender) {
    if (ER && wave == 15) {
      if (GOX && lane == 0)
        __hip_atomic_fetch_add(reinterpret_cast<unsigned*>(myws + OFF_GO), 1u, __ATOMIC_RELAXED, __HIP_MEMORY_SCOPE_AGENT);
      const unsigned g = global_max(true);
      if (lane == 0) { lds_st(ctl + C_GMAX * 4, g); lds_st(ctl + C_DEAD * 4, 1u); }
    } else if (ER && wave >= 1 && (wave - 1) * NSWG + (int)blockIdx.x < 512) {
      const int it = (wave - 1) * NSWG + blockIdx.x;
      const int src = it & 7, c64 = (it >> 3) & 31, hu = it >> 8;
      const int lr = rank * 128 + hu * 64 + (lane & 3) * 16;
      const u64* fl = reinterpret_cast<const u64*>(myws + OFF_EOF) +
                      ((src * 4 + lr / TM) * 16 + (c64 >> 1)) * 8 + (c64 & 1) * 4 + (lr % TM) / 80;
      while (__builtin_amdgcn_readfirstlane(lds_ld(ctl + C_DEAD * 4)) == 0u) __builtin_amdgcn_s_sleep(16);
      const u64 t0 = __builtin_amdgcn_s_memrealtime();
      bool dead = false;
      u64 v;
      for (unsigned spins = 1;; ++spins) {
        v = __hip_atomic_load(fl, __ATOMIC_RELAXED, __HIP_MEMORY_SCOPE_SYSTEM);
        if (__all((unsigned)(v >> 32) >= call)) break;
        if ((spins & 15) == 0 &&
            (__hip_atomic_load(tmo, __ATOMIC_RELAXED, __HIP_MEMORY_SCOPE_AGENT) == call ||
             __builtin_amdgcn_s_memrealtime() - t0 > DEADLINE_TICKS)) {
          if (lane == 0) __hip_atomic_store(tmo, call, __ATOMIC_RELAXED, __HIP_MEMORY_SCOPE_AGENT);
          dead = true;
          break;
        }
        __builtin_amdgcn_s_sleep(32);
      }
      const __amdgpu_buffer_rsrc_t rs = __builtin_amdgcn_make_buffer_rsrc(myws, 0, 0x7fffffff, 0x00020000);
      const unsigned so = (unsigned)OFF_ES + (unsigned)((src * 8 + hu * 4) * 32 + c64) * 2048;
      u32x4 h[4][2];
#pragma unroll
      for (int g = 0; g < 4; ++g) {
        h[g][0] = __builtin_amdgcn_raw_buffer_load_b128(rs, lane * 16, so + g * 65536, 17);
        h[g][1] = __builtin_amdgcn_raw_buffer_load_b128(rs, lane * 16 + 1024, so + g * 65536, 17);
      }
      const float amax = __uint_as_float(lds_ld(ctl + C_GMAX * 4));
      dead |= __float_as_uint(amax) == ~0u;
      const float scale = amax / 127.0f;
      const float inv = scale > 0.f ? 1.0f / scale : 0.f;
      const unsigned here = (unsigned)__ballot(dead || (lr % 80) / 16 < (int)(v & 7u)) & (ER >> (hu * 4));
      gchar* ob = (gchar*)P.out_all[rank] + (size_t)(hu * 64 + (lane & 15)) * NTOT + src * NLOC + c64 * 64 + (lane >> 4) * 16;
#pragma unroll
      for (int g = 0; g < 4; ++g) {
        if (!((here >> g) & 1u)) continue;
        const float step = __uint_as_float((unsigned)__builtin_amdgcn_readlane((int)v, g));
        u32x4 o;
#pragma unroll
        for (int j = 0; j < 4; ++j) {
          unsigned pk = 0;
#pragma unroll
          for (int nt = 0; nt < 4; ++nt) {
            const int w = (int)h[g][j >> 1][(j & 1) * 2 + (nt >> 1)];
            const float y = (float)((nt & 1) ? w >> 16 : (w << 16) >> 16) * step;
            float q = rintf(y * inv);
            q = fminf(fmaxf(q, -127.f), 127.f);
            const int qi = dead ? -128 : (int)q;
            pk |= ((unsigned)qi & 0xffu) << (8 * nt);
          }
          o[j] = pk;
        }
        st16(ob + (size_t)g * 16 * NTOT, o);
      }
      asm volatile("s_waitcnt vmcnt(0)" ::: "memory");
      TS(24);
    } else if (wave == 0 && blockIdx.x < NCOL) {
      const char* rp = myws + OFF_REP + lane * 16;
      const u64 t0 = __builtin_amdgcn_s_memrealtime();
      unsigned m;
      for (unsigned spins = 1;; ++spins) {
        u32x4 r0, r1;
        asm volatile("global_load_dwordx4 %0, %2, off sc0 sc1\n\tglobal_load_dwordx4 %1, %2, off offset:1024 sc0 sc1\n\t"
                     "s_waitcnt vmcnt(0)" : "=&v"(r0), "=&v"(r1) : "v"(rp) : "memory");
        if (lane >= (NCW - 128) / 2) r1 = r0;
        m = max(max(r0[0], r0[2]), max(r1[0], r1[2]));
        if (__all(r0[1] == call && r0[3] == call && r1[1] == call && r1[3] == call)) break;
        if ((spins & 15) == 0 && __builtin_amdgcn_s_memrealtime() - t0 > DEADLINE_TICKS) {
          if (lane == 0) __hip_atomic_store(tmo, call, __ATOMIC_RELAXED, __HIP_MEMORY_SCOPE_AGENT);
          m = ~0u;
          break;
        }
        __builtin_amdgcn_s_sleep(CSL);
      }
      TS(20);
#pragma unroll
      for (int o = 32; o >= 1; o >>= 1) m = max(m, (unsigned)__shfl_xor((int)m, o));
      __hip_atomic_store((gu64*)(slp + OFF_SLOTS + (lane >> 3) * 512) + rank, ((u64)call << 32) | m, __ATOMIC_RELAXED,
                         __HIP_MEMORY_SCOPE_SYSTEM);
    }
    return;
  }
  {
  int b2 = blockIdx.x, w2 = wave;
  asm volatile("" : "+s"(b2), "+s"(w2));
  const int bx2 = b2 & 7, bk2 = ((b2 >> 3) + 26) & 31;
  const int ti = bk2 < 24 ? 2 * (bk2 % 6) + (bx2 & 1) : 12;
  const int tj = 4 * (bx2 >> 1) + (bk2 < 24 ? bk2 / 6 : 2 * (bx2 & 1) + ((bk2 - 24) & 1));
  const int wm = w2 & 3, wn = (w2 >> 2) & 1;
  const bool mine = lane == rows_of((5 * ti) >> 1) >> 2 || lane == rows_of((5 * ti + 2) >> 1) >> 2 ||
                    lane == rows_of(min((5 * ti + 4) >> 1, 31)) >> 2;
  if (wave == 8) {
    unsigned m = lds_ld(ctl + (lane < 8 ? C_WMAX + lane : C_DEAD) * 4);
#pragma unroll
    for (int o = 8; o >= 1; o >>= 1) m = max(m, (unsigned)__shfl_xor((int)m, o));
    if (lane == 0)
      __hip_atomic_store((gu64*)(myws + OFF_REP) + (blockIdx.x - NSWG), ((u64)call << 32) | m, __ATOMIC_RELAXED,
                         __HIP_MEMORY_SCOPE_SYSTEM);
  }
  int ne = 0;
  if (ER && wave < 8 && ti * TM + wm * 80 < 1024) {
    if (wave == 0) TS(25);
    float tm = __uint_as_float(lds_ld(ctl + (C_WMAX + (lane & 7)) * 4));
#pragma unroll
    for (int o = 4; o >= 1; o >>= 1) tm = fmaxf(tm, __shfl_xor(tm, o));
    const unsigned sb = __float_as_uint(fmaxf(tm, 1e-30f) / 32000.0f) & ~7u;
    const float k16 = 1.0f / __uint_as_float(sb);
#pragma unroll
    for (int mt = 0; mt < 5; ++mt) {
      const int row0 = ti * TM + wm * 80 + mt * 16;
      if (ne != mt || __builtin_amdgcn_readfirstlane(lds_ld(ctl + C_ECNT * 4)) != 0u) continue;
      ne = mt + 1;
      if (row0 >= 1024 || !((ER >> ((row0 >> 4) & 7)) & 1)) continue;
      gchar* ob = (gchar*)(((u64)(unsigned)__builtin_amdgcn_readlane((int)((u64)(uintptr_t)slp >> 32), row0 >> 7) << 32) |
                           (unsigned)__builtin_amdgcn_readlane((int)(u64)(uintptr_t)slp, row0 >> 7));
      ob += OFF_ES + (size_t)((rank * 8 + ((row0 & 127) >> 4)) * 32 + tj * 2 + wn) * 2048 + lane * 16;
      u32x4 h[2];
#pragma unroll
      for (int j = 0; j < 4; ++j)
#pragma unroll
        for (int n2 = 0; n2 < 2; ++n2)
          h[j >> 1][(j & 1) * 2 + n2] = __builtin_amdgcn_perm((unsigned)(int)rintf(acc[mt][2 * n2 + 1][j] * k16),
                                                              (unsigned)(int)rintf(acc[mt][2 * n2][j] * k16), 0x05040100u);
      st16(ob, h[0]);
      if (EOTH == 2) asm volatile("s_waitcnt vmcnt(0)" ::: "memory");
      st16<1024>(ob, h[1]);
      asm volatile("s_waitcnt vmcnt(0)" ::: "memory");
    }
    if (lane == (ti * TM + wm * 80) >> 7 || lane == min(ti * TM + wm * 80 + 79, 1023) >> 7)
      __hip_atomic_store((gu64*)(slp + OFF_EOF) + ((rank * 4 + ti) * 16 + tj) * 8 + w2, ((u64)call << 32) | sb | (unsigned)ne,
                         __ATOMIC_RELAXED, __HIP_MEMORY_SCOPE_SYSTEM);
    TS(26);
    TV(27, ne);
  }
  if (wave == 8) {
    if (IDL) __builtin_amdgcn_s_sleep(IDL);
    const unsigned g = global_max(SPOLL && ER && ti < 4);
    TS(22);
    if (lane == 0) { lds_st(ctl + C_GMAX * 4, g); lds_st(ctl + C_ECNT * 4, 1u); }
  }
  if (SPOLL && ER && wave == 9 && ti < 4) {
    typedef __attribute__((ext_vector_type(16))) unsigned u32x16;
    const char* sp = myws + OFF_SLOTS + bx2 * 512;
    while (__builtin_amdgcn_readfirstlane(lds_ld(ctl + C_ECNT * 4)) == 0u) {
      u32x16 v;
      asm volatile("s_load_dwordx16 %0, %1, 0x0 glc\n\ts_waitcnt lgkmcnt(0)" : "=s"(v) : "s"(sp) : "memory");
      unsigned c = ~0u, g = 0;
#pragma unroll
      for (int i = 0; i < 8; ++i) { c = min(c, v[2 * i + 1]); g = max(g, v[2 * i]); }
      if (c >= call) {
        if (lane == 0) { lds_st(ctl + C_GMAX * 4, g); lds_st(ctl + C_ECNT * 4, 1u); }
        TS(34);
        break;
      }
      __builtin_amdgcn_s_sleep(SSL);
    }
  }
  if (wave < 8)
    while (__builtin_amdgcn_readfirstlane(lds_ld(ctl + C_ECNT * 4)) == 0u) __builtin_amdgcn_s_sleep(1);
  if (wave == 0) TS(14);
  if (wave < 8) {
    const float amax = __uint_as_float(lds_ld(ctl + C_GMAX * 4));
    const bool dead = __float_as_uint(amax) == ~0u;
    const float scale = amax / 127.0f;
    const float inv = scale > 0.f ? 1.0f / scale : 0.f;
    const int col = rank * NLOC + tj * 128 + wn * 64 + (lane >> 4) * 16;
#pragma unroll
    for (int mt = 0; mt < 5; ++mt) {
      const int lr0 = ti * TM + wm * 80 + mt * 16;
      if (lr0 >= 4096 || (mt < ne && lr0 < 1024 && ((ER >> ((lr0 >> 4) & 7)) & 1))) continue;
      const int row0 = rows_of(lr0 >> 7) * 128 + (lr0 & 127);
      const int owner = row0 >> 9;
      gchar* ob = (gchar*)(((u64)(unsigned)__builtin_amdgcn_readlane((int)(outp >> 32), owner) << 32) |
                           (unsigned)__builtin_amdgcn_readlane((int)outp, owner));
      u32x4 o;
#pragma unroll
      for (int j = 0; j < 4; ++j) {
        unsigned pk = 0;
#pragma unroll
        for (int nt = 0; nt < 4; ++nt) {
          float q = rintf(acc[mt][nt][j] * inv);
          q = fminf(fmaxf(q, -127.f), 127.f);
          int qi = dead ? -128 : (int)q;
          pk |= ((unsigned)qi & 0xffu) << (8 * nt);
        }
        o[j] = pk;
      }
      st16(ob + (size_t)((row0 & 511) + (lane & 15)) * NTOT + col, o);
      asm volatile("s_waitcnt vmcnt(0)" ::: "memory");
    }
    if (wave == 0) TS(17);
    asm volatile("s_waitcnt vmcnt(0)" ::: "memory");
    if (wave == 0) TS(18);
  }
  asm volatile("s_waitcnt lgkmcnt(0)" ::: "memory");
  __syncthreads();
  if (wave == 0) {
    TS(15);
    if (mine)
      __hip_atomic_store((gu32*)(slp + OFF_OUTF) + (rank * NTI + ti) * 16 + tj, call, __ATOMIC_RELAXED, __HIP_MEMORY_SCOPE_SYSTEM);
    if (blockIdx.x == 48) {
      const int tlo = (1024 + 384 * rank) / TM, thi = (1407 + 384 * rank) / TM;
      const __amdgpu_buffer_rsrc_t rs = __builtin_amdgcn_make_buffer_rsrc(myws, 0, 0x7fffffff, 0x00020000);
      const u64 t0 = __builtin_amdgcn_s_memrealtime();
      for (unsigned spins = 1;; ++spins) {
        bool ok = true;
        asm volatile("" ::: "memory");
#pragma unroll
        for (int j = 0; j < 3; ++j) {
          const int tt = j < 2 ? min(tlo + 2 * j + (lane >> 5), thi) : (2 * rank + (lane >> 5)) / 5;
          const u32x4 v = __builtin_amdgcn_raw_buffer_load_b128(rs, ((((lane >> 2) & 7) * NTI + tt) * 4 + (lane & 3)) * 16, (unsigned)OFF_OUTF, 17);
          ok &= min(min(v[0], v[1]), min(v[2], v[3])) >= call;
        }
        if (__all(ok)) break;
        if ((spins & 15) == 0 &&
            (__hip_atomic_load(tmo, __ATOMIC_RELAXED, __HIP_MEMORY_SCOPE_AGENT) == call ||
             __builtin_amdgcn_s_memrealtime() - t0 > DEADLINE_TICKS)) {
          if (lane == 0) {
            __hip_atomic_store(tmo, call, __ATOMIC_RELAXED, __HIP_MEMORY_SCOPE_AGENT);
            ((gchar*)(P.out_all[rank]))[rank * NLOC] = (char)-128;
          }
          break;
        }
        __builtin_amdgcn_s_sleep(1);
      }
      __builtin_amdgcn_fence(__ATOMIC_ACQUIRE, "");
      TS(16);
      }
    }
  }
}

extern "C" void kernel_launch(int rank, int nranks, ncclComm_t comm, hipStream_t stream,
                              void* const* d_in, const int* in_sizes, int n_in, void* d_out,
                              int out_size, const int* in_dtypes, int out_dtype,
                              void* const* d_out_all, void* const* d_ws_all, size_t ws_bytes) {
  if (nranks != NR || ws_bytes < WS_NEED || n_in < 2) {
    fprintf(stderr, "unexpected configuration\n");
    return;
  }
  Params P;
  P.x = static_cast<const float*>(d_in[0]);
  P.w = static_cast<const float*>(d_in[1]);
  P.out_all = d_out_all;
  P.ws_all = d_ws_all;
  P.rank = rank;
  hipLaunchKernelGGL(fused_gemm_q8, dim3(NWG), dim3(NTHREADS), 0, stream, P);
  HIP_CHECK(hipGetLastError());
}
```
